```python
import numpy as np

kernel_name = "gpu_gemm_ar_m4096_k4096_n16384_bf16_reluqe4m3"


ABSMAX_RTOL = 0.09

COMPUTE_DTYPE = "bfloat16"
EPILOGUE = "relu_quant_e4m3"
IN_QUANT = None
OUT_QUANT = "e4m3"
IN_DTYPES = {"x": "bfloat16", "w_mat": "bfloat16"}
WORLD_SIZE = 8

def _to_bf16(x):
    u = np.ascontiguousarray(x, dtype=np.float32).view(np.uint32)
    return ((u + 0x7FFF + ((u >> 16) & 1)) & 0xFFFF0000).view(np.float32)

def setup_inputs():

    x = _to_bf16(np.random.randn(4096, 4096).astype(np.float32))
    w_mat = _to_bf16((np.random.randn(4096, 16384) / np.sqrt(4096)).astype(np.float32))
    return {"x": x, "w_mat": w_mat}

def _snap_fp8(x, mant_bits, max_abs, min_normal):
    a = np.abs(x.astype(np.float64))
    _m, _e = np.frexp(a)
    q = np.where(a >= min_normal, np.ldexp(1.0, _e - 1 - mant_bits),
                 min_normal * 2.0 ** -mant_bits)
    snapped = np.minimum(np.round(a / q) * q, max_abs)
    return (np.sign(x) * snapped).astype(np.float32)
def _quant_dequant(y, amax):
    scale = amax / 448.0
    q = _snap_fp8(y / scale, 3, 448.0, 2.0 ** -6)
    return (q * scale).astype(np.float32)
def _epilogue(y):
    y = np.maximum(y, 0.0)
    return _quant_dequant(y, np.abs(y).max())

def reference(x, w_mat):
    y = (x @ w_mat).astype(np.float32)
    return _epilogue(y)
```

```cpp
#include <hip/hip_runtime.h>
#include <rccl/rccl.h>
#include <cstdio>

typedef unsigned u32;
typedef unsigned long long u64;
typedef u32 u32x4 __attribute__((ext_vector_type(4)));
typedef u32 u32x2 __attribute__((ext_vector_type(2)));
typedef unsigned short u16x2 __attribute__((ext_vector_type(2)));
typedef short bf16x8 __attribute__((ext_vector_type(8)));
typedef float f32x4 __attribute__((ext_vector_type(4)));
typedef __amdgpu_buffer_rsrc_t rsrc_t;
#define LAS __attribute__((address_space(3)))
#define GAS __attribute__((address_space(1)))
#define DI __device__ __forceinline__
#define RFL(x) __builtin_amdgcn_readfirstlane(x)
#define CFENCE() asm volatile("" ::: "memory")
#define VMWAIT0() asm volatile("s_waitcnt vmcnt(0)" ::: "memory")

namespace {
constexpr int NR = 8, MM = 4096, KS = 512, NN = 16384, NWG = 256;
constexpr int NBLK = 768;
constexpr u32 SYS = 17, AGT = 16;
constexpr u64 DEADLINE = 50000000ull;
constexpr u32 O_CALL = 0;
constexpr u32 O_TMO = 32768;
constexpr u32 O_TICK = O_TMO + 128;
constexpr u32 O_GT = O_TICK + 1024;
constexpr u32 O_LT = O_GT + 128;
constexpr u32 O_RF = O_LT + 2048;
constexpr u32 O_FX = O_RF + NR * NBLK * 4;
constexpr u32 O_FW = O_FX + 16 * 64 * 16;
constexpr u32 O_OF = O_FW + 16 * 64 * 8;
constexpr u32 O_DBG = 512 * 1024;
constexpr u32 O_OBX = 1u << 20;
constexpr u32 O_OBW = O_OBX + 512 * 8192;
constexpr u32 O_IMG = O_OBW + 2048 * 8192;
constexpr u32 O_OM = O_IMG + 64 * 96 * 8192;
constexpr u32 O_OV = O_OM + 256 * 4096;
constexpr u32 WS_USED = O_OV + 256 * 32768;
static_assert(O_OF + 256 * 64 <= O_DBG, "control block overflow");
constexpr int NS = 2, SLOTB = 49152;
constexpr u32 L_RING = 0;
constexpr u32 L_STG = 32768;
constexpr u32 L_MASK = NS * SLOTB;
constexpr u32 L_LUTC = L_MASK + 7 * 4096;
constexpr u32 L_LUTE = L_LUTC + 64;
constexpr u32 L_CTL = L_LUTE + 256;
enum { C_CALL = 0, C_ABORT, C_FULL0, C_FULL1, C_FULL2, C_FREE0, C_FREE1, C_FREE2, C_WGMAX, C_CMAX, C_CMASK, C_CVAL, C_GVALID, C_GMAX,
       C_MRDY, C_PROG, C_DFREE0, C_DFREE1, C_VSZ = 24, C_ORDER = 32, C_DSZ = 96, C_WORDS = 160 };
constexpr u32 LDS_BYTES = L_CTL + C_WORDS * 4;
static_assert(LDS_BYTES <= 163840 && LDS_BYTES > 81920, "one workgroup per CU");

struct Params { const unsigned short* x; const unsigned short* w; unsigned char* out; char* ws[NR]; int rank; int pad; };

DI u64 now() { return __builtin_amdgcn_s_memrealtime(); }
DI rsrc_t mkrs(const void* p, u32 bytes) { return __builtin_amdgcn_make_buffer_rsrc((void*)p, (short)0, (int)bytes, 0x00020000); }
template <u32 AUX> DI void st128(u32x4 v, rsrc_t rs, u32 off) { __builtin_amdgcn_raw_buffer_store_b128(v, rs, (int)off, 0, AUX); }
DI u32 prm(u32 a, u32 b, u32 s) { return __builtin_amdgcn_perm(a, b, s); }
DI u32 bfi(u32 m, u32 a, u32 b) { return (m & a) | (~m & b); }
DI u32 lds_ld(LAS u32* p) { return __hip_atomic_load(p, __ATOMIC_RELAXED, __HIP_MEMORY_SCOPE_WORKGROUP); }
DI void lds_st(LAS u32* p, u32 v) { __hip_atomic_store(p, v, __ATOMIC_RELAXED, __HIP_MEMORY_SCOPE_WORKGROUP); }
DI u32 lds_add(LAS u32* p, u32 v) { return __hip_atomic_fetch_add(p, v, __ATOMIC_RELAXED, __HIP_MEMORY_SCOPE_WORKGROUP); }
DI u32 g_ld_agent(const char* p) { return __hip_atomic_load((GAS u32*)p, __ATOMIC_RELAXED, __HIP_MEMORY_SCOPE_AGENT); }
DI u32 g_ld_sys(const char* p) { return __hip_atomic_load((GAS u32*)p, __ATOMIC_RELAXED, __HIP_MEMORY_SCOPE_SYSTEM); }
DI void g_st_agent(char* p, u32 v) { __hip_atomic_store((GAS u32*)p, v, __ATOMIC_RELAXED, __HIP_MEMORY_SCOPE_AGENT); }
DI void g_st_sys(char* p, u32 v) { __hip_atomic_store((GAS u32*)p, v, __ATOMIC_RELAXED, __HIP_MEMORY_SCOPE_SYSTEM); }

struct Ctx {
  LAS unsigned char* lds; LAS u32* ctl; char* ws; u64 t0; u32 call; int lane, wg, rank;
};
DI void give_up(const Ctx& c) { if (c.lane == 0) { lds_st(c.ctl + C_ABORT, 1u); if (c.call) g_st_agent(c.ws + O_TMO, c.call); } }
DI bool expired(const Ctx& c) {
  if (RFL(lds_ld(c.ctl + C_ABORT)) != 0u || RFL(g_ld_agent(c.ws + O_TMO)) == c.call || now() - c.t0 > DEADLINE) { give_up(c); return true; }
  return false;
}
DI u32 lds_wait(const Ctx& c, int word, u32 target) {
  u32 v, spins = 0;
  for (;;) {
    v = RFL(lds_ld(c.ctl + word));
    if (v >= target) break;
    if ((++spins & 63u) == 0u && (RFL(lds_ld(c.ctl + C_ABORT)) != 0u || now() - c.t0 > DEADLINE + DEADLINE / 4)) { give_up(c); break; }
    __builtin_amdgcn_s_sleep(1);
  }
  CFENCE();
  return v;
}
#define STAMP(c, i) do {} while (0)
DI void dma4(rsrc_t rs, u32 voff, u32 soff, u32 ldsdst) {
  u32 keep;
  asm volatile("s_nop 4\n\ts_mov_b32 %0, m0\n\ts_mov_b32 m0, %1\n\ts_nop 0\n\t"
               "buffer_load_dwordx4 %2, %3, %4 offen sc0 sc1 lds\n\t"
               "buffer_load_dwordx4 %2, %3, %4 offen offset:1024 sc0 sc1 lds\n\t"
               "buffer_load_dwordx4 %2, %3, %4 offen offset:2048 sc0 sc1 lds\n\t"
               "buffer_load_dwordx4 %2, %3, %4 offen offset:3072 sc0 sc1 lds\n\t"
               "s_mov_b32 m0, %0"
               : "=&s"(keep) : "s"(ldsdst), "v"(voff), "s"(rs), "s"(soff) : "memory");
}
DI void dma2(rsrc_t rs, u32 voff, u32 soff, u32 ldsdst) {
  u32 keep;
  asm volatile("s_nop 4\n\ts_mov_b32 %0, m0\n\ts_mov_b32 m0, %1\n\ts_nop 0\n\t"
               "buffer_load_dwordx4 %2, %3, %4 offen sc0 sc1 lds\n\t"
               "buffer_load_dwordx4 %2, %3, %4 offen offset:1024 sc0 sc1 lds\n\t"
               "s_mov_b32 m0, %0"
               : "=&s"(keep) : "s"(ldsdst), "v"(voff), "s"(rs), "s"(soff) : "memory");
}
DI u32 wave_incl_scan(u32 v, int lane) {
#pragma unroll
  for (int d = 1; d < 64; d <<= 1) { u32 t = __shfl_up(v, d); if (lane >= d) v += t; }
  return v;
}
DI u32 wave_max(u32 v) {
#pragma unroll
  for (int d = 32; d >= 1; d >>= 1) { u32 t = __shfl_xor(v, d); v = v > t ? v : t; }
  return v;
}
DI u32 row_bytes(u32 V) { return (((V + 7u) >> 3) + 3u) & ~3u; }
DI u32 sent_bytes(u32 V) { return (7u * row_bytes(V) + 15u) & ~15u; }
DI void image_flag(const Ctx& c, int kstep, int r) {
  if (c.lane == 0) g_st_agent(c.ws + (r < 64 ? O_FX + ((r >> 2) * 64 + kstep) * 16 + (r & 3) * 4 : O_FW + (((r - 64) >> 1) * 64 + kstep) * 8 + (r & 1) * 4), c.call);
}

DI void role_packer(Ctx& c, const Params& p, int pw) {
  const int lane = c.lane;
  const rsrc_t rsL = mkrs(c.ws, WS_USED);
  char* peer = p.ws[lane & 7];
  for (int i = 0; i < 4; ++i) {
    const int q = i * 768 + pw * 256 + c.wg;
    if (q >= 2560) break;
    const int j = q / 320, r = q - j * 320;
    const bool isx = r < 64;
    const int dest = (r - 64) & 7, ni = (r - 64) >> 4, kh = ((r - 64) >> 3) & 1;
    u32 lo[16], hi[16];
    if (isx) {
      u32x4 d[8];
      const unsigned short* src = p.x + (size_t)(64 * r + (lane & 15)) * KS + 64 * j + 8 * (lane >> 4);
#pragma unroll
      for (int t = 0; t < 8; ++t) d[t] = *(const u32x4*)(src + (size_t)(16 * (t & 3)) * KS + 32 * (t >> 2));
#pragma unroll
      for (int t = 0; t < 8; ++t) {
        lo[2 * t] = prm(d[t].y, d[t].x, 0x06040200u); lo[2 * t + 1] = prm(d[t].w, d[t].z, 0x06040200u);
        hi[2 * t] = prm(d[t].y, d[t].x, 0x07050301u); hi[2 * t + 1] = prm(d[t].w, d[t].z, 0x07050301u);
      }
    } else {
      u32x2 d[2][8];
      const unsigned short* src = p.w + (size_t)(64 * j + 32 * kh + 8 * (lane >> 4)) * NN + dest * 2048 + ni * 128 + 4 * (lane & 15);
#pragma unroll
      for (int h = 0; h < 2; ++h)
#pragma unroll
        for (int k = 0; k < 8; ++k) d[h][k] = *(const u32x2*)(src + (size_t)k * NN + 64 * h);
#pragma unroll
      for (int h = 0; h < 2; ++h)
#pragma unroll
        for (int cc = 0; cc < 2; ++cc)
#pragma unroll
          for (int u = 0; u < 2; ++u) {
            const u32 a0 = d[h][4 * u][cc], a1 = d[h][4 * u + 1][cc], a2 = d[h][4 * u + 2][cc], a3 = d[h][4 * u + 3][cc];
            const u32 l0 = prm(a1, a0, 0x06020400u), l1 = prm(a3, a2, 0x06020400u), h0 = prm(a1, a0, 0x07030501u), h1 = prm(a3, a2, 0x07030501u);
            const int t = 4 * h + 2 * cc;
            lo[2 * t + u] = prm(l1, l0, 0x05040100u); lo[2 * t + 2 + u] = prm(l1, l0, 0x07060302u);
            hi[2 * t + u] = prm(h1, h0, 0x05040100u); hi[2 * t + 2 + u] = prm(h1, h0, 0x07060302u);
          }
    }
    const int lr = isx ? r : 64 + 2 * ni + kh;
    const bool own = isx || dest == c.rank, send = isx || dest != c.rank;
    u32 info = 0;
    if (send) {
      const u32 slot = (isx ? O_OBX + (u32)(j * 64 + r) * 8192u : O_OBW + (u32)((dest * 8 + j) * 32 + 2 * ni + kh) * 8192u);
      u16x2 m0 = {0, 0}, m1 = {0, 0};
#pragma unroll
      for (int k = 0; k < 16; ++k) {
        m0 = __builtin_elementwise_max(m0, __builtin_bit_cast(u16x2, hi[k] & 0x007F007Fu));
        m1 = __builtin_elementwise_max(m1, __builtin_bit_cast(u16x2, hi[k] & 0x7F007F00u));
      }
      u32 T = max(max((u32)m0.x, (u32)m0.y), max((u32)m1.x >> 8, (u32)m1.y >> 8));
      T = RFL(wave_max(T));
      const u32 base = T > 7u ? T - 7u : 0u, base4 = base * 0x01010101u;
      u32 cd[16], all = 0xFFFFFFFFu;
#pragma unroll
      for (int k = 0; k < 16; ++k) { cd[k] = (hi[k] | 0x80808080u) - base4; all &= cd[k]; }
      u32 cnt = 0;
      if (__any((~all & 0x80808080u) != 0u)) {
#pragma unroll
        for (int k = 0; k < 16; ++k) {
          const u32 em = ~cd[k] & 0x80808080u;
          if (__any(em != 0u)) {
#pragma unroll
            for (int b = 0; b < 4; ++b) {
              const bool e = (em >> (8 * b + 7)) & 1u;
              const u64 m = __ballot(e);
              const u32 pos = cnt + __builtin_amdgcn_mbcnt_hi((u32)(m >> 32), __builtin_amdgcn_mbcnt_lo((u32)m, 0u));
              if (e && pos < 30u)
                __builtin_amdgcn_raw_buffer_store_b32((u32)(k * 4 + b) << 14 | (u32)lane << 8 | ((hi[k] >> (8 * b)) & 255u), rsL, (int)(slot + 6144u + pos * 4u), 0, SYS);
              cnt += (u32)__popcll(m);
            }
          }
        }
      }
      if (cnt <= 30u) {
#pragma unroll
        for (int hh = 0; hh < 2; ++hh) {
          u32x4 nb;
#pragma unroll
          for (int t = 0; t < 4; ++t) {
            const int k = 8 * hh + 2 * t;
            const u32 e0 = bfi(0x80808080u, hi[k], cd[k]) & 0x87878787u, e1 = bfi(0x80808080u, hi[k + 1], cd[k + 1]) & 0x87878787u;
            nb[t] = e0 | __builtin_amdgcn_alignbit(e1, e1, 28);
          }
          const u32x4 a = {lo[8 * hh], lo[8 * hh + 1], lo[8 * hh + 2], lo[8 * hh + 3]}, b = {lo[8 * hh + 4], lo[8 * hh + 5], lo[8 * hh + 6], lo[8 * hh + 7]};
          st128<SYS>(a, rsL, lane * 16 + (slot + hh * 3072));
          st128<SYS>(b, rsL, lane * 16 + (slot + hh * 3072 + 1024));
          st128<SYS>(nb, rsL, lane * 16 + (slot + hh * 3072 + 2048));
        }
        info = base << 5 | cnt;
      } else {
#pragma unroll
        for (int t = 0; t < 8; ++t) {
          const u32x4 v = {prm(hi[2 * t], lo[2 * t], 0x05010400u), prm(hi[2 * t], lo[2 * t], 0x07030602u),
                           prm(hi[2 * t + 1], lo[2 * t + 1], 0x05010400u), prm(hi[2 * t + 1], lo[2 * t + 1], 0x07030602u)};
          st128<SYS>(v, rsL, lane * 16 + (slot + t * 1024));
        }
        info = 31u;
      }
    }
    if (i == 0) c.call = lds_wait(c, C_CALL, 1u);
    if (send) {
      VMWAIT0();
      if (lane < NR && lane != c.rank && (isx || lane == dest)) g_st_sys(peer + O_RF + (c.rank * NBLK + j * 96 + lr) * 4, c.call << 12 | info);
    }
    if (own) {
      const u32 img = O_IMG + (u32)((j * 8 + c.rank) * 96 + lr) * 8192u;
#pragma unroll
      for (int t = 0; t < 8; ++t) {
        const u32x4 v = {prm(hi[2 * t], lo[2 * t], 0x05010400u), prm(hi[2 * t], lo[2 * t], 0x07030602u),
                         prm(hi[2 * t + 1], lo[2 * t + 1], 0x05010400u), prm(hi[2 * t + 1], lo[2 * t + 1], 0x07030602u)};
        st128<AGT>(v, rsL, lane * 16 + (img + t * 1024));
      }
      VMWAIT0();
      image_flag(c, j * 8 + c.rank, lr);
    }
    if (i == 0 && pw == 0) STAMP(c, 1);
  }
  if (pw == 0) STAMP(c, 2);
}

struct Half { u32x4 v[4]; };
DI u32 ob_slot(int b, int me) { const int j = b / 96, r = b - j * 96; return r < 64 ? O_OBX + (u32)(j * 64 + r) * 8192u : O_OBW + (u32)((me * 8 + j) * 32 + r - 64) * 8192u; }
DI void pull_half(Half& d, rsrc_t rsS, u32 slot, int hh, bool raw, int lane) {
  if (raw) {
#pragma unroll
    for (int t = 0; t < 4; ++t) d.v[t] = __builtin_amdgcn_raw_buffer_load_b128(rsS, lane * 16, (int)(slot + hh * 4096 + t * 1024), SYS);
  } else {
#pragma unroll
    for (int t = 0; t < 3; ++t) d.v[t] = __builtin_amdgcn_raw_buffer_load_b128(rsS, lane * 16, (int)(slot + hh * 3072 + t * 1024), SYS);
  }
}
DI void decode_half(const Half& d, u32 info, u32 exc, int hh, rsrc_t rsL, u32 img, int lane) {
  const u32 cnt = info & 31u;
  if (cnt == 31u) {
#pragma unroll
    for (int t = 0; t < 4; ++t) st128<AGT>(d.v[t], rsL, lane * 16 + (img + hh * 4096 + t * 1024));
    return;
  }
  const u32 base4 = ((info >> 5) & 127u) * 0x01010101u;
  u32 hi[8];
#pragma unroll
  for (int t = 0; t < 4; ++t) {
    const u32 n = d.v[2][t];
    hi[2 * t] = (n & 0x87878787u) + base4;
    hi[2 * t + 1] = (__builtin_amdgcn_alignbit(n, n, 4) & 0x87878787u) + base4;
  }
  for (u32 e = 0; e < cnt; ++e) {
    const u32 rec = __builtin_amdgcn_readlane(exc, e);
    const u32 k = (rec >> 16) & 15u, sh = ((rec >> 14) & 3u) * 8u;
    const u32 m = ((rec >> 8) & 63u) == (u32)lane ? 255u << sh : 0u, val = (rec & 255u) << sh;
#pragma unroll
    for (int i = 0; i < 8; ++i) hi[i] = bfi(k == (u32)(8 * hh + i) ? m : 0u, val, hi[i]);
  }
#pragma unroll
  for (int t = 0; t < 4; ++t) {
    const u32 l0 = d.v[t >> 1][2 * (t & 1)], l1 = d.v[t >> 1][2 * (t & 1) + 1];
    const u32x4 v = {prm(hi[2 * t], l0, 0x05010400u), prm(hi[2 * t], l0, 0x07030602u), prm(hi[2 * t + 1], l1, 0x05010400u), prm(hi[2 * t + 1], l1, 0x07030602u)};
    st128<AGT>(v, rsL, lane * 16 + (img + hh * 4096 + t * 1024));
  }
}
DI u32 wait_rf(const Ctx& c, const char* f, u32 v) {
  while ((int)((v & ~0xFFFu) - (c.call << 12)) < 0) {
    if (expired(c)) break;
    __builtin_amdgcn_s_sleep(2);
    v = RFL(g_ld_sys(f));
  }
  CFENCE();
  return v & 0xFFFu;
}
DI u32 wait_of(const Ctx& c, const char* f) {
  u32 v;
  while ((v = RFL(g_ld_sys(f))) < c.call) { if (expired(c)) break; __builtin_amdgcn_s_sleep(2); }
  CFENCE();
  return v;
}
DI void role_puller(Ctx& c, const Params& p) {
  const int lane = c.lane;
  c.call = lds_wait(c, C_CALL, 1u);
  const rsrc_t rsL = mkrs(c.ws, WS_USED);
  {
    const int link = c.wg % 7, src = (c.rank + 1 + link) & 7;
    const u32 nw = (NWG - link + 6) / 7, tbase = (c.call - 1u) * (NBLK + 2u * nw);
    const rsrc_t rsS = mkrs(p.ws[src], WS_USED);
    GAS u32* tk = (GAS u32*)(c.ws + O_TICK + link * 128);
    const char* rf = c.ws + O_RF + src * NBLK * 4;
    u32 t = 0;
    if (lane == 0) t = __hip_atomic_fetch_add(tk, 2u, __ATOMIC_RELAXED, __HIP_MEMORY_SCOPE_AGENT);
    u32 B = RFL(t) - tbase, B1 = B + 1u, info = 0, exc = 0;
    int pk = -1, pr = 0;
    Half A, Bh;
    if (B < NBLK) {
      info = wait_rf(c, rf + B * 4, RFL(g_ld_sys(rf + B * 4)));
      STAMP(c, 3);
      const u32 slot = ob_slot(B, c.rank);
      pull_half(A, rsS, slot, 0, (info & 31u) == 31u, lane);
      if ((u32)lane < (info & 31u)) exc = __builtin_amdgcn_raw_buffer_load_b32(rsS, lane * 4, (int)(slot + 6144u), SYS);
    }
    while (B < NBLK) {
      const int j = B / 96, r = B - j * 96, kstep = j * 8 + src;
      const u32 img = O_IMG + (u32)(kstep * 96 + r) * 8192u;
      VMWAIT0();
      if (pk >= 0) image_flag(c, pk, pr);
      pull_half(Bh, rsS, ob_slot(B, c.rank), 1, (info & 31u) == 31u, lane);
      u32 f1 = 0;
      if (B1 < NBLK) f1 = g_ld_sys(rf + B1 * 4);
      decode_half(A, info, exc, 0, rsL, img, lane);
      VMWAIT0();
      u32 info1 = 0, exc1 = 0;
      if (B1 < NBLK) {
        info1 = wait_rf(c, rf + B1 * 4, RFL(f1));
        const u32 slot = ob_slot(B1, c.rank);
        pull_half(A, rsS, slot, 0, (info1 & 31u) == 31u, lane);
        if ((u32)lane < (info1 & 31u)) exc1 = __builtin_amdgcn_raw_buffer_load_b32(rsS, lane * 4, (int)(slot + 6144u), SYS);
      }
      if (lane == 0) t = __hip_atomic_fetch_add(tk, 1u, __ATOMIC_RELAXED, __HIP_MEMORY_SCOPE_AGENT);
      decode_half(Bh, info, exc, 1, rsL, img, lane);
      pk = kstep; pr = r; info = info1; exc = exc1; B = B1; B1 = RFL(t) - tbase;
    }
    VMWAIT0();
    if (pk >= 0) image_flag(c, pk, pr);
  }
  STAMP(c, 4);
  const char* of = c.ws + O_OF + c.wg * 64;
  u32 tot[7];
#pragma unroll
  for (int i = 0; i < 7; ++i) {
    const int s = (c.rank + 1 + (c.wg + i) % 7) & 7;
    wait_of(c, of + s * 4);
    if (i == 0) STAMP(c, 5);
    const rsrc_t rs = mkrs(p.ws[s] + O_OM + c.wg * 4096, 4096);
    const u32 dst = (u32)(size_t)c.lds + L_MASK + i * 4096;
    dma2(rs, lane * 16, 0, dst); VMWAIT0();
    dma2(rs, lane * 16, 2048, dst + 2048); VMWAIT0();
    u32 v = 0;
#pragma unroll
    for (int k = 0; k < 4; ++k) {
      const u32x4 m = *(LAS u32x4*)(c.lds + L_MASK + i * 4096 + lane * 64 + k * 16);
      v += __popc(m.x) + __popc(m.y) + __popc(m.z) + __popc(m.w);
    }
    v += __shfl_xor(v, 1); v += __shfl_xor(v, 2); v += __shfl_xor(v, 4);
    u32 sb = sent_bytes(v);
    u32 inc = sb;
    { u32 u = __shfl_up(inc, 8); if (lane >= 8) inc += u; u = __shfl_up(inc, 16); if (lane >= 16) inc += u; u = __shfl_up(inc, 32); if (lane >= 32) inc += u; }
    if ((lane & 7) == 0) lds_st(c.ctl + C_DSZ + i * 8 + (lane >> 3), inc - sb);
    tot[i] = __builtin_amdgcn_readlane(inc, 63);
    CFENCE();
    if (lane == 0) lds_st(c.ctl + C_MRDY, i + 1);
  }
  STAMP(c, 6);
#pragma unroll
  for (int i = 0; i < 7; ++i) {
    const int s = (c.rank + 1 + (c.wg + i) % 7) & 7;
    wait_of(c, of + 32 + s * 4);
    if (i == 0) STAMP(c, 7);
    if (i >= 2) lds_wait(c, C_DFREE0 + (i & 1), 8u * (i >> 1));
    const rsrc_t rs = mkrs(p.ws[s] + O_OV + c.wg * 32768, tot[i]);
    const u32 dst = (u32)(size_t)c.lds + L_STG + (i & 1) * 32768;
    for (u32 q = 0; q < tot[i]; q += 2048u) {
      dma2(rs, lane * 16, q, dst + q); VMWAIT0();
      if (lane == 0) lds_st(c.ctl + C_PROG, i * 65536u + q + 2048u);
    }
    if (lane == 0) lds_st(c.ctl + C_PROG, i * 65536u + 65535u);
  }
  STAMP(c, 8);
}

DI void role_sched(Ctx& c, const Params& p, int mi, int ni) {
  const int lane = c.lane;
  u32 t = 0;
  if (lane == 0) t = __hip_atomic_fetch_add((GAS u32*)(c.ws + O_CALL + c.wg * 128), 1u, __ATOMIC_RELAXED, __HIP_MEMORY_SCOPE_AGENT) + 1u;
  c.call = RFL(t);
  if (lane == 0) lds_st(c.ctl + C_CALL, c.call);
  const rsrc_t rsL = mkrs(c.ws, WS_USED);
  u64 issued = 0; u32 n = 0;
  while (n < 64u) {
    const u32x4 a = __builtin_amdgcn_raw_buffer_load_b128(rsL, lane * 16, (int)(O_FX + mi * 1024), SYS);
    const u32x2 b = __builtin_amdgcn_raw_buffer_load_b64(rsL, lane * 8, (int)(O_FW + ni * 512), SYS);
    const bool rdy = min(min(min(a.x, a.y), min(a.z, a.w)), min(b.x, b.y)) >= c.call;
    u64 m = __ballot(rdy) & ~issued;
    if (m == 0) {
      if (!expired(c)) { __builtin_amdgcn_s_sleep(2); CFENCE(); continue; }
      m = ~issued;
    }
    if ((m >> lane) & 1) lds_st(c.ctl + C_ORDER + n + __builtin_amdgcn_mbcnt_hi((u32)(m >> 32), __builtin_amdgcn_mbcnt_lo((u32)m, 0u)), lane + 1);
    issued |= m; n += (u32)__popcll(m);
    CFENCE();
  }
  STAMP(c, 9);
  if (c.wg == 0) {
    u32 mx;
    for (;;) {
      const u32x4 a = __builtin_amdgcn_raw_buffer_load_b128(rsL, lane * 16, (int)O_LT, SYS);
      const u32x4 b = __builtin_amdgcn_raw_buffer_load_b128(rsL, lane * 16, (int)(O_LT + 1024), SYS);
      mx = max(max(a.x, a.z), max(b.x, b.z));
      if (__all(min(min(a.y, a.w), min(b.y, b.w)) >= c.call) || expired(c)) break;
      __builtin_amdgcn_s_sleep(1); CFENCE();
    }
    mx = wave_max(mx);
    if (lane < NR) __hip_atomic_store((GAS u64*)(p.ws[lane] + O_GT + c.rank * 8), (u64)c.call << 32 | mx, __ATOMIC_RELAXED, __HIP_MEMORY_SCOPE_SYSTEM);
  }
  lds_wait(c, C_CMAX, 8u);
  u32 g;
  for (;;) {
    const u64 v = __hip_atomic_load((GAS u64*)(c.ws + O_GT + (lane & 7) * 8), __ATOMIC_RELAXED, __HIP_MEMORY_SCOPE_SYSTEM);
    g = (u32)v;
    if (__all((u32)(v >> 32) >= c.call) || expired(c)) break;
    __builtin_amdgcn_s_sleep(1);
  }
  g = wave_max(g);
  if (lane == 0) { lds_st(c.ctl + C_GMAX, g); CFENCE(); lds_st(c.ctl + C_GVALID, 1u); }
  STAMP(c, 10);
}

DI void role_loader(Ctx& c, int li, int mi, int ni) {
  const rsrc_t rsL = mkrs(c.ws, WS_USED);
  const u32 part = O_IMG + (li < 2 ? mi * 32768 + li * 16384 : 64 * 8192 + ni * 16384);
  const u32 voff = c.lane * 16, ldsb = (u32)(size_t)c.lds + L_RING + li * 16384;
  u32 issued = 0, pub = 0;
  while (pub < 64u) {
    bool go = false; u32 o = 0;
    if (issued < 64u && issued - pub < 2u) {
      const u32 slot = issued % NS;
      if (issued == pub) { o = lds_wait(c, C_ORDER + issued, 1u); lds_wait(c, C_FREE0 + slot, 8u * (issued / NS)); go = true; }
      else { o = RFL(lds_ld(c.ctl + C_ORDER + issued)); go = o != 0u && RFL(lds_ld(c.ctl + C_FREE0 + slot)) >= 8u * (issued / NS); CFENCE(); }
      if (go) {
        const u32 so = part + ((o - 1u) & 63u) * (96u * 8192u), ld = ldsb + slot * SLOTB;
        dma4(rsL, voff, so, ld); dma4(rsL, voff, so + 4096u, ld + 4096u); dma4(rsL, voff, so + 8192u, ld + 8192u); dma4(rsL, voff, so + 12288u, ld + 12288u);
        ++issued;
      }
    }
    if (!go || issued - pub == 2u) {
      if (issued - pub == 2u) asm volatile("s_waitcnt vmcnt(16)" ::: "memory"); else VMWAIT0();
      if (c.lane == 0) lds_add(c.ctl + C_FULL0 + pub % NS, 1u);
      ++pub;
    }
  }
}

DI void role_mfma(Ctx& c, const Params& p, int wv, int mi, int ni) {
  const int lane = c.lane, mw = wv >> 1, h = wv & 1;
  f32x4 acc[4][4];
#pragma unroll
  for (int g = 0; g < 4; ++g)
#pragma unroll
    for (int b = 0; b < 4; ++b) acc[g][b] = (f32x4){0.f, 0.f, 0.f, 0.f};
  LAS unsigned char* xb = c.lds + L_RING + mw * 8192 + lane * 16;
  LAS unsigned char* wb = c.lds + L_RING + 32768 + h * 4096 + lane * 16;
  for (u32 n = 0; n < 64u; ++n) {
    const u32 slot = n % NS;
    lds_wait(c, C_FULL0 + slot, 3u * (n / NS + 1u));
#pragma unroll
    for (int kh = 0; kh < 2; ++kh) {
      bf16x8 xf[4], wf[4];
#pragma unroll
      for (int g = 0; g < 4; ++g) wf[g] = *(LAS bf16x8*)(wb + slot * SLOTB + kh * 8192 + g * 1024);
#pragma unroll
      for (int b = 0; b < 4; ++b) xf[b] = *(LAS bf16x8*)(xb + slot * SLOTB + kh * 4096 + b * 1024);
      if (kh == 1) { CFENCE(); if (lane == 0) lds_add(c.ctl + C_FREE0 + slot, 1u); CFENCE(); }
#pragma unroll
      for (int b = 0; b < 4; ++b)
#pragma unroll
        for (int g = 0; g < 4; ++g) acc[g][b] = __builtin_amdgcn_mfma_f32_16x16x32_bf16(wf[g], xf[b], acc[g][b], 0, 0, 0);
    }
  }
  if (wv == 0) STAMP(c, 11);
  c.call = RFL(lds_ld(c.ctl + C_CALL));
  char* peer = p.ws[lane & 7];
  const rsrc_t rsL = mkrs(c.ws, WS_USED);
  float mx = 0.f; u32 nz[2] = {0u, 0u};
#pragma unroll
  for (int e = 63; e >= 0; --e) {
    const float y = acc[e & 3][e >> 4][(e >> 2) & 3];
    mx = fmaxf(mx, y);
    nz[e >> 5] = __builtin_amdgcn_alignbit(nz[e >> 5], __float_as_uint(y), 31);
  }
  nz[0] = ~nz[0]; nz[1] = ~nz[1];
  const u32 cnt = __popc(nz[0]) + __popc(nz[1]);
  const u32 inc = wave_incl_scan(cnt, lane), V = __builtin_amdgcn_readlane(inc, 63);
  const u32 wmx = wave_max(__float_as_uint(mx));
  if (lane == 0) {
    lds_st(c.ctl + C_VSZ + wv, sent_bytes(V));
    __hip_atomic_fetch_max(c.ctl + C_WGMAX, wmx, __ATOMIC_RELAXED, __HIP_MEMORY_SCOPE_WORKGROUP);
    CFENCE();
    if (lds_add(c.ctl + C_CMAX, 1u) == 7u)
      __hip_atomic_store((GAS u64*)(c.ws + O_LT + c.wg * 8), (u64)c.call << 32 | lds_ld(c.ctl + C_WGMAX), __ATOMIC_RELAXED, __HIP_MEMORY_SCOPE_AGENT);
  }
  __builtin_amdgcn_raw_buffer_store_b64((u32x2){nz[0], nz[1]}, rsL, lane * 8, (int)(O_OM + c.wg * 4096 + wv * 512), SYS);
  VMWAIT0();
  {
    u32 t = 0;
    if (lane == 0) t = lds_add(c.ctl + C_CMASK, 1u);
    if (RFL(t) == 7u && lane < NR && lane != c.rank) g_st_sys(peer + O_OF + c.wg * 64 + c.rank * 4, c.call);
  }
  lds_wait(c, C_CMAX, 8u);
  LAS unsigned char* st = c.lds + wv * 4096;
#pragma unroll
  for (int k = 0; k < 4; ++k) *(LAS u32x4*)(st + k * 1024 + lane * 16) = (u32x4){0u, 0u, 0u, 0u};
  u32 woff = 0;
  for (int k = 0; k < wv; ++k) woff += RFL(lds_ld(c.ctl + C_VSZ + k));
  lds_wait(c, C_GVALID, 1u);
  const u32 tm = g_ld_agent(c.ws + O_TMO);
  const float amax = __uint_as_float(RFL(lds_ld(c.ctl + C_GMAX)));
  const float inv = amax > 0.f ? 448.0f / amax : 0.f;
  u32 q[16];
#pragma unroll
  for (int D = 0; D < 16; ++D) {
    const int b = D >> 2, r = D & 3;
    int v = __builtin_amdgcn_cvt_pk_fp8_f32(fmaxf(acc[0][b][r], 0.f) * inv, fmaxf(acc[1][b][r], 0.f) * inv, 0, false);
    v = __builtin_amdgcn_cvt_pk_fp8_f32(fmaxf(acc[2][b][r], 0.f) * inv, fmaxf(acc[3][b][r], 0.f) * inv, v, true);
    q[D] = (u32)v;
  }
  {
    u32 o = inc - cnt;
#pragma unroll
    for (int D = 0; D < 16; ++D) {
      const u32 nib = (nz[D >> 3] >> (4 * (D & 7))) & 15u;
      const u32 c4 = prm(0u, q[D], *(LAS u32*)(c.lds + L_LUTC + nib * 4));
      const u64 v = (u64)c4 << ((o & 3u) * 8u);
      LAS u32* d = (LAS u32*)(st + (o & ~3u));
      __hip_atomic_fetch_or(d, (u32)v, __ATOMIC_RELAXED, __HIP_MEMORY_SCOPE_WORKGROUP);
      if ((u32)(v >> 32)) __hip_atomic_fetch_or(d + 1, (u32)(v >> 32), __ATOMIC_RELAXED, __HIP_MEMORY_SCOPE_WORKGROUP);
      o += __popc(nib);
    }
    CFENCE();
    const u32 ncol = row_bytes(V) >> 2, sb = sent_bytes(V);
    for (u32 col = lane; col < ncol; col += 64u) {
      LAS u32* s = (LAS u32*)st + col;
      const u32 d7 = s[7u * ncol];
#pragma unroll
      for (u32 k = 0; k < 7u; ++k) s[k * ncol] |= (d7 << (7u - k)) & 0x80808080u;
    }
    CFENCE();
    for (u32 k = lane * 16; k < sb; k += 1024u)
      st128<SYS>(*(LAS u32x4*)(st + k), rsL, k + (O_OV + c.wg * 32768 + woff));
    VMWAIT0();
    u32 t = 0;
    if (lane == 0) t = lds_add(c.ctl + C_CVAL, 1u);
    if (RFL(t) == 7u && lane < NR && lane != c.rank) g_st_sys(peer + O_OF + c.wg * 64 + 32 + c.rank * 4, c.call);
  }
  if (wv == 0) STAMP(c, 12);
  const rsrc_t rsO = mkrs(p.out, (u32)MM * NN);
  const u32 obase = (u32)(mi * 256 + mw * 64 + (lane & 15)) * NN + ni * 128 + h * 64 + (lane >> 4) * 16;
  {
    const bool bad = tm == c.call || lds_ld(c.ctl + C_ABORT) != 0u;
#pragma unroll
    for (int b = 0; b < 4; ++b) {
      u32x4 v = {q[4 * b], q[4 * b + 1], q[4 * b + 2], q[4 * b + 3]};
      if (bad) v = (u32x4){0x7F7F7F7Fu, 0x7F7F7F7Fu, 0x7F7F7F7Fu, 0x7F7F7F7Fu};
      st128<AGT>(v, rsO, (obase + b * 16 * NN) + c.rank * 2048);
    }
  }
  for (int i = 0; i < 7; ++i) {
    const int s = (c.rank + 1 + (c.wg + i) % 7) & 7;
    lds_wait(c, C_MRDY, i + 1);
    const u32x2 m = *(LAS u32x2*)(c.lds + L_MASK + i * 4096 + wv * 512 + lane * 8);
    const u32 cn = __popc(m.x) + __popc(m.y), in2 = wave_incl_scan(cn, lane), V2 = __builtin_amdgcn_readlane(in2, 63);
    const u32 wo = RFL(lds_ld(c.ctl + C_DSZ + i * 8 + wv)), ncol = row_bytes(V2) >> 2;
    lds_wait(c, C_PROG, i * 65536u + wo + sent_bytes(V2));
    const u32 tm2 = g_ld_agent(c.ws + O_TMO);
    for (u32 col = lane; col < ncol; col += 64u) {
      LAS u32* sg = (LAS u32*)(c.lds + L_STG + (i & 1) * 32768 + wo) + col;
      LAS u32* d = (LAS u32*)st + col;
      u32 d7 = 0;
#pragma unroll
      for (u32 k = 0; k < 7u; ++k) { const u32 v = sg[k * ncol]; d7 |= (v & 0x80808080u) >> (7u - k); d[k * ncol] = v & 0x7F7F7F7Fu; }
      d[7u * ncol] = d7;
    }
    CFENCE();
    if (lane == 0) lds_add(c.ctl + C_DFREE0 + (i & 1), 1u);
    CFENCE();
    u32 o = in2 - cn, ov[16];
#pragma unroll
    for (int D = 0; D < 16; ++D) {
      const u32 nib = (m[D >> 3] >> (4 * (D & 7))) & 15u;
      const u32 sel = *(LAS u32*)(c.lds + L_LUTE + nib * 16 + (o & 3u) * 4);
      LAS u32* d = (LAS u32*)(st + (o & ~3u));
      ov[D] = prm(d[1], d[0], sel);
      o += __popc(nib);
    }
    const bool bad = tm2 == c.call || lds_ld(c.ctl + C_ABORT) != 0u;
#pragma unroll
    for (int b = 0; b < 4; ++b) {
      u32x4 v = {ov[4 * b], ov[4 * b + 1], ov[4 * b + 2], ov[4 * b + 3]};
      if (bad) v = (u32x4){0x7F7F7F7Fu, 0x7F7F7F7Fu, 0x7F7F7F7Fu, 0x7F7F7F7Fu};
      st128<AGT>(v, rsO, (obase + b * 16 * NN) + s * 2048);
    }
    CFENCE();
  }
  if (wv == 0) { VMWAIT0(); STAMP(c, 13); }
}

__global__ __launch_bounds__(1024, 4) void fused_gemm_ar(Params p) {
  extern __shared__ __attribute__((aligned(16))) unsigned char smem[];
  Ctx c;
  c.lds = (LAS unsigned char*)smem; c.ctl = (LAS u32*)(c.lds + L_CTL); c.ws = p.ws[p.rank]; c.t0 = now(); c.call = 0;
  c.lane = threadIdx.x & 63; c.wg = blockIdx.x; c.rank = p.rank;
  const int wv = RFL(threadIdx.x >> 6), tid = threadIdx.x;
  if (tid < C_WORDS) c.ctl[tid] = 0u;
  else if (tid >= 192 && tid < 256) {
    const u32 nib = (tid - 192) >> 2, a = tid & 3; u32 s = 0, k = a;
    for (int g = 0; g < 4; ++g) { if ((nib >> g) & 1u) s |= k++ << (8 * g); else s |= 0x0Cu << (8 * g); }
    *(LAS u32*)(c.lds + L_LUTE + (tid - 192) * 4) = s;
  } else if (tid >= 256 && tid < 272) {
    const u32 nib = tid - 256; u32 s = 0x0C0C0C0Cu, k = 0;
    for (u32 g = 0; g < 4; ++g) if ((nib >> g) & 1u) { s = (s & ~(255u << (8 * k))) | g << (8 * k); ++k; }
    *(LAS u32*)(c.lds + L_LUTC + nib * 4) = s;
  }
  asm volatile("s_waitcnt lgkmcnt(0)" ::: "memory");
  __builtin_amdgcn_s_barrier();
  CFENCE();
  const int xcd = c.wg & 7, idx = c.wg >> 3, mi = (xcd >> 1) * 4 + (idx & 3), ni = (xcd & 1) * 8 + (idx >> 2);
  if (wv == 11) STAMP(c, 0);
  if (wv < 8) role_mfma(c, p, wv, mi, ni);
  else if (wv < 11) role_loader(c, wv - 8, mi, ni);
  else if (wv == 11) role_sched(c, p, mi, ni);
  else if (wv == 12) role_puller(c, p);
  else role_packer(c, p, wv - 13);
}
}

extern "C" void kernel_launch(int rank, int nranks, ncclComm_t comm, hipStream_t stream,
                              void* const* d_in, const int* in_sizes, int n_in,
                              void* d_out, int out_size, const int* in_dtypes, int out_dtype,
                              void* const* d_out_all, void* const* d_ws_all, size_t ws_bytes) {
  if (nranks != NR || (unsigned)rank >= (unsigned)NR || n_in < 2 || ws_bytes < WS_USED || in_sizes[0] != MM * KS || in_sizes[1] != KS * NN || out_size != MM * NN) {
    fprintf(stderr, "kernel_launch: unexpected problem shape\n");
    return;
  }
  static bool init = false;
  if (!init) {
    hipError_t e = hipFuncSetAttribute((const void*)fused_gemm_ar, hipFuncAttributeMaxDynamicSharedMemorySize, (int)LDS_BYTES);
    if (e != hipSuccess) { fprintf(stderr, "hipFuncSetAttribute: %s\n", hipGetErrorString(e)); return; }
    init = true;
  }
  Params p;
  p.x = (const unsigned short*)d_in[0]; p.w = (const unsigned short*)d_in[1]; p.out = (unsigned char*)d_out;
  for (int i = 0; i < NR; ++i) p.ws[i] = (char*)d_ws_all[i];
  p.rank = rank; p.pad = 0;
  fused_gemm_ar<<<NWG, 1024, LDS_BYTES, stream>>>(p);
}
```

```cpp
#include <hip/hip_runtime.h>
#include <rccl/rccl.h>
#include <cstdio>

typedef unsigned u32;
typedef unsigned long long u64;
typedef u32 u32x4 __attribute__((ext_vector_type(4)));
typedef u32 u32x2 __attribute__((ext_vector_type(2)));
typedef unsigned short u16x2 __attribute__((ext_vector_type(2)));
typedef short bf16x8 __attribute__((ext_vector_type(8)));
typedef float f32x4 __attribute__((ext_vector_type(4)));
typedef __amdgpu_buffer_rsrc_t rsrc_t;
#define LAS __attribute__((address_space(3)))
#define GAS __attribute__((address_space(1)))
#define DI __device__ __forceinline__
#define RFL(x) __builtin_amdgcn_readfirstlane(x)
#define ALL(x) (__builtin_amdgcn_ballot_w64(!(x)) == 0ull)
#define ANY(x) (__builtin_amdgcn_ballot_w64(x) != 0ull)
#define CFENCE() asm volatile("" ::: "memory")
#define VMWAIT0() asm volatile("s_waitcnt vmcnt(0)" ::: "memory")

namespace {
constexpr int NR = 8, MM = 4096, KS = 512, NN = 16384, NWG = 256;
constexpr int NBLK = 768;
constexpr u32 SYS = 17, AGT = 16;
constexpr u64 DEADLINE = 50000000ull;
constexpr u32 O_CALL = 0;
constexpr u32 O_TMO = 32768;
constexpr u32 O_TICK = O_TMO + 128;
constexpr u32 O_GT = O_TICK + 3072;
constexpr u32 O_LT = O_GT + 128;
constexpr u32 O_RF = O_LT + 2048;
constexpr u32 O_FX = O_RF + NR * NBLK * 4;
constexpr u32 O_FW = O_FX + 16 * 64 * 16;
constexpr u32 O_OF = O_FW + 16 * 64 * 8;
constexpr u32 F_MF = 0;
constexpr u32 F_MI = 1024;
constexpr u32 F_VF = 2048;
constexpr u32 O_DBG = 512 * 1024;
constexpr u32 O_OBX = 1u << 20;
constexpr u32 O_OBW = O_OBX + 512 * 8192;
constexpr u32 O_IMG = O_OBW + 2048 * 8192;
constexpr u32 O_OM = O_IMG + 64 * 96 * 8192;
constexpr u32 O_OV = O_OM + 256 * 4096;
constexpr u32 O_MIN = O_OV + 512 * 16384;
constexpr u32 WS_USED = O_MIN + NR * 256 * 4096;
static_assert(O_OF + NR * 4096 <= O_DBG, "control block overflow");
constexpr u32 T_MASK = NBLK, T_VAL = T_MASK + 256, T_END = T_VAL + 512;
constexpr u32 NS = 5, SLOTB = 24576;
constexpr u32 L_STG = 32768;
constexpr u32 L_IN = NS * SLOTB;
constexpr u32 L_LUTC = L_IN + 4 * 8192;
constexpr u32 L_LUTE = L_LUTC + 64;
constexpr u32 L_JUNK = L_LUTE + 256;
constexpr u32 L_REL = L_JUNK + 256;
constexpr u32 L_RFC = L_REL + 2048;
constexpr u32 L_CTL = L_RFC + 4096;
enum { C_CALL = 0, C_ABORT, C_WGMAX, C_CMAX, C_CMASK, C_CVAL0, C_CVAL1, C_GVALID, C_GMAX, C_PSTART, C_PROG,
       C_FULL0 = 12, C_DFREE0 = 24, C_IFREE0 = 28, C_VSZ = 32, C_QI = 40, C_QV = 48, C_QW = 56, C_ORDER = 64, C_WORDS = 128 };
#define PLEAD 4096
#define WIN1 2
#define WIN2 4
#define T_HALF 768
#define CH 1024
constexpr u32 LDS_BYTES = L_CTL + C_WORDS * 4;
static_assert(LDS_BYTES <= 163840 && LDS_BYTES > 81920 && L_STG + 4 * 16384 <= L_IN, "one workgroup per CU");

struct Params { const unsigned short* x; const unsigned short* w; unsigned char* out; char* ws[NR]; int rank; int pad; };

DI u64 now() { return __builtin_amdgcn_s_memrealtime(); }
DI rsrc_t mkrs(const void* p, u32 bytes) { return __builtin_amdgcn_make_buffer_rsrc((void*)p, (short)0, (int)bytes, 0x00020000); }
template <u32 AUX> DI void st128(u32x4 v, rsrc_t rs, u32 off) { __builtin_amdgcn_raw_buffer_store_b128(v, rs, (int)off, 0, AUX); }
DI u32 prm(u32 a, u32 b, u32 s) { return __builtin_amdgcn_perm(a, b, s); }
DI u32 bfi(u32 m, u32 a, u32 b) { return (m & a) | (~m & b); }
DI u32 lds_ld(LAS u32* p) { return __hip_atomic_load(p, __ATOMIC_RELAXED, __HIP_MEMORY_SCOPE_WORKGROUP); }
DI void lds_st(LAS u32* p, u32 v) { __hip_atomic_store(p, v, __ATOMIC_RELAXED, __HIP_MEMORY_SCOPE_WORKGROUP); }
DI u32 lds_add(LAS u32* p, u32 v) { return __hip_atomic_fetch_add(p, v, __ATOMIC_RELAXED, __HIP_MEMORY_SCOPE_WORKGROUP); }
DI u32 g_ld_agent(const char* p) { return __hip_atomic_load((GAS u32*)p, __ATOMIC_RELAXED, __HIP_MEMORY_SCOPE_AGENT); }
DI u32 g_ld_sys(const char* p) { return __hip_atomic_load((GAS u32*)p, __ATOMIC_RELAXED, __HIP_MEMORY_SCOPE_SYSTEM); }
DI void g_st_agent(char* p, u32 v) { __hip_atomic_store((GAS u32*)p, v, __ATOMIC_RELAXED, __HIP_MEMORY_SCOPE_AGENT); }
DI void g_st_sys(char* p, u32 v) { __hip_atomic_store((GAS u32*)p, v, __ATOMIC_RELAXED, __HIP_MEMORY_SCOPE_SYSTEM); }

struct Ctx {
  LAS unsigned char* lds; LAS u32* ctl; char* ws; u64 t0; u32 call; int lane, wg, rank;
};
DI void give_up(const Ctx& c) { if (c.lane == 0) { lds_st(c.ctl + C_ABORT, 1u); if (c.call) g_st_agent(c.ws + O_TMO, c.call); } }
DI bool expired(const Ctx& c, u32& spins) {
  if ((++spins & 15u) != 0u) return false;
  if (RFL(lds_ld(c.ctl + C_ABORT)) != 0u || RFL(g_ld_agent(c.ws + O_TMO)) == c.call || now() - c.t0 > DEADLINE) { give_up(c); return true; }
  return false;
}
DI u32 q_wait(const Ctx& c, int q, u32 k) {
  u32 v, spins = 0;
  for (;;) {
    v = RFL(lds_ld(c.ctl + q + (k & 7u)));
    if ((v >> 23) == ((k + 1u) & 511u)) break;
    if ((++spins & 63u) == 0u && (RFL(lds_ld(c.ctl + C_ABORT)) != 0u || now() - c.t0 > DEADLINE + DEADLINE / 4)) { give_up(c); v = ~0u; break; }
    __builtin_amdgcn_s_sleep(1);
  }
  CFENCE();
  return v;
}
DI u32 lds_wait(const Ctx& c, int word, u32 target) {
  u32 v, spins = 0;
  for (;;) {
    v = RFL(lds_ld(c.ctl + word));
    if (v >= target) break;
    if ((++spins & 63u) == 0u && (RFL(lds_ld(c.ctl + C_ABORT)) != 0u || now() - c.t0 > DEADLINE + DEADLINE / 4)) { give_up(c); break; }
    __builtin_amdgcn_s_sleep(1);
  }
  CFENCE();
  return v;
}
#define STAMP(c, i) do {} while (0)
#define ACC0(v)
#define TIC(t)
#define TOC(v, t)
#define DUMP(c, i, v)
DI void dma4(rsrc_t rs, u32 voff, u32 soff, u32 ldsdst) {
  u32 keep;
  asm volatile("s_nop 4\n\ts_mov_b32 %0, m0\n\ts_mov_b32 m0, %1\n\ts_nop 0\n\t"
               "buffer_load_dwordx4 %2, %3, %4 offen sc0 sc1 lds\n\t"
               "buffer_load_dwordx4 %2, %3, %4 offen offset:1024 sc0 sc1 lds\n\t"
               "buffer_load_dwordx4 %2, %3, %4 offen offset:2048 sc0 sc1 lds\n\t"
               "buffer_load_dwordx4 %2, %3, %4 offen offset:3072 sc0 sc1 lds\n\t"
               "s_mov_b32 m0, %0"
               : "=&s"(keep) : "s"(RFL(ldsdst)), "v"(voff), "s"(rs), "s"(RFL(soff)) : "memory");
}
DI void dma1(rsrc_t rs, u32 voff, u32 soff, u32 ldsdst) {
  u32 keep;
  asm volatile("s_nop 4\n\ts_mov_b32 %0, m0\n\ts_mov_b32 m0, %1\n\ts_nop 0\n\t"
               "buffer_load_dwordx4 %2, %3, %4 offen sc0 sc1 lds\n\t"
               "s_mov_b32 m0, %0"
               : "=&s"(keep) : "s"(RFL(ldsdst)), "v"(voff), "s"(rs), "s"(RFL(soff)) : "memory");
}
DI void dma2(rsrc_t rs, u32 voff, u32 soff, u32 ldsdst) {
  u32 keep;
  asm volatile("s_nop 4\n\ts_mov_b32 %0, m0\n\ts_mov_b32 m0, %1\n\ts_nop 0\n\t"
               "buffer_load_dwordx4 %2, %3, %4 offen sc0 sc1 lds\n\t"
               "buffer_load_dwordx4 %2, %3, %4 offen offset:1024 sc0 sc1 lds\n\t"
               "s_mov_b32 m0, %0"
               : "=&s"(keep) : "s"(RFL(ldsdst)), "v"(voff), "s"(rs), "s"(RFL(soff)) : "memory");
}
template <int CTRL, int ROWS> DI u32 dpp0(u32 v) { return (u32)__builtin_amdgcn_update_dpp(0, (int)v, CTRL, ROWS, 0xF, false); }
DI u32 wave_incl_scan(u32 v) {
  v += dpp0<0x111, 0xF>(v); v += dpp0<0x112, 0xF>(v); v += dpp0<0x114, 0xF>(v); v += dpp0<0x118, 0xF>(v);
  v += dpp0<0x142, 0xA>(v); v += dpp0<0x143, 0xC>(v);
  return v;
}
DI u32 wave_max(u32 v) {
  v = max(v, dpp0<0x111, 0xF>(v)); v = max(v, dpp0<0x112, 0xF>(v)); v = max(v, dpp0<0x114, 0xF>(v)); v = max(v, dpp0<0x118, 0xF>(v));
  v = max(v, dpp0<0x142, 0xA>(v)); v = max(v, dpp0<0x143, 0xC>(v));
  return __builtin_amdgcn_readlane(v, 63);
}
DI int tile_mi(int wg) { return ((wg & 7) >> 1) * 4 + ((wg >> 3) & 3); }
DI int tile_ni(int wg) { return (wg & 1) * 8 + (wg >> 5); }
DI int wg_late(int wg) { return (wg >> 4) & 1; }
DI int wg_link(int wg) { return (((wg & 15) | (wg >> 5) << 4) + 3 * wg_late(wg)) % 7; }
DI u32 link_wgs(int link, int late) { return (u32)(127 + 7 - (link + 7 - 3 * late) % 7) / 7u; }
DI u32 row_bytes(u32 V) { return (((V + 7u) >> 3) + 3u) & ~3u; }
DI u32 sent_bytes(u32 V) { return (7u * row_bytes(V) + 15u) & ~15u; }
DI void image_flag(const Ctx& c, int kstep, int r) {
  if (c.lane == 0) g_st_agent(c.ws + (r < 64 ? O_FX + ((r >> 2) * 64 + kstep) * 16 + (r & 3) * 4 : O_FW + (((r - 64) >> 1) * 64 + kstep) * 8 + (r & 1) * 4), c.call);
}

#define PACK_T0 150
#define PACK_DT 60
DI void pack_inputs(Ctx& c, const Params& p, int wv) {
  const int lane = c.lane;
  const rsrc_t rsL = mkrs(c.ws, WS_USED), rsX = mkrs(p.x, MM * KS * 2), rsW = mkrs(p.w, KS * NN * 2);
  char* peer = p.ws[lane & 7];
  for (int q = wv * 256 + c.wg; q < 2560; q += 2048) {
    const int j = q / 320, r = q - j * 320;
    const bool isx = r < 64;
    if (q >= 64) { while (now() - c.t0 < (u64)(PACK_T0 + PACK_DT * j)) __builtin_amdgcn_s_sleep(8); }
    const int dest = (r - 64) & 7, ni = (r - 64) >> 4, kh = ((r - 64) >> 3) & 1;
    u32 lo[16], hi[16];
    if (isx) {
      u32x4 d[8];
      const int vo = ((64 * r + (lane & 15)) * KS + 64 * j + 8 * (lane >> 4)) * 2;
#pragma unroll
      for (int t = 0; t < 8; ++t) d[t] = __builtin_amdgcn_raw_buffer_load_b128(rsX, vo + 64 * (t >> 2), 16 * (t & 3) * KS * 2, 0);
      __builtin_amdgcn_sched_barrier(0);
#pragma unroll
      for (int t = 0; t < 8; ++t) {
        lo[2 * t] = prm(d[t].y, d[t].x, 0x06040200u); lo[2 * t + 1] = prm(d[t].w, d[t].z, 0x06040200u);
        hi[2 * t] = prm(d[t].y, d[t].x, 0x07050301u); hi[2 * t + 1] = prm(d[t].w, d[t].z, 0x07050301u);
      }
    } else {
      u32x2 d[2][8];
      const int vo = ((64 * j + 32 * kh + 8 * (lane >> 4)) * NN + dest * 2048 + ni * 128 + 4 * (lane & 15)) * 2;
#pragma unroll
      for (int k = 0; k < 8; ++k)
#pragma unroll
        for (int h = 0; h < 2; ++h) d[h][k] = __builtin_amdgcn_raw_buffer_load_b64(rsW, vo + 128 * h, k * NN * 2, 0);
      __builtin_amdgcn_sched_barrier(0);
#pragma unroll
      for (int h = 0; h < 2; ++h)
#pragma unroll
        for (int cc = 0; cc < 2; ++cc)
#pragma unroll
          for (int u = 0; u < 2; ++u) {
            const u32 a0 = d[h][4 * u][cc], a1 = d[h][4 * u + 1][cc], a2 = d[h][4 * u + 2][cc], a3 = d[h][4 * u + 3][cc];
            const u32 l0 = prm(a1, a0, 0x06020400u), l1 = prm(a3, a2, 0x06020400u), h0 = prm(a1, a0, 0x07030501u), h1 = prm(a3, a2, 0x07030501u);
            const int t = 4 * h + 2 * cc;
            lo[2 * t + u] = prm(l1, l0, 0x05040100u); lo[2 * t + 2 + u] = prm(l1, l0, 0x07060302u);
            hi[2 * t + u] = prm(h1, h0, 0x05040100u); hi[2 * t + 2 + u] = prm(h1, h0, 0x07060302u);
          }
    }
    const int lr = isx ? r : 64 + 2 * ni + kh;
    const bool own = isx || dest == c.rank, send = isx || dest != c.rank;
    u32 info = 0;
    if (send) {
      const u32 slot = (isx ? O_OBX + (u32)(j * 64 + r) * 8192u : O_OBW + (u32)((dest * 8 + j) * 32 + 2 * ni + kh) * 8192u);
      u16x2 m0 = {0, 0}, m1 = {0, 0};
#pragma unroll
      for (int k = 0; k < 16; ++k) {
        m0 = __builtin_elementwise_max(m0, __builtin_bit_cast(u16x2, hi[k] & 0x007F007Fu));
        m1 = __builtin_elementwise_max(m1, __builtin_bit_cast(u16x2, hi[k] & 0x7F007F00u));
      }
      u32 T = max(max((u32)m0.x, (u32)m0.y), max((u32)m1.x >> 8, (u32)m1.y >> 8));
      T = wave_max(T);
      const u32 base = T > 7u ? T - 7u : 0u, base4 = base * 0x01010101u;
      u32 cd[16], all = 0xFFFFFFFFu;
#pragma unroll
      for (int k = 0; k < 16; ++k) { cd[k] = (hi[k] | 0x80808080u) - base4; all &= cd[k]; }
      u32 cnt = 0;
      if (ANY((~all & 0x80808080u) != 0u)) {
#pragma unroll
        for (int k = 0; k < 16; ++k) {
          const u32 em = ~cd[k] & 0x80808080u;
          if (ANY(em != 0u)) {
#pragma unroll
            for (int b = 0; b < 4; ++b) {
              const bool e = (em >> (8 * b + 7)) & 1u;
              const u64 m = __builtin_amdgcn_ballot_w64(e);
              const u32 pos = cnt + __builtin_amdgcn_mbcnt_hi((u32)(m >> 32), __builtin_amdgcn_mbcnt_lo((u32)m, 0u));
              if (e && pos < 30u)
                __builtin_amdgcn_raw_buffer_store_b32((u32)(k * 4 + b) << 14 | (u32)lane << 8 | ((hi[k] >> (8 * b)) & 255u), rsL, (int)(slot + 6144u + pos * 4u), 0, SYS);
              cnt += (u32)__popcll(m);
            }
          }
        }
      }
      if (cnt <= 30u) {
#pragma unroll
        for (int hh = 0; hh < 2; ++hh) {
          u32x4 nb;
#pragma unroll
          for (int t = 0; t < 4; ++t) {
            const int k = 8 * hh + 2 * t;
            const u32 e0 = bfi(0x80808080u, hi[k], cd[k]) & 0x87878787u, e1 = bfi(0x80808080u, hi[k + 1], cd[k + 1]) & 0x87878787u;
            nb[t] = e0 | __builtin_amdgcn_alignbit(e1, e1, 28);
          }
          const u32x4 a = {lo[8 * hh], lo[8 * hh + 1], lo[8 * hh + 2], lo[8 * hh + 3]}, b = {lo[8 * hh + 4], lo[8 * hh + 5], lo[8 * hh + 6], lo[8 * hh + 7]};
          st128<SYS>(a, rsL, lane * 16 + (slot + hh * 3072));
          st128<SYS>(b, rsL, lane * 16 + (slot + hh * 3072 + 1024));
          st128<SYS>(nb, rsL, lane * 16 + (slot + hh * 3072 + 2048));
        }
        info = base << 5 | cnt;
      } else {
#pragma unroll
        for (int t = 0; t < 8; ++t) {
          const u32x4 v = {prm(hi[2 * t], lo[2 * t], 0x05010400u), prm(hi[2 * t], lo[2 * t], 0x07030602u),
                           prm(hi[2 * t + 1], lo[2 * t + 1], 0x05010400u), prm(hi[2 * t + 1], lo[2 * t + 1], 0x07030602u)};
          st128<SYS>(v, rsL, lane * 16 + (slot + t * 1024));
        }
        info = 31u;
      }
    }
    if (own) {
      const u32 img = O_IMG + (u32)((j * 8 + c.rank) * 96 + lr) * 8192u;
#pragma unroll
      for (int t = 0; t < 8; ++t) {
        const u32x4 v = {prm(hi[2 * t], lo[2 * t], 0x05010400u), prm(hi[2 * t], lo[2 * t], 0x07030602u),
                         prm(hi[2 * t + 1], lo[2 * t + 1], 0x05010400u), prm(hi[2 * t + 1], lo[2 * t + 1], 0x07030602u)};
        st128<AGT>(v, rsL, lane * 16 + (img + t * 1024));
      }
    }
    if (q < 2048) c.call = lds_wait(c, C_CALL, 1u);
    VMWAIT0();
    if (send && lane < NR && lane != c.rank && (isx || lane == dest)) g_st_sys(peer + O_RF + (c.rank * NBLK + j * 96 + lr) * 4, c.call << 12 | info);
    if (own) image_flag(c, j * 8 + c.rank, lr);
    if (q < 256) STAMP(c, 1);
    if (q >= 2048) STAMP(c, 2);
  }
}

DI u32 ob_slot(int b, int me) { const int j = b / 96, r = b - j * 96; return r < 64 ? O_OBX + (u32)(j * 64 + r) * 8192u : O_OBW + (u32)((me * 8 + j) * 32 + r - 64) * 8192u; }
DI u32 in_bytes(u32 info) { const u32 cnt = info & 31u; return cnt == 31u ? 8192u : 6144u + ((cnt + 3u) >> 2) * 16u; }
DI int pull_order(u32 B, int link) {
  if (B < 96u) return B < 64u ? (int)((B + 8u * link) & 63u) : (int)B;
  u32 j, r, late;
  if (B < 224u) { j = 1u + ((B - 96u) >> 6); r = B & 31u; late = 0u; if (((B - 96u) & 32u) == 0u) return (int)(j * 96u + 64u + r); }
  else if (B < 704u) {
    j = (B - 224u) / 96u + 3u; r = B - 224u - (j - 3u) * 96u;
    if (r < 32u) return (int)(j * 96u + 64u + r);
    late = (r >> 2) & 1u; r = (r & 3u) | ((r - 32u) >> 3) << 2; j -= 2u * late;
  } else { j = 6u + ((B - 704u) >> 5); r = B & 31u; late = 1u; }
  r = (r + 4u * link) & 31u;
  return (int)(j * 96u + ((r & 7u) | late << 3 | (r >> 3) << 4));
}
DI u32 early_first(u32 t) { return (t & 15u) | (t >> 7) << 4 | ((t >> 4) & 7u) << 5; }
DI void role_feeder(Ctx& c) {
  const int lane = c.lane;
  c.call = lds_wait(c, C_CALL, 1u);
  const int link = wg_link(c.wg), src = (c.rank + 1 + link) & 7;
  const u32 nl = link_wgs(link, 1), na = link_wgs(link, 0) + nl;
  u32 kv = 0, ph = 0, tbase = (c.call - 1u) * (T_HALF + na), tend = T_HALF;
  bool miss = true;
  const rsrc_t rsL = mkrs(c.ws, WS_USED);
  u32 fb = O_RF + src * NBLK * 4, B = 0;
  for (u32 k = 0;; ++k) {
    const u32 ahead = B >= T_HALF && B < T_MASK + 128u ? 2u : 1u;
    if (k >= ahead) lds_wait(c, C_PSTART, k + 1u - ahead);
    for (;;) {
      if (miss) dma4(rsL, lane * 16, fb, (u32)(size_t)c.lds + L_RFC);
      u32 t = 0;
      if (lane == 0) t = __hip_atomic_fetch_add((GAS u32*)(c.ws + O_TICK + ph * 1024 + link * 128), 1u, __ATOMIC_RELAXED, __HIP_MEMORY_SCOPE_AGENT);
      VMWAIT0();
      B = RFL(t) - tbase;
      if (B < tend || ph == 2u) break;
      if (ph == 0u && wg_late(c.wg)) { ph = 1u; tbase = (c.call - 1u) * (T_MASK - T_HALF + nl) - T_HALF; tend = T_MASK; }
      else {
        STAMP(c, 6);
        ph = 2u; tbase = (c.call - 1u) * (T_END - T_MASK + na) - T_MASK; tend = T_END; fb = O_OF + src * 4096; miss = true;
        lds_wait(c, C_CMAX, 8u);
      }
    }
    if (B >= T_END) {
      if (lane < 2) { lds_st(c.ctl + C_QI + ((k + lane) & 7u), (k + lane + 1u) << 23 | 0x7FFFFFu); lds_st(c.ctl + C_QV + ((kv + lane) & 7u), (kv + lane + 1u) << 23 | 0x7FFFFFu); }
      break;
    }
    u32 item, fo;
    if (B < T_MASK) { item = pull_order(B, link); fo = item * 4; }
    else if (B < T_VAL) { const u32 tile = early_first(B - T_MASK); item = T_MASK + tile; fo = F_MF + tile * 4; }
    else {
      const u32 ht = early_first((B - T_VAL) >> 1) * 2u + (B & 1u);
      item = T_VAL + ht; fo = (lane & 1) ? F_MI + (ht >> 1) * 4 : F_VF + ht * 4;
    }
    u32 v = lds_ld((LAS u32*)(c.lds + L_RFC + fo)), spins = 0;
    miss = false;
    while (!ALL((int)((v & ~0xFFFu) - (c.call << 12)) >= 0)) {
      miss = true;
      if (expired(c, spins)) break;
      if (spins > 1u) __builtin_amdgcn_s_sleep(1);
      v = g_ld_sys(c.ws + fb + fo);
    }
    if (lane == 0) {
      if (B >= T_VAL) lds_st(c.ctl + C_QV + (kv & 7u), (kv + 1u) << 23 | (k & 2047u) << 12 | (item - T_VAL));
      lds_st(c.ctl + C_QW + (k & 7u), B >= T_HALF && B < T_MASK + 128u);
      lds_st(c.ctl + C_QI + (k & 7u), (k + 1u) << 23 | item << 12 | (v & 0xFFFu));
    }
    if (B >= T_VAL) ++kv;
  }
}
DI void role_decoder(Ctx& c, u32 d) {
  const int lane = c.lane, src = (c.rank + 1 + wg_link(c.wg)) & 7;
  c.call = lds_wait(c, C_CALL, 1u);
  const rsrc_t rsL = mkrs(c.ws, WS_USED);
  ACC0(a_dr); ACC0(a_dc);
  for (u32 k = d;; k += 2u) {
    const u32 e = q_wait(c, C_QI, k), lb = (e >> 12) & 2047u, cnt = e & 31u;
    if (lb >= T_VAL) break;
    LAS unsigned char* sl = c.lds + L_IN + (k & 3u) * 8192u;
    if (lb >= T_MASK) {
      lds_wait(c, C_PROG, k * 32768u + 4096u);
      u32x4 v[4];
#pragma unroll
      for (int t = 0; t < 4; ++t) v[t] = *(LAS u32x4*)(sl + t * 1024 + lane * 16);
      CFENCE(); if (lane == 0) lds_add(c.ctl + C_IFREE0 + (k & 3u), 1u); CFENCE();
#pragma unroll
      for (int t = 0; t < 4; ++t) st128<AGT>(v[t], rsL, O_MIN + (src * 256 + lb - T_MASK) * 4096u + t * 1024 + lane * 16);
      VMWAIT0();
      if (lane == 0) g_st_agent(c.ws + O_OF + src * 4096 + F_MI + (lb - T_MASK) * 4, c.call << 12);
      continue;
    }
    const int j = lb / 96, r = lb - j * 96, kstep = j * 8 + src;
    const u32 img = O_IMG + (u32)(kstep * 96 + r) * 8192u + lane * 16;
    lds_wait(c, C_PROG, k * 32768u + in_bytes(e));
    if (k < 2u) STAMP(c, 26 + k);
    TIC(tz);
    if (cnt == 31u) {
      u32x4 v[8];
#pragma unroll
      for (int t = 0; t < 8; ++t) v[t] = *(LAS u32x4*)(sl + t * 1024 + lane * 16);
      CFENCE(); if (lane == 0) lds_add(c.ctl + C_IFREE0 + (k & 3u), 1u); CFENCE();
#pragma unroll
      for (int t = 0; t < 8; ++t) st128<AGT>(v[t], rsL, img + t * 1024);
    } else {
      u32x4 v[6];
#pragma unroll
      for (int t = 0; t < 6; ++t) v[t] = *(LAS u32x4*)(sl + t * 1024 + lane * 16);
      const u32 exc = *(LAS u32*)(sl + 6144 + (lane & 31) * 4);
      CFENCE(); if (lane == 0) lds_add(c.ctl + C_IFREE0 + (k & 3u), 1u); CFENCE();
      const u32 base4 = ((e >> 5) & 127u) * 0x01010101u;
      u32 hi[16];
#pragma unroll
      for (int t = 0; t < 8; ++t) {
        const u32 n = v[3 * (t >> 2) + 2][t & 3];
        hi[2 * t] = (n & 0x87878787u) + base4;
        hi[2 * t + 1] = (__builtin_amdgcn_alignbit(n, n, 4) & 0x87878787u) + base4;
      }
      for (u32 x = 0; x < cnt; ++x) {
        const u32 rec = __builtin_amdgcn_readlane(exc, x);
        const u32 kk = (rec >> 16) & 15u, sh = ((rec >> 14) & 3u) * 8u;
        const u32 m = ((rec >> 8) & 63u) == (u32)lane ? 255u << sh : 0u, val = (rec & 255u) << sh;
#pragma unroll
        for (int i = 0; i < 16; ++i) hi[i] = bfi(kk == (u32)i ? m : 0u, val, hi[i]);
      }
#pragma unroll
      for (int t = 0; t < 8; ++t) {
        const u32 l0 = v[3 * (t >> 2) + ((t >> 1) & 1)][2 * (t & 1)], l1 = v[3 * (t >> 2) + ((t >> 1) & 1)][2 * (t & 1) + 1];
        const u32x4 o = {prm(hi[2 * t], l0, 0x05010400u), prm(hi[2 * t], l0, 0x07030602u), prm(hi[2 * t + 1], l1, 0x05010400u), prm(hi[2 * t + 1], l1, 0x07030602u)};
        st128<AGT>(o, rsL, img + t * 1024);
      }
    }
    TIC(td);
    VMWAIT0();
    image_flag(c, kstep, r);
  }
}
constexpr int WMAX = WIN2 - 1;
struct Stream {
  u32 pub = 0, pend[WMAX];
  DI void init() { for (int k = 0; k < WMAX; ++k) pend[k] = 0u; }
  template <int W> DI void step(const Ctx& c, rsrc_t rs, u32 soff, u32 dst, u32 prog) {
    if (CH == 1024 || c.lane < CH / 16) dma1(rs, c.lane * 16, soff, dst);
    asm volatile("s_waitcnt vmcnt(%0)" :: "n"(W - 1) : "memory");
    pub = max(pub, pend[WMAX - (W - 1)]);
    if (c.lane == 0) lds_st(c.ctl + C_PROG, pub);
    for (int k = 0; k + 1 < WMAX; ++k) pend[k] = pend[k + 1];
    pend[WMAX - 1] = prog;
  }
  DI void flush(const Ctx& c) {
    VMWAIT0();
    const u32 v = pend[WMAX - 1];
    if (c.lane == 0) lds_st(c.ctl + C_PROG, v);
    pub = v;
  }
  DI void wait(const Ctx& c, int w, u32 target) {
    if (RFL(lds_ld(c.ctl + w)) < target) { flush(c); lds_wait(c, w, target); }
    CFENCE();
  }
};
DI void role_puller(Ctx& c, const Params& p) {
  __builtin_amdgcn_s_setprio(3);
  const u32 ldsb = (u32)(size_t)c.lds;
  const char* sws = p.ws[(c.rank + 1 + wg_link(c.wg)) & 7];
  Stream sm; sm.init();
  u32 kv = 0;
  for (u32 k = 0;; ++k) {
    u32 e = RFL(lds_ld(c.ctl + C_QI + (k & 7u)));
    if ((e >> 23) != ((k + 1u) & 511u)) { TIC(tf); sm.flush(c); e = q_wait(c, C_QI, k);
    }
    const u32 item = (e >> 12) & 2047u;
    if (item >= T_END) break;
    u32 so, tot, dst;
    if (item < T_VAL) {
      if (k >= 4u) sm.wait(c, C_IFREE0 + (k & 3u), k >> 2);
      dst = L_IN + (k & 3u) * 8192u;
      if (item < T_MASK) { so = ob_slot(item, c.rank); tot = in_bytes(e); } else { so = O_OM + (item - T_MASK) * 4096u; tot = 4096u; }
    } else {
      if (kv >= 4u) sm.wait(c, C_DFREE0 + (kv & 3u), kv & ~3u);
      dst = L_STG + (kv & 3u) * 16384u; so = O_OV + (item - T_VAL) * 16384u; tot = min((e & 0xFFFu) * 16u, 16384u);
      ++kv;
    }
    so = RFL(so); tot = RFL(tot); dst = RFL(dst);
    const rsrc_t rs = mkrs(sws + so, tot);
    const bool wide = RFL(lds_ld(c.ctl + C_QW + (k & 7u))) != 0u;
    const u32 lead = wide ? 2048u : PLEAD, qs = tot > lead ? (tot - lead) & ~(CH - 1u) : 0u;
    for (u32 q = 0; q < tot; q += CH) {
      if (q == qs && c.lane == 0) lds_st(c.ctl + C_PSTART, k + 1u);
      if (wide) sm.step<WIN2>(c, rs, q, ldsb + dst + q, k * 32768u + q + CH);
      else sm.step<WIN1>(c, rs, q, ldsb + dst + q, k * 32768u + q + CH);
    }
    if (tot == 0u && c.lane == 0) lds_st(c.ctl + C_PSTART, k + 1u);

  }
  sm.flush(c);
  STAMP(c, 8);
  DUMP(c, 39, sm.a_fl + 1);
}

DI void role_sched(Ctx& c, const Params& p, int mi, int ni) {
  const int lane = c.lane;
  u32 t = 0;
  if (lane == 0) t = __hip_atomic_fetch_add((GAS u32*)(c.ws + O_CALL + c.wg * 128), 1u, __ATOMIC_RELAXED, __HIP_MEMORY_SCOPE_AGENT) + 1u;
  c.call = RFL(t);
  if (lane == 0) lds_st(c.ctl + C_CALL, c.call);
  const rsrc_t rsL = mkrs(c.ws, WS_USED);
  u64 issued = 0; u32 n = 0, spins = 0;
  const u32 pfo = O_IMG + (lane < 32 ? mi * 32768 + (ni & 7) * 4096 + lane * 128 : 64 * 8192 + ni * 16384 + (mi & 3) * 4096 + (lane - 32) * 128);
  while (n < 64u) {
    const u32x4 a = __builtin_amdgcn_raw_buffer_load_b128(rsL, lane * 16, (int)(O_FX + mi * 1024), SYS);
    const u32x2 b = __builtin_amdgcn_raw_buffer_load_b64(rsL, lane * 8, (int)(O_FW + ni * 512), SYS);
    const bool rdy = min(min(min(a.x, a.y), min(a.z, a.w)), min(b.x, b.y)) >= c.call;
    u64 m = __builtin_amdgcn_ballot_w64(rdy) & ~issued;
    if (m == 0) {
      if (!expired(c, spins)) { __builtin_amdgcn_s_sleep(1); CFENCE(); continue; }
      m = ~issued;
    }
    if ((m >> lane) & 1) lds_st(c.ctl + C_ORDER + n + __builtin_amdgcn_mbcnt_hi((u32)(m >> 32), __builtin_amdgcn_mbcnt_lo((u32)m, 0u)), lane + 1);
    issued |= m;
    n += (u32)__popcll(m);
    for (u64 mm = m; mm; mm &= mm - 1) {
      u32 keep;
      asm volatile("s_nop 4\n\ts_mov_b32 %0, m0\n\ts_mov_b32 m0, %1\n\ts_nop 0\n\tbuffer_load_dword %2, %3, %4 offen sc1 lds\n\ts_mov_b32 m0, %0"
                   : "=&s"(keep) : "s"((u32)(size_t)c.lds + L_JUNK), "v"(pfo), "s"(rsL), "s"(RFL((u32)__builtin_ctzll(mm) * (96u * 8192u))) : "memory");
    }
    CFENCE();
  }
  STAMP(c, 9);
  if (c.wg == 0) {
    u32 mx;
    for (;;) {
      const u32x4 a = __builtin_amdgcn_raw_buffer_load_b128(rsL, lane * 16, (int)O_LT, SYS);
      const u32x4 b = __builtin_amdgcn_raw_buffer_load_b128(rsL, lane * 16, (int)(O_LT + 1024), SYS);
      mx = max(max(a.x, a.z), max(b.x, b.z));
      if (ALL(min(min(a.y, a.w), min(b.y, b.w)) >= c.call) || expired(c, spins)) break;
      CFENCE();
    }
    STAMP(c, 16);
    mx = wave_max(mx);
    if (lane < NR) __hip_atomic_store((GAS u64*)(p.ws[lane] + O_GT + c.rank * 8), (u64)c.call << 32 | mx, __ATOMIC_RELAXED, __HIP_MEMORY_SCOPE_SYSTEM);
  }
  lds_wait(c, C_CMAX, 8u);
  u32 g;
  for (;;) {
    const u64 v = __hip_atomic_load((GAS u64*)(c.ws + O_GT + (lane & 7) * 8), __ATOMIC_RELAXED, __HIP_MEMORY_SCOPE_SYSTEM);
    g = (u32)v;
    if (ALL((u32)(v >> 32) >= c.call) || expired(c, spins)) break;
    __builtin_amdgcn_s_sleep(1);
  }
  g = wave_max(g);
  if (lane == 0) { lds_st(c.ctl + C_GMAX, g); CFENCE(); lds_st(c.ctl + C_GVALID, 1u); }
  STAMP(c, 10);
}

DI void role_loader(Ctx& c, u32 li, int mi, int ni) {
  __builtin_amdgcn_s_setprio(2);
  const rsrc_t rsL = mkrs(c.ws, WS_USED);
  const u32 voff = c.lane * 16, ldsb = (u32)(size_t)c.lds;
  u32 pend = 0;
  for (u32 m = li; m < 128u; m += 3u) {
    const u32 slot = m % NS, kh = m & 1u;
    u32 o, spins = 0;
    for (;;) {
      const u32 vo = lds_ld(c.ctl + C_ORDER + (m >> 1)), vf = lds_ld((LAS u32*)(c.lds + L_REL + (c.lane & 7) * 256));
      o = RFL(vo);
      if (o != 0u && ALL(vf + (NS - 1u) >= m)) break;
      if (pend) { VMWAIT0(); if (c.lane == 0) lds_st(c.ctl + C_FULL0 + (pend - 1u) % NS, pend); pend = 0; continue; }
      if ((++spins & 63u) == 0u && (RFL(lds_ld(c.ctl + C_ABORT)) != 0u || now() - c.t0 > DEADLINE + DEADLINE / 4)) { give_up(c); break; }
      __builtin_amdgcn_s_sleep(1);
    }
    CFENCE();
    o = RFL(o);
    const u32 so = O_IMG + ((o - 1u) & 63u) * (96u * 8192u), sx = so + mi * 32768 + kh * 4096u, sw = so + 64 * 8192 + ni * 16384 + kh * 8192u, ld = ldsb + slot * SLOTB;
    dma4(rsL, voff, sw, ld + 16384u); dma4(rsL, voff, sw + 4096u, ld + 20480u);
    dma4(rsL, voff, sx, ld); dma4(rsL, voff, sx + 8192u, ld + 4096u); dma4(rsL, voff, sx + 16384u, ld + 8192u); dma4(rsL, voff, sx + 24576u, ld + 12288u);
    if (pend) { asm volatile("s_waitcnt vmcnt(24)" ::: "memory"); if (c.lane == 0) lds_st(c.ctl + C_FULL0 + (pend - 1u) % NS, pend); }
    pend = m + 1u;
  }
  VMWAIT0();
  if (c.lane == 0) lds_st(c.ctl + C_FULL0 + (pend - 1u) % NS, pend);
}

DI void role_mfma(Ctx& c, const Params& p, int wv, int mi, int ni) {
  const int lane = c.lane, mw = wv >> 1, h = wv & 1;
  pack_inputs(c, p, wv);
  f32x4 acc[4][4];
#pragma unroll
  for (int g = 0; g < 4; ++g)
#pragma unroll
    for (int b = 0; b < 4; ++b) acc[g][b] = (f32x4){0.f, 0.f, 0.f, 0.f};
  if (wv == 0) STAMP(c, 15);
  bf16x8 x0, x1, x2, x3, wa0, wa1, wa2, wa3, wb0, wb1, wb2, wb3;
  u32 vx = (u32)(size_t)c.lds + mw * 4096 + lane * 16, vw = (u32)(size_t)c.lds + 16384 + h * 4096 + lane * 16;
  u32 vf = (u32)(size_t)(c.ctl + C_FULL0 + 2), done = 1, sl = 1, seen = 0, vs;
  const u32 relad = (u32)(size_t)c.lds + L_REL + wv * 256 + lane * 4;
#define MF(a, w, x) "v_mfma_f32_16x16x32_bf16 " a ", " w ", " x ", " a "\n\t"
#define HSTEP(c0, c1, c2, c3, n0, n1, n2, n3, m) do { \
    if (__builtin_expect(seen <= (m) + 1u, 0)) lds_wait(c, C_FULL0 + sl, (m) + 2u); \
    asm volatile(MF("%0", "%9", "%8") "ds_read_b128 %4, %13\n\t" MF("%1", "%10", "%8") "ds_read_b128 %5, %13 offset:1024\n\t" \
                 MF("%2", "%11", "%8") "ds_read_b128 %6, %13 offset:2048\n\t" MF("%3", "%12", "%8") "ds_read_b128 %7, %13 offset:3072" \
                 : "+v"(acc[0][0]), "+v"(acc[1][0]), "+v"(acc[2][0]), "+v"(acc[3][0]), "+v"(n0), "+v"(n1), "+v"(n2), "+v"(n3) \
                 : "v"(x0), "v"(c0), "v"(c1), "v"(c2), "v"(c3), "v"(vw) : "memory"); \
    const u32 d = sl == NS - 1u ? -(NS - 1u) * SLOTB : SLOTB, d4 = sl == NS - 2u ? -(NS - 1u) * 4u : 4u; \
    sl = sl == NS - 1u ? 0u : sl + 1u; \
    asm volatile("s_waitcnt lgkmcnt(7)\n\t" MF("%0", "%7", "%6") "ds_read_b128 %4, %11\n\t" MF("%1", "%8", "%6") "ds_read_b32 %5, %12\n\t" \
                 MF("%2", "%9", "%6") MF("%3", "%10", "%6") \
                 : "+v"(acc[0][1]), "+v"(acc[1][1]), "+v"(acc[2][1]), "+v"(acc[3][1]), "+v"(x0), "=&v"(vs) \
                 : "v"(x1), "v"(c0), "v"(c1), "v"(c2), "v"(c3), "v"(vx), "v"(vf) : "memory"); \
    asm volatile("s_waitcnt lgkmcnt(8)\n\t" MF("%0", "%8", "%7") "ds_read_b128 %4, %12 offset:1024\n\t" MF("%1", "%9", "%7") "v_add_u32 %5, %13, %5\n\t" \
                 MF("%2", "%10", "%7") "v_add_u32 %6, %14, %6\n\t" MF("%3", "%11", "%7") \
                 : "+v"(acc[0][2]), "+v"(acc[1][2]), "+v"(acc[2][2]), "+v"(acc[3][2]), "+v"(x1), "+v"(vw), "+v"(vf) \
                 : "v"(x2), "v"(c0), "v"(c1), "v"(c2), "v"(c3), "v"(vx), "s"(d), "s"(d4) : "memory"); \
    asm volatile("s_waitcnt lgkmcnt(8)\n\t" MF("%0", "%9", "%5") "ds_read_b128 %4, %6 offset:2048\n\tv_add_u32 %7, 1, %7\n\t" MF("%1", "%10", "%5") \
                 "s_waitcnt lgkmcnt(2)\n\tv_readfirstlane_b32 %8, %13\n\t" MF("%2", "%11", "%5") MF("%3", "%12", "%5") \
                 "ds_read_b128 %5, %6 offset:3072\n\tds_write_b32 %14, %7\n\tv_add_u32 %6, %15, %6" \
                 : "+v"(acc[0][3]), "+v"(acc[1][3]), "+v"(acc[2][3]), "+v"(acc[3][3]), "+v"(x2), "+v"(x3), "+v"(vx), "+v"(done), "=s"(seen) \
                 : "v"(c0), "v"(c1), "v"(c2), "v"(c3), "v"(vs), "v"(relad), "s"(d) : "memory"); \
  } while (0)
  ACC0(ck0); ACC0(rt0);
  lds_wait(c, C_FULL0, 1u);
  asm volatile("ds_read_b128 %0, %9\n\tds_read_b128 %1, %9 offset:1024\n\tds_read_b128 %2, %9 offset:2048\n\tds_read_b128 %3, %9 offset:3072\n\t"
               "ds_read_b128 %4, %8\n\tds_read_b128 %5, %8 offset:1024\n\tds_read_b128 %6, %8 offset:2048\n\tds_read_b128 %7, %8 offset:3072\n\t"
               "ds_write_b32 %10, %11\n\ts_waitcnt lgkmcnt(0)"
               : "=&v"(wa0), "=&v"(wa1), "=&v"(wa2), "=&v"(wa3), "=&v"(x0), "=&v"(x1), "=&v"(x2), "=&v"(x3) : "v"(vx), "v"(vw), "v"(relad), "v"(done) : "memory");
  vx += SLOTB; vw += SLOTB;
  wb0 = wa0; wb1 = wa0; wb2 = wa0; wb3 = wa0;
  asm volatile("s_nop 1" : "+v"(vx), "+v"(vw), "+v"(wb0), "+v"(wb1), "+v"(wb2), "+v"(wb3));
  for (u32 m = 0; m < 126u; m += 2u) {
    HSTEP(wa0, wa1, wa2, wa3, wb0, wb1, wb2, wb3, m);
    HSTEP(wb0, wb1, wb2, wb3, wa0, wa1, wa2, wa3, m + 1u);
  }
  HSTEP(wa0, wa1, wa2, wa3, wb0, wb1, wb2, wb3, 126u);
#define MF4(b, x) asm volatile(MF("%0", "%5", "%4") MF("%1", "%6", "%4") MF("%2", "%7", "%4") MF("%3", "%8", "%4") \
    : "+v"(acc[0][b]), "+v"(acc[1][b]), "+v"(acc[2][b]), "+v"(acc[3][b]) : "v"(x), "v"(wb0), "v"(wb1), "v"(wb2), "v"(wb3))
  asm volatile("s_waitcnt lgkmcnt(0)" ::: "memory");
  MF4(0, x0); MF4(1, x1); MF4(2, x2); MF4(3, x3);
  asm volatile("s_nop 7\n\ts_nop 7" : "+v"(acc[0][2]), "+v"(acc[1][2]), "+v"(acc[2][2]), "+v"(acc[3][2]), "+v"(acc[0][3]), "+v"(acc[1][3]), "+v"(acc[2][3]), "+v"(acc[3][3]),
               "+v"(acc[0][0]), "+v"(acc[1][0]), "+v"(acc[2][0]), "+v"(acc[3][0]), "+v"(acc[0][1]), "+v"(acc[1][1]), "+v"(acc[2][1]), "+v"(acc[3][1]));
  if (wv == 0) { STAMP(c, 11); DUMP(c, 34, (__builtin_amdgcn_s_memtime() - ck0)); DUMP(c, 35, now() - rt0); }
  char* peer = p.ws[lane & 7];
  const rsrc_t rsL = mkrs(c.ws, WS_USED);
  {
    float mx = 0.f;
#pragma unroll
    for (int e = 0; e < 64; e += 2) mx = fmaxf(fmaxf(mx, acc[e & 3][e >> 4][(e >> 2) & 3]), acc[(e & 3) + 1][e >> 4][(e >> 2) & 3]);
    const u32 wmx = wave_max(__float_as_uint(mx));
    if (lane == 0) {
      __hip_atomic_fetch_max(c.ctl + C_WGMAX, wmx, __ATOMIC_RELAXED, __HIP_MEMORY_SCOPE_WORKGROUP);
      CFENCE();
      if (lds_add(c.ctl + C_CMAX, 1u) == 7u)
        __hip_atomic_store((GAS u64*)(c.ws + O_LT + c.wg * 8), (u64)c.call << 32 | lds_ld(c.ctl + C_WGMAX), __ATOMIC_RELAXED, __HIP_MEMORY_SCOPE_AGENT);
    }
  }
  u32 nz[2] = {0u, 0u};
#pragma unroll
  for (int e = 63; e >= 0; --e) nz[e >> 5] = __builtin_amdgcn_alignbit(nz[e >> 5], __float_as_uint(acc[e & 3][e >> 4][(e >> 2) & 3]), 31);
  nz[0] = ~nz[0]; nz[1] = ~nz[1];
  const u32 cnt = __popc(nz[0]) + __popc(nz[1]);
  const u32 inc = wave_incl_scan(cnt), V = __builtin_amdgcn_readlane(inc, 63);
  if (lane == 0) lds_st(c.ctl + C_VSZ + wv, sent_bytes(V));
  __builtin_amdgcn_raw_buffer_store_b64((u32x2){nz[0], nz[1]}, rsL, lane * 8, (int)(O_OM + c.wg * 4096 + wv * 512), SYS);
  VMWAIT0();
  const u32 zs = wv >> 2;
  {
    u32 t = 0;
    if (lane == 0) t = lds_add(c.ctl + C_CMASK, 1u);
    if (RFL(t) == 7u && lane < NR && lane != c.rank) g_st_sys(peer + O_OF + c.rank * 4096 + F_MF + c.wg * 4, c.call << 12);
  }
  lds_wait(c, C_CMASK, 8u);
  LAS unsigned char* st = c.lds + wv * 4096;
#pragma unroll
  for (int k = 0; k < 4; ++k) *(LAS u32x4*)(st + k * 1024 + lane * 16) = (u32x4){0u, 0u, 0u, 0u};
  u32x4 hdr;
  hdr.x = 16u; hdr.y = hdr.x + RFL(lds_ld(c.ctl + C_VSZ + zs * 4)); hdr.z = hdr.y + RFL(lds_ld(c.ctl + C_VSZ + zs * 4 + 1)); hdr.w = hdr.z + RFL(lds_ld(c.ctl + C_VSZ + zs * 4 + 2));
  const u32 vtot = hdr.w + RFL(lds_ld(c.ctl + C_VSZ + zs * 4 + 3)), woff = (wv & 2) ? ((wv & 1) ? hdr.w : hdr.z) : ((wv & 1) ? hdr.y : hdr.x);
  const u32 ovb = O_OV + (c.wg * 2 + zs) * 16384;
  if ((wv & 3) == 0 && lane == 0) st128<SYS>(hdr, rsL, ovb);
  lds_wait(c, C_GVALID, 1u);
  const u32 tm = g_ld_agent(c.ws + O_TMO);
  const float amax = __uint_as_float(RFL(lds_ld(c.ctl + C_GMAX)));
  const float inv = amax > 0.f ? 448.0f / amax : 0.f;
  u32 q[16];
#pragma unroll
  for (int D = 0; D < 16; ++D) {
    const int b = D >> 2, r = D & 3;
    int v = __builtin_amdgcn_cvt_pk_fp8_f32(fmaxf(acc[0][b][r], 0.f) * inv, fmaxf(acc[1][b][r], 0.f) * inv, 0, false);
    v = __builtin_amdgcn_cvt_pk_fp8_f32(fmaxf(acc[2][b][r], 0.f) * inv, fmaxf(acc[3][b][r], 0.f) * inv, v, true);
    q[D] = (u32)v;
  }
  {
    u32 o = inc - cnt;
#pragma unroll
    for (int D = 0; D < 16; ++D) {
      const u32 nib = (nz[D >> 3] >> (4 * (D & 7))) & 15u;
      const u32 c4 = prm(0u, q[D], *(LAS u32*)(c.lds + L_LUTC + nib * 4));
      const u64 v = (u64)c4 << ((o & 3u) * 8u);
      LAS u32* d = (LAS u32*)(st + (o & ~3u));
      __hip_atomic_fetch_or(d, (u32)v, __ATOMIC_RELAXED, __HIP_MEMORY_SCOPE_WORKGROUP);
      if ((u32)(v >> 32)) __hip_atomic_fetch_or(d + 1, (u32)(v >> 32), __ATOMIC_RELAXED, __HIP_MEMORY_SCOPE_WORKGROUP);
      o += __popc(nib);
    }
    CFENCE();
    const u32 ncol = row_bytes(V) >> 2, sb = sent_bytes(V);
    for (u32 col = lane; col < ncol; col += 64u) {
      LAS u32* s = (LAS u32*)st + col;
      const u32 d7 = s[7u * ncol];
#pragma unroll
      for (u32 k = 0; k < 7u; ++k) s[k * ncol] |= (d7 << (7u - k)) & 0x80808080u;
    }
    CFENCE();
    for (u32 k = lane * 16; k < sb; k += 1024u)
      st128<SYS>(*(LAS u32x4*)(st + k), rsL, k + (ovb + woff));
    VMWAIT0();
    u32 t = 0;
    if (lane == 0) t = lds_add(c.ctl + C_CVAL0 + zs, 1u);
    if (RFL(t) == 3u && lane < NR && lane != c.rank) g_st_sys(peer + O_OF + c.rank * 4096 + F_VF + (c.wg * 2 + zs) * 4, c.call << 12 | vtot >> 4);
  }
  if (wv == 0) STAMP(c, 12);
  const rsrc_t rsO = mkrs(p.out, (u32)MM * NN);
  const u32 obase = (u32)(mi * 256 + mw * 64 + (lane & 15)) * NN + ni * 128 + h * 64 + (lane >> 4) * 16;
  {
    const bool bad = tm == c.call || lds_ld(c.ctl + C_ABORT) != 0u;
#pragma unroll
    for (int b = 0; b < 4; ++b) {
      u32x4 v = {q[4 * b], q[4 * b + 1], q[4 * b + 2], q[4 * b + 3]};
      if (bad) v = (u32x4){0x7F7F7F7Fu, 0x7F7F7F7Fu, 0x7F7F7F7Fu, 0x7F7F7F7Fu};
      st128<AGT>(v, rsO, (obase + b * 16 * NN) + c.rank * 2048);
    }
  }
  const u32 s = (c.rank + 1 + wg_link(c.wg)) & 7;
  for (u32 kv = zs;; kv += 2u) {
    const u32 e = q_wait(c, C_QV, kv), ht = e & 0xFFFu, pb = ((e >> 12) & 2047u) * 32768u;
    if (ht >= 512u) break;
    const u32 dt = ht >> 1, dw = (ht & 1u) * 4u + (wv & 3);
    const u32x2 m = __builtin_amdgcn_raw_buffer_load_b64(rsL, lane * 8, (int)(O_MIN + (s * 256 + dt) * 4096u + dw * 512u), AGT);
    const u32 dbase = (u32)(tile_mi(dt) * 256 + (dw >> 1) * 64 + (lane & 15)) * NN + tile_ni(dt) * 128 + (dw & 1) * 64 + (lane >> 4) * 16;
    const u32 cn = __popc(m.x) + __popc(m.y), in2 = wave_incl_scan(cn), V2 = __builtin_amdgcn_readlane(in2, 63), ncol = row_bytes(V2) >> 2;
    LAS unsigned char* sl = c.lds + L_STG + (kv & 3u) * 16384u;
    lds_wait(c, C_PROG, pb + 16u);
    const u32 wo = min(RFL(*(LAS u32*)(sl + (wv & 3) * 4)) & ~15u, 16384u - 3584u);
    lds_wait(c, C_PROG, pb + wo + sent_bytes(V2));
    const u32 tm2 = g_ld_agent(c.ws + O_TMO);
    for (u32 col = lane; col < ncol; col += 64u) {
      LAS u32* sg = (LAS u32*)(sl + wo) + col;
      LAS u32* d = (LAS u32*)st + col;
      u32 d7 = 0;
#pragma unroll
      for (u32 k = 0; k < 7u; ++k) { const u32 v = sg[k * ncol]; d7 |= (v & 0x80808080u) >> (7u - k); d[k * ncol] = v & 0x7F7F7F7Fu; }
      d[7u * ncol] = d7;
    }
    CFENCE();
    if (lane == 0) lds_add(c.ctl + C_DFREE0 + (kv & 3u), 1u);
    CFENCE();
    u32 o = in2 - cn, ov[16];
#pragma unroll
    for (int D = 0; D < 16; ++D) {
      const u32 nib = (m[D >> 3] >> (4 * (D & 7))) & 15u;
      const u32 sel = *(LAS u32*)(c.lds + L_LUTE + nib * 16 + (o & 3u) * 4);
      LAS u32* d = (LAS u32*)(st + (o & ~3u));
      ov[D] = prm(d[1], d[0], sel);
      o += __popc(nib);
    }
    const bool bad = tm2 == c.call || lds_ld(c.ctl + C_ABORT) != 0u;
#pragma unroll
    for (int k = 0; k < 4; ++k) {
      u32x4 v = {ov[4 * k], ov[4 * k + 1], ov[4 * k + 2], ov[4 * k + 3]};
      if (bad) v = (u32x4){0x7F7F7F7Fu, 0x7F7F7F7Fu, 0x7F7F7F7Fu, 0x7F7F7F7Fu};
      st128<AGT>(v, rsO, (dbase + k * 16 * NN) + s * 2048);
    }
    CFENCE();
  }
  if ((wv & 3) == 0) { VMWAIT0(); STAMP(c, 13 + zs); }
}

__global__ __launch_bounds__(1024, 4) void fused_gemm_ar(Params p) {
  extern __shared__ __attribute__((aligned(16))) unsigned char smem[];
  Ctx c;
  c.lds = (LAS unsigned char*)smem; c.ctl = (LAS u32*)(c.lds + L_CTL); c.ws = p.ws[p.rank]; c.t0 = now(); c.call = 0;
  c.lane = threadIdx.x & 63; c.wg = blockIdx.x; c.rank = p.rank;
  const int wv = RFL(threadIdx.x >> 6), tid = threadIdx.x;
  if (tid >= 512) *(LAS u32*)(c.lds + L_REL + (tid - 512) * 4) = 0u;
  if (tid < C_WORDS) c.ctl[tid] = 0u;
  else if (tid >= 192 && tid < 256) {
    const u32 nib = (tid - 192) >> 2, a = tid & 3; u32 s = 0, k = a;
    for (int g = 0; g < 4; ++g) { if ((nib >> g) & 1u) s |= k++ << (8 * g); else s |= 0x0Cu << (8 * g); }
    *(LAS u32*)(c.lds + L_LUTE + (tid - 192) * 4) = s;
  } else if (tid >= 256 && tid < 272) {
    const u32 nib = tid - 256; u32 s = 0x0C0C0C0Cu, k = 0;
    for (u32 g = 0; g < 4; ++g) if ((nib >> g) & 1u) { s = (s & ~(255u << (8 * k))) | g << (8 * k); ++k; }
    *(LAS u32*)(c.lds + L_LUTC + nib * 4) = s;
  }
  asm volatile("s_waitcnt lgkmcnt(0)" ::: "memory");
  __builtin_amdgcn_s_barrier();
  CFENCE();
  const int mi = tile_mi(c.wg), ni = tile_ni(c.wg);
  if (wv == 11) STAMP(c, 0);
  if (wv < 8) role_mfma(c, p, wv, mi, ni);
  else if (wv < 11) role_loader(c, wv - 8, mi, ni);
  else if (wv == 11) role_sched(c, p, mi, ni);
  else if (wv == 12) role_puller(c, p);
  else if (wv == 13) role_feeder(c);
  else role_decoder(c, wv - 14);
}
}

extern "C" void kernel_launch(int rank, int nranks, ncclComm_t comm, hipStream_t stream,
                              void* const* d_in, const int* in_sizes, int n_in,
                              void* d_out, int out_size, const int* in_dtypes, int out_dtype,
                              void* const* d_out_all, void* const* d_ws_all, size_t ws_bytes) {
  if (nranks != NR || (unsigned)rank >= (unsigned)NR || n_in < 2 || ws_bytes < WS_USED || in_sizes[0] != MM * KS || in_sizes[1] != KS * NN || out_size != MM * NN) {
    fprintf(stderr, "kernel_launch: unexpected problem shape\n");
    return;
  }
  static bool init = false;
  if (!init) {
    hipError_t e = hipFuncSetAttribute((const void*)fused_gemm_ar, hipFuncAttributeMaxDynamicSharedMemorySize, (int)LDS_BYTES);
    if (e != hipSuccess) { fprintf(stderr, "hipFuncSetAttribute: %s\n", hipGetErrorString(e)); return; }
    init = true;
  }
  Params p;
  p.x = (const unsigned short*)d_in[0]; p.w = (const unsigned short*)d_in[1]; p.out = (unsigned char*)d_out;
  for (int i = 0; i < NR; ++i) p.ws[i] = (char*)d_ws_all[i];
  p.rank = rank; p.pad = 0;
  fused_gemm_ar<<<NWG, 1024, LDS_BYTES, stream>>>(p);
}
```

```cpp
#include <hip/hip_runtime.h>
#include <rccl/rccl.h>
#include <cstdio>

typedef unsigned u32;
typedef unsigned long long u64;
typedef u32 u32x4 __attribute__((ext_vector_type(4)));
typedef u32 u32x2 __attribute__((ext_vector_type(2)));
typedef short bf16x8 __attribute__((ext_vector_type(8)));
typedef float f32x4 __attribute__((ext_vector_type(4)));
typedef __amdgpu_buffer_rsrc_t rsrc_t;
#define LAS __attribute__((address_space(3)))
#define GAS __attribute__((address_space(1)))
#define DI __device__ __forceinline__
#define RFL(x) __builtin_amdgcn_readfirstlane(x)
#define ALL(x) (__builtin_amdgcn_ballot_w64(!(x)) == 0ull)
#define ANY(x) (__builtin_amdgcn_ballot_w64(x) != 0ull)
#define CFENCE() asm volatile("" ::: "memory")
#define VMWAIT0() asm volatile("s_waitcnt vmcnt(0)" ::: "memory")

namespace {
constexpr int NR = 8, MM = 4096, KS = 512, NN = 16384, NWG = 256;
constexpr int NBLK = 768;
constexpr u32 SYS = 17, AGT = 16;
constexpr int LEADER = 16;
constexpr u64 DEADLINE = 50000000ull;
constexpr u32 O_CALL = 0;
constexpr u32 O_TMO = 32768;
constexpr u32 O_TICK = O_TMO + 128;
constexpr u32 O_MQ = O_TICK + 3072;
constexpr u32 O_GT = O_MQ + 128;
constexpr u32 O_LT = O_GT + 128;
constexpr u32 O_RF = O_LT + 2048;
constexpr u32 O_FX = O_RF + NR * NBLK * 4;
constexpr u32 O_FW = O_FX + 16 * 64 * 16;
constexpr u32 O_OF = O_FW + 16 * 64 * 8;
constexpr u32 F_MF = 0;
constexpr u32 F_MI = 1024;
constexpr u32 F_VF = 2048;
constexpr u32 O_DBG = 512 * 1024;
constexpr u32 O_OBX = 1u << 20;
constexpr u32 O_OBW = O_OBX + 512 * 8192;
constexpr u32 O_IMG = O_OBW + 2048 * 8192;
constexpr u32 O_OM = O_IMG + 64 * 96 * 8192;
constexpr u32 O_OV = O_OM + 256 * 4096;
constexpr u32 O_MIN = O_OV + 512 * 16384;
constexpr u32 WS_USED = O_MIN + NR * 256 * 4096;
static_assert(O_OF + NR * 4096 <= O_DBG, "control block overflow");
constexpr u32 T_MASK = NBLK, T_VAL = T_MASK + 256, T_END = T_VAL + 512;
enum { C_CALL = 0, C_ABORT, C_WGMAX, C_CMAX, C_CMASK, C_CVAL0, C_CVAL1, C_GVALID, C_GMAX, C_PSTART, C_PROG, C_PACKED,
       C_FULL0 = 12, C_LEAD = 17, C_CSZ0 = 18, C_DFREE0 = 24, C_IFREE0 = 28, C_VSZ = 32, C_QI = 40, C_QV = 48, C_ORDER = 64, C_WORDS = 128 };
constexpr u32 L_LUTC = C_WORDS * 4;
constexpr u32 L_GT = L_LUTC + 128;
constexpr u32 L_LUTE = L_LUTC + 256;
constexpr u32 L_REL = 1024;
constexpr u32 L_RFC = L_REL + 2048;
constexpr u32 NS = 5, SLOTB = 24576;
constexpr u32 L_RING = L_RFC + 4096;
constexpr u32 L_STG = L_RING + 32768;
constexpr u32 L_IN = L_RING + NS * SLOTB;
constexpr u32 LDS_BYTES = L_IN + 4 * 8192;
static_assert(LDS_BYTES <= 163840 && LDS_BYTES > 81920 && L_STG + 4 * 16384 <= L_IN, "one workgroup per CU");
#define PLEAD 4096
#define WIN1 2
#define WIN2 4
#define T_HALF 672
#define PACK_T0 100
#define PACK_DT 40
#define NRAW 20
#define T_W1 728
#define T_W2 T_END
#define KFIX 6u
#define T_W0 128

struct Params { const unsigned short* x; const unsigned short* w; unsigned char* out; char* ws[NR]; int rank; int pad; };

DI u64 now() { return __builtin_amdgcn_s_memrealtime(); }
DI rsrc_t mkrs(const void* p, u32 bytes) { return __builtin_amdgcn_make_buffer_rsrc((void*)p, (short)0, (int)bytes, 0x00020000); }
template <u32 AUX> DI void st128(u32x4 v, rsrc_t rs, u32 off) { __builtin_amdgcn_raw_buffer_store_b128(v, rs, (int)off, 0, AUX); }
DI void st32(u32 v, rsrc_t rs, u32 off) { __builtin_amdgcn_raw_buffer_store_b32(v, rs, (int)off, 0, SYS); }
DI u32 prm(u32 a, u32 b, u32 s) { return __builtin_amdgcn_perm(a, b, s); }
DI u32 bfi(u32 m, u32 a, u32 b) { return (m & a) | (~m & b); }
DI u32 lds_ld(LAS u32* p) { return __hip_atomic_load(p, __ATOMIC_RELAXED, __HIP_MEMORY_SCOPE_WORKGROUP); }
DI void lds_st(LAS u32* p, u32 v) { __hip_atomic_store(p, v, __ATOMIC_RELAXED, __HIP_MEMORY_SCOPE_WORKGROUP); }
DI u32 lds_add(LAS u32* p, u32 v) { return __hip_atomic_fetch_add(p, v, __ATOMIC_RELAXED, __HIP_MEMORY_SCOPE_WORKGROUP); }
DI void lds_or(LAS unsigned char* p, u32 v) { __hip_atomic_fetch_or((LAS u32*)p, v, __ATOMIC_RELAXED, __HIP_MEMORY_SCOPE_WORKGROUP); }
DI u32 g_ld_agent(const char* p) { return __hip_atomic_load((GAS u32*)p, __ATOMIC_RELAXED, __HIP_MEMORY_SCOPE_AGENT); }
DI u32 g_ld_sys(const char* p) { return __hip_atomic_load((GAS u32*)p, __ATOMIC_RELAXED, __HIP_MEMORY_SCOPE_SYSTEM); }
DI void g_st_agent(char* p, u32 v) { __hip_atomic_store((GAS u32*)p, v, __ATOMIC_RELAXED, __HIP_MEMORY_SCOPE_AGENT); }
DI void g_st_sys(char* p, u32 v) { __hip_atomic_store((GAS u32*)p, v, __ATOMIC_RELAXED, __HIP_MEMORY_SCOPE_SYSTEM); }

struct Ctx {
  LAS unsigned char* lds; LAS u32* ctl; char* ws; u64 t0; u32 call; int lane, wg, rank;
};
DI void give_up(const Ctx& c) { if (c.lane == 0) { lds_st(c.ctl + C_ABORT, 1u); if (c.call) g_st_agent(c.ws + O_TMO, c.call); } }
DI bool expired(const Ctx& c, u32& spins) {
  if ((++spins & 15u) != 0u) return false;
  if (RFL(lds_ld(c.ctl + C_ABORT)) != 0u || RFL(g_ld_agent(c.ws + O_TMO)) == c.call || now() - c.t0 > DEADLINE) { give_up(c); return true; }
  return false;
}
DI bool lds_expired(const Ctx& c, u32& spins) {
  if ((++spins & 63u) == 0u && (RFL(lds_ld(c.ctl + C_ABORT)) != 0u || now() - c.t0 > DEADLINE + DEADLINE / 4)) { give_up(c); return true; }
  __builtin_amdgcn_s_sleep(1);
  return false;
}
DI u32 lds_wait(const Ctx& c, int word, u32 target) {
  u32 v, spins = 0;
  while ((v = RFL(lds_ld(c.ctl + word))) < target) if (lds_expired(c, spins)) break;
  CFENCE();
  return v;
}
DI u32 q_wait(const Ctx& c, int q, u32 k) {
  u32 v, spins = 0;
  while (((v = RFL(lds_ld(c.ctl + q + (k & 7u)))) >> 23) != ((k + 1u) & 511u)) if (lds_expired(c, spins)) { v = ~0u; break; }
  CFENCE();
  return RFL(v);
}
#define STAMP(c, i) do {} while (0)
DI void dma4(rsrc_t rs, u32 voff, u32 soff, u32 ldsdst) {
  u32 keep;
  asm volatile("s_nop 4\n\ts_mov_b32 %0, m0\n\ts_mov_b32 m0, %1\n\ts_nop 0\n\t"
               "buffer_load_dwordx4 %2, %3, %4 offen sc0 sc1 lds\n\t"
               "buffer_load_dwordx4 %2, %3, %4 offen offset:1024 sc0 sc1 lds\n\t"
               "buffer_load_dwordx4 %2, %3, %4 offen offset:2048 sc0 sc1 lds\n\t"
               "buffer_load_dwordx4 %2, %3, %4 offen offset:3072 sc0 sc1 lds\n\t"
               "s_mov_b32 m0, %0"
               : "=&s"(keep) : "s"(RFL(ldsdst)), "v"(voff), "s"(rs), "s"(RFL(soff)) : "memory");
}
DI void dma1(rsrc_t rs, u32 voff, u32 soff, u32 ldsdst) {
  u32 keep;
  asm volatile("s_nop 4\n\ts_mov_b32 %0, m0\n\ts_mov_b32 m0, %1\n\ts_nop 0\n\t"
               "buffer_load_dwordx4 %2, %3, %4 offen sc0 sc1 lds\n\t"
               "s_mov_b32 m0, %0"
               : "=&s"(keep) : "s"(RFL(ldsdst)), "v"(voff), "s"(rs), "s"(RFL(soff)) : "memory");
}
template <int CTRL, int ROWS> DI u32 dpp0(u32 v) { return (u32)__builtin_amdgcn_update_dpp(0, (int)v, CTRL, ROWS, 0xF, false); }
DI u32 wave_incl_scan(u32 v) {
  v += dpp0<0x111, 0xF>(v); v += dpp0<0x112, 0xF>(v); v += dpp0<0x114, 0xF>(v); v += dpp0<0x118, 0xF>(v);
  v += dpp0<0x142, 0xA>(v); v += dpp0<0x143, 0xC>(v);
  return v;
}
DI u32 wave_max(u32 v) {
  v = max(v, dpp0<0x111, 0xF>(v)); v = max(v, dpp0<0x112, 0xF>(v)); v = max(v, dpp0<0x114, 0xF>(v)); v = max(v, dpp0<0x118, 0xF>(v));
  v = max(v, dpp0<0x142, 0xA>(v)); v = max(v, dpp0<0x143, 0xC>(v));
  return __builtin_amdgcn_readlane(v, 63);
}
DI int tile_mi(int wg) { return ((wg & 7) >> 1) * 4 + ((wg >> 3) & 3); }
DI int tile_ni(int wg) { return (wg & 1) * 8 + (wg >> 5); }
DI int wg_late(int wg) { return (wg >> 4) & 1; }
DI int wg_link(int wg) { return (((wg & 15) | (wg >> 5) << 4) + 3 * wg_late(wg)) % 7; }
DI u32 link_wgs(int link, int late) { return (u32)(127 + 7 - (link + 7 - 3 * late) % 7) / 7u; }
DI u32 row_bytes(u32 V) { return (((V + 7u) >> 3) + 3u) & ~3u; }
DI void image_flag(const Ctx& c, int kstep, int r) {
  if (c.lane == 0) g_st_agent(c.ws + (r < 64 ? O_FX + ((r >> 2) * 64 + kstep) * 16 + (r & 3) * 4 : O_FW + (((r - 64) >> 1) * 64 + kstep) * 8 + (r & 1) * 4), c.call);
}

constexpr u32 B1 = 0x01010101u;
DI u32 nib4(u32 m, int k) { return (k ? m >> (4 * k - 2) : m << 2) & 0x3Cu; }
DI u32 to_nib(u32 bits) { return __builtin_amdgcn_udot4(bits, 0x08040201u, 0u, false); }
DI u32 put_sel(const Ctx& c, u32 n4) { return *(LAS u32*)(c.lds + L_LUTC + n4); }
DI void put4(LAS unsigned char* dst, u32& o, u32& carry, u32 v, u32 n4, u32 sel) {
  const u64 w = (u64)prm(0u, v, sel) << ((o & 3u) * 8u);
  lds_or(dst + (o & ~3u), (u32)w | carry);
  carry = (u32)(w >> 32);
  o += __popc(n4);
}
DI u32 get4(const Ctx& c, LAS unsigned char* src, u32& o, u32 n4) {
  const u32 sel = *(LAS u32*)(c.lds + L_LUTE + (((o << 6) & 0xC0u) | n4));
  LAS u32* d = (LAS u32*)(src + (o & ~3u));
  o += __popc(n4);
  return prm(d[1], d[0], sel);
}
struct Lv { u32 rows2, nc2, nc3; };
template <u32 R> DI bool enc_tail(const Ctx& c, u32 V2, LAS unsigned char* s, u32 cap, Lv& g, u32& l2a, u32& l2b, u32& l3) {
  const u32 nc2 = (V2 + 4u * R - 1u) / (4u * R), b2 = R * nc2 * 4u;
  u32 e[R], sel[R], nb3 = 0;
  l2a = 0u; l2b = 0u;
#pragma unroll
  for (u32 r = 0; r < R; ++r) e[r] = *(LAS u32*)(s + (r * nc2 + c.lane) * 4u);
#pragma unroll
  for (u32 r = 0; r < R; ++r) {
    e[r] = (u32)c.lane < nc2 ? e[r] : 0u;
    const u32 lo3 = e[r] & 7u * B1, b3 = (e[r] >> 3) & B1;
    const u32 c2 = bfi((b3 << 8) - b3, prm(0x00000002u, 0x03000000u, lo3), prm(0u, 1u, lo3));
    if (r < 4u) l2a |= c2 << (2u * r); else l2b |= c2 << (2u * r - 8u);
    nb3 |= to_nib((c2 | c2 >> 1) & B1) << (4u * r);
  }
  nb3 = ~nb3 & (R == 8u ? ~0u : 0xFFFFu);
#pragma unroll
  for (u32 r = 0; r < R; ++r) sel[r] = put_sel(c, nib4(nb3, r));
  const u32 cn3 = __popc(nb3), in3 = wave_incl_scan(cn3), V3 = __builtin_amdgcn_readlane(in3, 63);
  g.rows2 = R; g.nc2 = nc2; g.nc3 = (V3 + 7u) >> 3;
  if (V3 > 512u || b2 + g.nc3 * 8u + 8u > cap) return false;
  u32 o = in3 - cn3, carry = 0;
#pragma unroll
  for (u32 r = 0; r < R; ++r) put4(s + b2, o, carry, e[r], nib4(nb3, r), sel[r]);
  lds_or(s + b2 + (o & ~3u), carry);
  CFENCE();
  l3 = *(LAS u32*)(s + b2 + c.lane * 4u) | *(LAS u32*)(s + b2 + (g.nc3 + c.lane) * 4u) << 4;
  return true;
}
template <u32 N, u32 MAX2> DI bool enc_levels(const Ctx& c, u32 (&n)[N], LAS unsigned char* s, u32 cap, Lv& g, u32& l2a, u32& l2b, u32& l3) {
  u32 nb[2] = {0u, 0u}, c1[N], sel[N];
#pragma unroll
  for (u32 k = 0; k < N; ++k) {
    const u32 m = ((n[k] + 3u * B1) >> 4) & B1;
    c1[k] = n[k] & (m + (m << 1));
    nb[k >> 3] |= to_nib(m) << (4u * (k & 7u));
  }
  nb[0] = ~nb[0]; nb[1] = ~nb[1] & (N == 16u ? ~0u : (1u << (4u * (N & 7u))) - 1u);
#pragma unroll
  for (u32 k = 0; k < N; ++k) sel[k] = put_sel(c, nib4(nb[k >> 3], k & 7u));
  const u32 cnt = __popc(nb[0]) + __popc(nb[1]), inc = wave_incl_scan(cnt), V2 = __builtin_amdgcn_readlane(inc, 63);
  if (V2 > MAX2 || V2 + 40u > cap) return false;
  u32 o = inc - cnt, carry = 0;
#pragma unroll
  for (u32 k = 0; k < N; ++k) put4(s, o, carry, n[k], nib4(nb[k >> 3], k & 7u), sel[k]);
  lds_or(s + (o & ~3u), carry);
  CFENCE();
#pragma unroll
  for (u32 k = 0; k < N; ++k) n[k] = c1[k];
  if (MAX2 > 1024u && V2 > 1024u) return enc_tail<8>(c, V2, s, cap, g, l2a, l2b, l3);
  return enc_tail<4>(c, V2, s, cap, g, l2a, l2b, l3);
}
template <u32 R> DI void dec_tail(const Ctx& c, u32 V2, LAS unsigned char* in, LAS unsigned char* s, u32 cap) {
  const u32 nc2 = (V2 + 4u * R - 1u) / (4u * R), b2 = R * nc2 * 4u;
  const bool col = (u32)c.lane < nc2;
  u32 l2[2] = {*(LAS u32*)(in + c.lane * 4u), R == 8u ? *(LAS u32*)(in + (nc2 + c.lane) * 4u) : 0u}, nb3 = 0, e[R];
  const u32 l3 = *(LAS u32*)(in + R * nc2 + c.lane * 4u);
  l2[0] = col ? l2[0] : ~0u; l2[1] = col ? l2[1] : ~0u;
#pragma unroll
  for (u32 r = 0; r < R; ++r) { const u32 w = l2[r >> 2] >> (2u * (r & 3u)); nb3 |= to_nib((w | w >> 1) & B1) << (4u * r); }
  nb3 = ~nb3 & (R == 8u ? ~0u : 0xFFFFu);
  const u32 cn3 = __popc(nb3), in3 = wave_incl_scan(cn3), V3 = __builtin_amdgcn_readlane(in3, 63), nc3 = (V3 + 7u) >> 3;
  if (V3 > 512u || b2 + nc3 * 8u + 8u > cap) return;
  if ((u32)c.lane < nc3) { *(LAS u32*)(s + b2 + c.lane * 4u) = l3 & 15u * B1; *(LAS u32*)(s + b2 + (nc3 + c.lane) * 4u) = (l3 >> 4) & 15u * B1; }
  CFENCE();
  u32 o = in3 - cn3;
#pragma unroll
  for (u32 r = 0; r < R; ++r) e[r] = prm(0u, 0x0B0C0000u, (l2[r >> 2] >> (2u * (r & 3u))) & 3u * B1) | get4(c, s + b2, o, nib4(nb3, r));
  if (col) {
#pragma unroll
    for (u32 r = 0; r < R; ++r) *(LAS u32*)(s + (r * nc2 + c.lane) * 4u) = e[r];
  }
}
template <u32 N, u32 MAX2> DI void dec_levels(const Ctx& c, u32 (&n)[N], LAS unsigned char* in, LAS unsigned char* s, u32 cap) {
  u32 nb[2] = {0u, 0u};
#pragma unroll
  for (u32 k = 0; k < N; ++k) nb[k >> 3] |= to_nib((n[k] | n[k] >> 1) & B1) << (4u * (k & 7u));
  nb[0] = ~nb[0]; nb[1] = ~nb[1] & (N == 16u ? ~0u : (1u << (4u * (N & 7u))) - 1u);
  const u32 cnt = __popc(nb[0]) + __popc(nb[1]), inc = wave_incl_scan(cnt), V2 = __builtin_amdgcn_readlane(inc, 63);
  if (V2 > MAX2 || V2 + 40u > cap) return;
  if (MAX2 > 1024u && V2 > 1024u) dec_tail<8>(c, V2, in, s, cap); else dec_tail<4>(c, V2, in, s, cap);
  CFENCE();
  u32 o = inc - cnt;
#pragma unroll
  for (u32 k = 0; k < N; ++k) n[k] = prm(0u, 0x0F0E0D00u, n[k]) | get4(c, s, o, nib4(nb[k >> 3], k & 7u));
}
DI u32 pick_top(u32 sum, u32 cnt) { return (u32)((float)sum * __builtin_amdgcn_rcpf((float)max(cnt, 1u)) + 1.83f); }

DI u32x4 piece(const u32 (&lo)[16], const u32 (&hi)[16], int t) {
  return (u32x4){prm(hi[2 * t], lo[2 * t], 0x05010400u), prm(hi[2 * t], lo[2 * t], 0x07030602u),
                 prm(hi[2 * t + 1], lo[2 * t + 1], 0x05010400u), prm(hi[2 * t + 1], lo[2 * t + 1], 0x07030602u)};
}
DI void pack_inputs(Ctx& c, const Params& p, int wv) {
  const int lane = c.lane;
  const rsrc_t rsL = mkrs(c.ws, WS_USED), rsX = mkrs(p.x, MM * KS * 2), rsW = mkrs(p.w, KS * NN * 2);
  char* peer = p.ws[lane & 7];
  LAS unsigned char* st = c.lds + L_RING + wv * 4096;
  const int hi7 = (c.wg & 15) | (c.wg >> 5) << 4, late = wg_late(c.wg);
  int q = wv == 0 ? hi7 * 2 + late : late ? 128 + wv * 128 + hi7 : 2560;
  if (q < 320) __builtin_amdgcn_s_setprio(3);
  for (; q < 2560; q = q < 256 ? (late ? 1152 + hi7 : 2560) : q + 1024) {
    const int j = q / 320, r = q - j * 320;
    const bool isx = r < 64;
    if (j) { __builtin_amdgcn_s_setprio(0); while (now() - c.t0 < (u64)(PACK_T0 + PACK_DT * j)) __builtin_amdgcn_s_sleep(8); }
    const int dest = (r - 64) & 7, ni = (r - 64) >> 4, kh = ((r - 64) >> 3) & 1;
    u32 lo[16], hi[16];
    *(LAS u32x4*)(st + lane * 16) = (u32x4){0u, 0u, 0u, 0u}; *(LAS u32x4*)(st + 1024 + lane * 16) = (u32x4){0u, 0u, 0u, 0u};
    if (isx) {
      u32x4 d[8];
      const int vo = ((64 * r + (lane & 15)) * KS + 64 * j + 8 * (lane >> 4)) * 2;
#pragma unroll
      for (int t = 0; t < 8; ++t) d[t] = __builtin_amdgcn_raw_buffer_load_b128(rsX, vo + 64 * (t >> 2), 16 * (t & 3) * KS * 2, 0);
      __builtin_amdgcn_sched_barrier(0);
#pragma unroll
      for (int t = 0; t < 8; ++t) {
        lo[2 * t] = prm(d[t].y, d[t].x, 0x06040200u); lo[2 * t + 1] = prm(d[t].w, d[t].z, 0x06040200u);
        hi[2 * t] = prm(d[t].y, d[t].x, 0x07050301u); hi[2 * t + 1] = prm(d[t].w, d[t].z, 0x07050301u);
      }
    } else {
      u32x2 d[2][8];
      const int vo = ((64 * j + 32 * kh + 8 * (lane >> 4)) * NN + dest * 2048 + ni * 128 + 4 * (lane & 15)) * 2;
#pragma unroll
      for (int k = 0; k < 8; ++k)
#pragma unroll
        for (int h = 0; h < 2; ++h) d[h][k] = __builtin_amdgcn_raw_buffer_load_b64(rsW, vo + 128 * h, k * NN * 2, 0);
      __builtin_amdgcn_sched_barrier(0);
#pragma unroll
      for (int h = 0; h < 2; ++h)
#pragma unroll
        for (int cc = 0; cc < 2; ++cc)
#pragma unroll
          for (int u = 0; u < 2; ++u) {
            const u32 a0 = d[h][4 * u][cc], a1 = d[h][4 * u + 1][cc], a2 = d[h][4 * u + 2][cc], a3 = d[h][4 * u + 3][cc];
            const u32 l0 = prm(a1, a0, 0x06020400u), l1 = prm(a3, a2, 0x06020400u), h0 = prm(a1, a0, 0x07030501u), h1 = prm(a3, a2, 0x07030501u);
            const int t = 4 * h + 2 * cc;
            lo[2 * t + u] = prm(l1, l0, 0x05040100u); lo[2 * t + 2 + u] = prm(l1, l0, 0x07060302u);
            hi[2 * t + u] = prm(h1, h0, 0x05040100u); hi[2 * t + 2 + u] = prm(h1, h0, 0x07060302u);
          }
    }
    const int lr = isx ? r : 64 + 2 * ni + kh;
    const bool own = isx || dest == c.rank, send = isx || dest != c.rank;
    if (own) {
      const u32 img = O_IMG + (u32)((j * 8 + c.rank) * 96 + lr) * 8192u + lane * 16;
#pragma unroll
      for (int t = 0; t < 8; ++t) st128<AGT>(piece(lo, hi, t), rsL, img + t * 1024);
    }
    u32 tot = 8192u;
    if (send) {
      const u32 slot = (isx ? O_OBX + (u32)(j * 64 + r) * 8192u : O_OBW + (u32)((dest * 8 + j) * 32 + 2 * ni + kh) * 8192u);
      u32 n[16], se = 0, any = 0, cnt = 0;
      if (q >= NRAW) {
#pragma unroll
      for (int k = 0; k < 16; ++k) { n[k] = bfi(B1, lo[k] >> 7, hi[k] << 1); se = __builtin_amdgcn_sad_u8(n[k], 0u, se); }
      const u32 base = min(max(pick_top(__builtin_amdgcn_readlane(wave_incl_scan(se), 63), 4096u), 13u) - 13u, 240u);
      const u32 bl = (base & 127u) * B1, bx = base < 128u ? 0x80u * B1 : 0u;
#pragma unroll
      for (int k = 0; k < 16; ++k) { n[k] = (((n[k] | 0x80u * B1) - bl) ^ (n[k] & 0x80u * B1)) ^ bx; any |= n[k]; }
      if (ANY((any & 0xF0u * B1) != 0u)) {
#pragma unroll
        for (int k = 0; k < 16; ++k) if (ANY((n[k] & 0xF0u * B1) != 0u)) {
#pragma unroll
          for (int b = 0; b < 4; ++b) {
            const u32 v = (n[k] >> (8 * b)) & 255u;
            const u64 m = __builtin_amdgcn_ballot_w64(v > 15u);
            const u32 pos = cnt + __builtin_amdgcn_mbcnt_hi((u32)(m >> 32), __builtin_amdgcn_mbcnt_lo((u32)m, 0u));
            if (v > 15u && pos < 31u) st32((u32)(k * 4 + b) << 14 | (u32)lane << 8 | ((v + base) & 255u), rsL, slot + 5124u + pos * 4u);
            cnt += (u32)__popcll(m);
          }
        }
      }
#pragma unroll
      for (int k = 0; k < 16; ++k) n[k] = ((n[k] & 15u * B1) + 2u * B1) & 15u * B1;
      Lv g; u32 l2a, l2b, l3;
      if (cnt <= 31u && enc_levels<16, 2048>(c, n, st, 2048u, g, l2a, l2b, l3)) {
#pragma unroll
        for (int t = 0; t < 4; ++t)
          st128<SYS>((u32x4){bfi(0x80u * B1, hi[4 * t], lo[4 * t]), bfi(0x80u * B1, hi[4 * t + 1], lo[4 * t + 1]),
                             bfi(0x80u * B1, hi[4 * t + 2], lo[4 * t + 2]), bfi(0x80u * B1, hi[4 * t + 3], lo[4 * t + 3])}, rsL, slot + t * 1024 + lane * 16);
        u32x4 c1;
#pragma unroll
        for (int t = 0; t < 4; ++t) c1[t] = n[4 * t] | n[4 * t + 1] << 2 | n[4 * t + 2] << 4 | n[4 * t + 3] << 6;
        st128<SYS>(c1, rsL, slot + 4096u + lane * 16);
        const u32 o2 = 5120u + ((cnt + 4u) & ~3u) * 4u;
        if (lane == 0) st32(base | cnt << 8, rsL, slot + 5120u);
        if ((u32)lane < g.nc2) { st32(l2a, rsL, slot + o2 + lane * 4); if (g.rows2 == 8u) st32(l2b, rsL, slot + o2 + (g.nc2 + lane) * 4u); }
        if ((u32)lane < g.nc3) st32(l3, rsL, slot + o2 + g.rows2 * g.nc2 + lane * 4);
        tot = (o2 + g.rows2 * g.nc2 + g.nc3 * 4u + 15u) & ~15u;
      } else VMWAIT0();
      }
      if (tot == 8192u) {
        asm volatile("" : "+v"(lo[0]), "+v"(lo[2]), "+v"(lo[4]), "+v"(lo[6]), "+v"(lo[8]), "+v"(lo[10]), "+v"(lo[12]), "+v"(lo[14]),
                          "+v"(lo[1]), "+v"(lo[3]), "+v"(lo[5]), "+v"(lo[7]), "+v"(lo[9]), "+v"(lo[11]), "+v"(lo[13]), "+v"(lo[15]));
#pragma unroll
        for (int t = 0; t < 8; ++t) st128<SYS>(piece(lo, hi, t), rsL, slot + t * 1024 + lane * 16);
      }
    }
    if (!c.call) c.call = lds_wait(c, C_CALL, 1u);
    VMWAIT0();
    if (send && lane < NR && lane != c.rank && (isx || lane == dest)) g_st_sys(peer + O_RF + (c.rank * NBLK + j * 96 + lr) * 4, c.call << 12 | tot >> 4);
    if (own) image_flag(c, j * 8 + c.rank, lr);
    if (q < 64) STAMP(c, 1);
    if (q >= 2048) STAMP(c, 2);
  }
  __builtin_amdgcn_s_setprio(0);
  c.call = lds_wait(c, C_CALL, 1u);
  if (lane == 0) lds_add(c.ctl + C_PACKED, 1u);
}

DI u32 ob_slot(int b, int me) { const int j = b / 96, r = b - j * 96; return r < 64 ? O_OBX + (u32)(j * 64 + r) * 8192u : O_OBW + (u32)((me * 8 + j) * 32 + r - 64) * 8192u; }
#define LAGS {0, 1, 2, 2}
struct Order {
  unsigned short lb[NBLK];
  constexpr Order() : lb() {
    constexpr int lag[4] = LAGS;
    int n = 0;
    for (int r = 0; r < 96; ++r) lb[n++] = (unsigned short)r;
    for (int j = 1; j < 12; ++j) {
      if (j < 8) for (int r = 0; r < 32; ++r) lb[n++] = (unsigned short)(j * 96 + 64 + r);
      for (int t = 0; t < 4; ++t)
        for (int g = 0; g < 4; ++g) {
          const int jj = j - lag[g];
          if (jj >= 1 && jj < 8) for (int b = 0; b < 4; ++b) lb[n++] = (unsigned short)(jj * 96 + t * 16 + g * 4 + b);
        }
    }
  }
};
__constant__ Order ORDER = Order();
DI int pull_order(u32 B, int link) {
  constexpr int lag[4] = LAGS;
  const u32 lb = ORDER.lb[B], r = lb % 96u, sw = (link & 4) && ((r & 8u) ? lag[2] == lag[3] : lag[0] == lag[1]) ? 4u : 0u;
  return (int)(B >= 96u && r < 64u ? lb - r + (((r + 16u * link) & 63u) ^ sw) : lb);
}
DI u32 early_first(u32 t) { return (t & 15u) | (t >> 7) << 4 | ((t >> 4) & 7u) << 5; }
DI void role_feeder(Ctx& c) {
  const int lane = c.lane;
  c.call = lds_wait(c, C_CALL, 1u);
  const int link = wg_link(c.wg), src = (c.rank + 1 + link) & 7;
  const u32 nl = link_wgs(link, 1), ne = link_wgs(link, 0), na = ne + nl, late = wg_late(c.wg);
  u32 kv = 0, ph = 0, tbase = (c.call - 1u) * (T_HALF - KFIX * ne + na) - KFIX * ne, tend = T_HALF;
  bool miss = true;
  const rsrc_t rsL = mkrs(c.ws, WS_USED);
  u32 fb = O_RF + src * NBLK * 4, B = 0;
  if (late) lds_wait(c, C_PACKED, 8u);
  for (u32 k = 0;; ++k) {
    const u32 ahead = B < T_W0 || (B >= T_HALF && B < T_MASK + 128u) ? 2u : 1u;
    if (k >= ahead) lds_wait(c, C_PSTART, k + 1u - ahead);
    if (k < KFIX && !late) {
      B = (u32)((c.wg & 15) | (c.wg >> 5) << 4) / 7u + k * ne;
      if (miss) { dma4(rsL, lane * 16, fb, (u32)(size_t)c.lds + L_RFC); VMWAIT0(); }
    } else for (;;) {
      if (miss) dma4(rsL, lane * 16, fb, (u32)(size_t)c.lds + L_RFC);
      u32 t = 0;
      if (lane == 0) t = __hip_atomic_fetch_add((GAS u32*)(c.ws + O_TICK + ph * 1024 + link * 128), 1u, __ATOMIC_RELAXED, __HIP_MEMORY_SCOPE_AGENT);
      VMWAIT0();
      B = RFL(t) - tbase;
      if (B < tend || ph == 2u) break;
      if (ph == 0u && late) { ph = 1u; tbase = (c.call - 1u) * (T_MASK - T_HALF + nl) - T_HALF; tend = T_MASK; }
      else {
        STAMP(c, 6);
        ph = 2u; tbase = (c.call - 1u) * (T_END - T_MASK + na) - T_MASK; tend = T_END; fb = O_OF + src * 4096; miss = true;
        lds_wait(c, C_CMAX, 8u);
        if (c.wg == LEADER) lds_wait(c, C_LEAD, 1u);
      }
    }
    if (B >= T_END) {
      if (lane < 2) { lds_st(c.ctl + C_QI + ((k + lane) & 7u), (k + lane + 1u) << 23 | 0x7FFFFFu); lds_st(c.ctl + C_QV + ((kv + lane) & 7u), (kv + lane + 1u) << 23 | 0x7FFFFFu); }
      break;
    }
    u32 item, fo;
    if (B < T_MASK) { item = pull_order(B, link); fo = item * 4; }
    else if (B < T_VAL) { item = T_MASK; fo = F_MF + (B - T_MASK) * 4; }
    else {
      const u32 ht = early_first((B - T_VAL) >> 1) * 2u + (B & 1u);
      item = T_VAL + ht; fo = (lane & 1) ? F_MI + (ht >> 1) * 4 : F_VF + ht * 4;
    }
    u32 v = lds_ld((LAS u32*)(c.lds + L_RFC + fo)), spins = 0;
    miss = false;
    while (!ALL((int)((v & ~0xFFFu) - (c.call << 12)) >= 0)) {
      miss = true;
      if (expired(c, spins)) break;
      if (spins > 1u) __builtin_amdgcn_s_sleep(1);
      v = g_ld_sys(c.ws + fb + fo);
    }
    if (item == T_MASK) item += v & 255u;
    if (lane == 0) {
      if (B >= T_VAL) lds_st(c.ctl + C_QV + (kv & 7u), (kv + 1u) << 23 | (k & 2047u) << 12 | (item - T_VAL));
      const u32 wide = B < T_W0 || (B >= min(T_HALF, T_W1) && B < T_MASK + 128u) || B >= T_W2;
      lds_st(c.ctl + C_QI + (k & 7u), (k + 1u) << 23 | item << 12 | wide << 11 | (v & 0x7FFu));
    }
    if (B >= T_VAL) ++kv;
  }
}
DI void role_decoder(Ctx& c, u32 d) {
  const int lane = c.lane, src = (c.rank + 1 + wg_link(c.wg)) & 7;
  __builtin_amdgcn_s_setprio(1);
  c.call = lds_wait(c, C_CALL, 1u);
  const rsrc_t rsL = mkrs(c.ws, WS_USED);
  for (u32 k = d;; k += 2u) {
    const u32 e = q_wait(c, C_QI, k), lb = (e >> 12) & 2047u, tot = min((e & 0x7FFu) * 16u, 8192u);
    if (lb >= T_VAL) break;
    LAS unsigned char* sl = c.lds + L_IN + (k & 3u) * 8192u;
    if (lb >= T_MASK) {
      lds_wait(c, C_PROG, k * 32768u + 4096u);
      u32x4 v[4];
#pragma unroll
      for (int t = 0; t < 4; ++t) v[t] = *(LAS u32x4*)(sl + t * 1024 + lane * 16);
      CFENCE(); if (lane == 0) lds_add(c.ctl + C_IFREE0 + (k & 3u), 1u); CFENCE();
#pragma unroll
      for (int t = 0; t < 4; ++t) st128<AGT>(v[t], rsL, O_MIN + (src * 256 + lb - T_MASK) * 4096u + t * 1024 + lane * 16);
      VMWAIT0();
      if (lane == 0) g_st_agent(c.ws + O_OF + src * 4096 + F_MI + (lb - T_MASK) * 4, c.call << 12);
      continue;
    }
    const int j = lb / 96, r = lb - j * 96, kstep = j * 8 + src;
    const u32 img = O_IMG + (u32)(kstep * 96 + r) * 8192u + lane * 16;
    lds_wait(c, C_PROG, k * 32768u + tot);
    if (k < 2u) STAMP(c, 26 + k);
    if (tot == 8192u) {
      u32x4 v[8];
#pragma unroll
      for (int t = 0; t < 8; ++t) v[t] = *(LAS u32x4*)(sl + t * 1024 + lane * 16);
      CFENCE(); if (lane == 0) lds_add(c.ctl + C_IFREE0 + (k & 3u), 1u); CFENCE();
#pragma unroll
      for (int t = 0; t < 8; ++t) st128<AGT>(v[t], rsL, img + t * 1024);
    } else {
      const u32x4 c1 = *(LAS u32x4*)(sl + 4096 + lane * 16);
      const u32 exc = *(LAS u32*)(sl + 5120 + (lane & 31) * 4);
      u32x4 sm[4];
#pragma unroll
      for (int t = 0; t < 4; ++t) sm[t] = *(LAS u32x4*)(sl + t * 1024 + lane * 16);
      const u32 hd = RFL(exc), cnt = (hd >> 8) & 31u, base4 = min(hd & 255u, 240u) * B1;
      u32 n[16], lo[16], hi[16];
#pragma unroll
      for (int i = 0; i < 16; ++i) n[i] = (c1[i >> 2] >> (2 * (i & 3))) & 3u * B1;
      dec_levels<16, 2048>(c, n, sl + 5120u + ((cnt + 4u) & ~3u) * 4u, sl + 6144u, 2048u);
      CFENCE(); if (lane == 0) lds_add(c.ctl + C_IFREE0 + (k & 3u), 1u); CFENCE();
#pragma unroll
      for (int i = 0; i < 16; ++i) n[i] = ((n[i] + 14u * B1) & 15u * B1) + base4;
      for (u32 x = 1; x <= cnt; ++x) {
        const u32 rec = __builtin_amdgcn_readlane(exc, x);
        const u32 kk = (rec >> 16) & 15u, sh = ((rec >> 14) & 3u) * 8u;
        const u32 m = ((rec >> 8) & 63u) == (u32)lane ? 255u << sh : 0u, val = (rec & 255u) << sh;
#pragma unroll
        for (int i = 0; i < 16; ++i) n[i] = bfi(kk == (u32)i ? m : 0u, val, n[i]);
      }
#pragma unroll
      for (int i = 0; i < 16; ++i) { hi[i] = bfi(0x80u * B1, sm[i >> 2][i & 3], n[i] >> 1); lo[i] = bfi(0x80u * B1, n[i] << 7, sm[i >> 2][i & 3]); }
#pragma unroll
      for (int t = 0; t < 8; ++t) st128<AGT>(piece(lo, hi, t), rsL, img + t * 1024);
    }
    VMWAIT0();
    image_flag(c, kstep, r);
  }
}
constexpr int WMAX = WIN2 - 1;
struct Stream {
  u32 pub = 0, pend[WMAX] = {};
  template <int W> DI void landed(const Ctx& c, u32 prog) {
    asm volatile("s_waitcnt vmcnt(%0)" :: "n"(W - 1) : "memory");
    pub = max(pub, pend[WMAX - (W - 1)]);
    if (c.lane == 0) lds_st(c.ctl + C_PROG, pub);
    for (int k = 0; k + 1 < WMAX; ++k) pend[k] = pend[k + 1];
    pend[WMAX - 1] = prog;
  }
  DI void flush(const Ctx& c) {
    VMWAIT0();
    pub = pend[WMAX - 1];
    if (c.lane == 0) lds_st(c.ctl + C_PROG, pub);
  }
};
DI void role_puller(Ctx& c, const Params& p) {
  __builtin_amdgcn_s_setprio(3);
  const u32 ldsb = (u32)(size_t)c.lds;
  const char* sws = p.ws[(c.rank + 1 + wg_link(c.wg)) & 7];
  Stream sm;
  u32 kv = 0;
  u32 cw = 0;
  for (u32 k = 0;; ++k) {
    u32 e = __builtin_amdgcn_readlane(cw, 0), fi = __builtin_amdgcn_readlane(cw, 1), fd = __builtin_amdgcn_readlane(cw, 2);
    if ((e >> 23) != ((k + 1u) & 511u)) {
      e = RFL(lds_ld(c.ctl + C_QI + (k & 7u)));
      if ((e >> 23) != ((k + 1u) & 511u)) {
        sm.flush(c);
        e = q_wait(c, C_QI, k);
      }
    }
    CFENCE();
    const u32 item = (e >> 12) & 2047u;
    if (item >= T_END) break;
    u32 so, tot = (e & 0x7FFu) * 16u, dst;
    if (item < T_VAL) {
      if (fi < k >> 2) { sm.flush(c); lds_wait(c, C_IFREE0 + (k & 3u), k >> 2); }
      dst = L_IN + (k & 3u) * 8192u;
      if (item < T_MASK) { so = ob_slot(item, c.rank); tot = min(tot, 8192u); } else { so = O_OM + (item - T_MASK) * 4096u; tot = 4096u; }
    } else {
      if (fd < (kv & ~3u)) { sm.flush(c); lds_wait(c, C_DFREE0 + (kv & 3u), kv & ~3u); }
      dst = L_STG + (kv & 3u) * 16384u; so = O_OV + (item - T_VAL) * 16384u; tot = min(tot, 16384u);
      ++kv;
    }
    so = RFL(so); tot = RFL(max(tot, 16u)); dst = RFL(dst);
    const rsrc_t rs = mkrs(sws + so, tot);
    const bool wide = (e >> 11) & 1u;
    const u32 lead = wide ? 2048u : PLEAD, qs = tot > lead ? (tot - lead) & ~1023u : 0u;
    for (u32 q = 0; q < tot; q += 1024u) {
      dma1(rs, c.lane * 16, q, ldsb + dst + q);
      if (q == qs && c.lane == 0) lds_st(c.ctl + C_PSTART, k + 1u);
      if (q + 1024u >= tot) cw = lds_ld(c.ctl + (c.lane == 0 ? C_QI + ((k + 1u) & 7u) : c.lane == 1 ? C_IFREE0 + ((k + 1u) & 3u) : C_DFREE0 + (kv & 3u)));
      if (wide) sm.landed<WIN2>(c, k * 32768u + q + 1024u); else sm.landed<WIN1>(c, k * 32768u + q + 1024u);
    }
  }
  sm.flush(c);
  STAMP(c, 8);
}

DI void role_sched(Ctx& c, const Params& p, int mi, int ni) {
  const int lane = c.lane;
  u32 t = 0;
  if (lane == 0) t = __hip_atomic_fetch_add((GAS u32*)(c.ws + O_CALL + c.wg * 128), 1u, __ATOMIC_RELAXED, __HIP_MEMORY_SCOPE_AGENT) + 1u;
  c.call = RFL(t);
  if (lane == 0) lds_st(c.ctl + C_CALL, c.call);
  const rsrc_t rsL = mkrs(c.ws, WS_USED);
  u64 issued = 0; u32 n = 0, spins = 0;
  while (n < 64u) {
    const u32x4 a = __builtin_amdgcn_raw_buffer_load_b128(rsL, lane * 16, (int)(O_FX + mi * 1024), SYS);
    const u32x2 b = __builtin_amdgcn_raw_buffer_load_b64(rsL, lane * 8, (int)(O_FW + ni * 512), SYS);
    const bool rdy = min(min(min(a.x, a.y), min(a.z, a.w)), min(b.x, b.y)) >= c.call;
    u64 m = __builtin_amdgcn_ballot_w64(rdy) & ~issued;
    if (m == 0) {
      if (!expired(c, spins)) { __builtin_amdgcn_s_sleep(1); CFENCE(); continue; }
      m = ~issued;
    }
    if ((m >> lane) & 1) lds_st(c.ctl + C_ORDER + n + __builtin_amdgcn_mbcnt_hi((u32)(m >> 32), __builtin_amdgcn_mbcnt_lo((u32)m, 0u)), lane + 1);
    issued |= m;
    n += (u32)__popcll(m);
    CFENCE();
  }
  STAMP(c, 9);
  if (c.wg == LEADER) {
    u32 mx;
    for (;;) {
      const u32x4 a = __builtin_amdgcn_raw_buffer_load_b128(rsL, lane * 16, (int)O_LT, SYS);
      const u32x4 b = __builtin_amdgcn_raw_buffer_load_b128(rsL, lane * 16, (int)(O_LT + 1024), SYS);
      mx = max(max(a.x, a.z), max(b.x, b.z));
      if (ALL(min(min(a.y, a.w), min(b.y, b.w)) >= c.call) || expired(c, spins)) break;
      CFENCE();
    }
    STAMP(c, 16);
    mx = wave_max(mx);
    if (lane < NR) __hip_atomic_store((GAS u64*)(p.ws[lane] + O_GT + c.rank * 8), (u64)c.call << 32 | mx, __ATOMIC_RELAXED, __HIP_MEMORY_SCOPE_SYSTEM);
    if (lane == 0) lds_st(c.ctl + C_LEAD, 1u);
  }
  lds_wait(c, C_CMAX, 8u);
  const rsrc_t rsG = mkrs(c.ws + O_GT, 64u);
  u32 g;
  for (;;) {
    if (lane < 4) dma1(rsG, lane * 16, 0u, (u32)(size_t)c.lds + L_GT);
    asm volatile("s_waitcnt vmcnt(5)" ::: "memory");
    const u64 v = __hip_atomic_load((LAS u64*)(c.lds + L_GT) + (lane & 7), __ATOMIC_RELAXED, __HIP_MEMORY_SCOPE_WORKGROUP);
    g = (u32)v;
    if (ALL((u32)(v >> 32) >= c.call) || expired(c, spins)) break;
    __builtin_amdgcn_s_sleep(2);
  }
  g = wave_max(g);
  if (lane == 0) { lds_st(c.ctl + C_GMAX, g); CFENCE(); lds_st(c.ctl + C_GVALID, 1u); }
  STAMP(c, 10);
}

DI void role_loader(Ctx& c, u32 li, int mi, int ni) {
  __builtin_amdgcn_s_setprio(2);
  const rsrc_t rsL = mkrs(c.ws, WS_USED);
  const u32 voff = c.lane * 16, ldsb = (u32)(size_t)c.lds + L_RING;
  u32 pend = 0;
  lds_wait(c, C_PACKED, 8u);
  for (u32 m = li; m < 128u; m += 3u) {
    const u32 slot = m % NS, kh = m & 1u;
    u32 o, spins = 0;
    for (;;) {
      const u32 vo = lds_ld(c.ctl + C_ORDER + (m >> 1)), vf = lds_ld((LAS u32*)(c.lds + L_REL + (c.lane & 7) * 256));
      o = RFL(vo);
      if (o != 0u && ALL(vf + (NS - 1u) >= m)) break;
      if (pend) { VMWAIT0(); if (c.lane == 0) lds_st(c.ctl + C_FULL0 + (pend - 1u) % NS, pend); pend = 0; continue; }
      if (lds_expired(c, spins)) break;
    }
    CFENCE();
    o = RFL(o);
    const u32 so = O_IMG + ((o - 1u) & 63u) * (96u * 8192u), sx = so + mi * 32768 + kh * 4096u, sw = so + 64 * 8192 + ni * 16384 + kh * 8192u, ld = ldsb + slot * SLOTB;
    dma4(rsL, voff, sw, ld + 16384u);
    if (pend) { asm volatile("s_waitcnt vmcnt(4)" ::: "memory"); if (c.lane == 0) lds_st(c.ctl + C_FULL0 + (pend - 1u) % NS, pend); }
    dma4(rsL, voff, sw + 4096u, ld + 20480u);
    dma4(rsL, voff, sx, ld); dma4(rsL, voff, sx + 8192u, ld + 4096u); dma4(rsL, voff, sx + 16384u, ld + 8192u); dma4(rsL, voff, sx + 24576u, ld + 12288u);
    pend = m + 1u;
  }
  VMWAIT0();
  if (c.lane == 0) lds_st(c.ctl + C_FULL0 + (pend - 1u) % NS, pend);
}

struct Geo { u32 nc, sz; };
DI Geo geo(u32 V) { return V <= 2048u ? Geo{row_bytes(V) >> 2, 8u * row_bytes(V)} : Geo{64u, (V + 3u) & ~3u}; }

DI void role_mfma(Ctx& c, const Params& p, int wv, int mi, int ni) {
  const int lane = c.lane, mw = wv >> 1, h = wv & 1;
  pack_inputs(c, p, wv);
  f32x4 acc[4][4];
#pragma unroll
  for (int g = 0; g < 4; ++g)
#pragma unroll
    for (int b = 0; b < 4; ++b) acc[g][b] = (f32x4){0.f, 0.f, 0.f, 0.f};
  if (wv == 0) STAMP(c, 15);
  bf16x8 x0, x1, x2, x3, wa0, wa1, wa2, wa3, wb0, wb1, wb2, wb3;
  u32 vx = (u32)(size_t)c.lds + L_RING + mw * 4096 + lane * 16, vw = (u32)(size_t)c.lds + L_RING + 16384 + h * 4096 + lane * 16;
  u32 vf = (u32)(size_t)(c.ctl + C_FULL0 + 2), done = 1, sl = 1, seen = 0, vs;
  const u32 relad = (u32)(size_t)c.lds + L_REL + wv * 256 + lane * 4;
#define MF(a, w, x) "v_mfma_f32_16x16x32_bf16 " a ", " w ", " x ", " a "\n\t"
#define HSTEP(c0, c1, c2, c3, n0, n1, n2, n3, m) do { \
    if (__builtin_expect(seen <= (m) + 1u, 0)) lds_wait(c, C_FULL0 + sl, (m) + 2u); \
    asm volatile(MF("%0", "%9", "%8") "ds_read_b128 %4, %13\n\t" MF("%1", "%10", "%8") "ds_read_b128 %5, %13 offset:1024\n\t" \
                 MF("%2", "%11", "%8") "ds_read_b128 %6, %13 offset:2048\n\t" MF("%3", "%12", "%8") "ds_read_b128 %7, %13 offset:3072" \
                 : "+v"(acc[0][0]), "+v"(acc[1][0]), "+v"(acc[2][0]), "+v"(acc[3][0]), "+v"(n0), "+v"(n1), "+v"(n2), "+v"(n3) \
                 : "v"(x0), "v"(c0), "v"(c1), "v"(c2), "v"(c3), "v"(vw) : "memory"); \
    const u32 d = sl == NS - 1u ? -(NS - 1u) * SLOTB : SLOTB, d4 = sl == NS - 2u ? -(NS - 1u) * 4u : 4u; \
    sl = sl == NS - 1u ? 0u : sl + 1u; \
    asm volatile("s_waitcnt lgkmcnt(7)\n\t" MF("%0", "%7", "%6") "ds_read_b128 %4, %11\n\t" MF("%1", "%8", "%6") "ds_read_b32 %5, %12\n\t" \
                 MF("%2", "%9", "%6") MF("%3", "%10", "%6") \
                 : "+v"(acc[0][1]), "+v"(acc[1][1]), "+v"(acc[2][1]), "+v"(acc[3][1]), "+v"(x0), "=&v"(vs) \
                 : "v"(x1), "v"(c0), "v"(c1), "v"(c2), "v"(c3), "v"(vx), "v"(vf) : "memory"); \
    asm volatile("s_waitcnt lgkmcnt(8)\n\t" MF("%0", "%8", "%7") "ds_read_b128 %4, %12 offset:1024\n\t" MF("%1", "%9", "%7") "v_add_u32 %5, %13, %5\n\t" \
                 MF("%2", "%10", "%7") "v_add_u32 %6, %14, %6\n\t" MF("%3", "%11", "%7") \
                 : "+v"(acc[0][2]), "+v"(acc[1][2]), "+v"(acc[2][2]), "+v"(acc[3][2]), "+v"(x1), "+v"(vw), "+v"(vf) \
                 : "v"(x2), "v"(c0), "v"(c1), "v"(c2), "v"(c3), "v"(vx), "s"(d), "s"(d4) : "memory"); \
    asm volatile("s_waitcnt lgkmcnt(8)\n\t" MF("%0", "%9", "%5") "ds_read_b128 %4, %6 offset:2048\n\tv_add_u32 %7, 1, %7\n\t" MF("%1", "%10", "%5") \
                 "s_waitcnt lgkmcnt(2)\n\tv_readfirstlane_b32 %8, %13\n\t" MF("%2", "%11", "%5") MF("%3", "%12", "%5") \
                 "ds_read_b128 %5, %6 offset:3072\n\tds_write_b32 %14, %7\n\tv_add_u32 %6, %15, %6" \
                 : "+v"(acc[0][3]), "+v"(acc[1][3]), "+v"(acc[2][3]), "+v"(acc[3][3]), "+v"(x2), "+v"(x3), "+v"(vx), "+v"(done), "=&s"(seen) \
                 : "v"(c0), "v"(c1), "v"(c2), "v"(c3), "v"(vs), "v"(relad), "s"(d) : "memory"); \
  } while (0)
  lds_wait(c, C_FULL0, 1u);
  asm volatile("ds_read_b128 %0, %9\n\tds_read_b128 %1, %9 offset:1024\n\tds_read_b128 %2, %9 offset:2048\n\tds_read_b128 %3, %9 offset:3072\n\t"
               "ds_read_b128 %4, %8\n\tds_read_b128 %5, %8 offset:1024\n\tds_read_b128 %6, %8 offset:2048\n\tds_read_b128 %7, %8 offset:3072\n\t"
               "ds_write_b32 %10, %11\n\ts_waitcnt lgkmcnt(0)"
               : "=&v"(wa0), "=&v"(wa1), "=&v"(wa2), "=&v"(wa3), "=&v"(x0), "=&v"(x1), "=&v"(x2), "=&v"(x3) : "v"(vx), "v"(vw), "v"(relad), "v"(done) : "memory");
  vx += SLOTB; vw += SLOTB;
  wb0 = wa0; wb1 = wa0; wb2 = wa0; wb3 = wa0;
  asm volatile("s_nop 1" : "+v"(vx), "+v"(vw), "+v"(wb0), "+v"(wb1), "+v"(wb2), "+v"(wb3));
  for (u32 m = 0; m < 126u; m += 2u) {
    HSTEP(wa0, wa1, wa2, wa3, wb0, wb1, wb2, wb3, m);
    HSTEP(wb0, wb1, wb2, wb3, wa0, wa1, wa2, wa3, m + 1u);
  }
  HSTEP(wa0, wa1, wa2, wa3, wb0, wb1, wb2, wb3, 126u);
#define MF4(b, x) asm volatile(MF("%0", "%5", "%4") MF("%1", "%6", "%4") MF("%2", "%7", "%4") MF("%3", "%8", "%4") \
    : "+v"(acc[0][b]), "+v"(acc[1][b]), "+v"(acc[2][b]), "+v"(acc[3][b]) : "v"(x), "v"(wb0), "v"(wb1), "v"(wb2), "v"(wb3))
  asm volatile("s_waitcnt lgkmcnt(0)" ::: "memory");
  MF4(0, x0); MF4(1, x1); MF4(2, x2); MF4(3, x3);
  asm volatile("s_nop 7\n\ts_nop 7" : "+v"(acc[0][2]), "+v"(acc[1][2]), "+v"(acc[2][2]), "+v"(acc[3][2]), "+v"(acc[0][3]), "+v"(acc[1][3]), "+v"(acc[2][3]), "+v"(acc[3][3]),
               "+v"(acc[0][0]), "+v"(acc[1][0]), "+v"(acc[2][0]), "+v"(acc[3][0]), "+v"(acc[0][1]), "+v"(acc[1][1]), "+v"(acc[2][1]), "+v"(acc[3][1]));
  if (wv == 0) STAMP(c, 11);
  char* peer = p.ws[lane & 7];
  const rsrc_t rsL = mkrs(c.ws, WS_USED);
  u32 mq = 0;
  {
    float mx = 0.f;
#pragma unroll
    for (int e = 0; e < 64; e += 2) mx = fmaxf(fmaxf(mx, acc[e & 3][e >> 4][(e >> 2) & 3]), acc[(e & 3) + 1][e >> 4][(e >> 2) & 3]);
    const u32 wmx = wave_max(__float_as_uint(mx));
    if (lane == 0) {
      __hip_atomic_fetch_max(c.ctl + C_WGMAX, wmx, __ATOMIC_RELAXED, __HIP_MEMORY_SCOPE_WORKGROUP);
      CFENCE();
      if (lds_add(c.ctl + C_CMAX, 1u) == 7u) {
        __hip_atomic_store((GAS u64*)(c.ws + O_LT + c.wg * 8), (u64)c.call << 32 | lds_ld(c.ctl + C_WGMAX), __ATOMIC_RELAXED, __HIP_MEMORY_SCOPE_AGENT);
        mq = 1u;
      }
    }
  }
  u32 nz[2] = {0u, 0u};
#pragma unroll
  for (int e = 63; e >= 0; --e) nz[e >> 5] = __builtin_amdgcn_alignbit(nz[e >> 5], __float_as_uint(acc[e & 3][e >> 4][(e >> 2) & 3]), 31);
  nz[0] = ~nz[0]; nz[1] = ~nz[1];
  __builtin_amdgcn_raw_buffer_store_b64((u32x2){nz[0], nz[1]}, rsL, lane * 8, (int)(O_OM + c.wg * 4096 + wv * 512), SYS);
  if (mq) mq = __hip_atomic_fetch_add((GAS u32*)(c.ws + O_MQ), 1u, __ATOMIC_RELAXED, __HIP_MEMORY_SCOPE_AGENT) | 0x80000000u;
  VMWAIT0();
  const u32 zs = wv >> 2;
  if (lane == 0) lds_add(c.ctl + C_CMASK, 1u);
  mq = RFL(mq);
  if (mq) {
    lds_wait(c, C_CMASK, 8u);
    if (lane < NR && lane != c.rank) g_st_sys(peer + O_OF + c.rank * 4096 + F_MF + (mq & 255u) * 4, c.call << 12 | c.wg);
  }
  const u32 cnt = __popc(nz[0]) + __popc(nz[1]);
  const u32 inc = wave_incl_scan(cnt), V = __builtin_amdgcn_readlane(inc, 63);
  lds_wait(c, C_CMAX, 8u);
  LAS unsigned char* st = c.lds + L_RING + wv * 4096;
#pragma unroll
  for (int k = 0; k < 4; ++k) *(LAS u32x4*)(st + k * 1024 + lane * 16) = (u32x4){0u, 0u, 0u, 0u};
  u32 sel0[16];
#pragma unroll
  for (int D = 0; D < 16; ++D) sel0[D] = put_sel(c, nib4(nz[D >> 3], D & 7));
#pragma unroll
  for (int g = 0; g < 4; ++g)
#pragma unroll
    for (int b = 0; b < 4; ++b) {
#pragma unroll
      for (int r = 0; r < 4; ++r) acc[g][b][r] = __builtin_amdgcn_fmed3f(acc[g][b][r], 0.f, __builtin_inff());
      asm volatile("" : "+v"(acc[g][b]));
    }
  lds_wait(c, C_GVALID, 1u);
  if (wv == 0) STAMP(c, 31);
  const u32 tm = g_ld_agent(c.ws + O_TMO);
  const float amax = __uint_as_float(RFL(lds_ld(c.ctl + C_GMAX)));
  const float inv = amax > 0.f ? 448.0f / amax : 0.f;
  u32 q[16];
#pragma unroll
  for (int b = 0; b < 4; ++b) {
    const f32x4 a0 = acc[0][b] * inv, a1 = acc[1][b] * inv, a2 = acc[2][b] * inv, a3 = acc[3][b] * inv;
#pragma unroll
    for (int r = 0; r < 4; ++r)
      q[4 * b + r] = (u32)__builtin_amdgcn_cvt_pk_fp8_f32(a2[r], a3[r], __builtin_amdgcn_cvt_pk_fp8_f32(a0[r], a1[r], 0, false), true);
  }
  {
    u32 o = inc - cnt, se = 0, carry = 0;
#pragma unroll
    for (int D = 0; D < 16; ++D) { put4(st, o, carry, q[D], nib4(nz[D >> 3], D & 7), sel0[D]); se = __builtin_amdgcn_sad_u8(q[D] & 0x78u * B1, 0u, se); }
    lds_or(st + (o & ~3u), carry);
    const Geo G = geo(V);
    const u32 top = min(pick_top(__builtin_amdgcn_readlane(wave_incl_scan(se), 63) >> 3, V), 15u), used = G.sz >> 2;
    if (V + lane < G.sz) st[V + lane] = (unsigned char)(top << 3);
    CFENCE();
    if (wv == 0) STAMP(c, 28);
    u32 d[9], n[9], l2a, l2b, l3, sb;
    const bool in8 = (u32)lane < G.nc, in9 = 8u * G.nc + lane < used;
#pragma unroll
    for (u32 k = 0; k < 9u; ++k) d[k] = *(LAS u32*)(st + (k * G.nc + lane) * 4u);
#pragma unroll
    for (u32 k = 0; k < 9u; ++k) {
      d[k] = (k < 8u ? in8 : in9) ? d[k] : (top << 3) * B1;
      n[k] = (((d[k] >> 3) & 15u * B1) + (15u - top) * B1) & 15u * B1;
    }
    Lv g;
    const bool shrt = V <= 2304u && enc_levels<9, 1024>(c, n, st + G.sz, 4096u - G.sz, g, l2a, l2b, l3);
    CFENCE();
    if (wv == 0) STAMP(c, 29);
    if (shrt) {
      LAS u32* s = (LAS u32*)st + lane;
#pragma unroll
      for (u32 k = 0; k < 9u; ++k) d[k] = (d[k] & 7u * B1) | n[k] << 3;
      if (in8) {
        s[0] = d[0] | ((d[5] << 5) & 0xE0u * B1); s[G.nc] = d[1] | ((d[6] << 5) & 0xE0u * B1); s[2u * G.nc] = d[2] | ((d[7] << 5) & 0xE0u * B1);
        s[3u * G.nc] = d[3] | ((d[5] << 2) & 0x60u * B1) | ((d[7] << 4) & 0x80u * B1);
        s[4u * G.nc] = d[4] | ((d[6] << 2) & 0x60u * B1) | ((d[7] << 3) & 0x80u * B1);
      }
      if (in9) s[5u * G.nc] = d[8];
      s += used - 3u * G.nc;
      if ((u32)lane < g.nc2) s[0] = l2a;
      if ((u32)lane < g.nc3) s[g.nc2] = l3;
      sb = (G.sz - 12u * G.nc + (g.nc2 + g.nc3) * 4u + 15u) & ~15u;
    } else {
      const u32 nc = row_bytes(V) >> 2;
      for (u32 col = lane; col < nc; col += 64u) {
        LAS u32* s = (LAS u32*)st + col;
        const u32 d7 = s[7u * nc];
#pragma unroll
        for (u32 k = 0; k < 7u; ++k) s[k * nc] |= (d7 << (7u - k)) & 0x80u * B1;
      }
      sb = (28u * nc + 15u) & ~15u;
    }
    if (lane == 0) { lds_st(c.ctl + C_VSZ + wv, sb | top << 16 | (u32)shrt << 20); CFENCE(); lds_add(c.ctl + C_CSZ0 + zs, 1u); }
    lds_wait(c, C_CSZ0 + zs, 4u);
    if (wv == 0) STAMP(c, 30);
    const u32x4 z4 = *(LAS u32x4*)(c.ctl + C_VSZ + zs * 4);
    u32x4 hdr;
    hdr.x = z4.x + 16u; hdr.y = z4.y + (hdr.x & 0xFFFFu); hdr.z = z4.z + (hdr.y & 0xFFFFu); hdr.w = z4.w + (hdr.z & 0xFFFFu);
    const u32 ovb = O_OV + (c.wg * 2 + zs) * 16384, woff = (((wv & 2) ? ((wv & 1) ? hdr.w : hdr.z) : ((wv & 1) ? hdr.y : hdr.x)) & 0xFFFFu) - sb;
    if ((wv & 3) == 0 && lane == 0) st128<SYS>(hdr, rsL, ovb);
    for (u32 k = lane * 16; k < sb; k += 1024u)
      st128<SYS>(*(LAS u32x4*)(st + k), rsL, k + (ovb + woff));
    VMWAIT0();
    u32 t = 0;
    if (lane == 0) t = lds_add(c.ctl + C_CVAL0 + zs, 1u);
    if (RFL(t) == 3u && lane < NR && lane != c.rank) g_st_sys(peer + O_OF + c.rank * 4096 + F_VF + (c.wg * 2 + zs) * 4, c.call << 12 | (hdr.w & 0xFFFFu) >> 4);
  }
  if (wv == 0) STAMP(c, 12);
  const rsrc_t rsO = mkrs(p.out, (u32)MM * NN);
  const u32x4 nan4 = {0x7Fu * B1, 0x7Fu * B1, 0x7Fu * B1, 0x7Fu * B1};
  {
    const u32 obase = (u32)(mi * 256 + mw * 64 + (lane & 15)) * NN + c.rank * 2048 + ni * 128 + h * 64 + (lane >> 4) * 16;
    const bool bad = tm == c.call || lds_ld(c.ctl + C_ABORT) != 0u;
#pragma unroll
    for (int b = 0; b < 4; ++b) st128<AGT>(bad ? nan4 : (u32x4){q[4 * b], q[4 * b + 1], q[4 * b + 2], q[4 * b + 3]}, rsO, obase + b * 16 * NN);
  }
  const u32 src = (c.rank + 1 + wg_link(c.wg)) & 7;
  for (u32 kv = zs;; kv += 2u) {
    const u32 e = q_wait(c, C_QV, kv), ht = e & 0xFFFu, pb = ((e >> 12) & 2047u) * 32768u;
    if (ht >= 512u) break;
    const u32 dt = ht >> 1, dw = (ht & 1u) * 4u + (wv & 3);
    const u32x2 m = __builtin_amdgcn_raw_buffer_load_b64(rsL, lane * 8, (int)(O_MIN + (src * 256 + dt) * 4096u + dw * 512u), AGT);
    const u32 dbase = (u32)(tile_mi(dt) * 256 + (dw >> 1) * 64 + (lane & 15)) * NN + src * 2048 + tile_ni(dt) * 128 + (dw & 1) * 64 + (lane >> 4) * 16;
    const u32 cn = __popc(m.x) + __popc(m.y), in2 = wave_incl_scan(cn), V2 = __builtin_amdgcn_readlane(in2, 63);
    LAS unsigned char* sl = c.lds + L_STG + (kv & 3u) * 16384u;
    lds_wait(c, C_PROG, pb + 16u);
    const u32 hw = RFL(*(LAS u32*)(sl + (wv & 3) * 4)), hp = RFL(*(LAS u32*)(sl + ((wv + 3) & 3) * 4));
    const u32 wo = (wv & 3) ? min(hp & 0xFFF0u, 16384u - 3584u) : 16u, top = (hw >> 16) & 15u;
    lds_wait(c, C_PROG, pb + (hw & 0xFFFFu));
    const u32 tm2 = g_ld_agent(c.ws + O_TMO);
    if ((hw >> 20) & 1u) {
      const Geo G = geo(min(V2, 2304u));
      const bool in8 = (u32)lane < G.nc, in9 = 8u * G.nc + lane < G.sz >> 2;
      LAS u32* sg = (LAS u32*)(sl + wo) + lane;
      u32 y[9], n[9];
#pragma unroll
      for (u32 k = 0; k < 6u; ++k) y[k < 5u ? k : 8u] = sg[k * G.nc];
#pragma unroll
      for (u32 k = 0; k < 5u; ++k) y[k] = in8 ? y[k] : ~0u;
      y[8] = in9 ? y[8] : ~0u;
      y[5] = ((y[0] >> 5) & 7u * B1) | ((y[3] >> 2) & 0x18u * B1);
      y[6] = ((y[1] >> 5) & 7u * B1) | ((y[4] >> 2) & 0x18u * B1);
      y[7] = ((y[2] >> 5) & 7u * B1) | ((y[3] >> 4) & 8u * B1) | ((y[4] >> 3) & 0x10u * B1);
#pragma unroll
      for (u32 k = 0; k < 9u; ++k) n[k] = (y[k] >> 3) & 3u * B1;
      dec_levels<9, 1024>(c, n, sl + wo + G.sz - 12u * G.nc, st + G.sz, 4096u - G.sz);
#pragma unroll
      for (u32 k = 0; k < 9u; ++k) y[k] = (y[k] & 7u * B1) | ((n[k] + (top + 1u) * B1) & 15u * B1) << 3;
      CFENCE();
      if (in8) {
#pragma unroll
        for (u32 k = 0; k < 8u; ++k) *(LAS u32*)(st + (k * G.nc + lane) * 4u) = y[k];
      }
      if (in9) *(LAS u32*)(st + (8u * G.nc + lane) * 4u) = y[8];
    } else {
      const u32 nc = row_bytes(V2) >> 2;
      for (u32 col = lane; col < nc; col += 64u) {
        LAS u32* sg = (LAS u32*)(sl + wo) + col;
        LAS u32* d = (LAS u32*)st + col;
        u32 d7 = 0;
#pragma unroll
        for (u32 k = 0; k < 7u; ++k) { const u32 v = sg[k * nc]; d7 |= (v & 0x80u * B1) >> (7u - k); d[k * nc] = v & 0x7Fu * B1; }
        d[7u * nc] = d7;
      }
    }
    CFENCE();
    if (lane == 0) lds_add(c.ctl + C_DFREE0 + (kv & 3u), 1u);
    CFENCE();
    u32 o = in2 - cn, ov[16];
#pragma unroll
    for (int D = 0; D < 16; ++D) ov[D] = get4(c, st, o, nib4(m[D >> 3], D & 7));
    const bool bad = tm2 == c.call || lds_ld(c.ctl + C_ABORT) != 0u;
#pragma unroll
    for (int k = 0; k < 4; ++k) st128<AGT>(bad ? nan4 : (u32x4){ov[4 * k], ov[4 * k + 1], ov[4 * k + 2], ov[4 * k + 3]}, rsO, dbase + k * 16 * NN);
    CFENCE();
  }
}

__global__ __launch_bounds__(1024, 4) void fused_gemm_ar(Params p) {
  extern __shared__ __attribute__((aligned(16))) unsigned char smem[];
  Ctx c;
  c.lds = (LAS unsigned char*)smem; c.ctl = (LAS u32*)c.lds; c.ws = p.ws[p.rank]; c.t0 = now(); c.call = 0;
  c.lane = threadIdx.x & 63; c.wg = blockIdx.x; c.rank = p.rank;
  const int wv = RFL(threadIdx.x >> 6), tid = threadIdx.x;
  if (tid >= 512) *(LAS u32*)(c.lds + L_REL + (tid - 512) * 4) = 0u;
  else if (tid < C_WORDS || (tid >= L_GT / 4 && tid < L_GT / 4 + 16)) c.ctl[tid] = 0u;
  else if (tid >= 192 && tid < 256) {
    const u32 nib = tid & 15; u32 s = 0, k = (tid - 192) >> 4;
    for (int g = 0; g < 4; ++g) { if ((nib >> g) & 1u) s |= k++ << (8 * g); else s |= 0x0Cu << (8 * g); }
    *(LAS u32*)(c.lds + L_LUTE + (tid - 192) * 4) = s;
  } else if (tid >= 256 && tid < 272) {
    const u32 nib = tid - 256; u32 s = 0x0C0C0C0Cu, k = 0;
    for (u32 g = 0; g < 4; ++g) if ((nib >> g) & 1u) { s = (s & ~(255u << (8 * k))) | g << (8 * k); ++k; }
    *(LAS u32*)(c.lds + L_LUTC + nib * 4) = s;
  }
  asm volatile("s_waitcnt lgkmcnt(0)" ::: "memory");
  __builtin_amdgcn_s_barrier();
  CFENCE();
  const int mi = tile_mi(c.wg), ni = tile_ni(c.wg);
  if (wv == 11) STAMP(c, 0);
  if (wv < 8) role_mfma(c, p, wv, mi, ni);
  else if (wv < 11) role_loader(c, wv - 8, mi, ni);
  else if (wv == 11) role_sched(c, p, mi, ni);
  else if (wv == 12) role_puller(c, p);
  else if (wv == 13) role_feeder(c);
  else role_decoder(c, wv - 14);
}
}

extern "C" void kernel_launch(int rank, int nranks, ncclComm_t comm, hipStream_t stream,
                              void* const* d_in, const int* in_sizes, int n_in,
                              void* d_out, int out_size, const int* in_dtypes, int out_dtype,
                              void* const* d_out_all, void* const* d_ws_all, size_t ws_bytes) {
  if (nranks != NR || (unsigned)rank >= (unsigned)NR || n_in < 2 || ws_bytes < WS_USED || in_sizes[0] != MM * KS || in_sizes[1] != KS * NN || out_size != MM * NN) {
    fprintf(stderr, "kernel_launch: unexpected problem shape\n");
    return;
  }
  static bool init = false;
  if (!init) {
    hipError_t e = hipFuncSetAttribute((const void*)fused_gemm_ar, hipFuncAttributeMaxDynamicSharedMemorySize, (int)LDS_BYTES);
    if (e != hipSuccess) { fprintf(stderr, "hipFuncSetAttribute: %s\n", hipGetErrorString(e)); return; }
    init = true;
  }
  Params p;
  p.x = (const unsigned short*)d_in[0]; p.w = (const unsigned short*)d_in[1]; p.out = (unsigned char*)d_out;
  for (int i = 0; i < NR; ++i) p.ws[i] = (char*)d_ws_all[i];
  p.rank = rank; p.pad = 0;
  fused_gemm_ar<<<NWG, 1024, LDS_BYTES, stream>>>(p);
}
```

```cpp
#include <hip/hip_runtime.h>
#include <rccl/rccl.h>
#include <cstdio>

typedef unsigned u32;
typedef unsigned long long u64;
typedef u32 u32x4 __attribute__((ext_vector_type(4)));
typedef u32 u32x2 __attribute__((ext_vector_type(2)));
typedef unsigned short u16x2 __attribute__((ext_vector_type(2)));
typedef short bf16x8 __attribute__((ext_vector_type(8)));
typedef float f32x4 __attribute__((ext_vector_type(4)));
typedef __amdgpu_buffer_rsrc_t rsrc_t;
#define LAS __attribute__((address_space(3)))
#define GAS __attribute__((address_space(1)))
#define DI __device__ __forceinline__
#define RFL(x) __builtin_amdgcn_readfirstlane(x)
#define ALL(x) (__builtin_amdgcn_ballot_w64(!(x)) == 0ull)
#define ANY(x) (__builtin_amdgcn_ballot_w64(x) != 0ull)
#define CFENCE() asm volatile("" ::: "memory")
#define VMWAIT0() asm volatile("s_waitcnt vmcnt(0)" ::: "memory")

namespace {
constexpr int NR = 8, MM = 4096, KS = 512, NN = 16384, NWG = 256;
constexpr int NBLK = 768;
constexpr u32 SYS = 17, AGT = 16;
constexpr int LEADER = 16;
constexpr u64 DEADLINE = 50000000ull;
constexpr u32 O_CALL = 0;
constexpr u32 O_TMO = 32768;
constexpr u32 O_TICK = O_TMO + 128;
constexpr u32 O_MQ = O_TICK + 3072;
constexpr u32 O_GT = O_MQ + 128;
constexpr u32 O_LT = O_GT + 128;
constexpr u32 O_RF = O_LT + 2048;
constexpr u32 O_FX = O_RF + NR * NBLK * 4;
constexpr u32 O_FW = O_FX + 16 * 64 * 16;
constexpr u32 O_OF = O_FW + 16 * 64 * 8;
constexpr u32 F_MF = 0;
constexpr u32 F_MI = 2048;
constexpr u32 F_VF = 4096;
constexpr u32 O_DBG = 512 * 1024;
constexpr u32 O_OBX = 1u << 20;
constexpr u32 O_OBW = O_OBX + 512 * 8192;
constexpr u32 O_IMG = O_OBW + 2048 * 8192;
constexpr u32 O_OM = O_IMG + 64 * 96 * 8192;
constexpr u32 O_OV = O_OM + 512 * 8192;
constexpr u32 O_MIN = O_OV + 512 * 16384;
constexpr u32 WS_USED = O_MIN + NR * 512 * 8192;
static_assert(O_OF + NR * 8192 <= O_DBG, "control block overflow");
constexpr u32 T_MASK = NBLK, T_VAL = T_MASK + 512, T_END = T_VAL + 512;
enum { C_CALL = 0, C_ABORT, C_WGMAX, C_CMAX, C_SPARE, C_CVAL0, C_CVAL1, C_GVALID, C_GMAX, C_PSTART, C_PROG, C_PACKED,
       C_FULL0 = 12, C_LEAD = 17, C_CSZ0 = 18, C_CSZ2 = 20, C_CPRE0 = 22, C_DFREE0 = 24, C_IFREE0 = 28, C_VSZ = 32, C_QI = 40, C_QV = 48, C_MQ0 = 56, C_ORDER = 64, C_WORDS = 128 };
constexpr u32 L_LUTC = C_WORDS * 4;
constexpr u32 L_GT = L_LUTC + 128;
constexpr u32 L_PEER = L_GT + 64;
constexpr u32 L_LUTE = L_LUTC + 256;
constexpr u32 L_REL = 1024;
constexpr u32 L_RFC = L_REL + 2048;
constexpr u32 NS = 5, SLOTB = 24576;
constexpr u32 L_RING = L_RFC + 4096;
constexpr u32 L_STG = L_RING + 32768;
constexpr u32 L_IN = L_RING + NS * SLOTB;
constexpr u32 LDS_BYTES = L_IN + 4 * 8192;
static_assert(LDS_BYTES <= 163840 && LDS_BYTES > 81920 && L_STG + 5 * 16384 <= L_IN, "one workgroup per CU");
#define PLEAD 2048
#define WIN1 2
#define WIN2 4
#define T_HALF 640
#define PACK_T0 100
#define PACK_DT 40
#define NRAW 20
#define T_W1 728
#define T_W2 T_END
#define T_WA 0u
#define G0FREE 1
#define T_AH (T_END - 80u)
#define GATEM 10
#define SCHED_Q 3u
#define PMED 5u
#define OUTAUX 0u
#define KFIX 6u
#define T_W0 128

struct Params { const unsigned short* x; const unsigned short* w; unsigned char* out; char* ws[NR]; int rank; int pad; };

DI u64 now() { return __builtin_amdgcn_s_memrealtime(); }
DI rsrc_t mkrs(const void* p, u32 bytes) { return __builtin_amdgcn_make_buffer_rsrc((void*)p, (short)0, (int)bytes, 0x00020000); }
template <u32 AUX> DI void st128(u32x4 v, rsrc_t rs, u32 off) { __builtin_amdgcn_raw_buffer_store_b128(v, rs, (int)off, 0, AUX); }
DI void st32(u32 v, rsrc_t rs, u32 off) { __builtin_amdgcn_raw_buffer_store_b32(v, rs, (int)off, 0, SYS); }
DI u32 prm(u32 a, u32 b, u32 s) { return __builtin_amdgcn_perm(a, b, s); }
DI u32 bfi(u32 m, u32 a, u32 b) { return (m & a) | (~m & b); }
DI u32 lds_ld(LAS u32* p) { return __hip_atomic_load(p, __ATOMIC_RELAXED, __HIP_MEMORY_SCOPE_WORKGROUP); }
DI void lds_st(LAS u32* p, u32 v) { __hip_atomic_store(p, v, __ATOMIC_RELAXED, __HIP_MEMORY_SCOPE_WORKGROUP); }
DI u32 lds_add(LAS u32* p, u32 v) { return __hip_atomic_fetch_add(p, v, __ATOMIC_RELAXED, __HIP_MEMORY_SCOPE_WORKGROUP); }
DI void lds_or(LAS unsigned char* p, u32 v) { __hip_atomic_fetch_or((LAS u32*)p, v, __ATOMIC_RELAXED, __HIP_MEMORY_SCOPE_WORKGROUP); }
DI u32 g_ld_agent(const char* p) { return __hip_atomic_load((GAS u32*)p, __ATOMIC_RELAXED, __HIP_MEMORY_SCOPE_AGENT); }
DI u32 g_ld_sys(const char* p) { return __hip_atomic_load((GAS u32*)p, __ATOMIC_RELAXED, __HIP_MEMORY_SCOPE_SYSTEM); }
DI void g_st_agent(char* p, u32 v) { __hip_atomic_store((GAS u32*)p, v, __ATOMIC_RELAXED, __HIP_MEMORY_SCOPE_AGENT); }
DI void g_st_sys(char* p, u32 v) { __hip_atomic_store((GAS u32*)p, v, __ATOMIC_RELAXED, __HIP_MEMORY_SCOPE_SYSTEM); }

struct Ctx {
  LAS unsigned char* lds; LAS u32* ctl; char* ws; u64 t0; u32 call; int lane, wg, rank;
};
DI void give_up(const Ctx& c) { if (c.lane == 0) { lds_st(c.ctl + C_ABORT, 1u); if (c.call) g_st_agent(c.ws + O_TMO, c.call); } }
DI bool expired(const Ctx& c, u32& spins) {
  if ((++spins & 15u) != 0u) return false;
  if (RFL(lds_ld(c.ctl + C_ABORT)) != 0u || RFL(g_ld_agent(c.ws + O_TMO)) == c.call || now() - c.t0 > DEADLINE) { give_up(c); return true; }
  return false;
}
DI bool lds_expired(const Ctx& c, u32& spins) {
  if ((++spins & 63u) == 0u && (RFL(lds_ld(c.ctl + C_ABORT)) != 0u || now() - c.t0 > DEADLINE + DEADLINE / 4)) { give_up(c); return true; }
  __builtin_amdgcn_s_sleep(1);
  return false;
}
DI u32 lds_wait(const Ctx& c, int word, u32 target) {
  u32 v, spins = 0;
  while ((v = RFL(lds_ld(c.ctl + word))) < target) if (lds_expired(c, spins)) break;
  CFENCE();
  return v;
}
DI u32 q_wait(const Ctx& c, int q, u32 k) {
  u32 v, spins = 0;
  while (((v = RFL(lds_ld(c.ctl + q + (k & 7u)))) >> 23) != ((k + 1u) & 511u)) if (lds_expired(c, spins)) { v = ~0u; break; }
  CFENCE();
  return RFL(v);
}
#define STAMP(c, i) do {} while (0)
DI void dma4(rsrc_t rs, u32 voff, u32 soff, u32 ldsdst) {
  u32 keep;
  asm volatile("s_nop 4\n\ts_mov_b32 %0, m0\n\ts_mov_b32 m0, %1\n\ts_nop 0\n\t"
               "buffer_load_dwordx4 %2, %3, %4 offen sc0 sc1 lds\n\t"
               "buffer_load_dwordx4 %2, %3, %4 offen offset:1024 sc0 sc1 lds\n\t"
               "buffer_load_dwordx4 %2, %3, %4 offen offset:2048 sc0 sc1 lds\n\t"
               "buffer_load_dwordx4 %2, %3, %4 offen offset:3072 sc0 sc1 lds\n\t"
               "s_mov_b32 m0, %0"
               : "=&s"(keep) : "s"(RFL(ldsdst)), "v"(voff), "s"(rs), "s"(RFL(soff)) : "memory");
}
DI void dma1(rsrc_t rs, u32 voff, u32 soff, u32 ldsdst) {
  u32 keep;
  asm volatile("s_nop 4\n\ts_mov_b32 %0, m0\n\ts_mov_b32 m0, %1\n\ts_nop 0\n\t"
               "buffer_load_dwordx4 %2, %3, %4 offen sc0 sc1 lds\n\t"
               "s_mov_b32 m0, %0"
               : "=&s"(keep) : "s"(RFL(ldsdst)), "v"(voff), "s"(rs), "s"(RFL(soff)) : "memory");
}
template <int CTRL, int ROWS> DI u32 dpp0(u32 v) { return (u32)__builtin_amdgcn_update_dpp(0, (int)v, CTRL, ROWS, 0xF, false); }
DI u32 wave_incl_scan(u32 v) {
  v += dpp0<0x111, 0xF>(v); v += dpp0<0x112, 0xF>(v); v += dpp0<0x114, 0xF>(v); v += dpp0<0x118, 0xF>(v);
  v += dpp0<0x142, 0xA>(v); v += dpp0<0x143, 0xC>(v);
  return v;
}
DI u32 wave_max(u32 v) {
  v = max(v, dpp0<0x111, 0xF>(v)); v = max(v, dpp0<0x112, 0xF>(v)); v = max(v, dpp0<0x114, 0xF>(v)); v = max(v, dpp0<0x118, 0xF>(v));
  v = max(v, dpp0<0x142, 0xA>(v)); v = max(v, dpp0<0x143, 0xC>(v));
  return __builtin_amdgcn_readlane(v, 63);
}
DI int tile_mi(int wg) { return ((wg & 7) >> 1) * 4 + ((wg >> 3) & 3); }
DI int tile_ni(int wg) { return (wg & 1) * 8 + (wg >> 5); }
DI int wg_late(int wg) { return (wg >> 4) & 1; }
DI int wg_link(int wg) { return (((wg & 15) | (wg >> 5) << 4) + 3 * wg_late(wg)) % 7; }
DI u32 link_wgs(int link, int late) { return (u32)(127 + 7 - (link + 7 - 3 * late) % 7) / 7u; }
DI u32 row_bytes(u32 V) { return (((V + 7u) >> 3) + 3u) & ~3u; }
DI void tell_peers(const Ctx& c, u32 off, u32 v) { if (c.lane < NR && c.lane != c.rank) g_st_sys(*(char* LAS*)(c.lds + L_PEER + c.lane * 8) + off, v); }
DI void image_flag(const Ctx& c, int kstep, int r) {
  if (c.lane == 0) g_st_agent(c.ws + (r < 64 ? O_FX + ((r >> 2) * 64 + kstep) * 16 + (r & 3) * 4 : O_FW + (((r - 64) >> 1) * 64 + kstep) * 8 + (r & 1) * 4), c.call);
}

constexpr u32 B1 = 0x01010101u;
DI u32 nib4(u32 m, int k) { return (k ? m >> (4 * k - 2) : m << 2) & 0x3Cu; }
DI u32 to_nib(u32 bits) { return __builtin_amdgcn_udot4(bits, 0x08040201u, 0u, false); }
DI u32 put_sel(const Ctx& c, u32 n4) { return *(LAS u32*)(c.lds + L_LUTC + n4); }
DI u32 ldsa(LAS unsigned char* p) { return (u32)(size_t)p; }
DI void or_at(u32 a, u32 v) { lds_or((LAS unsigned char*)(size_t)(a & ~3u), v); }
DI u32 put4(u32& a, u32& carry, u32 v, u32 n4, u32 sel) {
  const u32 c4 = prm(0u, v, sel);
  const u64 w = (u64)c4 << ((a << 3) & 24u);
  or_at(a, (u32)w | carry);
  carry = (u32)(w >> 32);
  a += __popc(n4);
  return c4;
}
template <u32 N> DI void get_all(const Ctx& c, u32 a, const u32* m, u32 (&x)[N]) {
  u32 sel[N], hi[N];
#pragma unroll
  for (u32 k = 0; k < N; ++k) {
    const u32 n4 = nib4(m[k >> 3], k & 7u);
    sel[k] = *(LAS u32*)(c.lds + L_LUTE + (((a << 6) & 0xC0u) | n4));
    LAS u32* d = (LAS u32*)(size_t)(a & ~3u);
    x[k] = d[0]; hi[k] = d[1];
    a += __popc(n4);
  }
  __builtin_amdgcn_sched_barrier(0);
#pragma unroll
  for (u32 k = 0; k < N; ++k) x[k] = prm(hi[k], x[k], sel[k]);
}
struct Lv { u32 rows2, nc2, nc3; };
template <u32 R> DI bool enc_tail(const Ctx& c, u32 V2, LAS unsigned char* s, u32 cap, Lv& g, u32& l2a, u32& l2b, u32& l3) {
  const u32 nc2 = (V2 + 4u * R - 1u) / (4u * R), b2 = R * nc2 * 4u;
  u32 e[R], sel[R], nb3 = 0;
  l2a = 0u; l2b = 0u;
#pragma unroll
  for (u32 r = 0; r < R; ++r) e[r] = *(LAS u32*)(s + (r * nc2 + c.lane) * 4u);
#pragma unroll
  for (u32 r = 0; r < R; ++r) {
    e[r] = (u32)c.lane < nc2 ? e[r] : 0u;
    const u32 lo3 = e[r] & 7u * B1, b3 = (e[r] >> 3) & B1;
    const u32 c2 = bfi((b3 << 8) - b3, prm(0x00000002u, 0x03000000u, lo3), prm(0u, 1u, lo3));
    if (r < 4u) l2a |= c2 << (2u * r); else l2b |= c2 << (2u * r - 8u);
    nb3 |= to_nib((c2 | c2 >> 1) & B1) << (4u * r);
  }
  nb3 = ~nb3 & (R == 8u ? ~0u : 0xFFFFu);
#pragma unroll
  for (u32 r = 0; r < R; ++r) sel[r] = put_sel(c, nib4(nb3, r));
  const u32 cn3 = __popc(nb3), in3 = wave_incl_scan(cn3), V3 = __builtin_amdgcn_readlane(in3, 63);
  g.rows2 = R; g.nc2 = nc2; g.nc3 = (V3 + 7u) >> 3;
  if (V3 > 512u || b2 + g.nc3 * 8u + 8u > cap) return false;
  u32 a = ldsa(s) + b2 + in3 - cn3, carry = 0;
#pragma unroll
  for (u32 r = 0; r < R; ++r) put4(a, carry, e[r], nib4(nb3, r), sel[r]);
  or_at(a, carry);
  CFENCE();
  l3 = *(LAS u32*)(s + b2 + c.lane * 4u) | *(LAS u32*)(s + b2 + (g.nc3 + c.lane) * 4u) << 4;
  return true;
}
template <u32 N, u32 MAX2> DI bool enc_levels(const Ctx& c, u32 (&n)[N], LAS unsigned char* s, u32 cap, Lv& g, u32& l2a, u32& l2b, u32& l3) {
  u32 nb[2] = {0u, 0u}, c1[N], sel[N];
#pragma unroll
  for (u32 k = 0; k < N; ++k) {
    const u32 m = ((n[k] + 3u * B1) >> 4) & B1;
    c1[k] = n[k] & (m + (m << 1));
    nb[k >> 3] |= to_nib(m) << (4u * (k & 7u));
  }
  nb[0] = ~nb[0]; nb[1] = ~nb[1] & (N == 16u ? ~0u : (1u << (4u * (N & 7u))) - 1u);
#pragma unroll
  for (u32 k = 0; k < N; ++k) sel[k] = put_sel(c, nib4(nb[k >> 3], k & 7u));
  const u32 cnt = __popc(nb[0]) + __popc(nb[1]), inc = wave_incl_scan(cnt), V2 = __builtin_amdgcn_readlane(inc, 63);
  if (V2 > MAX2 || V2 + 40u > cap) return false;
  u32 a = ldsa(s) + inc - cnt, carry = 0;
#pragma unroll
  for (u32 k = 0; k < N; ++k) put4(a, carry, n[k], nib4(nb[k >> 3], k & 7u), sel[k]);
  or_at(a, carry);
  CFENCE();
#pragma unroll
  for (u32 k = 0; k < N; ++k) n[k] = c1[k];
  if (MAX2 > 1024u && V2 > 1024u) return enc_tail<8>(c, V2, s, cap, g, l2a, l2b, l3);
  return enc_tail<4>(c, V2, s, cap, g, l2a, l2b, l3);
}
template <u32 R> DI void dec_tail(const Ctx& c, u32 V2, LAS unsigned char* in, LAS unsigned char* s, u32 cap) {
  const u32 nc2 = (V2 + 4u * R - 1u) / (4u * R), b2 = R * nc2 * 4u;
  const bool col = (u32)c.lane < nc2;
  u32 l2[2] = {*(LAS u32*)(in + c.lane * 4u), R == 8u ? *(LAS u32*)(in + (nc2 + c.lane) * 4u) : 0u}, nb3 = 0, e[R];
  const u32 l3 = *(LAS u32*)(in + R * nc2 + c.lane * 4u);
  l2[0] = col ? l2[0] : ~0u; l2[1] = col ? l2[1] : ~0u;
#pragma unroll
  for (u32 r = 0; r < R; ++r) nb3 |= (to_nib((l2[r >> 2] | l2[r >> 2] >> 1) & B1 << (2u * (r & 3u))) >> (2u * (r & 3u))) << (4u * r);
  nb3 = ~nb3 & (R == 8u ? ~0u : 0xFFFFu);
  const u32 cn3 = __popc(nb3), in3 = wave_incl_scan(cn3), V3 = __builtin_amdgcn_readlane(in3, 63), nc3 = (V3 + 7u) >> 3;
  if (V3 > 512u || b2 + nc3 * 8u + 8u > cap) return;
  if ((u32)c.lane < nc3) {
    *(LAS u32*)(s + b2 + c.lane * 4u) = ((l3 & 15u * B1) + 14u * B1) & 15u * B1;
    *(LAS u32*)(s + b2 + (nc3 + c.lane) * 4u) = (((l3 >> 4) & 15u * B1) + 14u * B1) & 15u * B1;
  }
  CFENCE();
  get_all<R>(c, ldsa(s) + b2 + in3 - cn3, &nb3, e);
  if (col) {
#pragma unroll
    for (u32 r = 0; r < R; ++r) *(LAS u32*)(s + (r * nc2 + c.lane) * 4u) = e[r] | prm(0u, 0x090A0E00u, (l2[r >> 2] >> (2u * (r & 3u))) & 3u * B1);
  }
}
template <u32 N, u32 MAX2> DI void dec_levels(const Ctx& c, const u32* l1, u32 add, u32 (&o)[N], LAS unsigned char* in, LAS unsigned char* s, u32 cap) {
  u32 nb[2] = {0u, 0u};
#pragma unroll
  for (u32 k = 0; k < N; ++k) nb[k >> 3] |= (to_nib((l1[k >> 2] | l1[k >> 2] >> 1) & B1 << (2u * (k & 3u))) >> (2u * (k & 3u))) << (4u * (k & 7u));
  nb[0] = ~nb[0]; nb[1] = ~nb[1] & (N == 16u ? ~0u : (1u << (4u * (N & 7u))) - 1u);
  const u32 cnt = __popc(nb[0]) + __popc(nb[1]), inc = wave_incl_scan(cnt), V2 = __builtin_amdgcn_readlane(inc, 63);
  if (V2 <= MAX2 && V2 + 40u <= cap) {
    if (MAX2 > 1024u && V2 > 1024u) dec_tail<8>(c, V2, in, s, cap); else dec_tail<4>(c, V2, in, s, cap);
  }
  CFENCE();
  get_all<N>(c, ldsa(s) + inc - cnt, nb, o);
#pragma unroll
  for (u32 k = 0; k < N; ++k) o[k] += prm(0u, 0x0D0C0B00u, (l1[k >> 2] >> (2u * (k & 3u))) & 3u * B1) + add;
}
DI u32 pick_top(u32 sum, u32 cnt) { return (u32)((float)sum * __builtin_amdgcn_rcpf((float)max(cnt, 1u)) + 1.83f); }

DI u32x4 piece(const u32 (&lo)[16], const u32 (&hi)[16], int t) {
  return (u32x4){prm(hi[2 * t], lo[2 * t], 0x05010400u), prm(hi[2 * t], lo[2 * t], 0x07030602u),
                 prm(hi[2 * t + 1], lo[2 * t + 1], 0x05010400u), prm(hi[2 * t + 1], lo[2 * t + 1], 0x07030602u)};
}
DI void pack_inputs(Ctx& c, const Params& p, int wv) {
  const int lane = c.lane;
  const rsrc_t rsL = mkrs(c.ws, WS_USED), rsX = mkrs(p.x, MM * KS * 2), rsW = mkrs(p.w, KS * NN * 2);
  LAS unsigned char* st = c.lds + L_RING + wv * 4096;
  const int hi7 = (c.wg & 15) | (c.wg >> 5) << 4, late = wg_late(c.wg);
  int q = wv == 0 ? hi7 * 2 + late : late ? 128 + wv * 128 + hi7 : 2560;
  if (q < 320) __builtin_amdgcn_s_setprio(3);
  for (; q < 2560; q = q < 256 ? (late ? 1152 + hi7 : 2560) : q + 1024) {
    const int j = q / 320, r = q - j * 320;
    const bool isx = r < 64;
    if (j) { __builtin_amdgcn_s_setprio(0); while (now() - c.t0 < (u64)(PACK_T0 + PACK_DT * j)) __builtin_amdgcn_s_sleep(8); }
    const int dest = (r - 64) & 7, ni = (r - 64) >> 4, kh = ((r - 64) >> 3) & 1;
    u32 lo[16], hi[16];
    *(LAS u32x4*)(st + lane * 16) = (u32x4){0u, 0u, 0u, 0u}; *(LAS u32x4*)(st + 1024 + lane * 16) = (u32x4){0u, 0u, 0u, 0u};
    if (isx) {
      u32x4 d[8];
      const int vo = ((64 * r + (lane & 15)) * KS + 64 * j + 8 * (lane >> 4)) * 2;
#pragma unroll
      for (int t = 0; t < 8; ++t) d[t] = __builtin_amdgcn_raw_buffer_load_b128(rsX, vo + 64 * (t >> 2), 16 * (t & 3) * KS * 2, 0);
      __builtin_amdgcn_sched_barrier(0);
#pragma unroll
      for (int t = 0; t < 8; ++t) {
        lo[2 * t] = prm(d[t].y, d[t].x, 0x06040200u); lo[2 * t + 1] = prm(d[t].w, d[t].z, 0x06040200u);
        hi[2 * t] = prm(d[t].y, d[t].x, 0x07050301u); hi[2 * t + 1] = prm(d[t].w, d[t].z, 0x07050301u);
      }
    } else {
      u32x2 d[2][8];
      const int vo = ((64 * j + 32 * kh + 8 * (lane >> 4)) * NN + dest * 2048 + ni * 128 + 4 * (lane & 15)) * 2;
#pragma unroll
      for (int k = 0; k < 8; ++k)
#pragma unroll
        for (int h = 0; h < 2; ++h) d[h][k] = __builtin_amdgcn_raw_buffer_load_b64(rsW, vo + 128 * h, k * NN * 2, 0);
      __builtin_amdgcn_sched_barrier(0);
#pragma unroll
      for (int h = 0; h < 2; ++h)
#pragma unroll
        for (int cc = 0; cc < 2; ++cc)
#pragma unroll
          for (int u = 0; u < 2; ++u) {
            const u32 a0 = d[h][4 * u][cc], a1 = d[h][4 * u + 1][cc], a2 = d[h][4 * u + 2][cc], a3 = d[h][4 * u + 3][cc];
            const u32 l0 = prm(a1, a0, 0x06020400u), l1 = prm(a3, a2, 0x06020400u), h0 = prm(a1, a0, 0x07030501u), h1 = prm(a3, a2, 0x07030501u);
            const int t = 4 * h + 2 * cc;
            lo[2 * t + u] = prm(l1, l0, 0x05040100u); lo[2 * t + 2 + u] = prm(l1, l0, 0x07060302u);
            hi[2 * t + u] = prm(h1, h0, 0x05040100u); hi[2 * t + 2 + u] = prm(h1, h0, 0x07060302u);
          }
    }
    const int lr = isx ? r : 64 + 2 * ni + kh;
    const bool own = isx || dest == c.rank, send = isx || dest != c.rank;
    if (own) {
      const u32 img = O_IMG + (u32)((j * 8 + c.rank) * 96 + lr) * 8192u + lane * 16;
#pragma unroll
      for (int t = 0; t < 8; ++t) st128<AGT>(piece(lo, hi, t), rsL, img + t * 1024);
    }
    u32 tot = 8192u;
    if (send) {
      const u32 slot = (isx ? O_OBX + (u32)(j * 64 + r) * 8192u : O_OBW + (u32)((dest * 8 + j) * 32 + 2 * ni + kh) * 8192u);
      u32 n[16], se = 0, any = 0, cnt = 0;
      if (q >= NRAW) {
#pragma unroll
      for (int k = 0; k < 16; ++k) { n[k] = bfi(B1, lo[k] >> 7, hi[k] << 1); se = __builtin_amdgcn_sad_u8(n[k], 0u, se); }
      const u32 base = min(max(pick_top(__builtin_amdgcn_readlane(wave_incl_scan(se), 63), 4096u), 13u) - 13u, 240u);
      const u32 bl = (base & 127u) * B1, bx = base < 128u ? 0x80u * B1 : 0u;
#pragma unroll
      for (int k = 0; k < 16; ++k) { n[k] = (((n[k] | 0x80u * B1) - bl) ^ (n[k] & 0x80u * B1)) ^ bx; any |= n[k]; }
      if (ANY((any & 0xF0u * B1) != 0u)) {
#pragma unroll
        for (int k = 0; k < 16; ++k) if (ANY((n[k] & 0xF0u * B1) != 0u)) {
#pragma unroll
          for (int b = 0; b < 4; ++b) {
            const u32 v = (n[k] >> (8 * b)) & 255u;
            const u64 m = __builtin_amdgcn_ballot_w64(v > 15u);
            const u32 pos = cnt + __builtin_amdgcn_mbcnt_hi((u32)(m >> 32), __builtin_amdgcn_mbcnt_lo((u32)m, 0u));
            if (v > 15u && pos < 31u) st32((u32)(k * 4 + b) << 14 | (u32)lane << 8 | ((v + base) & 255u), rsL, slot + 5124u + pos * 4u);
            cnt += (u32)__popcll(m);
          }
        }
      }
#pragma unroll
      for (int k = 0; k < 16; ++k) n[k] = ((n[k] & 15u * B1) + 2u * B1) & 15u * B1;
      Lv g; u32 l2a, l2b, l3;
      if (cnt <= 31u && enc_levels<16, 2048>(c, n, st, 2048u, g, l2a, l2b, l3)) {
#pragma unroll
        for (int t = 0; t < 4; ++t)
          st128<SYS>((u32x4){bfi(0x80u * B1, hi[4 * t], lo[4 * t]), bfi(0x80u * B1, hi[4 * t + 1], lo[4 * t + 1]),
                             bfi(0x80u * B1, hi[4 * t + 2], lo[4 * t + 2]), bfi(0x80u * B1, hi[4 * t + 3], lo[4 * t + 3])}, rsL, slot + t * 1024 + lane * 16);
        u32x4 c1;
#pragma unroll
        for (int t = 0; t < 4; ++t) c1[t] = n[4 * t] | n[4 * t + 1] << 2 | n[4 * t + 2] << 4 | n[4 * t + 3] << 6;
        st128<SYS>(c1, rsL, slot + 4096u + lane * 16);
        const u32 o2 = 5120u + ((cnt + 4u) & ~3u) * 4u;
        if (lane == 0) st32(base | cnt << 8, rsL, slot + 5120u);
        if ((u32)lane < g.nc2) { st32(l2a, rsL, slot + o2 + lane * 4); if (g.rows2 == 8u) st32(l2b, rsL, slot + o2 + (g.nc2 + lane) * 4u); }
        if ((u32)lane < g.nc3) st32(l3, rsL, slot + o2 + g.rows2 * g.nc2 + lane * 4);
        tot = (o2 + g.rows2 * g.nc2 + g.nc3 * 4u + 15u) & ~15u;
      } else VMWAIT0();
      }
      if (tot == 8192u) {
        asm volatile("" : "+v"(lo[0]), "+v"(lo[2]), "+v"(lo[4]), "+v"(lo[6]), "+v"(lo[8]), "+v"(lo[10]), "+v"(lo[12]), "+v"(lo[14]),
                          "+v"(lo[1]), "+v"(lo[3]), "+v"(lo[5]), "+v"(lo[7]), "+v"(lo[9]), "+v"(lo[11]), "+v"(lo[13]), "+v"(lo[15]));
#pragma unroll
        for (int t = 0; t < 8; ++t) st128<SYS>(piece(lo, hi, t), rsL, slot + t * 1024 + lane * 16);
      }
    }
    if (!c.call) c.call = lds_wait(c, C_CALL, 1u);
    VMWAIT0();
    if (send && (isx || lane == dest)) tell_peers(c, O_RF + (c.rank * NBLK + j * 96 + lr) * 4, c.call << 20 | tot >> 4);
    if (own) image_flag(c, j * 8 + c.rank, lr);
    if (q < 64) STAMP(c, 1);
    if (q >= 2048) STAMP(c, 2);
  }
  __builtin_amdgcn_s_setprio(0);
  c.call = lds_wait(c, C_CALL, 1u);
  if (lane == 0) lds_add(c.ctl + C_PACKED, 1u);
}

DI u32 ob_slot(int b, int me) { const int j = b / 96, r = b - j * 96; return r < 64 ? O_OBX + (u32)(j * 64 + r) * 8192u : O_OBW + (u32)((me * 8 + j) * 32 + r - 64) * 8192u; }
#define LAGS {0, 2, 3, 4}
struct Order {
  unsigned short lb[NBLK];
  constexpr Order() : lb() {
    constexpr int lag[4] = LAGS;
    int n = 0;
    for (int r = 0; r < 96; ++r) lb[n++] = (unsigned short)r;
    for (int j = 1; j < 16; ++j) {
      if (j < 8) for (int r = 0; r < 32; ++r) lb[n++] = (unsigned short)(j * 96 + 64 + r);
      for (int t = 0; t < 4; ++t)
        for (int g = 0; g < 4; ++g) {
          const int jj = j - lag[g];
          if (jj >= 1 && jj < 8) for (int b = 0; b < 4; ++b) lb[n++] = (unsigned short)(jj * 96 + t * 16 + g * 4 + b);
        }
    }
  }
};
__constant__ Order ORDER = Order();
DI int pull_order(u32 B, int link) {
  constexpr int lag[4] = LAGS;
  const u32 lb = ORDER.lb[B], r = lb % 96u, sw = (link & 4) && ((r & 8u) ? lag[2] == lag[3] : lag[0] == lag[1]) ? 4u : 0u;
  return (int)(B >= 96u && r < 64u ? lb - r + (((r + 16u * link) & 63u) ^ sw) : lb);
}
DI u32 early_first(u32 t) { return (t & 15u) | (t >> 7) << 4 | ((t >> 4) & 7u) << 5; }
DI void role_feeder(Ctx& c) {
  const int lane = c.lane;
  __builtin_amdgcn_s_setprio(3);
  c.call = lds_wait(c, C_CALL, 1u);
  const int link = wg_link(c.wg), src = (c.rank + 1 + link) & 7;
  const u32 nl = link_wgs(link, 1), ne = link_wgs(link, 0), na = ne + nl, late = wg_late(c.wg);
  u32 n0 = 0;
  for (u32 i = link; i < 128u; i += 7u) n0 += (i & 8u) == 0u;
  u32 kv = 0, ph = 0, tbase = (c.call - 1u) * (T_HALF - KFIX * ne + na - n0) - KFIX * ne, tend = T_HALF;
  bool miss = true;
  const rsrc_t rsL = mkrs(c.ws, WS_USED);
  u32 fb = O_RF + src * NBLK * 4, B = 0;
  if (late) lds_wait(c, C_PACKED, 8u);
  for (u32 k = 0;; ++k) {
    const u32 ahead = B < T_AH ? 2u : 1u;
    if (k >= ahead) lds_wait(c, C_PSTART, k + 1u - ahead);
    if (k < KFIX && !late) {
      B = (u32)((c.wg & 15) | (c.wg >> 5) << 4) / 7u + k * ne;
      if (miss) { dma4(rsL, lane * 16, fb, (u32)(size_t)c.lds + L_RFC); VMWAIT0(); }
    } else for (;;) {
      if (G0FREE && ph == 0u && ((c.wg >> 3) & 3) == 0) {
        ph = 2u; tbase = (c.call - 1u) * (T_END - T_MASK + na) - T_MASK; tend = T_END; fb = O_OF + src * 8192; miss = true;
        lds_wait(c, C_CMAX, 8u);
      }
      if (miss) dma4(rsL, lane * 16, fb, (u32)(size_t)c.lds + L_RFC);
      u32 t = 0;
      if (lane == 0) t = __hip_atomic_fetch_add((GAS u32*)(c.ws + O_TICK + ph * 1024 + link * 128), 1u, __ATOMIC_RELAXED, __HIP_MEMORY_SCOPE_AGENT);
      VMWAIT0();
      B = RFL(t) - tbase;
      if (B < tend || ph == 2u) break;
      if (ph == 0u && late) { ph = 1u; tbase = (c.call - 1u) * (T_MASK - T_HALF + nl) - T_HALF; tend = T_MASK; }
      else {
        STAMP(c, 6);
        ph = 2u; tbase = (c.call - 1u) * (T_END - T_MASK + na) - T_MASK; tend = T_END; fb = O_OF + src * 8192; miss = true;
        if ((GATEM >> ((c.wg >> 3) & 3)) & 1) lds_wait(c, C_CMAX, 8u);
      }
    }
    if (B >= T_END) {
      if (lane < 2) lds_st(c.ctl + C_QI + ((k + lane) & 7u), (k + lane + 1u) << 23 | 0x7FFFFFu);
      if (kv >= 5u) lds_wait(c, C_DFREE0 + ((kv - 5u) & 3u), ((kv - 5u) & ~3u) + 4u);
      if (lane < 3) lds_st(c.ctl + C_QV + ((kv + lane) & 7u), (kv + lane + 1u) << 23 | 0x7FFFFFu);
      break;
    }
    u32 item, fo;
    if (B < T_MASK) { item = pull_order(B, link); fo = item * 4; }
    else if (B < T_VAL) { item = T_MASK; fo = F_MF + (B - T_MASK) * 4; }
    else {
      const u32 ht = early_first((B - T_VAL) >> 1) * 2u + (B & 1u);
      item = T_VAL + ht; fo = ((lane & 1) ? F_MI : F_VF) - F_MI + ht * 4;
      if (kv == 0u) fb += F_MI;
    }
    u32 v = B >= T_VAL && kv == 0u ? 0u : lds_ld((LAS u32*)(c.lds + L_RFC + fo)), spins = 0;
    miss = false;
    while (!ALL((int)((v & 0xFFF00000u) - (c.call << 20)) >= 0)) {
      miss = true;
      if (expired(c, spins)) break;
      if (spins > 1u) __builtin_amdgcn_s_sleep(1);
      v = g_ld_sys(c.ws + fb + fo);
    }
    if (item == T_MASK) item += (v >> 11) & 511u;
    if (lane == 0) {
      if (B >= T_VAL) lds_st(c.ctl + C_QV + (kv & 7u), (kv + 1u) << 23 | (k & 2047u) << 12 | (item - T_VAL));
      const u32 wide = B < T_W0 || (B >= min(T_HALF, T_W1) && B < T_MASK + T_WA) || B >= T_W2;
      lds_st(c.ctl + C_QI + (k & 7u), (k + 1u) << 23 | item << 12 | wide << 11 | (v & 0x7FFu));
    }
    if (B >= T_VAL) ++kv;
  }
}
DI void role_decoder(Ctx& c, u32 d) {
  const int lane = c.lane, src = (c.rank + 1 + wg_link(c.wg)) & 7;
  __builtin_amdgcn_s_setprio(1);
  c.call = lds_wait(c, C_CALL, 1u);
  const rsrc_t rsL = mkrs(c.ws, WS_USED);
  for (u32 k = d;; k += 2u) {
    const u32 e = q_wait(c, C_QI, k), lb = (e >> 12) & 2047u, tot = min((e & 0x7FFu) * 16u, 8192u);
    if (lb >= T_VAL) break;
    LAS unsigned char* sl = c.lds + L_IN + (k & 3u) * 8192u;
    if (lb >= T_MASK) {
      lds_wait(c, C_PROG, k * 32768u + tot);
      const u32x4 hd = *(LAS u32x4*)sl;
      u32x4 v[8], y[3];
#pragma unroll
      for (int t = 0; t < 3; ++t) y[t] = *(LAS u32x4*)(sl + min(t * 1024 + lane * 16, 2048));
#pragma unroll
      for (int t = 0; t < 8; ++t) v[t] = *(LAS u32x4*)(sl + min((t < 2 ? 2064u : hd[(t >> 1) - 1] & 0x1FF0u) + (t & 1) * 1024 + lane * 16, 8176u));
      CFENCE(); if (lane == 0) lds_add(c.ctl + C_IFREE0 + (k & 3u), 1u); CFENCE();
      const u32 inb = O_MIN + (src * 512 + lb - T_MASK) * 8192u + lane * 16;
      st128<AGT>(y[0], rsL, inb); st128<AGT>(y[1], rsL, inb + 1024u);
      if (lane == 0) st128<AGT>(y[2], rsL, inb + 2048u);
#pragma unroll
      for (int t = 0; t < 8; t += 2) {
        st128<AGT>(v[t], rsL, inb + 2064u + t * 640u);
        if (lane < 16 && (hd[t >> 1] & 0xFFFFu) - (t ? hd[(t >> 1) - 1] & 0xFFFFu : 2064u) > 1024u) st128<AGT>(v[t + 1], rsL, inb + 3088u + t * 640u);
      }
      VMWAIT0();
      if (lane == 0) g_st_agent(c.ws + O_OF + src * 8192 + F_MI + (lb - T_MASK) * 4, c.call << 20);
      continue;
    }
    const int j = lb / 96, r = lb - j * 96, kstep = j * 8 + src;
    const u32 img = O_IMG + (u32)(kstep * 96 + r) * 8192u + lane * 16;
    lds_wait(c, C_PROG, k * 32768u + tot);
    if (k < 2u) STAMP(c, 26 + k);
    if (tot == 8192u) {
      u32x4 v[8];
#pragma unroll
      for (int t = 0; t < 8; ++t) v[t] = *(LAS u32x4*)(sl + t * 1024 + lane * 16);
      CFENCE(); if (lane == 0) lds_add(c.ctl + C_IFREE0 + (k & 3u), 1u); CFENCE();
#pragma unroll
      for (int t = 0; t < 8; ++t) st128<AGT>(v[t], rsL, img + t * 1024);
    } else {
      const u32x4 c1 = *(LAS u32x4*)(sl + 4096 + lane * 16);
      const u32 exc = *(LAS u32*)(sl + 5120 + (lane & 31) * 4);
      u32x4 sm[4];
#pragma unroll
      for (int t = 0; t < 4; ++t) sm[t] = *(LAS u32x4*)(sl + t * 1024 + lane * 16);
      const u32 hd = RFL(exc), cnt = (hd >> 8) & 31u, base4 = min(hd & 255u, 240u) * B1;
      u32 n[16], lo[16], hi[16];
      const u32 l1[4] = {c1.x, c1.y, c1.z, c1.w};
      dec_levels<16, 2048>(c, l1, base4, n, sl + 5120u + ((cnt + 4u) & ~3u) * 4u, sl + 6144u, 2048u);
      CFENCE(); if (lane == 0) lds_add(c.ctl + C_IFREE0 + (k & 3u), 1u); CFENCE();
      for (u32 x = 1; x <= cnt; ++x) {
        const u32 rec = __builtin_amdgcn_readlane(exc, x);
        const u32 kk = (rec >> 16) & 15u, sh = ((rec >> 14) & 3u) * 8u;
        const u32 m = ((rec >> 8) & 63u) == (u32)lane ? 255u << sh : 0u, val = (rec & 255u) << sh;
#pragma unroll
        for (int i = 0; i < 16; ++i) n[i] = bfi(kk == (u32)i ? m : 0u, val, n[i]);
      }
#pragma unroll
      for (int i = 0; i < 16; ++i) { hi[i] = bfi(0x80u * B1, sm[i >> 2][i & 3], n[i] >> 1); lo[i] = bfi(0x80u * B1, n[i] << 7, sm[i >> 2][i & 3]); }
#pragma unroll
      for (int t = 0; t < 8; ++t) st128<AGT>(piece(lo, hi, t), rsL, img + t * 1024);
    }
    VMWAIT0();
    image_flag(c, kstep, r);
  }
}
constexpr int WMAX = WIN2 - 1;
#define WINM 3
struct Stream {
  u32 pub = 0, pend[WMAX] = {};
  template <int W> DI void landed(const Ctx& c, u32 prog) {
    asm volatile("s_waitcnt vmcnt(%0)" :: "n"(W - 1) : "memory");
    pub = max(pub, pend[WMAX - (W - 1)]);
    if (c.lane == 0) lds_st(c.ctl + C_PROG, pub);
    for (int k = 0; k + 1 < WMAX; ++k) pend[k] = pend[k + 1];
    pend[WMAX - 1] = prog;
  }
  DI void flush(const Ctx& c) {
    VMWAIT0();
    pub = pend[WMAX - 1];
    if (c.lane == 0) lds_st(c.ctl + C_PROG, pub);
  }
};
DI void role_puller(Ctx& c, const Params& p) {
  __builtin_amdgcn_s_setprio(3);
  const u32 ldsb = (u32)(size_t)c.lds;
  const char* sws = p.ws[(c.rank + 1 + wg_link(c.wg)) & 7];
  Stream sm;
  u32 kv = 0;
  u32 cw = 0, kp = 0;
  for (u32 k = 0;; ++k) {
    u32 e = __builtin_amdgcn_readlane(cw, 0), fi = __builtin_amdgcn_readlane(cw, 1), fd = __builtin_amdgcn_readlane(cw, 2);
    if ((e >> 23) != ((k + 1u) & 511u)) {
      e = RFL(lds_ld(c.ctl + C_QI + (k & 7u)));
      if ((e >> 23) != ((k + 1u) & 511u)) {
        sm.flush(c);
        e = q_wait(c, C_QI, k);
      }
    }
    CFENCE();
    const u32 item = (e >> 12) & 2047u;
    if (item >= T_END) break;
    u32 so, tot = (e & 0x7FFu) * 16u, dst;
    if (item < T_VAL) {
      if (fi < k >> 2) { sm.flush(c); lds_wait(c, C_IFREE0 + (k & 3u), k >> 2); }
      dst = L_IN + (k & 3u) * 8192u;
      so = item < T_MASK ? ob_slot(item, c.rank) : O_OM + (item - T_MASK) * 8192u; tot = min(tot, 8192u);
    } else {
      if (fd < (kv & ~3u)) { sm.flush(c); lds_wait(c, C_DFREE0 + (kv & 3u), kv & ~3u); }
      dst = L_STG + (kv & 3u) * 16384u; so = O_OV + (item - T_VAL) * 16384u; tot = min(tot, 16384u);
      ++kv;
    }
    kp += item >= T_MASK;
    so = RFL(so); tot = RFL(max(tot, 16u)); dst = RFL(dst);
    const rsrc_t rs = mkrs(sws + so, tot);
    const bool wide = (e >> 11) & 1u;
    const u32 lead = PLEAD, qs = tot > lead ? (tot - lead) & ~1023u : 0u;
    for (u32 q = 0; q < tot; q += 1024u) {
      dma1(rs, c.lane * 16, q, ldsb + dst + q);
      if (q == qs && c.lane == 0) lds_st(c.ctl + C_PSTART, k + 1u);
      if (q + 1024u >= tot) cw = lds_ld(c.ctl + (c.lane == 0 ? C_QI + ((k + 1u) & 7u) : c.lane == 1 ? C_IFREE0 + ((k + 1u) & 3u) : C_DFREE0 + (kv & 3u)));
      if (wide) sm.landed<WIN2>(c, k * 32768u + q + 1024u);
      else if (WINM != WIN1 && (item < T_MASK || (GATEM && kp < PMED))) sm.landed<WINM>(c, k * 32768u + q + 1024u);
      else sm.landed<WIN1>(c, k * 32768u + q + 1024u);
    }
  }
  sm.flush(c);
  STAMP(c, 8);
}

DI void role_sched(Ctx& c, const Params& p, int mi, int ni) {
  const int lane = c.lane;
  __builtin_amdgcn_s_setprio(3);
  u32 t = 0;
  if (lane == 0) t = __hip_atomic_fetch_add((GAS u32*)(c.ws + O_CALL + c.wg * 128), 1u, __ATOMIC_RELAXED, __HIP_MEMORY_SCOPE_AGENT) + 1u;
  c.call = RFL(t);
  if (lane == 0) lds_st(c.ctl + C_CALL, c.call);
  const rsrc_t rsL = mkrs(c.ws, WS_USED);
  u64 issued = 0; u32 n = 0, spins = 0;
  while (n < 64u) {
    if (2u * n >= RFL(lds_ld((LAS u32*)(c.lds + L_REL + 7 * 256))) + 2u * SCHED_Q) { if (!lds_expired(c, spins)) continue; }
    const u32x4 a = __builtin_amdgcn_raw_buffer_load_b128(rsL, lane * 16, (int)(O_FX + mi * 1024), SYS);
    const u32x2 b = __builtin_amdgcn_raw_buffer_load_b64(rsL, lane * 8, (int)(O_FW + ni * 512), SYS);
    const bool rdy = min(min(min(a.x, a.y), min(a.z, a.w)), min(b.x, b.y)) >= c.call;
    u64 m = __builtin_amdgcn_ballot_w64(rdy) & ~issued;
    if (m == 0) {
      if (!expired(c, spins)) { __builtin_amdgcn_s_sleep(1); CFENCE(); continue; }
      m = ~issued;
    }
    if ((m >> lane) & 1) lds_st(c.ctl + C_ORDER + n + __builtin_amdgcn_mbcnt_hi((u32)(m >> 32), __builtin_amdgcn_mbcnt_lo((u32)m, 0u)), lane + 1);
    issued |= m;
    n += (u32)__popcll(m);
    CFENCE();
  }
  STAMP(c, 9);
  if (c.wg == LEADER) {
    u32 mx;
    for (;;) {
      const u32x4 a = __builtin_amdgcn_raw_buffer_load_b128(rsL, lane * 16, (int)O_LT, SYS);
      const u32x4 b = __builtin_amdgcn_raw_buffer_load_b128(rsL, lane * 16, (int)(O_LT + 1024), SYS);
      mx = max(max(a.x, a.z), max(b.x, b.z));
      if (ALL(min(min(a.y, a.w), min(b.y, b.w)) >= c.call) || expired(c, spins)) break;
      CFENCE();
    }
    STAMP(c, 16);
    mx = wave_max(mx);
    if (lane < NR) __hip_atomic_store((GAS u64*)(*(char* LAS*)(c.lds + L_PEER + lane * 8) + O_GT + c.rank * 8), (u64)c.call << 32 | mx, __ATOMIC_RELAXED, __HIP_MEMORY_SCOPE_SYSTEM);
    if (lane == 0) lds_st(c.ctl + C_LEAD, 1u);
  }
  lds_wait(c, C_CMAX, 8u);
  const rsrc_t rsG = mkrs(c.ws + O_GT, 64u);
  u32 g;
  for (;;) {
    if (lane < 4) dma1(rsG, lane * 16, 0u, (u32)(size_t)c.lds + L_GT);
    asm volatile("s_waitcnt vmcnt(5)" ::: "memory");
    const u64 v = __hip_atomic_load((LAS u64*)(c.lds + L_GT) + (lane & 7), __ATOMIC_RELAXED, __HIP_MEMORY_SCOPE_WORKGROUP);
    g = (u32)v;
    if (ALL((u32)(v >> 32) >= c.call) || expired(c, spins)) break;
    __builtin_amdgcn_s_sleep(2);
  }
  g = wave_max(g);
  if (lane == 0) { lds_st(c.ctl + C_GMAX, g); CFENCE(); lds_st(c.ctl + C_GVALID, 1u); }
  STAMP(c, 10);
}

DI void role_loader(Ctx& c, u32 li, int mi, int ni) {
  __builtin_amdgcn_s_setprio(2);
  const rsrc_t rsL = mkrs(c.ws, WS_USED);
  const u32 voff = c.lane * 16, ldsb = (u32)(size_t)c.lds + L_RING;
  u32 pend = 0;
  lds_wait(c, C_PACKED, 8u);
  for (u32 m = li; m < 128u; m += 3u) {
    const u32 slot = m % NS, kh = m & 1u;
    u32 o, spins = 0;
    for (;;) {
      const u32 vo = lds_ld(c.ctl + C_ORDER + (m >> 1)), vf = lds_ld((LAS u32*)(c.lds + L_REL + (c.lane & 7) * 256));
      o = RFL(vo);
      if (o != 0u && ALL(vf + (NS - 1u) >= m)) break;
      if (pend) { VMWAIT0(); if (c.lane == 0) lds_st(c.ctl + C_FULL0 + (pend - 1u) % NS, pend); pend = 0; continue; }
      if (lds_expired(c, spins)) break;
    }
    CFENCE();
    o = RFL(o);
    const u32 so = O_IMG + ((o - 1u) & 63u) * (96u * 8192u), sx = so + mi * 32768 + kh * 4096u, sw = so + 64 * 8192 + ni * 16384 + kh * 8192u, ld = ldsb + slot * SLOTB;
    dma4(rsL, voff, sw, ld + 16384u);
    if (pend) { asm volatile("s_waitcnt vmcnt(4)" ::: "memory"); if (c.lane == 0) lds_st(c.ctl + C_FULL0 + (pend - 1u) % NS, pend); }
    dma4(rsL, voff, sw + 4096u, ld + 20480u);
    dma4(rsL, voff, sx, ld); dma4(rsL, voff, sx + 8192u, ld + 4096u); dma4(rsL, voff, sx + 16384u, ld + 8192u); dma4(rsL, voff, sx + 24576u, ld + 12288u);
    pend = m + 1u;
  }
  VMWAIT0();
  if (c.lane == 0) lds_st(c.ctl + C_FULL0 + (pend - 1u) % NS, pend);
}

struct Geo { u32 nc, sz; };
DI Geo geo(u32 V) { return V <= 2048u ? Geo{row_bytes(V) >> 2, 8u * row_bytes(V)} : Geo{64u, (min(V, 2304u) + 3u) & ~3u}; }
struct Scale { u32 r; int kap; };
DI Scale split(float inv) { const u32 ib = __float_as_uint(inv), r = ((ib & 0x7FFFFFu) + (1u << 19)) >> 20; return Scale{r & 7u, (int)(ib >> 23) - 247 + (int)(r >> 3)}; }
DI u32 low_exp(u32 o, u32 kap32, u32& pos) {
  const u32 t = o + kap32, ge = (t >> 5) & B1;
  pos = ((t + 31u * B1) >> 6) & B1;
  return t & ((ge << 5) - ge);
}
DI u32 sure(u32 m, u32 r8, u32 pos, u32& up) {
  const u32 x = m + r8;
  up = (~x >> 3) & B1;
  return (((x ^ 8u * B1) + 15u * B1) >> 4) & pos;
}
DI u32 place(const Ctx& c, int wv, int cnt, u32 first, u32 sb, u32 extra, u32x4& hdr) {
  if (c.lane == 0) { lds_st(c.ctl + C_VSZ + wv, sb | extra); CFENCE(); lds_add(c.ctl + cnt, 1u); }
  lds_wait(c, cnt, 4u);
  const u32x4 z = *(LAS u32x4*)(c.ctl + C_VSZ + (wv & 4));
  hdr.x = z.x + first; hdr.y = z.y + (hdr.x & 0xFFFFu); hdr.z = z.z + (hdr.y & 0xFFFFu); hdr.w = z.w + (hdr.z & 0xFFFFu);
  return (((wv & 2) ? ((wv & 1) ? hdr.w : hdr.z) : ((wv & 1) ? hdr.y : hdr.x)) & 0xFFFFu) - sb;
}

DI void role_mfma(Ctx& c, const Params& p, int wv, int mi, int ni) {
  const int lane = c.lane, mw = wv >> 1, h = wv & 1;
  pack_inputs(c, p, wv);
  f32x4 acc[4][4];
#pragma unroll
  for (int g = 0; g < 4; ++g)
#pragma unroll
    for (int b = 0; b < 4; ++b) acc[g][b] = (f32x4){0.f, 0.f, 0.f, 0.f};
  if (wv == 0) STAMP(c, 15);
  bf16x8 x0, x1, x2, x3, wa0, wa1, wa2, wa3, wb0, wb1, wb2, wb3;
  u32 vx = (u32)(size_t)c.lds + L_RING + mw * 4096 + lane * 16, vw = (u32)(size_t)c.lds + L_RING + 16384 + h * 4096 + lane * 16;
  u32 vf = (u32)(size_t)(c.ctl + C_FULL0 + 2), done = 1, sl = 1, seen = 0, vs;
  const u32 relad = (u32)(size_t)c.lds + L_REL + wv * 256 + lane * 4;
#define MF(a, w, x) "v_mfma_f32_16x16x32_bf16 " a ", " w ", " x ", " a "\n\t"
#define HSTEP(c0, c1, c2, c3, n0, n1, n2, n3, m) do { \
    if (__builtin_expect(seen <= (m) + 1u, 0)) lds_wait(c, C_FULL0 + sl, (m) + 2u); \
    asm volatile(MF("%0", "%9", "%8") "ds_read_b128 %4, %13\n\t" MF("%1", "%10", "%8") "ds_read_b128 %5, %13 offset:1024\n\t" \
                 MF("%2", "%11", "%8") "ds_read_b128 %6, %13 offset:2048\n\t" MF("%3", "%12", "%8") "ds_read_b128 %7, %13 offset:3072" \
                 : "+v"(acc[0][0]), "+v"(acc[1][0]), "+v"(acc[2][0]), "+v"(acc[3][0]), "+v"(n0), "+v"(n1), "+v"(n2), "+v"(n3) \
                 : "v"(x0), "v"(c0), "v"(c1), "v"(c2), "v"(c3), "v"(vw) : "memory"); \
    const u32 d = sl == NS - 1u ? -(NS - 1u) * SLOTB : SLOTB, d4 = sl == NS - 2u ? -(NS - 1u) * 4u : 4u; \
    sl = sl == NS - 1u ? 0u : sl + 1u; \
    asm volatile("s_waitcnt lgkmcnt(7)\n\t" MF("%0", "%7", "%6") "ds_read_b128 %4, %11\n\t" MF("%1", "%8", "%6") "ds_read_b32 %5, %12\n\t" \
                 MF("%2", "%9", "%6") MF("%3", "%10", "%6") \
                 : "+v"(acc[0][1]), "+v"(acc[1][1]), "+v"(acc[2][1]), "+v"(acc[3][1]), "+v"(x0), "=&v"(vs) \
                 : "v"(x1), "v"(c0), "v"(c1), "v"(c2), "v"(c3), "v"(vx), "v"(vf) : "memory"); \
    asm volatile("s_waitcnt lgkmcnt(8)\n\t" MF("%0", "%8", "%7") "ds_read_b128 %4, %12 offset:1024\n\t" MF("%1", "%9", "%7") "v_add_u32 %5, %13, %5\n\t" \
                 MF("%2", "%10", "%7") "v_add_u32 %6, %14, %6\n\t" MF("%3", "%11", "%7") \
                 : "+v"(acc[0][2]), "+v"(acc[1][2]), "+v"(acc[2][2]), "+v"(acc[3][2]), "+v"(x1), "+v"(vw), "+v"(vf) \
                 : "v"(x2), "v"(c0), "v"(c1), "v"(c2), "v"(c3), "v"(vx), "s"(d), "s"(d4) : "memory"); \
    asm volatile("s_waitcnt lgkmcnt(8)\n\t" MF("%0", "%9", "%5") "ds_read_b128 %4, %6 offset:2048\n\tv_add_u32 %7, 1, %7\n\t" MF("%1", "%10", "%5") \
                 "s_waitcnt lgkmcnt(2)\n\tv_readfirstlane_b32 %8, %13\n\t" MF("%2", "%11", "%5") MF("%3", "%12", "%5") \
                 "ds_read_b128 %5, %6 offset:3072\n\tds_write_b32 %14, %7\n\tv_add_u32 %6, %15, %6" \
                 : "+v"(acc[0][3]), "+v"(acc[1][3]), "+v"(acc[2][3]), "+v"(acc[3][3]), "+v"(x2), "+v"(x3), "+v"(vx), "+v"(done), "=&s"(seen) \
                 : "v"(c0), "v"(c1), "v"(c2), "v"(c3), "v"(vs), "v"(relad), "s"(d) : "memory"); \
  } while (0)
  lds_wait(c, C_FULL0, 1u);
  asm volatile("ds_read_b128 %0, %9\n\tds_read_b128 %1, %9 offset:1024\n\tds_read_b128 %2, %9 offset:2048\n\tds_read_b128 %3, %9 offset:3072\n\t"
               "ds_read_b128 %4, %8\n\tds_read_b128 %5, %8 offset:1024\n\tds_read_b128 %6, %8 offset:2048\n\tds_read_b128 %7, %8 offset:3072\n\t"
               "ds_write_b32 %10, %11\n\ts_waitcnt lgkmcnt(0)"
               : "=&v"(wa0), "=&v"(wa1), "=&v"(wa2), "=&v"(wa3), "=&v"(x0), "=&v"(x1), "=&v"(x2), "=&v"(x3) : "v"(vx), "v"(vw), "v"(relad), "v"(done) : "memory");
  vx += SLOTB; vw += SLOTB;
  wb0 = wa0; wb1 = wa0; wb2 = wa0; wb3 = wa0;
  asm volatile("s_nop 1" : "+v"(vx), "+v"(vw), "+v"(wb0), "+v"(wb1), "+v"(wb2), "+v"(wb3));
  for (u32 m = 0; m < 126u; m += 2u) {
    HSTEP(wa0, wa1, wa2, wa3, wb0, wb1, wb2, wb3, m);
    HSTEP(wb0, wb1, wb2, wb3, wa0, wa1, wa2, wa3, m + 1u);
  }
  HSTEP(wa0, wa1, wa2, wa3, wb0, wb1, wb2, wb3, 126u);
#define MF4(b, x) asm volatile(MF("%0", "%5", "%4") MF("%1", "%6", "%4") MF("%2", "%7", "%4") MF("%3", "%8", "%4") \
    : "+v"(acc[0][b]), "+v"(acc[1][b]), "+v"(acc[2][b]), "+v"(acc[3][b]) : "v"(x), "v"(wb0), "v"(wb1), "v"(wb2), "v"(wb3))
  asm volatile("s_waitcnt lgkmcnt(0)" ::: "memory");
  MF4(0, x0); MF4(1, x1); MF4(2, x2); MF4(3, x3);
  asm volatile("s_nop 7\n\ts_nop 7" : "+v"(acc[0][2]), "+v"(acc[1][2]), "+v"(acc[2][2]), "+v"(acc[3][2]), "+v"(acc[0][3]), "+v"(acc[1][3]), "+v"(acc[2][3]), "+v"(acc[3][3]),
               "+v"(acc[0][0]), "+v"(acc[1][0]), "+v"(acc[2][0]), "+v"(acc[3][0]), "+v"(acc[0][1]), "+v"(acc[1][1]), "+v"(acc[2][1]), "+v"(acc[3][1]));
  if (wv == 0) STAMP(c, 11);
  const rsrc_t rsL = mkrs(c.ws, WS_USED);
  {
    float mx = 0.f;
#pragma unroll
    for (int e = 0; e < 64; e += 2) mx = fmaxf(fmaxf(mx, acc[e & 3][e >> 4][(e >> 2) & 3]), acc[(e & 3) + 1][e >> 4][(e >> 2) & 3]);
    const u32 wmx = wave_max(__float_as_uint(mx));
    if (lane == 0) {
      __hip_atomic_fetch_max(c.ctl + C_WGMAX, wmx, __ATOMIC_RELAXED, __HIP_MEMORY_SCOPE_WORKGROUP);
      CFENCE();
      if (lds_add(c.ctl + C_CMAX, 1u) == 7u)
        __hip_atomic_store((GAS u64*)(c.ws + O_LT + c.wg * 8), (u64)c.call << 32 | lds_ld(c.ctl + C_WGMAX), __ATOMIC_RELAXED, __HIP_MEMORY_SCOPE_AGENT);
    }
  }
  const u32 zs = wv >> 2, ht = c.wg * 2 + zs;
  u32 nz[2] = {0u, 0u};
#pragma unroll
  for (int e = 63; e >= 0; --e) nz[e >> 5] = __builtin_amdgcn_alignbit(nz[e >> 5], __float_as_uint(acc[e & 3][e >> 4][(e >> 2) & 3]), 31);
  nz[0] = ~nz[0]; nz[1] = ~nz[1];
  __builtin_amdgcn_raw_buffer_store_b64((u32x2){nz[0], nz[1]}, rsL, lane * 8, (int)(O_OM + ht * 8192 + 16 + (wv & 3) * 512), SYS);
  u32 mqv = 0;
  if ((wv & 3) == 0) mqv = __builtin_amdgcn_raw_ptr_buffer_atomic_add_i32(1, mkrs(c.ws + O_MQ, 4u), lane * 4, 0, AGT);
  const u32 cnt = __popc(nz[0]) + __popc(nz[1]), inc = wave_incl_scan(cnt), V = __builtin_amdgcn_readlane(inc, 63);
  u32 o0 = ldsa(c.lds) + L_RING + wv * 4096 + inc - cnt;
  Geo G = geo(V);
  const bool in8 = (u32)lane < G.nc, in9 = 8u * G.nc + lane < G.sz >> 2;
  lds_wait(c, C_CMAX, 8u);
  LAS unsigned char* st = c.lds + L_RING + wv * 4096;
  u32 ob[9], base;
  bool hint;
  {
#pragma unroll
    for (int k = 0; k < 4; ++k) *(LAS u32x4*)(st + k * 1024 + lane * 16) = (u32x4){0u, 0u, 0u, 0u};
    u32 o = o0, se = 0, carry = 0;
#pragma unroll
    for (int z = 0; z < 16; z += 8) {
      u32 sel[8];
#pragma unroll
      for (int D = 0; D < 8; ++D) sel[D] = put_sel(c, nib4(nz[z >> 3], D));
#pragma unroll
      for (int D = 0; D < 8; ++D) {
        const int b = (z + D) >> 2, r = D & 3;
        const u32 h = prm(prm(__float_as_uint(acc[3][b][r]) << 1, __float_as_uint(acc[2][b][r]) << 1, 0x07030000u),
                          prm(__float_as_uint(acc[1][b][r]) << 1, __float_as_uint(acc[0][b][r]) << 1, 0x00000703u), 0x07060100u);
        se = __builtin_amdgcn_sad_u8(put4(o, carry, h, nib4(nz[z >> 3], D), sel[D]), 0u, se);
      }
    }
    or_at(o, carry);
    base = min(max(pick_top(__builtin_amdgcn_readlane(wave_incl_scan(se), 63), V), 13u) - 13u, 240u);
    if (V + lane < G.sz) st[V + lane] = (unsigned char)(base + 13u);
    CFENCE();
    u32 n[9], any = 0, l2a, l2b, l3, sb = 0;
    const u16x2 b2 = {(unsigned short)base, (unsigned short)base};
#pragma unroll
    for (u32 k = 0; k < 9u; ++k) ob[k] = *(LAS u32*)(st + (k * G.nc + lane) * 4u);
#pragma unroll
    for (u32 k = 0; k < 9u; ++k) {
      const u32 lo = __builtin_bit_cast(u32, __builtin_elementwise_sub_sat(__builtin_bit_cast(u16x2, ob[k] & 0x00FF00FFu), b2));
      const u32 hi = __builtin_bit_cast(u32, __builtin_elementwise_sub_sat(__builtin_bit_cast(u16x2, (ob[k] >> 8) & 0x00FF00FFu), b2));
      ob[k] = (k < 8u ? in8 : in9) ? lo | hi << 8 : 13u * B1;
      any |= ob[k];
      n[k] = (ob[k] + 2u * B1) & 15u * B1;
    }
    Lv g;
    hint = V <= 2304u && !ANY((any & 0xF0u * B1) != 0u) && enc_levels<9, 1024>(c, n, st + G.sz, 4096u - G.sz, g, l2a, l2b, l3);
    CFENCE();
    if (hint) {
      LAS u32* s = (LAS u32*)st + lane;
      if (in8) { s[0] = n[0] | n[1] << 2 | n[2] << 4 | n[3] << 6; s[G.nc] = n[4] | n[5] << 2 | n[6] << 4 | n[7] << 6; }
      if (in9) s[2u * G.nc] = n[8];
      s += (G.sz >> 2) - 6u * G.nc;
      if ((u32)lane < g.nc2) s[0] = l2a;
      if ((u32)lane < g.nc3) s[g.nc2] = l3;
      sb = (G.sz - 24u * G.nc + (g.nc2 + g.nc3) * 4u + 15u) & ~15u;
    }
    if ((wv & 3) == 0 && lane == 0) lds_st(c.ctl + C_MQ0 + zs, mqv);
    u32x4 hdr;
    const u32 omb = O_OM + ht * 8192, woff = place(c, wv, C_CSZ0 + zs, 2064u, sb, base << 16, hdr);
    if ((wv & 3) == 0 && lane == 0) st128<SYS>(hdr, rsL, omb);
    for (u32 k = lane * 16; k < sb; k += 1024u) st128<SYS>(*(LAS u32x4*)(st + k), rsL, k + (omb + woff));
    VMWAIT0();
    u32 t = 0;
    if (lane == 0) t = lds_add(c.ctl + C_CPRE0 + zs, 1u);
    if (RFL(t) == 3u) {
      const u32 mq = RFL(lds_ld(c.ctl + C_MQ0 + zs));
      tell_peers(c, O_OF + c.rank * 8192 + F_MF + (mq & 511u) * 4, c.call << 20 | ht << 11 | (hdr.w & 0xFFFFu) >> 4);
    }
  }
  if (wv == 0) STAMP(c, 28);
  asm volatile("" : "+v"(nz[0]), "+v"(nz[1]), "+v"(o0), "+s"(G.nc));
  lds_wait(c, C_CPRE0 + zs, 4u);
#pragma unroll
  for (int k = 0; k < 4; ++k) *(LAS u32x4*)(st + k * 1024 + lane * 16) = (u32x4){0u, 0u, 0u, 0u};
  u32 sel0[16];
#pragma unroll
  for (int D = 0; D < 16; ++D) sel0[D] = put_sel(c, nib4(nz[D >> 3], D & 7));
#pragma unroll
  for (int g = 0; g < 4; ++g)
#pragma unroll
    for (int b = 0; b < 4; ++b) {
#pragma unroll
      for (int r = 0; r < 4; ++r) acc[g][b][r] = __builtin_amdgcn_fmed3f(acc[g][b][r], 0.f, __builtin_inff());
      asm volatile("" : "+v"(acc[g][b]));
    }
  lds_wait(c, C_GVALID, 1u);
  if (wv == 0) STAMP(c, 31);
  const u32 tm = g_ld_agent(c.ws + O_TMO);
  const float amax = __uint_as_float(RFL(lds_ld(c.ctl + C_GMAX)));
  const float inv = amax > 0.f ? 448.0f / amax : 0.f;
  const Scale sc = split(__uint_as_float(RFL(__float_as_uint(inv))));
  const u32 r8 = (8u - sc.r) * B1;
  u32 q[16];
#pragma unroll
  for (int b = 0; b < 4; ++b) {
    const f32x4 a0 = acc[0][b] * inv, a1 = acc[1][b] * inv, a2 = acc[2][b] * inv, a3 = acc[3][b] * inv;
#pragma unroll
    for (int r = 0; r < 4; ++r)
      q[4 * b + r] = (u32)__builtin_amdgcn_cvt_pk_fp8_f32(a2[r], a3[r], __builtin_amdgcn_cvt_pk_fp8_f32(a0[r], a1[r], 0, false), true);
  }
  {
    u32 o = o0, carry = 0, sb;
#pragma unroll
    for (int D = 0; D < 16; ++D) put4(o, carry, q[D], nib4(nz[D >> 3], D & 7), sel0[D]);
    or_at(o, carry);
    const int kap = min(max((int)base + sc.kap, -16), 16);
    if (V + lane < G.sz) st[V + lane] = (unsigned char)((((u32)(13 + kap) + (sc.r == 7u)) & 15u) << 3 | ((sc.r + 1u) & 7u));
    CFENCE();
    u32 d[9], nb[2] = {0u, 0u}, bad = 0, nca = 0;
    if (hint) {
      u32 sel[9];
#pragma unroll
      for (u32 k = 0; k < 9u; ++k) d[k] = *(LAS u32*)(st + (k * G.nc + lane) * 4u);
#pragma unroll
      for (u32 k = 0; k < 9u; ++k) {
        u32 pos, up;
        const u32 u = low_exp(ob[k], (u32)(kap + 32) * B1, pos), df = ((d[k] >> 3) & 15u * B1) + 32u * B1 - u;
        d[k] &= 7u * B1;
        const u32 su = sure(d[k], r8, pos, up), w = ((df ^ 32u * B1) & ~B1) | ((df ^ up) & su);
        bad |= (k < 8u ? in8 : in9) ? w : 0u;
        nb[k >> 3] |= to_nib(su) << (4u * (k & 7u));
        ob[k] = df & B1;
      }
      nb[0] = in8 ? ~nb[0] : 0u; nb[1] = in9 ? ~nb[1] & 15u : 0u;
#pragma unroll
      for (u32 k = 0; k < 9u; ++k) sel[k] = put_sel(c, nib4(nb[k >> 3], k & 7u));
      const u32 cn = __popc(nb[0]) + __popc(nb[1]), in2 = wave_incl_scan(cn), Va = __builtin_amdgcn_readlane(in2, 63);
      nca = (Va + 31u) >> 5;
      hint = !ANY(bad != 0u) && G.sz + nca * 32u + 8u <= 4096u;
      if (hint) {
        o = ldsa(st) + G.sz + in2 - cn; carry = 0;
#pragma unroll
        for (u32 k = 0; k < 9u; ++k) put4(o, carry, ob[k], nib4(nb[k >> 3], k & 7u), sel[k]);
        or_at(o, carry);
      }
    }
    CFENCE();
    if (hint) {
      u32 bits = 0;
#pragma unroll
      for (u32 k = 0; k < 8u; ++k) bits |= *(LAS u32*)(st + G.sz + (k * nca + lane) * 4u) << k;
      CFENCE();
      LAS u32* s = (LAS u32*)st + lane;
      if (in8) {
        s[0] = d[0] | d[1] << 3 | ((d[6] << 6) & 0xC0u * B1); s[G.nc] = d[2] | d[3] << 3 | ((d[7] << 6) & 0xC0u * B1);
        s[2u * G.nc] = d[4] | d[5] << 3 | ((d[6] << 4) & 0x40u * B1) | ((d[7] << 5) & 0x80u * B1);
      }
      if (in9) s[3u * G.nc] = d[8];
      if ((u32)lane < nca) s[(G.sz >> 2) - 5u * G.nc] = bits;
      sb = (G.sz - 20u * G.nc + nca * 4u + 15u) & ~15u;
    } else {
      const u32 nc = row_bytes(V) >> 2;
      for (u32 col = lane; col < nc; col += 64u) {
        LAS u32* s = (LAS u32*)st + col;
        const u32 d7 = s[7u * nc];
#pragma unroll
        for (u32 k = 0; k < 7u; ++k) s[k * nc] |= (d7 << (7u - k)) & 0x80u * B1;
      }
      sb = (28u * nc + 15u) & ~15u;
    }
    u32x4 hdr;
    const u32 ovb = O_OV + ht * 16384, woff = place(c, wv, C_CSZ2 + zs, 16u, sb, (u32)hint << 16, hdr);
    if ((wv & 3) == 0 && lane == 0) st128<SYS>(hdr, rsL, ovb);
    for (u32 k = lane * 16; k < sb; k += 1024u) st128<SYS>(*(LAS u32x4*)(st + k), rsL, k + (ovb + woff));
    VMWAIT0();
    u32 t = 0;
    if (lane == 0) t = lds_add(c.ctl + C_CVAL0 + zs, 1u);
    if (RFL(t) == 3u) tell_peers(c, O_OF + c.rank * 8192 + F_VF + ht * 4, c.call << 20 | (hdr.w & 0xFFFFu) >> 4);
  }
  if (wv == 0) STAMP(c, 12);
  const rsrc_t rsO = mkrs(p.out, (u32)MM * NN);
  const u32x4 nan4 = {0x7Fu * B1, 0x7Fu * B1, 0x7Fu * B1, 0x7Fu * B1};
  {
    const u32 obase = (u32)(mi * 256 + mw * 64 + (lane & 15)) * NN + c.rank * 2048 + ni * 128 + h * 64 + (lane >> 4) * 16;
    const bool bad = tm == c.call || lds_ld(c.ctl + C_ABORT) != 0u;
#pragma unroll
    for (int b = 0; b < 4; ++b) st128<OUTAUX>(bad ? nan4 : (u32x4){q[4 * b], q[4 * b + 1], q[4 * b + 2], q[4 * b + 3]}, rsO, obase + b * 16 * NN);
  }
}

DI void role_outdec(Ctx& c, const Params& p, int wv) {
  const int lane = c.lane;
  __builtin_amdgcn_s_setprio(0);
  c.call = lds_wait(c, C_CALL, 1u);
  lds_wait(c, C_GVALID, 1u);
  const float amax = __uint_as_float(RFL(lds_ld(c.ctl + C_GMAX)));
  const Scale sc = split(__uint_as_float(RFL(__float_as_uint(amax > 0.f ? 448.0f / amax : 0.f))));
  const u32 r8 = (8u - sc.r) * B1;
  const rsrc_t rsL = mkrs(c.ws, WS_USED), rsO = mkrs(p.out, (u32)MM * NN);
  const u32x4 nan4 = {0x7Fu * B1, 0x7Fu * B1, 0x7Fu * B1, 0x7Fu * B1};
  LAS unsigned char* st = c.lds + L_RING + (wv < 8 ? wv * 4096 : 98304 + (wv - 8) * 4096);
  const u32 src = (c.rank + 1 + wg_link(c.wg)) & 7;
  for (u32 kv = wv >> 2;; kv += 3u) {
    const u32 e = q_wait(c, C_QV, kv), dh = e & 0xFFFu, pb = ((e >> 12) & 2047u) * 32768u;
    if (dh >= 512u) break;
    const u32 dt = dh >> 1, dw = (dh & 1u) * 4u + (wv & 3), inb = O_MIN + (src * 512 + dh) * 8192u;
    dma1(rsL, lane * 16, inb + 2064u + (wv & 3) * 1280u, (u32)(size_t)st + 2560u);
    if (lane < 16) dma1(rsL, lane * 16, inb + 3088u + (wv & 3) * 1280u, (u32)(size_t)st + 3584u);
    const u32x2 m = __builtin_amdgcn_raw_buffer_load_b64(rsL, lane * 8, (int)(inb + 16 + (wv & 3) * 512), AGT);
    const u32 he = __builtin_amdgcn_raw_buffer_load_b32(rsL, 0, (int)(inb + (wv & 3) * 4), AGT);
    const u32 dbase = (u32)(tile_mi(dt) * 256 + (dw >> 1) * 64 + (lane & 15)) * NN + src * 2048 + tile_ni(dt) * 128 + (dw & 1) * 64 + (lane >> 4) * 16;
    const u32 cn = __popc(m.x) + __popc(m.y), in2 = wave_incl_scan(cn), V2 = __builtin_amdgcn_readlane(in2, 63);
    const Geo G2 = geo(V2);
    const bool j8 = (u32)lane < G2.nc, j9 = 8u * G2.nc + lane < G2.sz >> 2;
    u32 u[9], pos[9];
    VMWAIT0();
    {
      LAS u32* s = (LAS u32*)(st + 2560) + lane;
      const u32 ra = s[0], rb = s[G2.nc], rc = s[2u * G2.nc], l1[3] = {j8 ? ra : ~0u, j8 ? rb : ~0u, j9 ? rc : ~0u};
      const u32 k32 = (u32)(min(max((int)((he >> 16) & 255u) + sc.kap, -16), 16) + 32) * B1;
      dec_levels<9, 1024>(c, l1, k32, u, st + 2560 + G2.sz - 24u * G2.nc, st, 2560u);
#pragma unroll
      for (u32 k = 0; k < 9u; ++k) u[k] = low_exp(u[k], 0u, pos[k]);
    }
    LAS unsigned char* sl = c.lds + L_STG + (kv & 3u) * 16384u;
    lds_wait(c, C_PROG, pb + 16u);
    const u32 hw = RFL(*(LAS u32*)(sl + (wv & 3) * 4)), hp = RFL(*(LAS u32*)(sl + ((wv + 3) & 3) * 4));
    const u32 wo = (wv & 3) ? min(hp & 0xFFF0u, 16384u - 3584u) : 16u;
    lds_wait(c, C_PROG, pb + (hw & 0xFFFFu));
    const u32 tm2 = g_ld_agent(c.ws + O_TMO);
    if ((hw >> 16) & 1u) {
      LAS u32* sg = (LAS u32*)(sl + wo) + lane;
      const u32 ra = sg[0], rb = sg[G2.nc], rc = sg[2u * G2.nc], bits = sg[(G2.sz >> 2) - 5u * G2.nc];
      u32 y[9], nb[2] = {0u, 0u};
      y[8] = sg[3u * G2.nc] & 7u * B1;
      y[0] = ra & 7u * B1; y[1] = (ra >> 3) & 7u * B1; y[2] = rb & 7u * B1; y[3] = (rb >> 3) & 7u * B1; y[4] = rc & 7u * B1; y[5] = (rc >> 3) & 7u * B1;
      y[6] = ((ra >> 6) & 3u * B1) | ((rc >> 4) & 4u * B1); y[7] = ((rb >> 6) & 3u * B1) | ((rc >> 5) & 4u * B1);
#pragma unroll
      for (u32 k = 0; k < 9u; ++k) {
        u32 up;
        const u32 su = sure(y[k], r8, pos[k], up);
        u[k] += up & su;
        nb[k >> 3] |= to_nib(su) << (4u * (k & 7u));
      }
      nb[0] = j8 ? ~nb[0] : 0u; nb[1] = j9 ? ~nb[1] & 15u : 0u;
      const u32 ca = __popc(nb[0]) + __popc(nb[1]), ia = wave_incl_scan(ca), nca = min((__builtin_amdgcn_readlane(ia, 63) + 31u) >> 5, 64u);
      if ((u32)lane < nca) {
#pragma unroll
        for (u32 k = 0; k < 8u; ++k) *(LAS u32*)(st + (k * nca + lane) * 4u) = (bits >> k) & B1;
      }
      CFENCE();
      u32 bb[9];
      get_all<9>(c, ldsa(st) + ia - ca, nb, bb);
#pragma unroll
      for (u32 k = 0; k < 9u; ++k) y[k] |= (u[k] + bb[k]) << 3;
      CFENCE();
      if (j8) {
#pragma unroll
        for (u32 k = 0; k < 8u; ++k) *(LAS u32*)(st + (k * G2.nc + lane) * 4u) = y[k];
      }
      if (j9) *(LAS u32*)(st + (8u * G2.nc + lane) * 4u) = y[8];
    } else {
      const u32 nc = row_bytes(V2) >> 2;
      for (u32 col = lane; col < nc; col += 64u) {
        LAS u32* sg = (LAS u32*)(sl + wo) + col;
        LAS u32* d = (LAS u32*)st + col;
        u32 d7 = 0;
#pragma unroll
        for (u32 k = 0; k < 7u; ++k) { const u32 v = sg[k * nc]; d7 |= (v & 0x80u * B1) >> (7u - k); d[k * nc] = v & 0x7Fu * B1; }
        d[7u * nc] = d7;
      }
    }
    CFENCE();
    if (lane == 0) lds_add(c.ctl + C_DFREE0 + (kv & 3u), 1u);
    CFENCE();
    u32 ov[16];
    const u32 mm[2] = {m.x, m.y};
    get_all<16>(c, ldsa(st) + in2 - cn, mm, ov);
    const bool bad = tm2 == c.call || lds_ld(c.ctl + C_ABORT) != 0u;
#pragma unroll
    for (int k = 0; k < 4; ++k) st128<OUTAUX>(bad ? nan4 : (u32x4){ov[4 * k], ov[4 * k + 1], ov[4 * k + 2], ov[4 * k + 3]}, rsO, dbase + k * 16 * NN);
    CFENCE();
  }
}

__global__ __launch_bounds__(1024, 4) void fused_gemm_ar(Params p) {
  extern __shared__ __attribute__((aligned(16))) unsigned char smem[];
  Ctx c;
  c.lds = (LAS unsigned char*)smem; c.ctl = (LAS u32*)c.lds; c.ws = p.ws[p.rank]; c.t0 = now(); c.call = 0;
  c.lane = threadIdx.x & 63; c.wg = blockIdx.x; c.rank = p.rank;
  const int wv = RFL(threadIdx.x >> 6), tid = threadIdx.x;
  if (tid >= 512) *(LAS u32*)(c.lds + L_REL + (tid - 512) * 4) = 0u;
  else if (tid < C_WORDS || (tid >= L_GT / 4 && tid < L_GT / 4 + 16)) c.ctl[tid] = 0u;
  else if (tid >= 192 && tid < 256) {
    const u32 nib = tid & 15; u32 s = 0, k = (tid - 192) >> 4;
    for (int g = 0; g < 4; ++g) { if ((nib >> g) & 1u) s |= k++ << (8 * g); else s |= 0x0Cu << (8 * g); }
    *(LAS u32*)(c.lds + L_LUTE + (tid - 192) * 4) = s;
  } else if (tid >= 288 && tid < 288 + NR) {
    u64 v = 0;
#pragma unroll
    for (int i = 0; i < NR; ++i) { u64 t = (u64)p.ws[i]; asm volatile("" : "+s"(t)); v = tid == 288 + i ? t : v; }
    *(LAS u64*)(c.lds + L_PEER + (tid - 288) * 8) = v;
  } else if (tid >= 256 && tid < 272) {
    const u32 nib = tid - 256; u32 s = 0x0C0C0C0Cu, k = 0;
    for (u32 g = 0; g < 4; ++g) if ((nib >> g) & 1u) { s = (s & ~(255u << (8 * k))) | g << (8 * k); ++k; }
    *(LAS u32*)(c.lds + L_LUTC + nib * 4) = s;
  }
  asm volatile("s_waitcnt lgkmcnt(0)" ::: "memory");
  __builtin_amdgcn_s_barrier();
  CFENCE();
  const int mi = tile_mi(c.wg), ni = tile_ni(c.wg);
  if (wv == 11) STAMP(c, 0);
  if (wv < 8) role_mfma(c, p, wv, mi, ni);
  else if (wv < 11) role_loader(c, wv - 8, mi, ni);
  else if (wv == 11) role_sched(c, p, mi, ni);
  else if (wv == 12) role_puller(c, p);
  else if (wv == 13) role_feeder(c);
  else role_decoder(c, wv - 14);
  if (wv < 12) role_outdec(c, p, wv);
}
}

extern "C" void kernel_launch(int rank, int nranks, ncclComm_t comm, hipStream_t stream,
                              void* const* d_in, const int* in_sizes, int n_in,
                              void* d_out, int out_size, const int* in_dtypes, int out_dtype,
                              void* const* d_out_all, void* const* d_ws_all, size_t ws_bytes) {
  if (nranks != NR || (unsigned)rank >= (unsigned)NR || n_in < 2 || ws_bytes < WS_USED || in_sizes[0] != MM * KS || in_sizes[1] != KS * NN || out_size != MM * NN) {
    fprintf(stderr, "kernel_launch: unexpected problem shape\n");
    return;
  }
  static bool init = false;
  if (!init) {
    hipError_t e = hipFuncSetAttribute((const void*)fused_gemm_ar, hipFuncAttributeMaxDynamicSharedMemorySize, (int)LDS_BYTES);
    if (e != hipSuccess) { fprintf(stderr, "hipFuncSetAttribute: %s\n", hipGetErrorString(e)); return; }
    init = true;
  }
  Params p;
  p.x = (const unsigned short*)d_in[0]; p.w = (const unsigned short*)d_in[1]; p.out = (unsigned char*)d_out;
  for (int i = 0; i < NR; ++i) p.ws[i] = (char*)d_ws_all[i];
  p.rank = rank; p.pad = 0;
  fused_gemm_ar<<<NWG, 1024, LDS_BYTES, stream>>>(p);
}
```

```cpp
#include <hip/hip_runtime.h>
#include <rccl/rccl.h>
#include <cstdio>

typedef unsigned u32;
typedef unsigned long long u64;
typedef u32 u32x4 __attribute__((ext_vector_type(4)));
typedef u32 u32x2 __attribute__((ext_vector_type(2)));
typedef unsigned short u16x2 __attribute__((ext_vector_type(2)));
typedef short bf16x8 __attribute__((ext_vector_type(8)));
typedef float f32x4 __attribute__((ext_vector_type(4)));
typedef __amdgpu_buffer_rsrc_t rsrc_t;
#define LAS __attribute__((address_space(3)))
#define GAS __attribute__((address_space(1)))
#define DI __device__ __forceinline__
#define RFL(x) __builtin_amdgcn_readfirstlane(x)
#define ALL(x) (__builtin_amdgcn_ballot_w64(!(x)) == 0ull)
#define ANY(x) (__builtin_amdgcn_ballot_w64(x) != 0ull)
#define CFENCE() asm volatile("" ::: "memory")
#define VMWAIT0() asm volatile("s_waitcnt vmcnt(0)" ::: "memory")

namespace {
constexpr int NR = 8, MM = 4096, KS = 512, NN = 16384, NWG = 256;
constexpr int NBLK = 768;
constexpr u32 SYS = 17, AGT = 16;
constexpr int LEADER = 16;
constexpr u64 DEADLINE = 50000000ull;
constexpr u32 O_CALL = 0;
constexpr u32 O_TMO = 32768;
constexpr u32 O_TICK = O_TMO + 128;
constexpr u32 O_MQ = O_TICK + 3072;
constexpr u32 O_GT = O_MQ + 128;
constexpr u32 O_LT = O_GT + 128;
constexpr u32 O_RF = O_LT + 2048;
constexpr u32 O_FX = O_RF + NR * NBLK * 4;
constexpr u32 O_FW = O_FX + 16 * 64 * 16;
constexpr u32 O_OF = O_FW + 16 * 64 * 8;
constexpr u32 F_MF = 0;
constexpr u32 F_MI = 2048;
constexpr u32 F_VF = 4096;
constexpr u32 O_DBG = 512 * 1024;
constexpr u32 O_OBX = 1u << 20;
constexpr u32 O_OBW = O_OBX + 512 * 8192;
constexpr u32 O_IMG = O_OBW + 2048 * 8192;
constexpr u32 O_OM = O_IMG + 64 * 96 * 8192;
constexpr u32 O_OV = O_OM + 512 * 8192;
constexpr u32 O_MIN = O_OV + 512 * 16384;
constexpr u32 WS_USED = O_MIN + NR * 512 * 8192;
static_assert(O_OF + NR * 8192 <= O_DBG, "control block overflow");
constexpr u32 T_MASK = NBLK, T_VAL = T_MASK + 512, T_END = T_VAL + 512;
enum { C_CALL = 0, C_ABORT, C_WGMAX, C_CMAX, C_DONE, C_CVAL0, C_CVAL1, C_GVALID, C_GMAX, C_PSTART, C_PROG, C_PACKED,
       C_FULL0 = 12, C_LEAD = 17, C_CSZ0 = 18, C_CSZ2 = 20, C_CPRE0 = 22, C_DFREE0 = 24, C_IFREE0 = 28, C_VSZ = 32, C_QI = 40, C_QV = 48, C_MQ0 = 56, C_ORDER = 64, C_WORDS = 128 };
constexpr u32 L_LUTC = C_WORDS * 4;
constexpr u32 L_GT = L_LUTC + 128;
constexpr u32 L_PEER = L_GT + 64;
constexpr u32 L_LUTE = L_LUTC + 256;
constexpr u32 L_REL = 1024;
constexpr u32 L_RFC = L_REL + 2048;
constexpr u32 NS = 5, SLOTB = 24576;
constexpr u32 L_RING = L_RFC + 4096;
constexpr u32 L_STG = L_RING + 32768;
constexpr u32 L_IN = L_RING + NS * SLOTB;
constexpr u32 LDS_BYTES = L_IN + 4 * 8192;
static_assert(LDS_BYTES <= 163840 && LDS_BYTES > 81920 && L_STG + 5 * 16384 <= L_IN, "one workgroup per CU");
#define PLEAD 2048
#define WIN1 2
#define WIN2 4
#define T_HALF 160
#define PACK_T0 200
#define PACK_DT 100
#define NRAW 20
#define T_W1 728
#define T_W2 T_END
#define T_WA 220u
#define G0FREE 1
#define T_AH (T_END - 80u)
#define GATEM 12
#define SCHED_Q 3u
#define PMED 5u
#define OUTAUX 0u
#define NDEC 3
#define DPRIO 1
#define KFIX 8u
#define T_W0 160

#define SPIN_FAST 16u
#define SLEEP_LONG 1
struct Params { const unsigned short* x; const unsigned short* w; unsigned char* out; char* ws[NR]; int rank; int pad; };

DI u64 now() { return __builtin_amdgcn_s_memrealtime(); }
DI rsrc_t mkrs(const void* p, u32 bytes) { return __builtin_amdgcn_make_buffer_rsrc((void*)p, (short)0, (int)bytes, 0x00020000); }
template <u32 AUX> DI void st128(u32x4 v, rsrc_t rs, u32 off) { __builtin_amdgcn_raw_buffer_store_b128(v, rs, (int)off, 0, AUX); }
DI void st32(u32 v, rsrc_t rs, u32 off) { __builtin_amdgcn_raw_buffer_store_b32(v, rs, (int)off, 0, SYS); }
DI u32 prm(u32 a, u32 b, u32 s) { return __builtin_amdgcn_perm(a, b, s); }
DI u32 bfi(u32 m, u32 a, u32 b) { return (m & a) | (~m & b); }
DI u32 lds_ld(LAS u32* p) { return __hip_atomic_load(p, __ATOMIC_RELAXED, __HIP_MEMORY_SCOPE_WORKGROUP); }
DI void lds_st(LAS u32* p, u32 v) { __hip_atomic_store(p, v, __ATOMIC_RELAXED, __HIP_MEMORY_SCOPE_WORKGROUP); }
DI u32 lds_add(LAS u32* p, u32 v) { return __hip_atomic_fetch_add(p, v, __ATOMIC_RELAXED, __HIP_MEMORY_SCOPE_WORKGROUP); }
DI void lds_or(LAS unsigned char* p, u32 v) { __hip_atomic_fetch_or((LAS u32*)p, v, __ATOMIC_RELAXED, __HIP_MEMORY_SCOPE_WORKGROUP); }
DI u32 g_ld_agent(const char* p) { return __hip_atomic_load((GAS u32*)p, __ATOMIC_RELAXED, __HIP_MEMORY_SCOPE_AGENT); }
DI u32 g_ld_sys(const char* p) { return __hip_atomic_load((GAS u32*)p, __ATOMIC_RELAXED, __HIP_MEMORY_SCOPE_SYSTEM); }
DI void g_st_agent(char* p, u32 v) { __hip_atomic_store((GAS u32*)p, v, __ATOMIC_RELAXED, __HIP_MEMORY_SCOPE_AGENT); }
DI void g_st_sys(char* p, u32 v) { __hip_atomic_store((GAS u32*)p, v, __ATOMIC_RELAXED, __HIP_MEMORY_SCOPE_SYSTEM); }

struct Ctx {
  LAS unsigned char* lds; LAS u32* ctl; char* ws; u64 t0; u32 call; int lane, wg, rank;
};
DI void give_up(const Ctx& c) { if (c.lane == 0) { lds_st(c.ctl + C_ABORT, 1u); if (c.call) g_st_agent(c.ws + O_TMO, c.call); } }
DI bool expired(const Ctx& c, u32& spins) {
  if ((++spins & 15u) != 0u) return false;
  if (RFL(lds_ld(c.ctl + C_ABORT)) != 0u || RFL(g_ld_agent(c.ws + O_TMO)) == c.call || now() - c.t0 > DEADLINE) { give_up(c); return true; }
  return false;
}
DI bool lds_expired(const Ctx& c, u32& spins) {
  if ((++spins & 63u) == 0u && (RFL(lds_ld(c.ctl + C_ABORT)) != 0u || now() - c.t0 > DEADLINE + DEADLINE / 4)) { give_up(c); return true; }
  if (spins < SPIN_FAST) __builtin_amdgcn_s_sleep(1); else __builtin_amdgcn_s_sleep(SLEEP_LONG);
  return false;
}
DI u32 lds_wait(const Ctx& c, int word, u32 target) {
  u32 v, spins = 0;
  while ((v = RFL(lds_ld(c.ctl + word))) < target) if (lds_expired(c, spins)) break;
  CFENCE();
  return v;
}
DI u32 q_wait(const Ctx& c, int q, u32 k) {
  u32 v, spins = 0;
  while (((((v = RFL(lds_ld(c.ctl + q + (k & 7u)))) >> 23) - (k + 1u)) & 256u) != 0u) if (lds_expired(c, spins)) { v = ~0u; break; }
  CFENCE();
  return RFL(v);
}
#define STAMP(c, i) do {} while (0)
DI void dma4(rsrc_t rs, u32 voff, u32 soff, u32 ldsdst) {
  u32 keep;
  asm volatile("s_nop 4\n\ts_mov_b32 %0, m0\n\ts_mov_b32 m0, %1\n\ts_nop 0\n\t"
               "buffer_load_dwordx4 %2, %3, %4 offen sc0 sc1 lds\n\t"
               "buffer_load_dwordx4 %2, %3, %4 offen offset:1024 sc0 sc1 lds\n\t"
               "buffer_load_dwordx4 %2, %3, %4 offen offset:2048 sc0 sc1 lds\n\t"
               "buffer_load_dwordx4 %2, %3, %4 offen offset:3072 sc0 sc1 lds\n\t"
               "s_mov_b32 m0, %0"
               : "=&s"(keep) : "s"(RFL(ldsdst)), "v"(voff), "s"(rs), "s"(RFL(soff)) : "memory");
}
DI void dma1(rsrc_t rs, u32 voff, u32 soff, u32 ldsdst) {
  u32 keep;
  asm volatile("s_nop 4\n\ts_mov_b32 %0, m0\n\ts_mov_b32 m0, %1\n\ts_nop 0\n\t"
               "buffer_load_dwordx4 %2, %3, %4 offen sc0 sc1 lds\n\t"
               "s_mov_b32 m0, %0"
               : "=&s"(keep) : "s"(RFL(ldsdst)), "v"(voff), "s"(rs), "s"(RFL(soff)) : "memory");
}
template <int CTRL, int ROWS> DI u32 dpp0(u32 v) { return (u32)__builtin_amdgcn_update_dpp(0, (int)v, CTRL, ROWS, 0xF, false); }
DI u32 wave_incl_scan(u32 v) {
  v += dpp0<0x111, 0xF>(v); v += dpp0<0x112, 0xF>(v); v += dpp0<0x114, 0xF>(v); v += dpp0<0x118, 0xF>(v);
  v += dpp0<0x142, 0xA>(v); v += dpp0<0x143, 0xC>(v);
  return v;
}
DI u32 wave_max(u32 v) {
  v = max(v, dpp0<0x111, 0xF>(v)); v = max(v, dpp0<0x112, 0xF>(v)); v = max(v, dpp0<0x114, 0xF>(v)); v = max(v, dpp0<0x118, 0xF>(v));
  v = max(v, dpp0<0x142, 0xA>(v)); v = max(v, dpp0<0x143, 0xC>(v));
  return __builtin_amdgcn_readlane(v, 63);
}
DI int tile_mi(int wg) { return ((wg & 7) >> 1) * 4 + ((wg >> 3) & 3); }
DI int tile_ni(int wg) { return (wg & 1) * 8 + (wg >> 5); }
DI int wg_late(int wg) { return (wg >> 4) & 1; }
DI int wg_link(int wg) { return (((wg & 15) | (wg >> 5) << 4) + 3 * wg_late(wg)) % 7; }
DI u32 link_wgs(int link, int late) { return (u32)(127 + 7 - (link + 7 - 3 * late) % 7) / 7u; }
DI u32 row_bytes(u32 V) { return (((V + 7u) >> 3) + 3u) & ~3u; }
DI void tell_peers(const Ctx& c, u32 off, u32 v) { if (c.lane < NR && c.lane != c.rank) g_st_sys(*(char* LAS*)(c.lds + L_PEER + c.lane * 8) + off, v); }
DI void image_flag(const Ctx& c, int kstep, int r) {
  if (c.lane == 0) g_st_agent(c.ws + (r < 64 ? O_FX + ((r >> 2) * 64 + kstep) * 16 + (r & 3) * 4 : O_FW + (((r - 64) >> 1) * 64 + kstep) * 8 + (r & 1) * 4), c.call);
}

constexpr u32 B1 = 0x01010101u;
DI u32 nib4(u32 m, int k) { return (k ? m >> (4 * k - 2) : m << 2) & 0x3Cu; }
DI u32 to_nib(u32 bits) { return __builtin_amdgcn_udot4(bits, 0x08040201u, 0u, false); }
DI u32 put_sel(const Ctx& c, u32 n4) { return *(LAS u32*)(c.lds + L_LUTC + n4); }
DI u32 ldsa(LAS unsigned char* p) { return (u32)(size_t)p; }
DI void or_at(u32 a, u32 v) { lds_or((LAS unsigned char*)(size_t)(a & ~3u), v); }
DI u32 put4(u32& a, u32& carry, u32 v, u32 n4, u32 sel) {
  const u32 c4 = prm(0u, v, sel);
  const u64 w = (u64)c4 << ((a << 3) & 24u);
  or_at(a, (u32)w | carry);
  carry = (u32)(w >> 32);
  a += __popc(n4);
  return c4;
}
template <u32 N> DI void get_all(const Ctx& c, u32 a, const u32* m, u32 (&x)[N]) {
  u32 sel[N], hi[N];
#pragma unroll
  for (u32 k = 0; k < N; ++k) {
    const u32 n4 = nib4(m[k >> 3], k & 7u);
    sel[k] = *(LAS u32*)(c.lds + L_LUTE + (((a << 6) & 0xC0u) | n4));
    LAS u32* d = (LAS u32*)(size_t)(a & ~3u);
    x[k] = d[0]; hi[k] = d[1];
    a += __popc(n4);
  }
  __builtin_amdgcn_sched_barrier(0);
#pragma unroll
  for (u32 k = 0; k < N; ++k) x[k] = prm(hi[k], x[k], sel[k]);
}
struct Lv { u32 rows2, nc2, nc3; };
template <u32 R> DI bool enc_tail(const Ctx& c, u32 V2, LAS unsigned char* s, u32 cap, Lv& g, u32& l2a, u32& l2b, u32& l3) {
  const u32 nc2 = (V2 + 4u * R - 1u) / (4u * R), b2 = R * nc2 * 4u;
  u32 e[R], sel[R], nb3 = 0;
  l2a = 0u; l2b = 0u;
#pragma unroll
  for (u32 r = 0; r < R; ++r) e[r] = *(LAS u32*)(s + (r * nc2 + c.lane) * 4u);
#pragma unroll
  for (u32 r = 0; r < R; ++r) {
    e[r] = (u32)c.lane < nc2 ? e[r] : 0u;
    const u32 lo3 = e[r] & 7u * B1, b3 = (e[r] >> 3) & B1;
    const u32 c2 = bfi((b3 << 8) - b3, prm(0x00000002u, 0x03000000u, lo3), prm(0u, 1u, lo3));
    if (r < 4u) l2a |= c2 << (2u * r); else l2b |= c2 << (2u * r - 8u);
    nb3 |= to_nib((c2 | c2 >> 1) & B1) << (4u * r);
  }
  nb3 = ~nb3 & (R == 8u ? ~0u : 0xFFFFu);
#pragma unroll
  for (u32 r = 0; r < R; ++r) sel[r] = put_sel(c, nib4(nb3, r));
  const u32 cn3 = __popc(nb3), in3 = wave_incl_scan(cn3), V3 = __builtin_amdgcn_readlane(in3, 63);
  g.rows2 = R; g.nc2 = nc2; g.nc3 = (V3 + 7u) >> 3;
  if (V3 > 512u || b2 + g.nc3 * 8u + 8u > cap) return false;
  u32 a = ldsa(s) + b2 + in3 - cn3, carry = 0;
#pragma unroll
  for (u32 r = 0; r < R; ++r) put4(a, carry, e[r], nib4(nb3, r), sel[r]);
  or_at(a, carry);
  CFENCE();
  l3 = *(LAS u32*)(s + b2 + c.lane * 4u) | *(LAS u32*)(s + b2 + (g.nc3 + c.lane) * 4u) << 4;
  return true;
}
template <u32 N, u32 MAX2> DI bool enc_levels(const Ctx& c, u32 (&n)[N], LAS unsigned char* s, u32 cap, Lv& g, u32& l2a, u32& l2b, u32& l3) {
  u32 nb[2] = {0u, 0u}, c1[N], sel[N];
#pragma unroll
  for (u32 k = 0; k < N; ++k) {
    const u32 m = ((n[k] + 3u * B1) >> 4) & B1;
    c1[k] = n[k] & (m + (m << 1));
    nb[k >> 3] |= to_nib(m) << (4u * (k & 7u));
  }
  nb[0] = ~nb[0]; nb[1] = ~nb[1] & (N == 16u ? ~0u : (1u << (4u * (N & 7u))) - 1u);
#pragma unroll
  for (u32 k = 0; k < N; ++k) sel[k] = put_sel(c, nib4(nb[k >> 3], k & 7u));
  const u32 cnt = __popc(nb[0]) + __popc(nb[1]), inc = wave_incl_scan(cnt), V2 = __builtin_amdgcn_readlane(inc, 63);
  if (V2 > MAX2 || V2 + 40u > cap) return false;
  u32 a = ldsa(s) + inc - cnt, carry = 0;
#pragma unroll
  for (u32 k = 0; k < N; ++k) put4(a, carry, n[k], nib4(nb[k >> 3], k & 7u), sel[k]);
  or_at(a, carry);
  CFENCE();
#pragma unroll
  for (u32 k = 0; k < N; ++k) n[k] = c1[k];
  if (MAX2 > 1024u && V2 > 1024u) return enc_tail<8>(c, V2, s, cap, g, l2a, l2b, l3);
  return enc_tail<4>(c, V2, s, cap, g, l2a, l2b, l3);
}
template <u32 R> DI void dec_tail(const Ctx& c, u32 V2, LAS unsigned char* in, LAS unsigned char* s, u32 cap) {
  const u32 nc2 = (V2 + 4u * R - 1u) / (4u * R), b2 = R * nc2 * 4u;
  const bool col = (u32)c.lane < nc2;
  u32 l2[2] = {*(LAS u32*)(in + c.lane * 4u), R == 8u ? *(LAS u32*)(in + (nc2 + c.lane) * 4u) : 0u}, nb3 = 0, e[R];
  const u32 l3 = *(LAS u32*)(in + R * nc2 + c.lane * 4u);
  l2[0] = col ? l2[0] : ~0u; l2[1] = col ? l2[1] : ~0u;
#pragma unroll
  for (u32 r = 0; r < R; ++r) nb3 |= (to_nib((l2[r >> 2] | l2[r >> 2] >> 1) & B1 << (2u * (r & 3u))) >> (2u * (r & 3u))) << (4u * r);
  nb3 = ~nb3 & (R == 8u ? ~0u : 0xFFFFu);
  const u32 cn3 = __popc(nb3), in3 = wave_incl_scan(cn3), V3 = __builtin_amdgcn_readlane(in3, 63), nc3 = (V3 + 7u) >> 3;
  if (V3 > 512u || b2 + nc3 * 8u + 8u > cap) return;
  if ((u32)c.lane < nc3) {
    *(LAS u32*)(s + b2 + c.lane * 4u) = ((l3 & 15u * B1) + 14u * B1) & 15u * B1;
    *(LAS u32*)(s + b2 + (nc3 + c.lane) * 4u) = (((l3 >> 4) & 15u * B1) + 14u * B1) & 15u * B1;
  }
  CFENCE();
  get_all<R>(c, ldsa(s) + b2 + in3 - cn3, &nb3, e);
  if (col) {
#pragma unroll
    for (u32 r = 0; r < R; ++r) *(LAS u32*)(s + (r * nc2 + c.lane) * 4u) = e[r] | prm(0u, 0x090A0E00u, (l2[r >> 2] >> (2u * (r & 3u))) & 3u * B1);
  }
}
template <u32 N, u32 MAX2> DI void dec_levels(const Ctx& c, const u32* l1, u32 add, u32 (&o)[N], LAS unsigned char* in, LAS unsigned char* s, u32 cap) {
  u32 nb[2] = {0u, 0u};
#pragma unroll
  for (u32 k = 0; k < N; ++k) nb[k >> 3] |= (to_nib((l1[k >> 2] | l1[k >> 2] >> 1) & B1 << (2u * (k & 3u))) >> (2u * (k & 3u))) << (4u * (k & 7u));
  nb[0] = ~nb[0]; nb[1] = ~nb[1] & (N == 16u ? ~0u : (1u << (4u * (N & 7u))) - 1u);
  const u32 cnt = __popc(nb[0]) + __popc(nb[1]), inc = wave_incl_scan(cnt), V2 = __builtin_amdgcn_readlane(inc, 63);
  if (V2 <= MAX2 && V2 + 40u <= cap) {
    if (MAX2 > 1024u && V2 > 1024u) dec_tail<8>(c, V2, in, s, cap); else dec_tail<4>(c, V2, in, s, cap);
  }
  CFENCE();
  get_all<N>(c, ldsa(s) + inc - cnt, nb, o);
#pragma unroll
  for (u32 k = 0; k < N; ++k) o[k] += prm(0u, 0x0D0C0B00u, (l1[k >> 2] >> (2u * (k & 3u))) & 3u * B1) + add;
}
DI u32 pick_top(u32 sum, u32 cnt) { return (u32)((float)sum * __builtin_amdgcn_rcpf((float)max(cnt, 1u)) + 1.83f); }

DI u32x4 piece(const u32 (&lo)[16], const u32 (&hi)[16], int t) {
  return (u32x4){prm(hi[2 * t], lo[2 * t], 0x05010400u), prm(hi[2 * t], lo[2 * t], 0x07030602u),
                 prm(hi[2 * t + 1], lo[2 * t + 1], 0x05010400u), prm(hi[2 * t + 1], lo[2 * t + 1], 0x07030602u)};
}
DI void pack_inputs(Ctx& c, const Params& p, int wv) {
  const int lane = c.lane;
  const rsrc_t rsL = mkrs(c.ws, WS_USED), rsX = mkrs(p.x, MM * KS * 2), rsW = mkrs(p.w, KS * NN * 2);
  LAS unsigned char* st = c.lds + L_RING + wv * 4096;
  const int hi7 = (c.wg & 15) | (c.wg >> 5) << 4, late = wg_late(c.wg);
  int q = wv == 0 ? hi7 * 2 + late : late ? 128 + wv * 128 + hi7 : 2560;
  if (q < 320) __builtin_amdgcn_s_setprio(3);
  for (; q < 2560; q = q < 256 ? (late ? 1152 + hi7 : 2560) : q + 1024) {
    int j = 0, r = q;
    j = q / 320; r = q - j * 320;
    if (j) { __builtin_amdgcn_s_setprio(0); while (now() - c.t0 < (u64)(PACK_T0 + PACK_DT * j)) __builtin_amdgcn_s_sleep(8); }
    const bool isx = r < 64;
    const int dest = (r - 64) & 7, ni = (r - 64) >> 4, kh = ((r - 64) >> 3) & 1;
    u32 lo[16], hi[16];
    *(LAS u32x4*)(st + lane * 16) = (u32x4){0u, 0u, 0u, 0u}; *(LAS u32x4*)(st + 1024 + lane * 16) = (u32x4){0u, 0u, 0u, 0u};
    if (isx) {
      u32x4 d[8];
      const int vo = ((64 * r + (lane & 15)) * KS + 64 * j + 8 * (lane >> 4)) * 2;
#pragma unroll
      for (int t = 0; t < 8; ++t) d[t] = __builtin_amdgcn_raw_buffer_load_b128(rsX, vo + 64 * (t >> 2), 16 * (t & 3) * KS * 2, 0);
      __builtin_amdgcn_sched_barrier(0);
#pragma unroll
      for (int t = 0; t < 8; ++t) {
        lo[2 * t] = prm(d[t].y, d[t].x, 0x06040200u); lo[2 * t + 1] = prm(d[t].w, d[t].z, 0x06040200u);
        hi[2 * t] = prm(d[t].y, d[t].x, 0x07050301u); hi[2 * t + 1] = prm(d[t].w, d[t].z, 0x07050301u);
      }
    } else {
      u32x2 d[2][8];
      const int vo = ((64 * j + 32 * kh + 8 * (lane >> 4)) * NN + dest * 2048 + ni * 128 + 4 * (lane & 15)) * 2;
#pragma unroll
      for (int k = 0; k < 8; ++k)
#pragma unroll
        for (int h = 0; h < 2; ++h) d[h][k] = __builtin_amdgcn_raw_buffer_load_b64(rsW, vo + 128 * h, k * NN * 2, 0);
      __builtin_amdgcn_sched_barrier(0);
#pragma unroll
      for (int h = 0; h < 2; ++h)
#pragma unroll
        for (int cc = 0; cc < 2; ++cc)
#pragma unroll
          for (int u = 0; u < 2; ++u) {
            const u32 a0 = d[h][4 * u][cc], a1 = d[h][4 * u + 1][cc], a2 = d[h][4 * u + 2][cc], a3 = d[h][4 * u + 3][cc];
            const u32 l0 = prm(a1, a0, 0x06020400u), l1 = prm(a3, a2, 0x06020400u), h0 = prm(a1, a0, 0x07030501u), h1 = prm(a3, a2, 0x07030501u);
            const int t = 4 * h + 2 * cc;
            lo[2 * t + u] = prm(l1, l0, 0x05040100u); lo[2 * t + 2 + u] = prm(l1, l0, 0x07060302u);
            hi[2 * t + u] = prm(h1, h0, 0x05040100u); hi[2 * t + 2 + u] = prm(h1, h0, 0x07060302u);
          }
    }
    const int lr = isx ? r : 64 + 2 * ni + kh;
    const bool own = isx || dest == c.rank, send = isx || dest != c.rank;
    if (own) {
      const u32 img = O_IMG + (u32)((j * 8 + c.rank) * 96 + lr) * 8192u + lane * 16;
#pragma unroll
      for (int t = 0; t < 8; ++t) st128<AGT>(piece(lo, hi, t), rsL, img + t * 1024);
    }
    u32 tot = 8192u;
    if (send) {
      const u32 slot = (isx ? O_OBX + (u32)(j * 64 + r) * 8192u : O_OBW + (u32)((dest * 8 + j) * 32 + 2 * ni + kh) * 8192u);
      u32 n[16], se = 0, any = 0, cnt = 0;
      if (q >= NRAW) {
#pragma unroll
      for (int k = 0; k < 16; ++k) { n[k] = bfi(B1, lo[k] >> 7, hi[k] << 1); se = __builtin_amdgcn_sad_u8(n[k], 0u, se); }
      const u32 base = min(max(pick_top(__builtin_amdgcn_readlane(wave_incl_scan(se), 63), 4096u), 13u) - 13u, 240u);
      const u32 bl = (base & 127u) * B1, bx = base < 128u ? 0x80u * B1 : 0u;
#pragma unroll
      for (int k = 0; k < 16; ++k) { n[k] = (((n[k] | 0x80u * B1) - bl) ^ (n[k] & 0x80u * B1)) ^ bx; any |= n[k]; }
      if (ANY((any & 0xF0u * B1) != 0u)) {
#pragma unroll
        for (int k = 0; k < 16; ++k) if (ANY((n[k] & 0xF0u * B1) != 0u)) {
#pragma unroll
          for (int b = 0; b < 4; ++b) {
            const u32 v = (n[k] >> (8 * b)) & 255u;
            const u64 m = __builtin_amdgcn_ballot_w64(v > 15u);
            const u32 pos = cnt + __builtin_amdgcn_mbcnt_hi((u32)(m >> 32), __builtin_amdgcn_mbcnt_lo((u32)m, 0u));
            if (v > 15u && pos < 31u) st32((u32)(k * 4 + b) << 14 | (u32)lane << 8 | ((v + base) & 255u), rsL, slot + 5124u + pos * 4u);
            cnt += (u32)__popcll(m);
          }
        }
      }
#pragma unroll
      for (int k = 0; k < 16; ++k) n[k] = ((n[k] & 15u * B1) + 2u * B1) & 15u * B1;
      Lv g; u32 l2a, l2b, l3;
      if (cnt <= 31u && enc_levels<16, 2048>(c, n, st, 2048u, g, l2a, l2b, l3)) {
#pragma unroll
        for (int t = 0; t < 4; ++t)
          st128<SYS>((u32x4){bfi(0x80u * B1, hi[4 * t], lo[4 * t]), bfi(0x80u * B1, hi[4 * t + 1], lo[4 * t + 1]),
                             bfi(0x80u * B1, hi[4 * t + 2], lo[4 * t + 2]), bfi(0x80u * B1, hi[4 * t + 3], lo[4 * t + 3])}, rsL, slot + t * 1024 + lane * 16);
        u32x4 c1;
#pragma unroll
        for (int t = 0; t < 4; ++t) c1[t] = n[4 * t] | n[4 * t + 1] << 2 | n[4 * t + 2] << 4 | n[4 * t + 3] << 6;
        st128<SYS>(c1, rsL, slot + 4096u + lane * 16);
        const u32 o2 = 5120u + ((cnt + 4u) & ~3u) * 4u;
        if (lane == 0) st32(base | cnt << 8, rsL, slot + 5120u);
        if ((u32)lane < g.nc2) { st32(l2a, rsL, slot + o2 + lane * 4); if (g.rows2 == 8u) st32(l2b, rsL, slot + o2 + (g.nc2 + lane) * 4u); }
        if ((u32)lane < g.nc3) st32(l3, rsL, slot + o2 + g.rows2 * g.nc2 + lane * 4);
        tot = (o2 + g.rows2 * g.nc2 + g.nc3 * 4u + 15u) & ~15u;
      } else VMWAIT0();
      }
      if (tot == 8192u) {
        asm volatile("" : "+v"(lo[0]), "+v"(lo[2]), "+v"(lo[4]), "+v"(lo[6]), "+v"(lo[8]), "+v"(lo[10]), "+v"(lo[12]), "+v"(lo[14]),
                          "+v"(lo[1]), "+v"(lo[3]), "+v"(lo[5]), "+v"(lo[7]), "+v"(lo[9]), "+v"(lo[11]), "+v"(lo[13]), "+v"(lo[15]));
#pragma unroll
        for (int t = 0; t < 8; ++t) st128<SYS>(piece(lo, hi, t), rsL, slot + t * 1024 + lane * 16);
      }
    }
    if (!c.call) c.call = lds_wait(c, C_CALL, 1u);
    VMWAIT0();
    if (send && (isx || lane == dest)) tell_peers(c, O_RF + (c.rank * NBLK + j * 96 + lr) * 4, c.call << 20 | tot >> 4);
    if (own) image_flag(c, j * 8 + c.rank, lr);
    if (q < 64) STAMP(c, 1);
    if (q >= 2048) STAMP(c, 2);
  }
  __builtin_amdgcn_s_setprio(0);
  c.call = lds_wait(c, C_CALL, 1u);
  if (lane == 0) lds_add(c.ctl + C_PACKED, 1u);
}

DI u32 ob_slot(int b, int me) { const int j = b / 96, r = b - j * 96; return r < 64 ? O_OBX + (u32)(j * 64 + r) * 8192u : O_OBW + (u32)((me * 8 + j) * 32 + r - 64) * 8192u; }
#define LEFT {0, 2, 4, 4}
struct Order {
  u32 lb[NBLK];
  constexpr Order() : lb() {
    constexpr int left[4] = LEFT, phase[4] = {6, 5, 1, 3};
    int n = 0, next[4] = {1, 1, 1, 1};
    for (int r = 0; r < 96; ++r) lb[n++] = (u32)r;
    for (int j = 1; j < 16; ++j) {
      bool go[4] = {};
      for (int g = 0; g < 4; ++g) go[g] = next[g] < 8 && (j > 7 || (j * (7 - left[g]) + phase[g]) / 7 >= next[g]) && !(j <= 7 && next[g] > 7 - left[g]);
      if (j < 8) for (int r = 0; r < 32; ++r) lb[n++] = (u32)(j * 96 + 64 + r);
      for (int u = 0; u < 4; ++u)
        for (int v = 0; v < 4; ++v) {
          const int t = j > 7 ? v : u, g = j > 7 ? u : v;
          if (go[g]) for (int b = 0; b < 4; ++b) lb[n++] = (u32)(next[g] * 96 + t * 16 + g * 4 + b);
        }
      for (int g = 0; g < 4; ++g) next[g] += go[g];
    }
  }
};
__constant__ Order ORDER = Order();
DI int pull_order(u32 B, int link) {
  const u32 lb = ORDER.lb[B], r = lb % 96u;
  return (int)(B >= 96u && r < 64u ? lb - r + ((r + 16u * link) & 63u) : lb);
}
DI u32 early_first(u32 t) { return (t & 15u) | (t >> 7) << 4 | ((t >> 4) & 7u) << 5; }
DI void role_feeder(Ctx& c) {
  const int lane = c.lane;
  __builtin_amdgcn_s_setprio(3);
  c.call = lds_wait(c, C_CALL, 1u);
  const int link = wg_link(c.wg), src = (c.rank + 1 + link) & 7;
  const u32 nl = link_wgs(link, 1), ne = link_wgs(link, 0), na = ne + nl, late = wg_late(c.wg);
  u32 n0 = 0;
  for (u32 i = link; i < 128u; i += 7u) n0 += (i & 8u) == 0u;
  u32 kv = 0, ph = 0, tbase = (c.call - 1u) * (T_HALF - KFIX * ne + na - n0) - KFIX * ne, tend = T_HALF;
  bool miss = true;
  const rsrc_t rsL = mkrs(c.ws, WS_USED);
  u32 fb = O_RF + src * NBLK * 4, B = 0;
  if (late) lds_wait(c, C_PACKED, 8u);
  for (u32 k = 0;; ++k) {
    const u32 ahead = B < T_AH ? 2u : 1u;
    if (k >= ahead) lds_wait(c, C_PSTART, k + 1u - ahead);
    if (k < KFIX && !late) {
      B = (u32)((c.wg & 15) | (c.wg >> 5) << 4) / 7u + k * ne;
      if (miss) { dma4(rsL, lane * 16, fb, (u32)(size_t)c.lds + L_RFC); VMWAIT0(); }
    } else for (;;) {
      if (G0FREE && ph == 0u && ((c.wg >> 3) & 3) == 0) {
        ph = 2u; tbase = (c.call - 1u) * (T_END - T_MASK + na) - T_MASK; tend = T_END; fb = O_OF + src * 8192; miss = true;
        lds_wait(c, C_CMAX, 8u);
      }
      if (miss) dma4(rsL, lane * 16, fb, (u32)(size_t)c.lds + L_RFC);
      u32 t = 0;
      if (lane == 0) t = __hip_atomic_fetch_add((GAS u32*)(c.ws + O_TICK + ph * 1024 + link * 128), 1u, __ATOMIC_RELAXED, __HIP_MEMORY_SCOPE_AGENT);
      VMWAIT0();
      B = RFL(t) - tbase;
      if (B < tend || ph == 2u) break;
      if (ph == 0u && late) { ph = 1u; tbase = (c.call - 1u) * (T_MASK - T_HALF + nl) - T_HALF; tend = T_MASK; }
      else {
        STAMP(c, 6);
        ph = 2u; tbase = (c.call - 1u) * (T_END - T_MASK + na) - T_MASK; tend = T_END; fb = O_OF + src * 8192; miss = true;
        if ((GATEM >> ((c.wg >> 3) & 3)) & 1) lds_wait(c, C_CMAX, 8u);
      }
    }
    if (B >= T_END) {
      if (lane < 3) lds_st(c.ctl + C_QI + ((k + lane) & 7u), (k + lane + 1u) << 23 | 0x7FFFFFu);
      if (kv >= 5u) lds_wait(c, C_DFREE0 + ((kv - 5u) & 3u), ((kv - 5u) & ~3u) + 4u);
      if (lane < 3) lds_st(c.ctl + C_QV + ((kv + lane) & 7u), (kv + lane + 1u) << 23 | 0x7FFFFFu);
      break;
    }
    u32 item, fo;
    if (B < T_MASK) { item = pull_order(B, link); fo = item * 4; }
    else if (B < T_VAL) { item = T_MASK; fo = F_MF + (B - T_MASK) * 4; }
    else {
      const u32 ht = early_first((B - T_VAL) >> 1) * 2u + (B & 1u);
      item = T_VAL + ht; fo = ((lane & 1) ? F_MI : F_VF) - F_MI + ht * 4;
      if (kv == 0u) fb += F_MI;
    }
    u32 v = B >= T_VAL && kv == 0u ? 0u : lds_ld((LAS u32*)(c.lds + L_RFC + fo)), spins = 0;
    miss = false;
    while (!ALL((int)((v & 0xFFF00000u) - (c.call << 20)) >= 0)) {
      miss = true;
      if (expired(c, spins)) break;
      if (spins > 1u) __builtin_amdgcn_s_sleep(1);
      v = g_ld_sys(c.ws + fb + fo);
    }
    if (item == T_MASK) item += (v >> 11) & 511u;
    if (lane == 0) {
      if (B >= T_VAL) lds_st(c.ctl + C_QV + (kv & 7u), (kv + 1u) << 23 | (k & 2047u) << 12 | (item - T_VAL));
      const u32 wide = B < T_W0 || (B >= min(T_HALF, T_W1) && B < T_MASK + T_WA) || B >= T_W2;
      lds_st(c.ctl + C_QI + (k & 7u), (k + 1u) << 23 | item << 12 | wide << 11 | (v & 0x7FFu));
    }
    if (B >= T_VAL) ++kv;
  }
}
DI void role_decoder(Ctx& c, u32 d) {
  const int lane = c.lane, src = (c.rank + 1 + wg_link(c.wg)) & 7;
  __builtin_amdgcn_s_setprio(DPRIO);
  c.call = lds_wait(c, C_CALL, 1u);
  const rsrc_t rsL = mkrs(c.ws, WS_USED);
  int pk = -1, pr = 0;
#define PUBLISH() do { if (pk >= 0) { VMWAIT0(); image_flag(c, pk, pr); pk = -1; } } while (0)
  for (u32 k = d;; k += NDEC) {
    if (pk >= 0 && RFL(lds_ld(c.ctl + C_PROG)) < (k + 1u) * 32768u - 28672u) PUBLISH();
    const u32 e = q_wait(c, C_QI, k), lb = (e >> 12) & 2047u, tot = min((e & 0x7FFu) * 16u, 8192u);
    if (lb >= T_MASK) PUBLISH();
    if (lb >= T_VAL) break;
    LAS unsigned char* sl = c.lds + L_IN + (k & 3u) * 8192u;
    if (lb >= T_MASK) {
      lds_wait(c, C_PROG, k * 32768u + tot);
      const u32x4 hd = *(LAS u32x4*)sl;
      u32x4 v[8], y[3];
#pragma unroll
      for (int t = 0; t < 3; ++t) y[t] = *(LAS u32x4*)(sl + min(t * 1024 + lane * 16, 2048));
#pragma unroll
      for (int t = 0; t < 8; ++t) v[t] = *(LAS u32x4*)(sl + min((t < 2 ? 2064u : hd[(t >> 1) - 1] & 0x1FF0u) + (t & 1) * 1024 + lane * 16, 8176u));
      CFENCE(); if (lane == 0) lds_add(c.ctl + C_IFREE0 + (k & 3u), 1u); CFENCE();
      const u32 inb = O_MIN + (src * 512 + lb - T_MASK) * 8192u + lane * 16;
      st128<AGT>(y[0], rsL, inb); st128<AGT>(y[1], rsL, inb + 1024u);
      if (lane == 0) st128<AGT>(y[2], rsL, inb + 2048u);
#pragma unroll
      for (int t = 0; t < 8; t += 2) {
        st128<AGT>(v[t], rsL, inb + 2064u + t * 640u);
        if (lane < 16 && (hd[t >> 1] & 0xFFFFu) - (t ? hd[(t >> 1) - 1] & 0xFFFFu : 2064u) > 1024u) st128<AGT>(v[t + 1], rsL, inb + 3088u + t * 640u);
      }
      VMWAIT0();
      if (lane == 0) g_st_agent(c.ws + O_OF + src * 8192 + F_MI + (lb - T_MASK) * 4, c.call << 20);
      continue;
    }
    const int j = lb / 96, r = lb - j * 96, kstep = j * 8 + src;
    const u32 img = O_IMG + (u32)(kstep * 96 + r) * 8192u + lane * 16;
    lds_wait(c, C_PROG, k * 32768u + tot);
    if (k < 2u) STAMP(c, 26 + k);
    if (tot == 8192u) {
      PUBLISH();
      u32x4 v[8];
#pragma unroll
      for (int t = 0; t < 8; ++t) v[t] = *(LAS u32x4*)(sl + t * 1024 + lane * 16);
      CFENCE(); if (lane == 0) lds_add(c.ctl + C_IFREE0 + (k & 3u), 1u); CFENCE();
#pragma unroll
      for (int t = 0; t < 8; ++t) st128<AGT>(v[t], rsL, img + t * 1024);
    } else {
      const u32x4 c1 = *(LAS u32x4*)(sl + 4096 + lane * 16);
      const u32 exc = *(LAS u32*)(sl + 5120 + (lane & 31) * 4);
      u32x4 sm[4];
#pragma unroll
      for (int t = 0; t < 4; ++t) sm[t] = *(LAS u32x4*)(sl + t * 1024 + lane * 16);
      const u32 hd = RFL(exc), cnt = (hd >> 8) & 31u, base4 = min(hd & 255u, 240u) * B1;
      u32 n[16], lo[16], hi[16];
      const u32 l1[4] = {c1.x, c1.y, c1.z, c1.w};
      dec_levels<16, 2048>(c, l1, base4, n, sl + 5120u + ((cnt + 4u) & ~3u) * 4u, sl + 6144u, 2048u);
      CFENCE(); if (lane == 0) lds_add(c.ctl + C_IFREE0 + (k & 3u), 1u); CFENCE();
      PUBLISH();
      for (u32 x = 1; x <= cnt; ++x) {
        const u32 rec = __builtin_amdgcn_readlane(exc, x);
        const u32 kk = (rec >> 16) & 15u, sh = ((rec >> 14) & 3u) * 8u;
        const u32 m = ((rec >> 8) & 63u) == (u32)lane ? 255u << sh : 0u, val = (rec & 255u) << sh;
#pragma unroll
        for (int i = 0; i < 16; ++i) n[i] = bfi(kk == (u32)i ? m : 0u, val, n[i]);
      }
#pragma unroll
      for (int i = 0; i < 16; ++i) { hi[i] = bfi(0x80u * B1, sm[i >> 2][i & 3], n[i] >> 1); lo[i] = bfi(0x80u * B1, n[i] << 7, sm[i >> 2][i & 3]); }
#pragma unroll
      for (int t = 0; t < 8; ++t) st128<AGT>(piece(lo, hi, t), rsL, img + t * 1024);
    }
    pk = kstep; pr = r;
  }
}
#define WIN0 WIN2
constexpr int WMAX = (WIN0 > WIN2 ? WIN0 : WIN2) - 1;
#define WINM 3
struct Stream {
  u32 pub = 0, pend[WMAX] = {};
  template <int W> DI void landed(const Ctx& c, u32 prog) {
    asm volatile("s_waitcnt vmcnt(%0)" :: "n"(W - 1) : "memory");
    pub = max(pub, pend[WMAX - (W - 1)]);
    if (c.lane == 0) lds_st(c.ctl + C_PROG, pub);
    for (int k = 0; k + 1 < WMAX; ++k) pend[k] = pend[k + 1];
    pend[WMAX - 1] = prog;
  }
  DI void flush(const Ctx& c) {
    VMWAIT0();
    pub = pend[WMAX - 1];
    if (c.lane == 0) lds_st(c.ctl + C_PROG, pub);
  }
};
DI void role_puller(Ctx& c, const Params& p) {
  __builtin_amdgcn_s_setprio(3);
  const u32 ldsb = (u32)(size_t)c.lds;
  const char* sws = p.ws[(c.rank + 1 + wg_link(c.wg)) & 7];
  Stream sm;
  u32 kv = 0;
  u32 cw = 0, kp = 0;
  for (u32 k = 0;; ++k) {
    u32 e = __builtin_amdgcn_readlane(cw, 0), fi = __builtin_amdgcn_readlane(cw, 1), fd = __builtin_amdgcn_readlane(cw, 2);
    if ((e >> 23) != ((k + 1u) & 511u)) {
      e = RFL(lds_ld(c.ctl + C_QI + (k & 7u)));
      if ((e >> 23) != ((k + 1u) & 511u)) {
        sm.flush(c);
        e = q_wait(c, C_QI, k);
      }
    }
    CFENCE();
    const u32 item = (e >> 12) & 2047u;
    if (item >= T_END) break;
    u32 so, tot = (e & 0x7FFu) * 16u, dst;
    if (item < T_VAL) {
      if (fi < k >> 2) { sm.flush(c); lds_wait(c, C_IFREE0 + (k & 3u), k >> 2); }
      dst = L_IN + (k & 3u) * 8192u;
      so = item < T_MASK ? ob_slot(item, c.rank) : O_OM + (item - T_MASK) * 8192u; tot = min(tot, 8192u);
    } else {
      if (fd < (kv & ~3u)) { sm.flush(c); lds_wait(c, C_DFREE0 + (kv & 3u), kv & ~3u); }
      dst = L_STG + (kv & 3u) * 16384u; so = O_OV + (item - T_VAL) * 16384u; tot = min(tot, 16384u);
      ++kv;
    }
    kp += item >= T_MASK;
    so = RFL(so); tot = RFL(max(tot, 16u)); dst = RFL(dst);
    const rsrc_t rs = mkrs(sws + so, tot);
    const bool wide = (e >> 11) & 1u;
    const u32 lead = PLEAD, qs = tot > lead ? (tot - lead) & ~1023u : 0u;
    constexpr u32 ch = 1024u;
    bool told = false;
    for (u32 q = 0; q < tot; q += ch) {
      dma1(rs, c.lane * 16, q, ldsb + dst + q);
      if (q >= qs && !told) { told = true; if (c.lane == 0) lds_st(c.ctl + C_PSTART, k + 1u); }
      if (q + ch >= tot) cw = lds_ld(c.ctl + (c.lane == 0 ? C_QI + ((k + 1u) & 7u) : c.lane == 1 ? C_IFREE0 + ((k + 1u) & 3u) : C_DFREE0 + (kv & 3u)));
      if (WIN0 != WIN2 && k < KFIX && !wg_late(c.wg)) sm.landed<WIN0>(c, k * 32768u + q + ch);
      else if (wide) sm.landed<WIN2>(c, k * 32768u + q + ch);
      else if (WINM != WIN1 && (item < T_MASK || (GATEM && kp < PMED))) sm.landed<WINM>(c, k * 32768u + q + ch);
      else sm.landed<WIN1>(c, k * 32768u + q + ch);
    }
  }
  sm.flush(c);
  STAMP(c, 8);
}

DI void role_sched(Ctx& c, const Params& p, int mi, int ni) {
  const int lane = c.lane;
  __builtin_amdgcn_s_setprio(3);
  u32 t = 0;
  if (lane == 0) t = __hip_atomic_fetch_add((GAS u32*)(c.ws + O_CALL + c.wg * 128), 1u, __ATOMIC_RELAXED, __HIP_MEMORY_SCOPE_AGENT) + 1u;
  c.call = RFL(t);
  if (lane == 0) lds_st(c.ctl + C_CALL, c.call);
  const rsrc_t rsL = mkrs(c.ws, WS_USED);
  u64 issued = 0; u32 n = 0, spins = 0;
  while (n < 64u) {
    if (2u * n >= RFL(lds_ld((LAS u32*)(c.lds + L_REL + 7 * 256))) + 2u * SCHED_Q) { if (!lds_expired(c, spins)) continue; }
    const u32x4 a = __builtin_amdgcn_raw_buffer_load_b128(rsL, lane * 16, (int)(O_FX + mi * 1024), SYS);
    const u32x2 b = __builtin_amdgcn_raw_buffer_load_b64(rsL, lane * 8, (int)(O_FW + ni * 512), SYS);
    const bool rdy = min(min(min(a.x, a.y), min(a.z, a.w)), min(b.x, b.y)) >= c.call;
    u64 m = __builtin_amdgcn_ballot_w64(rdy) & ~issued;
    if (m == 0) {
      if (!expired(c, spins)) { __builtin_amdgcn_s_sleep(1); CFENCE(); continue; }
      m = ~issued;
    }
    if ((m >> lane) & 1) lds_st(c.ctl + C_ORDER + n + __builtin_amdgcn_mbcnt_hi((u32)(m >> 32), __builtin_amdgcn_mbcnt_lo((u32)m, 0u)), lane + 1);
    issued |= m;
    n += (u32)__popcll(m);
    CFENCE();
  }
  STAMP(c, 9);
  if (c.wg == LEADER) {
    u32 mx;
    for (;;) {
      const u32x4 a = __builtin_amdgcn_raw_buffer_load_b128(rsL, lane * 16, (int)O_LT, SYS);
      const u32x4 b = __builtin_amdgcn_raw_buffer_load_b128(rsL, lane * 16, (int)(O_LT + 1024), SYS);
      mx = max(max(a.x, a.z), max(b.x, b.z));
      if (ALL(min(min(a.y, a.w), min(b.y, b.w)) >= c.call) || expired(c, spins)) break;
      CFENCE();
    }
    STAMP(c, 16);
    mx = wave_max(mx);
    if (lane < NR) __hip_atomic_store((GAS u64*)(*(char* LAS*)(c.lds + L_PEER + lane * 8) + O_GT + c.rank * 8), (u64)c.call << 32 | mx, __ATOMIC_RELAXED, __HIP_MEMORY_SCOPE_SYSTEM);
    if (lane == 0) lds_st(c.ctl + C_LEAD, 1u);
  }
  lds_wait(c, C_CMAX, 8u);
  const rsrc_t rsG = mkrs(c.ws + O_GT, 64u);
  u32 g;
  for (;;) {
    if (lane < 4) dma1(rsG, lane * 16, 0u, (u32)(size_t)c.lds + L_GT);
    asm volatile("s_waitcnt vmcnt(5)" ::: "memory");
    const u64 v = __hip_atomic_load((LAS u64*)(c.lds + L_GT) + (lane & 7), __ATOMIC_RELAXED, __HIP_MEMORY_SCOPE_WORKGROUP);
    g = (u32)v;
    if (ALL((u32)(v >> 32) >= c.call) || expired(c, spins)) break;
    __builtin_amdgcn_s_sleep(2);
  }
  g = wave_max(g);
  if (lane == 0) { lds_st(c.ctl + C_GMAX, g); CFENCE(); lds_st(c.ctl + C_GVALID, 1u); }
  STAMP(c, 10);
}

DI void role_loader(Ctx& c, u32 li, int mi, int ni) {
  __builtin_amdgcn_s_setprio(2);
  const rsrc_t rsL = mkrs(c.ws, WS_USED);
  const u32 voff = c.lane * 16, ldsb = (u32)(size_t)c.lds + L_RING;
  u32 pend = 0;
  lds_wait(c, C_PACKED, 8u);
  for (u32 m = li; m < 128u; m += 5u - NDEC) {
    const u32 slot = m % NS, kh = m & 1u;
    u32 o, spins = 0;
    for (;;) {
      const u32 vo = lds_ld(c.ctl + C_ORDER + (m >> 1)), vf = lds_ld((LAS u32*)(c.lds + L_REL + (c.lane & 7) * 256));
      o = RFL(vo);
      if (o != 0u && ALL(vf + (NS - 1u) >= m)) break;
      if (pend) { VMWAIT0(); if (c.lane == 0) lds_st(c.ctl + C_FULL0 + (pend - 1u) % NS, pend); pend = 0; continue; }
      if (lds_expired(c, spins)) break;
    }
    CFENCE();
    o = RFL(o);
    const u32 so = O_IMG + ((o - 1u) & 63u) * (96u * 8192u), sx = so + mi * 32768 + kh * 4096u, sw = so + 64 * 8192 + ni * 16384 + kh * 8192u, ld = ldsb + slot * SLOTB;
    dma4(rsL, voff, sw, ld + 16384u);
    if (pend) { asm volatile("s_waitcnt vmcnt(4)" ::: "memory"); if (c.lane == 0) lds_st(c.ctl + C_FULL0 + (pend - 1u) % NS, pend); }
    dma4(rsL, voff, sw + 4096u, ld + 20480u);
    dma4(rsL, voff, sx, ld); dma4(rsL, voff, sx + 8192u, ld + 4096u); dma4(rsL, voff, sx + 16384u, ld + 8192u); dma4(rsL, voff, sx + 24576u, ld + 12288u);
    pend = m + 1u;
  }
  VMWAIT0();
  if (c.lane == 0) lds_st(c.ctl + C_FULL0 + (pend - 1u) % NS, pend);
}

struct Geo { u32 nc, sz; };
DI Geo geo(u32 V) { return V <= 2048u ? Geo{row_bytes(V) >> 2, 8u * row_bytes(V)} : Geo{64u, (min(V, 2304u) + 3u) & ~3u}; }
struct Scale { u32 r; int kap; };
DI Scale split(float inv) { const u32 ib = __float_as_uint(inv), r = ((ib & 0x7FFFFFu) + (1u << 19)) >> 20; return Scale{r & 7u, (int)(ib >> 23) - 247 + (int)(r >> 3)}; }
DI u32 low_exp(u32 o, u32 kap32, u32& pos) {
  const u32 t = o + kap32, ge = (t >> 5) & B1;
  pos = ((t + 31u * B1) >> 6) & B1;
  return t & ((ge << 5) - ge);
}
DI u32 sure(u32 m, u32 r8, u32 pos, u32& up) {
  const u32 x = m + r8;
  up = (~x >> 3) & B1;
  return (((x ^ 8u * B1) + 15u * B1) >> 4) & pos;
}
DI u32 place(const Ctx& c, int wv, int cnt, u32 first, u32 sb, u32 extra, u32x4& hdr) {
  if (c.lane == 0) { lds_st(c.ctl + C_VSZ + wv, sb | extra); CFENCE(); lds_add(c.ctl + cnt, 1u); }
  lds_wait(c, cnt, 4u);
  const u32x4 z = *(LAS u32x4*)(c.ctl + C_VSZ + (wv & 4));
  hdr.x = z.x + first; hdr.y = z.y + (hdr.x & 0xFFFFu); hdr.z = z.z + (hdr.y & 0xFFFFu); hdr.w = z.w + (hdr.z & 0xFFFFu);
  return (((wv & 2) ? ((wv & 1) ? hdr.w : hdr.z) : ((wv & 1) ? hdr.y : hdr.x)) & 0xFFFFu) - sb;
}

DI void role_mfma(Ctx& c, const Params& p, int wv, int mi, int ni) {
  const int lane = c.lane, mw = wv >> 1, h = wv & 1;
  pack_inputs(c, p, wv);
  f32x4 acc[4][4];
#pragma unroll
  for (int g = 0; g < 4; ++g)
#pragma unroll
    for (int b = 0; b < 4; ++b) acc[g][b] = (f32x4){0.f, 0.f, 0.f, 0.f};
  if (wv == 0) STAMP(c, 15);
  bf16x8 x0, x1, x2, x3, wa0, wa1, wa2, wa3, wb0, wb1, wb2, wb3;
  u32 vx = (u32)(size_t)c.lds + L_RING + mw * 4096 + lane * 16, vw = (u32)(size_t)c.lds + L_RING + 16384 + h * 4096 + lane * 16;
  u32 vf = (u32)(size_t)(c.ctl + C_FULL0 + 2), done = 1, sl = 1, seen = 0, vs;
  const u32 relad = (u32)(size_t)c.lds + L_REL + wv * 256 + lane * 4;
#define MF(a, w, x) "v_mfma_f32_16x16x32_bf16 " a ", " w ", " x ", " a "\n\t"
#define HSTEP(c0, c1, c2, c3, n0, n1, n2, n3, m) do { \
    if (__builtin_expect(seen <= (m) + 1u, 0)) lds_wait(c, C_FULL0 + sl, (m) + 2u); \
    asm volatile(MF("%0", "%9", "%8") "ds_read_b128 %4, %13\n\t" MF("%1", "%10", "%8") "ds_read_b128 %5, %13 offset:1024\n\t" \
                 MF("%2", "%11", "%8") "ds_read_b128 %6, %13 offset:2048\n\t" MF("%3", "%12", "%8") "ds_read_b128 %7, %13 offset:3072" \
                 : "+v"(acc[0][0]), "+v"(acc[1][0]), "+v"(acc[2][0]), "+v"(acc[3][0]), "+v"(n0), "+v"(n1), "+v"(n2), "+v"(n3) \
                 : "v"(x0), "v"(c0), "v"(c1), "v"(c2), "v"(c3), "v"(vw) : "memory"); \
    const u32 d = sl == NS - 1u ? -(NS - 1u) * SLOTB : SLOTB, d4 = sl == NS - 2u ? -(NS - 1u) * 4u : 4u; \
    sl = sl == NS - 1u ? 0u : sl + 1u; \
    asm volatile("s_waitcnt lgkmcnt(7)\n\t" MF("%0", "%7", "%6") "ds_read_b128 %4, %11\n\t" MF("%1", "%8", "%6") "ds_read_b32 %5, %12\n\t" \
                 MF("%2", "%9", "%6") MF("%3", "%10", "%6") \
                 : "+v"(acc[0][1]), "+v"(acc[1][1]), "+v"(acc[2][1]), "+v"(acc[3][1]), "+v"(x0), "=&v"(vs) \
                 : "v"(x1), "v"(c0), "v"(c1), "v"(c2), "v"(c3), "v"(vx), "v"(vf) : "memory"); \
    asm volatile("s_waitcnt lgkmcnt(8)\n\t" MF("%0", "%8", "%7") "ds_read_b128 %4, %12 offset:1024\n\t" MF("%1", "%9", "%7") "v_add_u32 %5, %13, %5\n\t" \
                 MF("%2", "%10", "%7") "v_add_u32 %6, %14, %6\n\t" MF("%3", "%11", "%7") \
                 : "+v"(acc[0][2]), "+v"(acc[1][2]), "+v"(acc[2][2]), "+v"(acc[3][2]), "+v"(x1), "+v"(vw), "+v"(vf) \
                 : "v"(x2), "v"(c0), "v"(c1), "v"(c2), "v"(c3), "v"(vx), "s"(d), "s"(d4) : "memory"); \
    asm volatile("s_waitcnt lgkmcnt(8)\n\t" MF("%0", "%9", "%5") "ds_read_b128 %4, %6 offset:2048\n\tv_add_u32 %7, 1, %7\n\t" MF("%1", "%10", "%5") \
                 "s_waitcnt lgkmcnt(2)\n\tv_readfirstlane_b32 %8, %13\n\t" MF("%2", "%11", "%5") MF("%3", "%12", "%5") \
                 "ds_read_b128 %5, %6 offset:3072\n\tds_write_b32 %14, %7\n\tv_add_u32 %6, %15, %6" \
                 : "+v"(acc[0][3]), "+v"(acc[1][3]), "+v"(acc[2][3]), "+v"(acc[3][3]), "+v"(x2), "+v"(x3), "+v"(vx), "+v"(done), "=&s"(seen) \
                 : "v"(c0), "v"(c1), "v"(c2), "v"(c3), "v"(vs), "v"(relad), "s"(d) : "memory"); \
  } while (0)
  lds_wait(c, C_FULL0, 1u);
  asm volatile("ds_read_b128 %0, %9\n\tds_read_b128 %1, %9 offset:1024\n\tds_read_b128 %2, %9 offset:2048\n\tds_read_b128 %3, %9 offset:3072\n\t"
               "ds_read_b128 %4, %8\n\tds_read_b128 %5, %8 offset:1024\n\tds_read_b128 %6, %8 offset:2048\n\tds_read_b128 %7, %8 offset:3072\n\t"
               "ds_write_b32 %10, %11\n\ts_waitcnt lgkmcnt(0)"
               : "=&v"(wa0), "=&v"(wa1), "=&v"(wa2), "=&v"(wa3), "=&v"(x0), "=&v"(x1), "=&v"(x2), "=&v"(x3) : "v"(vx), "v"(vw), "v"(relad), "v"(done) : "memory");
  vx += SLOTB; vw += SLOTB;
  wb0 = wa0; wb1 = wa0; wb2 = wa0; wb3 = wa0;
  asm volatile("s_nop 1" : "+v"(vx), "+v"(vw), "+v"(wb0), "+v"(wb1), "+v"(wb2), "+v"(wb3));
  for (u32 m = 0; m < 126u; m += 2u) {
    HSTEP(wa0, wa1, wa2, wa3, wb0, wb1, wb2, wb3, m);
    HSTEP(wb0, wb1, wb2, wb3, wa0, wa1, wa2, wa3, m + 1u);
  }
  HSTEP(wa0, wa1, wa2, wa3, wb0, wb1, wb2, wb3, 126u);
#define MF4(b, x) asm volatile(MF("%0", "%5", "%4") MF("%1", "%6", "%4") MF("%2", "%7", "%4") MF("%3", "%8", "%4") \
    : "+v"(acc[0][b]), "+v"(acc[1][b]), "+v"(acc[2][b]), "+v"(acc[3][b]) : "v"(x), "v"(wb0), "v"(wb1), "v"(wb2), "v"(wb3))
  asm volatile("s_waitcnt lgkmcnt(0)" ::: "memory");
  MF4(0, x0); MF4(1, x1); MF4(2, x2); MF4(3, x3);
  asm volatile("s_nop 7\n\ts_nop 7" : "+v"(acc[0][2]), "+v"(acc[1][2]), "+v"(acc[2][2]), "+v"(acc[3][2]), "+v"(acc[0][3]), "+v"(acc[1][3]), "+v"(acc[2][3]), "+v"(acc[3][3]),
               "+v"(acc[0][0]), "+v"(acc[1][0]), "+v"(acc[2][0]), "+v"(acc[3][0]), "+v"(acc[0][1]), "+v"(acc[1][1]), "+v"(acc[2][1]), "+v"(acc[3][1]));
  if (wv == 0) STAMP(c, 11);
  const rsrc_t rsL = mkrs(c.ws, WS_USED);
  {
    float mx = 0.f;
#pragma unroll
    for (int e = 0; e < 64; e += 2) mx = fmaxf(fmaxf(mx, acc[e & 3][e >> 4][(e >> 2) & 3]), acc[(e & 3) + 1][e >> 4][(e >> 2) & 3]);
    const u32 wmx = wave_max(__float_as_uint(mx));
    if (lane == 0) {
      __hip_atomic_fetch_max(c.ctl + C_WGMAX, wmx, __ATOMIC_RELAXED, __HIP_MEMORY_SCOPE_WORKGROUP);
      CFENCE();
      if (lds_add(c.ctl + C_CMAX, 1u) == 7u)
        __hip_atomic_store((GAS u64*)(c.ws + O_LT + c.wg * 8), (u64)c.call << 32 | lds_ld(c.ctl + C_WGMAX), __ATOMIC_RELAXED, __HIP_MEMORY_SCOPE_AGENT);
    }
  }
  const u32 zs = wv >> 2, ht = c.wg * 2 + zs;
  u32 nz[2] = {0u, 0u};
#pragma unroll
  for (int e = 63; e >= 0; --e) nz[e >> 5] = __builtin_amdgcn_alignbit(nz[e >> 5], __float_as_uint(acc[e & 3][e >> 4][(e >> 2) & 3]), 31);
  nz[0] = ~nz[0]; nz[1] = ~nz[1];
  __builtin_amdgcn_raw_buffer_store_b64((u32x2){nz[0], nz[1]}, rsL, lane * 8, (int)(O_OM + ht * 8192 + 16 + (wv & 3) * 512), SYS);
  u32 mqv = 0;
  if ((wv & 3) == 0) mqv = __builtin_amdgcn_raw_ptr_buffer_atomic_add_i32(1, mkrs(c.ws + O_MQ, 4u), lane * 4, 0, AGT);
  const u32 cnt = __popc(nz[0]) + __popc(nz[1]), inc = wave_incl_scan(cnt), V = __builtin_amdgcn_readlane(inc, 63);
  u32 o0 = ldsa(c.lds) + L_RING + wv * 4096 + inc - cnt;
  Geo G = geo(V);
  const bool in8 = (u32)lane < G.nc, in9 = 8u * G.nc + lane < G.sz >> 2;
  lds_wait(c, C_CMAX, 8u);
  if (wv == 0) STAMP(c, 56);
  LAS unsigned char* st = c.lds + L_RING + wv * 4096;
  u32 ob[9], base;
  bool hint;
  {
#pragma unroll
    for (int k = 0; k < 4; ++k) *(LAS u32x4*)(st + k * 1024 + lane * 16) = (u32x4){0u, 0u, 0u, 0u};
    u32 o = o0, se = 0, carry = 0;
#pragma unroll
    for (int z = 0; z < 16; z += 8) {
      u32 sel[8];
#pragma unroll
      for (int D = 0; D < 8; ++D) sel[D] = put_sel(c, nib4(nz[z >> 3], D));
#pragma unroll
      for (int D = 0; D < 8; ++D) {
        const int b = (z + D) >> 2, r = D & 3;
        const u32 h = prm(prm(__float_as_uint(acc[3][b][r]) << 1, __float_as_uint(acc[2][b][r]) << 1, 0x07030000u),
                          prm(__float_as_uint(acc[1][b][r]) << 1, __float_as_uint(acc[0][b][r]) << 1, 0x00000703u), 0x07060100u);
        se = __builtin_amdgcn_sad_u8(put4(o, carry, h, nib4(nz[z >> 3], D), sel[D]), 0u, se);
      }
    }
    or_at(o, carry);
    base = min(max(pick_top(__builtin_amdgcn_readlane(wave_incl_scan(se), 63), V), 13u) - 13u, 240u);
    if (V + lane < G.sz) st[V + lane] = (unsigned char)(base + 13u);
    CFENCE();
    if (wv == 0) STAMP(c, 57);
    u32 n[9], any = 0, l2a, l2b, l3, sb = 0;
    const u16x2 b2 = {(unsigned short)base, (unsigned short)base};
#pragma unroll
    for (u32 k = 0; k < 9u; ++k) ob[k] = *(LAS u32*)(st + (k * G.nc + lane) * 4u);
#pragma unroll
    for (u32 k = 0; k < 9u; ++k) {
      const u32 lo = __builtin_bit_cast(u32, __builtin_elementwise_sub_sat(__builtin_bit_cast(u16x2, ob[k] & 0x00FF00FFu), b2));
      const u32 hi = __builtin_bit_cast(u32, __builtin_elementwise_sub_sat(__builtin_bit_cast(u16x2, (ob[k] >> 8) & 0x00FF00FFu), b2));
      ob[k] = (k < 8u ? in8 : in9) ? lo | hi << 8 : 13u * B1;
      any |= ob[k];
      n[k] = (ob[k] + 2u * B1) & 15u * B1;
    }
    Lv g;
    hint = V <= 2304u && !ANY((any & 0xF0u * B1) != 0u) && enc_levels<9, 1024>(c, n, st + G.sz, 4096u - G.sz, g, l2a, l2b, l3);
    CFENCE();
    if (wv == 0) STAMP(c, 58);
    if (hint) {
      LAS u32* s = (LAS u32*)st + lane;
      if (in8) { s[0] = n[0] | n[1] << 2 | n[2] << 4 | n[3] << 6; s[G.nc] = n[4] | n[5] << 2 | n[6] << 4 | n[7] << 6; }
      if (in9) s[2u * G.nc] = n[8];
      s += (G.sz >> 2) - 6u * G.nc;
      if ((u32)lane < g.nc2) s[0] = l2a;
      if ((u32)lane < g.nc3) s[g.nc2] = l3;
      sb = (G.sz - 24u * G.nc + (g.nc2 + g.nc3) * 4u + 15u) & ~15u;
    }
    if ((wv & 3) == 0 && lane == 0) lds_st(c.ctl + C_MQ0 + zs, mqv);
    u32x4 hdr;
    const u32 omb = O_OM + ht * 8192, woff = place(c, wv, C_CSZ0 + zs, 2064u, sb, base << 16, hdr);
    if (wv == 0) STAMP(c, 59);
    if ((wv & 3) == 0 && lane == 0) st128<SYS>(hdr, rsL, omb);
    for (u32 k = lane * 16; k < sb; k += 1024u) st128<SYS>(*(LAS u32x4*)(st + k), rsL, k + (omb + woff));
    VMWAIT0();
    u32 t = 0;
    if (lane == 0) t = lds_add(c.ctl + C_CPRE0 + zs, 1u);
    if (RFL(t) == 3u) {
      const u32 mq = RFL(lds_ld(c.ctl + C_MQ0 + zs));
      tell_peers(c, O_OF + c.rank * 8192 + F_MF + (mq & 511u) * 4, c.call << 20 | ht << 11 | (hdr.w & 0xFFFFu) >> 4);
    }
  }
  if (wv == 0) STAMP(c, 28);
  asm volatile("" : "+v"(nz[0]), "+v"(nz[1]), "+v"(o0), "+s"(G.nc));
  lds_wait(c, C_CPRE0 + zs, 4u);
#pragma unroll
  for (int k = 0; k < 4; ++k) *(LAS u32x4*)(st + k * 1024 + lane * 16) = (u32x4){0u, 0u, 0u, 0u};
  u32 sel0[16];
#pragma unroll
  for (int D = 0; D < 16; ++D) sel0[D] = put_sel(c, nib4(nz[D >> 3], D & 7));
#pragma unroll
  for (int g = 0; g < 4; ++g)
#pragma unroll
    for (int b = 0; b < 4; ++b) {
#pragma unroll
      for (int r = 0; r < 4; ++r) acc[g][b][r] = __builtin_amdgcn_fmed3f(acc[g][b][r], 0.f, __builtin_inff());
      asm volatile("" : "+v"(acc[g][b]));
    }
  lds_wait(c, C_GVALID, 1u);
  if (wv == 0) STAMP(c, 31);
  const u32 tm = g_ld_agent(c.ws + O_TMO);
  const float amax = __uint_as_float(RFL(lds_ld(c.ctl + C_GMAX)));
  const float inv = amax > 0.f ? 448.0f / amax : 0.f;
  const Scale sc = split(__uint_as_float(RFL(__float_as_uint(inv))));
  const u32 r8 = (8u - sc.r) * B1;
  u32 q[16];
#pragma unroll
  for (int b = 0; b < 4; ++b) {
    const f32x4 a0 = acc[0][b] * inv, a1 = acc[1][b] * inv, a2 = acc[2][b] * inv, a3 = acc[3][b] * inv;
#pragma unroll
    for (int r = 0; r < 4; ++r)
      q[4 * b + r] = (u32)__builtin_amdgcn_cvt_pk_fp8_f32(a2[r], a3[r], __builtin_amdgcn_cvt_pk_fp8_f32(a0[r], a1[r], 0, false), true);
  }
  {
    u32 o = o0, carry = 0, sb;
#pragma unroll
    for (int D = 0; D < 16; ++D) put4(o, carry, q[D], nib4(nz[D >> 3], D & 7), sel0[D]);
    or_at(o, carry);
    const int kap = min(max((int)base + sc.kap, -16), 16);
    if (V + lane < G.sz) st[V + lane] = (unsigned char)((((u32)(13 + kap) + (sc.r == 7u)) & 15u) << 3 | ((sc.r + 1u) & 7u));
    CFENCE();
    u32 d[9], nb[2] = {0u, 0u}, bad = 0, nca = 0;
    if (hint) {
      u32 sel[9];
#pragma unroll
      for (u32 k = 0; k < 9u; ++k) d[k] = *(LAS u32*)(st + (k * G.nc + lane) * 4u);
#pragma unroll
      for (u32 k = 0; k < 9u; ++k) {
        u32 pos, up;
        const u32 u = low_exp(ob[k], (u32)(kap + 32) * B1, pos), df = ((d[k] >> 3) & 15u * B1) + 32u * B1 - u;
        d[k] &= 7u * B1;
        const u32 su = sure(d[k], r8, pos, up), w = ((df ^ 32u * B1) & ~B1) | ((df ^ up) & su);
        bad |= (k < 8u ? in8 : in9) ? w : 0u;
        nb[k >> 3] |= to_nib(su) << (4u * (k & 7u));
        ob[k] = df & B1;
      }
      nb[0] = in8 ? ~nb[0] : 0u; nb[1] = in9 ? ~nb[1] & 15u : 0u;
#pragma unroll
      for (u32 k = 0; k < 9u; ++k) sel[k] = put_sel(c, nib4(nb[k >> 3], k & 7u));
      const u32 cn = __popc(nb[0]) + __popc(nb[1]), in2 = wave_incl_scan(cn), Va = __builtin_amdgcn_readlane(in2, 63);
      nca = (Va + 31u) >> 5;
      hint = !ANY(bad != 0u) && G.sz + nca * 32u + 8u <= 4096u;
      if (hint) {
        o = ldsa(st) + G.sz + in2 - cn; carry = 0;
#pragma unroll
        for (u32 k = 0; k < 9u; ++k) put4(o, carry, ob[k], nib4(nb[k >> 3], k & 7u), sel[k]);
        or_at(o, carry);
      }
    }
    CFENCE();
    if (hint) {
      u32 bits = 0;
#pragma unroll
      for (u32 k = 0; k < 8u; ++k) bits |= *(LAS u32*)(st + G.sz + (k * nca + lane) * 4u) << k;
      CFENCE();
      LAS u32* s = (LAS u32*)st + lane;
      if (in8) {
        s[0] = d[0] | d[1] << 3 | ((d[6] << 6) & 0xC0u * B1); s[G.nc] = d[2] | d[3] << 3 | ((d[7] << 6) & 0xC0u * B1);
        s[2u * G.nc] = d[4] | d[5] << 3 | ((d[6] << 4) & 0x40u * B1) | ((d[7] << 5) & 0x80u * B1);
      }
      if (in9) s[3u * G.nc] = d[8];
      if ((u32)lane < nca) s[(G.sz >> 2) - 5u * G.nc] = bits;
      sb = (G.sz - 20u * G.nc + nca * 4u + 15u) & ~15u;
    } else {
      const u32 nc = row_bytes(V) >> 2;
      for (u32 col = lane; col < nc; col += 64u) {
        LAS u32* s = (LAS u32*)st + col;
        const u32 d7 = s[7u * nc];
#pragma unroll
        for (u32 k = 0; k < 7u; ++k) s[k * nc] |= (d7 << (7u - k)) & 0x80u * B1;
      }
      sb = (28u * nc + 15u) & ~15u;
    }
    u32x4 hdr;
    const u32 ovb = O_OV + ht * 16384, woff = place(c, wv, C_CSZ2 + zs, 16u, sb, (u32)hint << 16, hdr);
    if ((wv & 3) == 0 && lane == 0) st128<SYS>(hdr, rsL, ovb);
    for (u32 k = lane * 16; k < sb; k += 1024u) st128<SYS>(*(LAS u32x4*)(st + k), rsL, k + (ovb + woff));
    VMWAIT0();
    u32 t = 0;
    if (lane == 0) t = lds_add(c.ctl + C_CVAL0 + zs, 1u);
    if (RFL(t) == 3u) tell_peers(c, O_OF + c.rank * 8192 + F_VF + ht * 4, c.call << 20 | (hdr.w & 0xFFFFu) >> 4);
  }
  if (wv == 0) STAMP(c, 12);
  const rsrc_t rsO = mkrs(p.out, (u32)MM * NN);
  const u32x4 nan4 = {0x7Fu * B1, 0x7Fu * B1, 0x7Fu * B1, 0x7Fu * B1};
  {
    const u32 obase = (u32)(mi * 256 + mw * 64 + (lane & 15)) * NN + c.rank * 2048 + ni * 128 + h * 64 + (lane >> 4) * 16;
    const bool bad = tm == c.call || lds_ld(c.ctl + C_ABORT) != 0u;
#pragma unroll
    for (int b = 0; b < 4; ++b) st128<OUTAUX>(bad ? nan4 : (u32x4){q[4 * b], q[4 * b + 1], q[4 * b + 2], q[4 * b + 3]}, rsO, obase + b * 16 * NN);
  }
}

DI void role_outdec(Ctx& c, const Params& p, int wv) {
  const int lane = c.lane;
  __builtin_amdgcn_s_setprio(0);
  c.call = lds_wait(c, C_CALL, 1u);
  lds_wait(c, C_GVALID, 1u);
  const float amax = __uint_as_float(RFL(lds_ld(c.ctl + C_GMAX)));
  const Scale sc = split(__uint_as_float(RFL(__float_as_uint(amax > 0.f ? 448.0f / amax : 0.f))));
  const u32 r8 = (8u - sc.r) * B1;
  const rsrc_t rsL = mkrs(c.ws, WS_USED), rsO = mkrs(p.out, (u32)MM * NN);
  const u32x4 nan4 = {0x7Fu * B1, 0x7Fu * B1, 0x7Fu * B1, 0x7Fu * B1};
  LAS unsigned char* st = c.lds + L_RING + (wv < 8 ? wv * 4096 : 98304 + (wv - 8) * 4096);
  const u32 src = (c.rank + 1 + wg_link(c.wg)) & 7;
  for (u32 kv = wv >> 2;; kv += 3u) {
    const u32 e = q_wait(c, C_QV, kv), dh = e & 0xFFFu, pb = ((e >> 12) & 2047u) * 32768u;
    if (dh >= 512u) break;
    const u32 dt = dh >> 1, dw = (dh & 1u) * 4u + (wv & 3), inb = O_MIN + (src * 512 + dh) * 8192u;
    dma1(rsL, lane * 16, inb + 2064u + (wv & 3) * 1280u, (u32)(size_t)st + 2560u);
    if (lane < 16) dma1(rsL, lane * 16, inb + 3088u + (wv & 3) * 1280u, (u32)(size_t)st + 3584u);
    const u32x2 m = __builtin_amdgcn_raw_buffer_load_b64(rsL, lane * 8, (int)(inb + 16 + (wv & 3) * 512), AGT);
    const u32 he = __builtin_amdgcn_raw_buffer_load_b32(rsL, 0, (int)(inb + (wv & 3) * 4), AGT);
    const u32 dbase = (u32)(tile_mi(dt) * 256 + (dw >> 1) * 64 + (lane & 15)) * NN + src * 2048 + tile_ni(dt) * 128 + (dw & 1) * 64 + (lane >> 4) * 16;
    const u32 cn = __popc(m.x) + __popc(m.y), in2 = wave_incl_scan(cn), V2 = __builtin_amdgcn_readlane(in2, 63);
    const Geo G2 = geo(V2);
    const bool j8 = (u32)lane < G2.nc, j9 = 8u * G2.nc + lane < G2.sz >> 2;
    u32 u[9], pos[9];
    VMWAIT0();
    {
      LAS u32* s = (LAS u32*)(st + 2560) + lane;
      const u32 ra = s[0], rb = s[G2.nc], rc = s[2u * G2.nc], l1[3] = {j8 ? ra : ~0u, j8 ? rb : ~0u, j9 ? rc : ~0u};
      const u32 k32 = (u32)(min(max((int)((he >> 16) & 255u) + sc.kap, -16), 16) + 32) * B1;
      dec_levels<9, 1024>(c, l1, k32, u, st + 2560 + G2.sz - 24u * G2.nc, st, 2560u);
#pragma unroll
      for (u32 k = 0; k < 9u; ++k) u[k] = low_exp(u[k], 0u, pos[k]);
    }
    LAS unsigned char* sl = c.lds + L_STG + (kv & 3u) * 16384u;
    lds_wait(c, C_PROG, pb + 16u);
    const u32 hw = RFL(*(LAS u32*)(sl + (wv & 3) * 4)), hp = RFL(*(LAS u32*)(sl + ((wv + 3) & 3) * 4));
    const u32 wo = (wv & 3) ? min(hp & 0xFFF0u, 16384u - 3584u) : 16u;
    lds_wait(c, C_PROG, pb + (hw & 0xFFFFu));
    const u32 tm2 = g_ld_agent(c.ws + O_TMO);
    if ((hw >> 16) & 1u) {
      LAS u32* sg = (LAS u32*)(sl + wo) + lane;
      const u32 ra = sg[0], rb = sg[G2.nc], rc = sg[2u * G2.nc], bits = sg[(G2.sz >> 2) - 5u * G2.nc];
      u32 y[9], nb[2] = {0u, 0u};
      y[8] = sg[3u * G2.nc] & 7u * B1;
      y[0] = ra & 7u * B1; y[1] = (ra >> 3) & 7u * B1; y[2] = rb & 7u * B1; y[3] = (rb >> 3) & 7u * B1; y[4] = rc & 7u * B1; y[5] = (rc >> 3) & 7u * B1;
      y[6] = ((ra >> 6) & 3u * B1) | ((rc >> 4) & 4u * B1); y[7] = ((rb >> 6) & 3u * B1) | ((rc >> 5) & 4u * B1);
#pragma unroll
      for (u32 k = 0; k < 9u; ++k) {
        u32 up;
        const u32 su = sure(y[k], r8, pos[k], up);
        u[k] += up & su;
        nb[k >> 3] |= to_nib(su) << (4u * (k & 7u));
      }
      nb[0] = j8 ? ~nb[0] : 0u; nb[1] = j9 ? ~nb[1] & 15u : 0u;
      const u32 ca = __popc(nb[0]) + __popc(nb[1]), ia = wave_incl_scan(ca), nca = min((__builtin_amdgcn_readlane(ia, 63) + 31u) >> 5, 64u);
      if ((u32)lane < nca) {
#pragma unroll
        for (u32 k = 0; k < 8u; ++k) *(LAS u32*)(st + (k * nca + lane) * 4u) = (bits >> k) & B1;
      }
      CFENCE();
      u32 bb[9];
      get_all<9>(c, ldsa(st) + ia - ca, nb, bb);
#pragma unroll
      for (u32 k = 0; k < 9u; ++k) y[k] |= (u[k] + bb[k]) << 3;
      CFENCE();
      if (j8) {
#pragma unroll
        for (u32 k = 0; k < 8u; ++k) *(LAS u32*)(st + (k * G2.nc + lane) * 4u) = y[k];
      }
      if (j9) *(LAS u32*)(st + (8u * G2.nc + lane) * 4u) = y[8];
    } else {
      const u32 nc = row_bytes(V2) >> 2;
      for (u32 col = lane; col < nc; col += 64u) {
        LAS u32* sg = (LAS u32*)(sl + wo) + col;
        LAS u32* d = (LAS u32*)st + col;
        u32 d7 = 0;
#pragma unroll
        for (u32 k = 0; k < 7u; ++k) { const u32 v = sg[k * nc]; d7 |= (v & 0x80u * B1) >> (7u - k); d[k * nc] = v & 0x7Fu * B1; }
        d[7u * nc] = d7;
      }
    }
    CFENCE();
    if (lane == 0) lds_add(c.ctl + C_DFREE0 + (kv & 3u), 1u);
    CFENCE();
    u32 ov[16];
    const u32 mm[2] = {m.x, m.y};
    get_all<16>(c, ldsa(st) + in2 - cn, mm, ov);
    const bool bad = tm2 == c.call || lds_ld(c.ctl + C_ABORT) != 0u;
#pragma unroll
    for (int k = 0; k < 4; ++k) st128<OUTAUX>(bad ? nan4 : (u32x4){ov[4 * k], ov[4 * k + 1], ov[4 * k + 2], ov[4 * k + 3]}, rsO, dbase + k * 16 * NN);
    CFENCE();
  }
}

__global__ __launch_bounds__(1024, 4) void fused_gemm_ar(Params p) {
  extern __shared__ __attribute__((aligned(16))) unsigned char smem[];
  Ctx c;
  c.lds = (LAS unsigned char*)smem; c.ctl = (LAS u32*)c.lds; c.ws = p.ws[p.rank]; c.t0 = now(); c.call = 0;
  c.lane = threadIdx.x & 63; c.wg = blockIdx.x; c.rank = p.rank;
  const int wv = RFL(threadIdx.x >> 6), tid = threadIdx.x;
  if (tid >= 512) *(LAS u32*)(c.lds + L_REL + (tid - 512) * 4) = 0u;
  else if (tid < C_WORDS || (tid >= L_GT / 4 && tid < L_GT / 4 + 16)) c.ctl[tid] = 0u;
  else if (tid >= 192 && tid < 256) {
    const u32 nib = tid & 15; u32 s = 0, k = (tid - 192) >> 4;
    for (int g = 0; g < 4; ++g) { if ((nib >> g) & 1u) s |= k++ << (8 * g); else s |= 0x0Cu << (8 * g); }
    *(LAS u32*)(c.lds + L_LUTE + (tid - 192) * 4) = s;
  } else if (tid >= 288 && tid < 288 + NR) {
    u64 v = 0;
#pragma unroll
    for (int i = 0; i < NR; ++i) { u64 t = (u64)p.ws[i]; asm volatile("" : "+s"(t)); v = tid == 288 + i ? t : v; }
    *(LAS u64*)(c.lds + L_PEER + (tid - 288) * 8) = v;
  } else if (tid >= 256 && tid < 272) {
    const u32 nib = tid - 256; u32 s = 0x0C0C0C0Cu, k = 0;
    for (u32 g = 0; g < 4; ++g) if ((nib >> g) & 1u) { s = (s & ~(255u << (8 * k))) | g << (8 * k); ++k; }
    *(LAS u32*)(c.lds + L_LUTC + nib * 4) = s;
  }
  asm volatile("s_waitcnt lgkmcnt(0)" ::: "memory");
  __builtin_amdgcn_s_barrier();
  CFENCE();
  const int mi = tile_mi(c.wg), ni = tile_ni(c.wg);
  if (wv == 11) STAMP(c, 0);
  if (wv < 8) role_mfma(c, p, wv, mi, ni);
  else if (wv < 13 - NDEC) role_loader(c, wv - 8, mi, ni);
  else if (wv == 11) role_sched(c, p, mi, ni);
  else if (wv == 12) role_puller(c, p);
  else if (wv == 13) role_feeder(c);
  else role_decoder(c, wv == 10 ? 2 : wv - 14);
  if (wv < 12) role_outdec(c, p, wv);
}
}

extern "C" void kernel_launch(int rank, int nranks, ncclComm_t comm, hipStream_t stream,
                              void* const* d_in, const int* in_sizes, int n_in,
                              void* d_out, int out_size, const int* in_dtypes, int out_dtype,
                              void* const* d_out_all, void* const* d_ws_all, size_t ws_bytes) {
  if (nranks != NR || (unsigned)rank >= (unsigned)NR || n_in < 2 || ws_bytes < WS_USED || in_sizes[0] != MM * KS || in_sizes[1] != KS * NN || out_size != MM * NN) {
    fprintf(stderr, "kernel_launch: unexpected problem shape\n");
    return;
  }
  static bool init = false;
  if (!init) {
    hipError_t e = hipFuncSetAttribute((const void*)fused_gemm_ar, hipFuncAttributeMaxDynamicSharedMemorySize, (int)LDS_BYTES);
    if (e != hipSuccess) { fprintf(stderr, "hipFuncSetAttribute: %s\n", hipGetErrorString(e)); return; }
    init = true;
  }
  Params p;
  p.x = (const unsigned short*)d_in[0]; p.w = (const unsigned short*)d_in[1]; p.out = (unsigned char*)d_out;
  for (int i = 0; i < NR; ++i) p.ws[i] = (char*)d_ws_all[i];
  p.rank = rank; p.pad = 0;
  fused_gemm_ar<<<NWG, 1024, LDS_BYTES, stream>>>(p);
}
```

```cpp
#include <hip/hip_runtime.h>
#include <rccl/rccl.h>
#include <cstdio>

typedef unsigned u32;
typedef unsigned long long u64;
typedef u32 u32x4 __attribute__((ext_vector_type(4)));
typedef u32 u32x2 __attribute__((ext_vector_type(2)));
typedef unsigned short u16x2 __attribute__((ext_vector_type(2)));
typedef short bf16x8 __attribute__((ext_vector_type(8)));
typedef float f32x4 __attribute__((ext_vector_type(4)));
typedef __amdgpu_buffer_rsrc_t rsrc_t;
#define LAS __attribute__((address_space(3)))
#define GAS __attribute__((address_space(1)))
#define DI __device__ __forceinline__
#define RFL(x) __builtin_amdgcn_readfirstlane(x)
#define ALL(x) (__builtin_amdgcn_ballot_w64(!(x)) == 0ull)
#define ANY(x) (__builtin_amdgcn_ballot_w64(x) != 0ull)
#define CFENCE() asm volatile("" ::: "memory")
#define VMWAIT0() asm volatile("s_waitcnt vmcnt(0)" ::: "memory")

namespace {
constexpr int NR = 8, MM = 4096, KS = 512, NN = 16384, NWG = 256;
constexpr int NBLK = 768;
constexpr u32 SYS = 17, AGT = 16;
constexpr int LEADER = 16;
constexpr u64 DEADLINE = 50000000ull;
constexpr u32 O_CALL = 0;
constexpr u32 O_TMO = 32768;
constexpr u32 O_TICK = O_TMO + 128;
constexpr u32 O_MQ = O_TICK + 3072;
constexpr u32 O_GT = O_MQ + 128;
constexpr u32 O_LT = O_GT + 128;
constexpr u32 O_RF = O_LT + 2048;
constexpr u32 O_FX = O_RF + NR * NBLK * 4;
constexpr u32 O_FW = O_FX + 16 * 64 * 16;
constexpr u32 O_OF = O_FW + 16 * 64 * 8;
constexpr u32 F_MF = 0;
constexpr u32 F_MI = 4096;
constexpr u32 F_VF = 6144;
constexpr u32 O_DBG = 512 * 1024;
constexpr u32 O_OBX = 1u << 20;
constexpr u32 O_OBW = O_OBX + 512 * 8192;
constexpr u32 O_IMG = O_OBW + 2048 * 8192;
constexpr u32 O_OM = O_IMG + 64 * 96 * 8192;
constexpr u32 O_OV = O_OM + 256 * 16384;
constexpr u32 O_MIN = O_OV + 512 * 16384;
constexpr u32 WS_USED = O_MIN + NR * 256 * 16384;
__host__ __device__ constexpr u32 piece_off(u32 pc) { return pc < 512u ? (pc >> 1) * 16384u + 4096u + (pc & 1u) * 6144u : (pc - 512u) * 16384u; }
static_assert(O_OF + NR * 8192 <= O_DBG, "control block overflow");
constexpr u32 T_MASK = NBLK, T_VAL = T_MASK + 768, T_END = T_VAL + 512;
enum { C_CALL = 0, C_ABORT, C_WGMAX, C_CMAX, C_CMASK, C_CVAL0, C_CVAL1, C_GVALID, C_GMAX, C_PSTART, C_PROG, C_PACKED,
       C_FULL0 = 12, C_LEAD = 17, C_CSZ0 = 18, C_CSZ2 = 20, C_CPRE0 = 22, C_DFREE0 = 24, C_IFREE0 = 28, C_VSZ = 32, C_QI = 40, C_QV = 48, C_MQ0 = 56, C_ORDER = 64, C_WORDS = 128 };
constexpr u32 L_LUTC = C_WORDS * 4;
constexpr u32 L_GT = L_LUTC + 128;
constexpr u32 L_PEER = L_GT + 64;
constexpr u32 L_LUTE = L_LUTC + 256;
constexpr u32 L_REL = 1024;
constexpr u32 L_RFC = L_REL + 2048;
constexpr u32 NS = 5, SLOTB = 24576;
constexpr u32 L_RING = L_RFC + 4096;
constexpr u32 L_STG = L_RING + 32768;
constexpr u32 L_IN = L_RING + NS * SLOTB;
constexpr u32 LDS_BYTES = L_IN + 4 * 8192;
static_assert(LDS_BYTES <= 163840 && LDS_BYTES > 81920 && L_STG + 5 * 16384 <= L_IN, "one workgroup per CU");
#define PLEAD 2048
#define WIN1 2
#define WIN2 4
#define T_HALF 352
#define PACK_T0 200
#define PACK_DT 100
#define NRAW 32
#define T_W1 728
#define T_W2 T_END
#define T_WA 330u
#define G0FREE 1
#define T_AH (T_END - 80u)
#define GATEM 12
#define SCHED_Q 3u
#define OUTAUX 0u
#define NDEC 3
#define DPRIO 1
#define T_PAIR 700u
#define NPUSH 12
#define KFIX 8u
#define T_W0 160

#define SPIN_FAST 16u
#define SLEEP_LONG 1
struct Params { const unsigned short* x; const unsigned short* w; unsigned char* out; char* ws[NR]; int rank; int pad; };

DI u64 now() { return __builtin_amdgcn_s_memrealtime(); }
DI rsrc_t mkrs(const void* p, u32 bytes) { return __builtin_amdgcn_make_buffer_rsrc((void*)p, (short)0, (int)bytes, 0x00020000); }
template <u32 AUX> DI void st128(u32x4 v, rsrc_t rs, u32 off) { __builtin_amdgcn_raw_buffer_store_b128(v, rs, (int)off, 0, AUX); }
DI void st32(u32 v, rsrc_t rs, u32 off) { __builtin_amdgcn_raw_buffer_store_b32(v, rs, (int)off, 0, SYS); }
DI u32 prm(u32 a, u32 b, u32 s) { return __builtin_amdgcn_perm(a, b, s); }
DI u32 bfi(u32 m, u32 a, u32 b) { return (m & a) | (~m & b); }
DI u32 lds_ld(LAS u32* p) { return __hip_atomic_load(p, __ATOMIC_RELAXED, __HIP_MEMORY_SCOPE_WORKGROUP); }
DI void lds_st(LAS u32* p, u32 v) { __hip_atomic_store(p, v, __ATOMIC_RELAXED, __HIP_MEMORY_SCOPE_WORKGROUP); }
DI u32 lds_add(LAS u32* p, u32 v) { return __hip_atomic_fetch_add(p, v, __ATOMIC_RELAXED, __HIP_MEMORY_SCOPE_WORKGROUP); }
DI void lds_or(LAS unsigned char* p, u32 v) { __hip_atomic_fetch_or((LAS u32*)p, v, __ATOMIC_RELAXED, __HIP_MEMORY_SCOPE_WORKGROUP); }
DI u32 g_ld_agent(const char* p) { return __hip_atomic_load((GAS u32*)p, __ATOMIC_RELAXED, __HIP_MEMORY_SCOPE_AGENT); }
DI u32 g_ld_sys(const char* p) { return __hip_atomic_load((GAS u32*)p, __ATOMIC_RELAXED, __HIP_MEMORY_SCOPE_SYSTEM); }
DI void g_st_agent(char* p, u32 v) { __hip_atomic_store((GAS u32*)p, v, __ATOMIC_RELAXED, __HIP_MEMORY_SCOPE_AGENT); }
DI void g_st_sys(char* p, u32 v) { __hip_atomic_store((GAS u32*)p, v, __ATOMIC_RELAXED, __HIP_MEMORY_SCOPE_SYSTEM); }

struct Ctx {
  LAS unsigned char* lds; LAS u32* ctl; char* ws; u64 t0; u32 call; int lane, wg, rank;
};
DI void give_up(const Ctx& c) { if (c.lane == 0) { lds_st(c.ctl + C_ABORT, 1u); if (c.call) g_st_agent(c.ws + O_TMO, c.call); } }
DI bool expired(const Ctx& c, u32& spins) {
  if ((++spins & 15u) != 0u) return false;
  if (RFL(lds_ld(c.ctl + C_ABORT)) != 0u || RFL(g_ld_agent(c.ws + O_TMO)) == c.call || now() - c.t0 > DEADLINE) { give_up(c); return true; }
  return false;
}
DI bool lds_expired(const Ctx& c, u32& spins) {
  if ((++spins & 63u) == 0u && (RFL(lds_ld(c.ctl + C_ABORT)) != 0u || now() - c.t0 > DEADLINE + DEADLINE / 4)) { give_up(c); return true; }
  if (spins < SPIN_FAST) __builtin_amdgcn_s_sleep(1); else __builtin_amdgcn_s_sleep(SLEEP_LONG);
  return false;
}
DI u32 lds_wait(const Ctx& c, int word, u32 target) {
  u32 v, spins = 0;
  while ((v = RFL(lds_ld(c.ctl + word))) < target) if (lds_expired(c, spins)) break;
  CFENCE();
  return v;
}
DI u32 q_wait(const Ctx& c, int q, u32 k) {
  u32 v, spins = 0;
  while (((((v = RFL(lds_ld(c.ctl + q + (k & 7u)))) >> 23) - (k + 1u)) & 256u) != 0u) if (lds_expired(c, spins)) { v = ~0u; break; }
  CFENCE();
  return RFL(v);
}
#define STAMP(c, i) do {} while (0)
DI void dma4(rsrc_t rs, u32 voff, u32 soff, u32 ldsdst) {
  u32 keep;
  asm volatile("s_nop 4\n\ts_mov_b32 %0, m0\n\ts_mov_b32 m0, %1\n\ts_nop 0\n\t"
               "buffer_load_dwordx4 %2, %3, %4 offen sc0 sc1 lds\n\t"
               "buffer_load_dwordx4 %2, %3, %4 offen offset:1024 sc0 sc1 lds\n\t"
               "buffer_load_dwordx4 %2, %3, %4 offen offset:2048 sc0 sc1 lds\n\t"
               "buffer_load_dwordx4 %2, %3, %4 offen offset:3072 sc0 sc1 lds\n\t"
               "s_mov_b32 m0, %0"
               : "=&s"(keep) : "s"(RFL(ldsdst)), "v"(voff), "s"(rs), "s"(RFL(soff)) : "memory");
}
DI void dma1(rsrc_t rs, u32 voff, u32 soff, u32 ldsdst) {
  u32 keep;
  asm volatile("s_nop 4\n\ts_mov_b32 %0, m0\n\ts_mov_b32 m0, %1\n\ts_nop 0\n\t"
               "buffer_load_dwordx4 %2, %3, %4 offen sc0 sc1 lds\n\t"
               "s_mov_b32 m0, %0"
               : "=&s"(keep) : "s"(RFL(ldsdst)), "v"(voff), "s"(rs), "s"(RFL(soff)) : "memory");
}
template <int CTRL, int ROWS> DI u32 dpp0(u32 v) { return (u32)__builtin_amdgcn_update_dpp(0, (int)v, CTRL, ROWS, 0xF, false); }
DI u32 wave_incl_scan(u32 v) {
  v += dpp0<0x111, 0xF>(v); v += dpp0<0x112, 0xF>(v); v += dpp0<0x114, 0xF>(v); v += dpp0<0x118, 0xF>(v);
  v += dpp0<0x142, 0xA>(v); v += dpp0<0x143, 0xC>(v);
  return v;
}
DI u32 wave_max(u32 v) {
  v = max(v, dpp0<0x111, 0xF>(v)); v = max(v, dpp0<0x112, 0xF>(v)); v = max(v, dpp0<0x114, 0xF>(v)); v = max(v, dpp0<0x118, 0xF>(v));
  v = max(v, dpp0<0x142, 0xA>(v)); v = max(v, dpp0<0x143, 0xC>(v));
  return __builtin_amdgcn_readlane(v, 63);
}
DI int tile_mi(int wg) { return ((wg & 7) >> 1) * 4 + ((wg >> 3) & 3); }
DI int tile_ni(int wg) { return (wg & 1) * 8 + (wg >> 5); }
DI int wg_late(int wg) { return (wg >> 4) & 1; }
DI int wg_link(int wg) { return (((wg & 15) | (wg >> 5) << 4) + 3 * wg_late(wg)) % 7; }
DI u32 link_wgs(int link, int late) { return (u32)(127 + 7 - (link + 7 - 3 * late) % 7) / 7u; }
DI u32 row_bytes(u32 V) { return (((V + 7u) >> 3) + 3u) & ~3u; }
DI void tell_peers(const Ctx& c, u32 off, u32 v) { if (c.lane < NR && c.lane != c.rank) g_st_sys(*(char* LAS*)(c.lds + L_PEER + c.lane * 8) + off, v); }
DI void image_flag(const Ctx& c, int kstep, int r) {
  if (c.lane == 0) g_st_agent(c.ws + (r < 64 ? O_FX + ((r >> 2) * 64 + kstep) * 16 + (r & 3) * 4 : O_FW + (((r - 64) >> 1) * 64 + kstep) * 8 + (r & 1) * 4), c.call);
}

constexpr u32 B1 = 0x01010101u;
DI u32 nib4(u32 m, int k) { return (k ? m >> (4 * k - 2) : m << 2) & 0x3Cu; }
DI u32 to_nib(u32 bits) { return __builtin_amdgcn_udot4(bits, 0x08040201u, 0u, false); }
DI u32 put_sel(const Ctx& c, u32 n4) { return *(LAS u32*)(c.lds + L_LUTC + n4); }
DI u32 ldsa(LAS unsigned char* p) { return (u32)(size_t)p; }
DI void or_at(u32 a, u32 v) { lds_or((LAS unsigned char*)(size_t)(a & ~3u), v); }
DI u32 put4(u32& a, u32& carry, u32 v, u32 n4, u32 sel) {
  const u32 c4 = prm(0u, v, sel);
  const u64 w = (u64)c4 << ((a << 3) & 24u);
  or_at(a, (u32)w | carry);
  carry = (u32)(w >> 32);
  a += __popc(n4);
  return c4;
}
template <u32 N> DI void get_all(const Ctx& c, u32 a, const u32* m, u32 (&x)[N]) {
  u32 sel[N], hi[N];
#pragma unroll
  for (u32 k = 0; k < N; ++k) {
    const u32 n4 = nib4(m[k >> 3], k & 7u);
    sel[k] = *(LAS u32*)(c.lds + L_LUTE + (((a << 6) & 0xC0u) | n4));
    LAS u32* d = (LAS u32*)(size_t)(a & ~3u);
    x[k] = d[0]; hi[k] = d[1];
    a += __popc(n4);
  }
  __builtin_amdgcn_sched_barrier(0);
#pragma unroll
  for (u32 k = 0; k < N; ++k) x[k] = prm(hi[k], x[k], sel[k]);
}
struct Lv { u32 rows2, nc2, nc3; };
template <u32 R> DI bool enc_tail(const Ctx& c, u32 V2, LAS unsigned char* s, u32 cap, Lv& g, u32& l2a, u32& l2b, u32& l3) {
  const u32 nc2 = (V2 + 4u * R - 1u) / (4u * R), b2 = R * nc2 * 4u;
  u32 e[R], sel[R], nb3 = 0;
  l2a = 0u; l2b = 0u;
#pragma unroll
  for (u32 r = 0; r < R; ++r) e[r] = *(LAS u32*)(s + (r * nc2 + c.lane) * 4u);
#pragma unroll
  for (u32 r = 0; r < R; ++r) {
    e[r] = (u32)c.lane < nc2 ? e[r] : 0u;
    const u32 lo3 = e[r] & 7u * B1, b3 = (e[r] >> 3) & B1;
    const u32 c2 = bfi((b3 << 8) - b3, prm(0x00000002u, 0x03000000u, lo3), prm(0u, 1u, lo3));
    if (r < 4u) l2a |= c2 << (2u * r); else l2b |= c2 << (2u * r - 8u);
    nb3 |= to_nib((c2 | c2 >> 1) & B1) << (4u * r);
  }
  nb3 = ~nb3 & (R == 8u ? ~0u : 0xFFFFu);
#pragma unroll
  for (u32 r = 0; r < R; ++r) sel[r] = put_sel(c, nib4(nb3, r));
  const u32 cn3 = __popc(nb3), in3 = wave_incl_scan(cn3), V3 = __builtin_amdgcn_readlane(in3, 63);
  g.rows2 = R; g.nc2 = nc2; g.nc3 = (V3 + 7u) >> 3;
  if (V3 > 512u || b2 + g.nc3 * 8u + 8u > cap) return false;
  u32 a = ldsa(s) + b2 + in3 - cn3, carry = 0;
#pragma unroll
  for (u32 r = 0; r < R; ++r) put4(a, carry, e[r], nib4(nb3, r), sel[r]);
  or_at(a, carry);
  CFENCE();
  l3 = *(LAS u32*)(s + b2 + c.lane * 4u) | *(LAS u32*)(s + b2 + (g.nc3 + c.lane) * 4u) << 4;
  return true;
}
template <u32 N, u32 MAX2> DI bool enc_levels(const Ctx& c, u32 (&n)[N], LAS unsigned char* s, u32 cap, Lv& g, u32& l2a, u32& l2b, u32& l3) {
  u32 nb[2] = {0u, 0u}, c1[N], sel[N];
#pragma unroll
  for (u32 k = 0; k < N; ++k) {
    const u32 m = ((n[k] + 3u * B1) >> 4) & B1;
    c1[k] = n[k] & (m + (m << 1));
    nb[k >> 3] |= to_nib(m) << (4u * (k & 7u));
  }
  nb[0] = ~nb[0]; nb[1] = ~nb[1] & (N == 16u ? ~0u : (1u << (4u * (N & 7u))) - 1u);
#pragma unroll
  for (u32 k = 0; k < N; ++k) sel[k] = put_sel(c, nib4(nb[k >> 3], k & 7u));
  const u32 cnt = __popc(nb[0]) + __popc(nb[1]), inc = wave_incl_scan(cnt), V2 = __builtin_amdgcn_readlane(inc, 63);
  if (V2 > MAX2 || V2 + 40u > cap) return false;
  u32 a = ldsa(s) + inc - cnt, carry = 0;
#pragma unroll
  for (u32 k = 0; k < N; ++k) put4(a, carry, n[k], nib4(nb[k >> 3], k & 7u), sel[k]);
  or_at(a, carry);
  CFENCE();
#pragma unroll
  for (u32 k = 0; k < N; ++k) n[k] = c1[k];
  if (MAX2 > 1024u && V2 > 1024u) return enc_tail<8>(c, V2, s, cap, g, l2a, l2b, l3);
  return enc_tail<4>(c, V2, s, cap, g, l2a, l2b, l3);
}
template <u32 R> DI void dec_tail(const Ctx& c, u32 V2, LAS unsigned char* in, LAS unsigned char* s, u32 cap) {
  const u32 nc2 = (V2 + 4u * R - 1u) / (4u * R), b2 = R * nc2 * 4u;
  const bool col = (u32)c.lane < nc2;
  u32 l2[2] = {*(LAS u32*)(in + c.lane * 4u), R == 8u ? *(LAS u32*)(in + (nc2 + c.lane) * 4u) : 0u}, nb3 = 0, e[R];
  const u32 l3 = *(LAS u32*)(in + R * nc2 + c.lane * 4u);
  l2[0] = col ? l2[0] : ~0u; l2[1] = col ? l2[1] : ~0u;
#pragma unroll
  for (u32 r = 0; r < R; ++r) nb3 |= (to_nib((l2[r >> 2] | l2[r >> 2] >> 1) & B1 << (2u * (r & 3u))) >> (2u * (r & 3u))) << (4u * r);
  nb3 = ~nb3 & (R == 8u ? ~0u : 0xFFFFu);
  const u32 cn3 = __popc(nb3), in3 = wave_incl_scan(cn3), V3 = __builtin_amdgcn_readlane(in3, 63), nc3 = (V3 + 7u) >> 3;
  if (V3 > 512u || b2 + nc3 * 8u + 8u > cap) return;
  if ((u32)c.lane < nc3) {
    *(LAS u32*)(s + b2 + c.lane * 4u) = ((l3 & 15u * B1) + 14u * B1) & 15u * B1;
    *(LAS u32*)(s + b2 + (nc3 + c.lane) * 4u) = (((l3 >> 4) & 15u * B1) + 14u * B1) & 15u * B1;
  }
  CFENCE();
  get_all<R>(c, ldsa(s) + b2 + in3 - cn3, &nb3, e);
  if (col) {
#pragma unroll
    for (u32 r = 0; r < R; ++r) *(LAS u32*)(s + (r * nc2 + c.lane) * 4u) = e[r] | prm(0u, 0x090A0E00u, (l2[r >> 2] >> (2u * (r & 3u))) & 3u * B1);
  }
}
template <u32 N, u32 MAX2> DI void dec_levels(const Ctx& c, const u32* l1, u32 add, u32 (&o)[N], LAS unsigned char* in, LAS unsigned char* s, u32 cap) {
  u32 nb[2] = {0u, 0u};
#pragma unroll
  for (u32 k = 0; k < N; ++k) nb[k >> 3] |= (to_nib((l1[k >> 2] | l1[k >> 2] >> 1) & B1 << (2u * (k & 3u))) >> (2u * (k & 3u))) << (4u * (k & 7u));
  nb[0] = ~nb[0]; nb[1] = ~nb[1] & (N == 16u ? ~0u : (1u << (4u * (N & 7u))) - 1u);
  const u32 cnt = __popc(nb[0]) + __popc(nb[1]), inc = wave_incl_scan(cnt), V2 = __builtin_amdgcn_readlane(inc, 63);
  if (V2 <= MAX2 && V2 + 40u <= cap) {
    if (MAX2 > 1024u && V2 > 1024u) dec_tail<8>(c, V2, in, s, cap); else dec_tail<4>(c, V2, in, s, cap);
  }
  CFENCE();
  get_all<N>(c, ldsa(s) + inc - cnt, nb, o);
#pragma unroll
  for (u32 k = 0; k < N; ++k) o[k] += prm(0u, 0x0D0C0B00u, (l1[k >> 2] >> (2u * (k & 3u))) & 3u * B1) + add;
}
DI u32 pick_top(u32 sum, u32 cnt) { return (u32)((float)sum * __builtin_amdgcn_rcpf((float)max(cnt, 1u)) + 1.83f); }

DI u32x4 piece(const u32 (&lo)[16], const u32 (&hi)[16], int t) {
  return (u32x4){prm(hi[2 * t], lo[2 * t], 0x05010400u), prm(hi[2 * t], lo[2 * t], 0x07030602u),
                 prm(hi[2 * t + 1], lo[2 * t + 1], 0x05010400u), prm(hi[2 * t + 1], lo[2 * t + 1], 0x07030602u)};
}
DI void pack_inputs(Ctx& c, const Params& p, int wv) {
  const int lane = c.lane;
  const rsrc_t rsL = mkrs(c.ws, WS_USED), rsX = mkrs(p.x, MM * KS * 2), rsW = mkrs(p.w, KS * NN * 2);
  LAS unsigned char* st = c.lds + L_RING + wv * 4096;
  const int hi7 = (c.wg & 15) | (c.wg >> 5) << 4, late = wg_late(c.wg);
  int q = wv == 0 ? hi7 * 2 + late : late ? 128 + wv * 128 + hi7 : 2560;
  if (q < 320) __builtin_amdgcn_s_setprio(3);
  if (q < NPUSH) {
    u32x4 v[8];
#pragma unroll
    for (int t = 0; t < 8; ++t) v[t] = __builtin_amdgcn_raw_buffer_load_b128(rsX, ((64 * q + (lane & 15)) * KS + 8 * (lane >> 4)) * 2 + 64 * (t >> 2), 16 * (t & 3) * KS * 2, 0);
#pragma unroll 1
    for (int l = 1; l <= NR; ++l) {
      const rsrc_t rsP = mkrs(p.ws[(c.rank + q + l) & 7], WS_USED);
#pragma unroll
      for (int t = 0; t < 8; ++t) st128<SYS>(v[t], rsP, O_IMG + (u32)(c.rank * 96 + q) * 8192u + t * 1024 + lane * 16);
      if (l == 6) asm volatile("s_waitcnt vmcnt(40)" ::: "memory");
    }
    c.call = lds_wait(c, C_CALL, 1u);
    VMWAIT0();
    tell_peers(c, O_FX + ((q >> 2) * 64 + c.rank) * 16 + (q & 3) * 4, c.call);
    image_flag(c, c.rank, q);
    q = late ? 1152 + hi7 : 2560;
  }
  for (; q < 2560; q = q < 256 ? (late ? 1152 + hi7 : 2560) : q + 1024) {
    int j = 0, r = q;
    j = q / 320; r = q - j * 320;
    if (j) { __builtin_amdgcn_s_setprio(0); while (now() - c.t0 < (u64)(PACK_T0 + PACK_DT * j)) __builtin_amdgcn_s_sleep(8); }
    const bool isx = r < 64;
    const int dest = (r - 64) & 7, ni = (r - 64) >> 4, kh = ((r - 64) >> 3) & 1;
    u32 lo[16], hi[16];
    *(LAS u32x4*)(st + lane * 16) = (u32x4){0u, 0u, 0u, 0u}; *(LAS u32x4*)(st + 1024 + lane * 16) = (u32x4){0u, 0u, 0u, 0u};
    if (isx) {
      u32x4 d[8];
      const int vo = ((64 * r + (lane & 15)) * KS + 64 * j + 8 * (lane >> 4)) * 2;
#pragma unroll
      for (int t = 0; t < 8; ++t) d[t] = __builtin_amdgcn_raw_buffer_load_b128(rsX, vo + 64 * (t >> 2), 16 * (t & 3) * KS * 2, 0);
      __builtin_amdgcn_sched_barrier(0);
#pragma unroll
      for (int t = 0; t < 8; ++t) {
        lo[2 * t] = prm(d[t].y, d[t].x, 0x06040200u); lo[2 * t + 1] = prm(d[t].w, d[t].z, 0x06040200u);
        hi[2 * t] = prm(d[t].y, d[t].x, 0x07050301u); hi[2 * t + 1] = prm(d[t].w, d[t].z, 0x07050301u);
      }
    } else {
      u32x2 d[2][8];
      const int vo = ((64 * j + 32 * kh + 8 * (lane >> 4)) * NN + dest * 2048 + ni * 128 + 4 * (lane & 15)) * 2;
#pragma unroll
      for (int k = 0; k < 8; ++k)
#pragma unroll
        for (int h = 0; h < 2; ++h) d[h][k] = __builtin_amdgcn_raw_buffer_load_b64(rsW, vo + 128 * h, k * NN * 2, 0);
      __builtin_amdgcn_sched_barrier(0);
#pragma unroll
      for (int h = 0; h < 2; ++h)
#pragma unroll
        for (int cc = 0; cc < 2; ++cc)
#pragma unroll
          for (int u = 0; u < 2; ++u) {
            const u32 a0 = d[h][4 * u][cc], a1 = d[h][4 * u + 1][cc], a2 = d[h][4 * u + 2][cc], a3 = d[h][4 * u + 3][cc];
            const u32 l0 = prm(a1, a0, 0x06020400u), l1 = prm(a3, a2, 0x06020400u), h0 = prm(a1, a0, 0x07030501u), h1 = prm(a3, a2, 0x07030501u);
            const int t = 4 * h + 2 * cc;
            lo[2 * t + u] = prm(l1, l0, 0x05040100u); lo[2 * t + 2 + u] = prm(l1, l0, 0x07060302u);
            hi[2 * t + u] = prm(h1, h0, 0x05040100u); hi[2 * t + 2 + u] = prm(h1, h0, 0x07060302u);
          }
    }
    const int lr = isx ? r : 64 + 2 * ni + kh;
    const bool own = isx || dest == c.rank, send = isx || dest != c.rank;
    if (own) {
      const u32 img = O_IMG + (u32)((j * 8 + c.rank) * 96 + lr) * 8192u + lane * 16;
#pragma unroll
      for (int t = 0; t < 8; ++t) st128<AGT>(piece(lo, hi, t), rsL, img + t * 1024);
    }
    u32 tot = 8192u;
    if (send) {
      const u32 slot = (isx ? O_OBX + (u32)(j * 64 + r) * 8192u : O_OBW + (u32)((dest * 8 + j) * 32 + 2 * ni + kh) * 8192u);
      u32 n[16], se = 0, any = 0, cnt = 0;
      if (q >= NRAW) {
#pragma unroll
      for (int k = 0; k < 16; ++k) { n[k] = bfi(B1, lo[k] >> 7, hi[k] << 1); se = __builtin_amdgcn_sad_u8(n[k], 0u, se); }
      const u32 base = min(max(pick_top(__builtin_amdgcn_readlane(wave_incl_scan(se), 63), 4096u), 13u) - 13u, 240u);
      const u32 bl = (base & 127u) * B1, bx = base < 128u ? 0x80u * B1 : 0u;
#pragma unroll
      for (int k = 0; k < 16; ++k) { n[k] = (((n[k] | 0x80u * B1) - bl) ^ (n[k] & 0x80u * B1)) ^ bx; any |= n[k]; }
      if (ANY((any & 0xF0u * B1) != 0u)) {
#pragma unroll
        for (int k = 0; k < 16; ++k) if (ANY((n[k] & 0xF0u * B1) != 0u)) {
#pragma unroll
          for (int b = 0; b < 4; ++b) {
            const u32 v = (n[k] >> (8 * b)) & 255u;
            const u64 m = __builtin_amdgcn_ballot_w64(v > 15u);
            const u32 pos = cnt + __builtin_amdgcn_mbcnt_hi((u32)(m >> 32), __builtin_amdgcn_mbcnt_lo((u32)m, 0u));
            if (v > 15u && pos < 31u) st32((u32)(k * 4 + b) << 14 | (u32)lane << 8 | ((v + base) & 255u), rsL, slot + 5124u + pos * 4u);
            cnt += (u32)__popcll(m);
          }
        }
      }
#pragma unroll
      for (int k = 0; k < 16; ++k) n[k] = ((n[k] & 15u * B1) + 2u * B1) & 15u * B1;
      Lv g; u32 l2a, l2b, l3;
      if (cnt <= 31u && enc_levels<16, 2048>(c, n, st, 2048u, g, l2a, l2b, l3)) {
#pragma unroll
        for (int t = 0; t < 4; ++t)
          st128<SYS>((u32x4){bfi(0x80u * B1, hi[4 * t], lo[4 * t]), bfi(0x80u * B1, hi[4 * t + 1], lo[4 * t + 1]),
                             bfi(0x80u * B1, hi[4 * t + 2], lo[4 * t + 2]), bfi(0x80u * B1, hi[4 * t + 3], lo[4 * t + 3])}, rsL, slot + t * 1024 + lane * 16);
        u32x4 c1;
#pragma unroll
        for (int t = 0; t < 4; ++t) c1[t] = n[4 * t] | n[4 * t + 1] << 2 | n[4 * t + 2] << 4 | n[4 * t + 3] << 6;
        st128<SYS>(c1, rsL, slot + 4096u + lane * 16);
        const u32 o2 = 5120u + ((cnt + 4u) & ~3u) * 4u;
        if (lane == 0) st32(base | cnt << 8, rsL, slot + 5120u);
        if ((u32)lane < g.nc2) { st32(l2a, rsL, slot + o2 + lane * 4); if (g.rows2 == 8u) st32(l2b, rsL, slot + o2 + (g.nc2 + lane) * 4u); }
        if ((u32)lane < g.nc3) st32(l3, rsL, slot + o2 + g.rows2 * g.nc2 + lane * 4);
        tot = (o2 + g.rows2 * g.nc2 + g.nc3 * 4u + 15u) & ~15u;
      } else VMWAIT0();
      }
      if (tot == 8192u) {
        asm volatile("" : "+v"(lo[0]), "+v"(lo[2]), "+v"(lo[4]), "+v"(lo[6]), "+v"(lo[8]), "+v"(lo[10]), "+v"(lo[12]), "+v"(lo[14]),
                          "+v"(lo[1]), "+v"(lo[3]), "+v"(lo[5]), "+v"(lo[7]), "+v"(lo[9]), "+v"(lo[11]), "+v"(lo[13]), "+v"(lo[15]));
#pragma unroll
        for (int t = 0; t < 8; ++t) st128<SYS>(piece(lo, hi, t), rsL, slot + t * 1024 + lane * 16);
      }
    }
    if (!c.call) c.call = lds_wait(c, C_CALL, 1u);
    VMWAIT0();
    if (send && (isx || lane == dest)) tell_peers(c, O_RF + (c.rank * NBLK + j * 96 + lr) * 4, c.call << 20 | tot >> 4);
    if (own) image_flag(c, j * 8 + c.rank, lr);
    if (q < 64) STAMP(c, 1);
    if (q >= 2048) STAMP(c, 2);
  }
  __builtin_amdgcn_s_setprio(0);
  c.call = lds_wait(c, C_CALL, 1u);
  if (lane == 0) lds_add(c.ctl + C_PACKED, 1u);
}

DI u32 ob_slot(int b, int me) { const int j = b / 96, r = b - j * 96; return r < 64 ? O_OBX + (u32)(j * 64 + r) * 8192u : O_OBW + (u32)((me * 8 + j) * 32 + r - 64) * 8192u; }
#define LEFT {0, 2, 4, 4}
struct Order {
  u32 lb[NBLK];
  constexpr Order() : lb() {
    constexpr int left[4] = LEFT, phase[4] = {6, 5, 1, 3};
    int n = 0, next[4] = {1, 1, 1, 1};
    for (int r = 0; r < 96; ++r) lb[n++] = (u32)r;
    for (int j = 1; j < 16; ++j) {
      bool go[4] = {};
      for (int g = 0; g < 4; ++g) go[g] = next[g] < 8 && (j > 7 || (j * (7 - left[g]) + phase[g]) / 7 >= next[g]) && !(j <= 7 && next[g] > 7 - left[g]);
      if (j < 8) for (int r = 0; r < 32; ++r) lb[n++] = (u32)(j * 96 + 64 + r);
      for (int u = 0; u < 4; ++u)
        for (int v = 0; v < 4; ++v) {
          const int t = j > 7 ? v : u, g = j > 7 ? u : v;
          if (go[g]) for (int b = 0; b < 4; ++b) lb[n++] = (u32)(next[g] * 96 + t * 16 + g * 4 + b);
        }
      for (int g = 0; g < 4; ++g) next[g] += go[g];
    }
  }
};
__constant__ Order ORDER = Order();
DI int pull_order(u32 B, int link) {
  const u32 lb = ORDER.lb[B], r = lb % 96u;
  return (int)(B >= 96u && r < 64u ? lb - r + ((r + 16u * link) & 63u) : lb);
}
DI u32 early_first(u32 t) { return (t & 15u) | (t >> 7) << 4 | ((t >> 4) & 7u) << 5; }
DI void role_feeder(Ctx& c) {
  const int lane = c.lane;
  __builtin_amdgcn_s_setprio(3);
  c.call = lds_wait(c, C_CALL, 1u);
  const int link = wg_link(c.wg), src = (c.rank + 1 + link) & 7;
  const u32 ne = link_wgs(link, 0), late = wg_late(c.wg), grp = (c.wg >> 3) & 3;
  GAS u32* tk = (GAS u32*)(c.ws + O_TICK + link * 128);
  u32 kv = 0, tbase = (c.call << 12) - KFIX * ne - NPUSH;
  const u32 stop = grp == 0u ? 0u : grp == 1u ? T_HALF : T_MASK;
  bool miss = true, out = false;
  const rsrc_t rsL = mkrs(c.ws, WS_USED);
  u32 fb = O_RF + src * NBLK * 4, B = 0, pair = 0;
  for (u32 k = 0;; ++k) {
    if (k == (late ? 0u : KFIX - 1u)) {
      if (lane < 2) __hip_atomic_fetch_max(tk + lane * 256, c.call << 12, __ATOMIC_RELAXED, __HIP_MEMORY_SCOPE_AGENT);
      VMWAIT0();
      if (late) lds_wait(c, C_PACKED, 8u);
    }
    const u32 ahead = B < T_AH ? 2u : 1u;
    if (k >= ahead) lds_wait(c, C_PSTART, k + 1u - ahead);
    if (k < KFIX && !late) {
      B = NPUSH + (u32)((c.wg & 15) | (c.wg >> 5) << 4) / 7u + k * ne;
      if (miss) { dma4(rsL, lane * 16, fb, (u32)(size_t)c.lds + L_RFC); VMWAIT0(); }
    } else if (pair) {
      --pair; ++B;
      if (miss) { dma4(rsL, lane * 16, fb, (u32)(size_t)c.lds + L_RFC); VMWAIT0(); }
    } else for (;;) {
      if (!out && B >= stop) {
        if (grp != 1u) lds_wait(c, C_CMAX, 8u);
        STAMP(c, 6);
        out = true; tk += 256; tbase = (c.call << 12) - T_MASK; fb = O_OF + src * 8192; miss = true;
      }
      if (miss) dma4(rsL, lane * 16, fb, (u32)(size_t)c.lds + L_RFC);
      u32 t = 0;
      pair = out && B < T_MASK + T_PAIR ? 1u : 0u;
      if (lane == 0) t = __hip_atomic_fetch_add(tk, pair + 1u, __ATOMIC_RELAXED, __HIP_MEMORY_SCOPE_AGENT);
      VMWAIT0();
      B = RFL(t) - tbase;
      if (B < T_MASK || out) break;
    }
    if (B >= T_END) {
      if (lane < 3) lds_st(c.ctl + C_QI + ((k + lane) & 7u), (k + lane + 1u) << 23 | 0x7FFFFFu);
      if (kv >= 5u) lds_wait(c, C_DFREE0 + ((kv - 5u) & 3u), ((kv - 5u) & ~3u) + 4u);
      if (lane < 3) lds_st(c.ctl + C_QV + ((kv + lane) & 7u), (kv + lane + 1u) << 23 | 0x7FFFFFu);
      break;
    }
    u32 item, fo;
    if (B < T_MASK) { item = pull_order(B, link); fo = item * 4; }
    else if (B < T_VAL) { item = T_MASK; fo = F_MF + (B - T_MASK) * 4; }
    else {
      const u32 ht = early_first((B - T_VAL) >> 1) * 2u + (B & 1u);
      item = T_VAL + ht; fo = ((lane & 1) ? F_MI : F_VF) - F_MI + ht * 4;
      if (kv == 0u) fb += F_MI;
    }
    u32 v = B >= T_VAL && kv == 0u ? 0u : lds_ld((LAS u32*)(c.lds + L_RFC + fo)), spins = 0;
    miss = false;
    while (!ALL((int)((v & 0xFFF00000u) - (c.call << 20)) >= 0)) {
      miss = true;
      if (expired(c, spins)) break;
      if (spins > 1u) __builtin_amdgcn_s_sleep(1);
      v = g_ld_sys(c.ws + fb + fo);
    }
    if (item == T_MASK) { item += min((v >> 9) & 1023u, 767u); v &= 511u; }
    if (lane == 0) {
      if (B >= T_VAL) lds_st(c.ctl + C_QV + (kv & 7u), (kv + 1u) << 23 | (k & 2047u) << 12 | (item - T_VAL));
      const u32 wide = B < T_W0 || (B >= T_HALF && B < T_MASK + T_WA);
      lds_st(c.ctl + C_QI + (k & 7u), (k + 1u) << 23 | item << 12 | wide << 11 | (v & 0x7FFu));
    }
    if (B >= T_VAL) ++kv;
  }
}
DI void role_decoder(Ctx& c, u32 d) {
  const int lane = c.lane, src = (c.rank + 1 + wg_link(c.wg)) & 7;
  __builtin_amdgcn_s_setprio(DPRIO);
  c.call = lds_wait(c, C_CALL, 1u);
  const rsrc_t rsL = mkrs(c.ws, WS_USED);
  for (u32 k = d;; k += NDEC) {
    const u32 e = q_wait(c, C_QI, k), lb = (e >> 12) & 2047u, tot = min((e & 0x7FFu) * 16u, 8192u);
    if (lb >= T_VAL) break;
    LAS unsigned char* sl = c.lds + L_IN + (k & 3u) * 8192u;
    if (lb >= T_MASK) {
      lds_wait(c, C_PROG, k * 32768u + tot);
      const u32 pc = lb - T_MASK, inb = O_MIN + src * 256 * 16384 + piece_off(pc) + lane * 16;
      const u32x4 hd = *(LAS u32x4*)sl;
      u32x4 v[8];
#pragma unroll
      for (int t = 0; t < 8; ++t)
        v[t] = *(LAS u32x4*)(sl + (pc >= 512u ? (t >> 1) * 1024u : min((t < 2 ? 16u : hd[(t >> 1) - 1] & 0x1FF0u) + (t & 1) * 1024u, 7152u)) + lane * 16);
      CFENCE(); if (lane == 0) lds_add(c.ctl + C_IFREE0 + (k & 3u), 1u); CFENCE();
      if (pc < 512u && lane == 0) st128<AGT>(hd, rsL, inb);
#pragma unroll
      for (int t = 0; t < 8; t += 2) {
        st128<AGT>(v[t], rsL, inb + (pc >= 512u ? t * 512u : 16u + t * 640u));
        if (pc < 512u && lane < 16 && (hd[t >> 1] & 0xFFFFu) - (t ? hd[(t >> 1) - 1] & 0xFFFFu : 16u) > 1024u) st128<AGT>(v[t + 1], rsL, inb + 1040u + t * 640u);
      }
      VMWAIT0();
      if ((u32)lane < (pc >= 512u ? 2u : 1u))
        __hip_atomic_fetch_add((GAS u32*)(c.ws + O_OF + src * 8192 + F_MI) + (pc >= 512u ? (pc - 512u) * 2u + lane : pc), 1u << 19, __ATOMIC_RELAXED, __HIP_MEMORY_SCOPE_AGENT);
      continue;
    }
    const int j = lb / 96, r = lb - j * 96, kstep = j * 8 + src;
    const u32 img = O_IMG + (u32)(kstep * 96 + r) * 8192u + lane * 16;
    lds_wait(c, C_PROG, k * 32768u + tot);
    if (k < 2u) STAMP(c, 26 + k);
    if (tot == 8192u) {
      u32x4 v[8];
#pragma unroll
      for (int t = 0; t < 8; ++t) v[t] = *(LAS u32x4*)(sl + t * 1024 + lane * 16);
      CFENCE(); if (lane == 0) lds_add(c.ctl + C_IFREE0 + (k & 3u), 1u); CFENCE();
#pragma unroll
      for (int t = 0; t < 8; ++t) st128<AGT>(v[t], rsL, img + t * 1024);
    } else {
      const u32x4 c1 = *(LAS u32x4*)(sl + 4096 + lane * 16);
      const u32 exc = *(LAS u32*)(sl + 5120 + (lane & 31) * 4);
      u32x4 sm[4];
#pragma unroll
      for (int t = 0; t < 4; ++t) sm[t] = *(LAS u32x4*)(sl + t * 1024 + lane * 16);
      const u32 hd = RFL(exc), cnt = (hd >> 8) & 31u, base4 = min(hd & 255u, 240u) * B1;
      u32 n[16], lo[16], hi[16];
      const u32 l1[4] = {c1.x, c1.y, c1.z, c1.w};
      dec_levels<16, 2048>(c, l1, base4, n, sl + 5120u + ((cnt + 4u) & ~3u) * 4u, sl + 6144u, 2048u);
      CFENCE(); if (lane == 0) lds_add(c.ctl + C_IFREE0 + (k & 3u), 1u); CFENCE();
      for (u32 x = 1; x <= cnt; ++x) {
        const u32 rec = __builtin_amdgcn_readlane(exc, x);
        const u32 kk = (rec >> 16) & 15u, sh = ((rec >> 14) & 3u) * 8u;
        const u32 m = ((rec >> 8) & 63u) == (u32)lane ? 255u << sh : 0u, val = (rec & 255u) << sh;
#pragma unroll
        for (int i = 0; i < 16; ++i) n[i] = bfi(kk == (u32)i ? m : 0u, val, n[i]);
      }
#pragma unroll
      for (int i = 0; i < 16; ++i) { hi[i] = bfi(0x80u * B1, sm[i >> 2][i & 3], n[i] >> 1); lo[i] = bfi(0x80u * B1, n[i] << 7, sm[i >> 2][i & 3]); }
#pragma unroll
      for (int t = 0; t < 8; ++t) st128<AGT>(piece(lo, hi, t), rsL, img + t * 1024);
    }
    VMWAIT0();
    image_flag(c, kstep, r);
  }
}
constexpr int WMAX = WIN2 - 1;
#define WINM 3
struct Stream {
  u32 pub = 0, pend[WMAX] = {};
  template <int W> DI void landed(const Ctx& c, u32 prog) {
    asm volatile("s_waitcnt vmcnt(%0)" :: "n"(W - 1) : "memory");
    pub = max(pub, pend[WMAX - (W - 1)]);
    if (c.lane == 0) lds_st(c.ctl + C_PROG, pub);
    for (int k = 0; k + 1 < WMAX; ++k) pend[k] = pend[k + 1];
    pend[WMAX - 1] = prog;
  }
  DI void flush(const Ctx& c) {
    VMWAIT0();
    pub = pend[WMAX - 1];
    if (c.lane == 0) lds_st(c.ctl + C_PROG, pub);
  }
};
DI void role_puller(Ctx& c, const Params& p) {
  __builtin_amdgcn_s_setprio(3);
  const u32 ldsb = (u32)(size_t)c.lds;
  const char* sws = p.ws[(c.rank + 1 + wg_link(c.wg)) & 7];
  Stream sm;
  u32 kv = 0;
  u32 cw = 0;
  for (u32 k = 0;; ++k) {
    u32 e = __builtin_amdgcn_readlane(cw, 0), fi = __builtin_amdgcn_readlane(cw, 1), fd = __builtin_amdgcn_readlane(cw, 2);
    if ((e >> 23) != ((k + 1u) & 511u)) {
      e = RFL(lds_ld(c.ctl + C_QI + (k & 7u)));
      if ((e >> 23) != ((k + 1u) & 511u)) {
        sm.flush(c);
        e = q_wait(c, C_QI, k);
      }
    }
    CFENCE();
    const u32 item = (e >> 12) & 2047u;
    if ((~e & 0x7FFFFFu) == 0u) break;
    u32 so, tot = (e & 0x7FFu) * 16u, dst;
    if (item < T_VAL) {
      if (fi < k >> 2) { sm.flush(c); lds_wait(c, C_IFREE0 + (k & 3u), k >> 2); }
      dst = L_IN + (k & 3u) * 8192u;
      so = item < T_MASK ? ob_slot(item, c.rank) : O_OM + piece_off(item - T_MASK); tot = min(tot, 8192u);
    } else {
      if (fd < (kv & ~3u)) { sm.flush(c); lds_wait(c, C_DFREE0 + (kv & 3u), kv & ~3u); }
      dst = L_STG + (kv & 3u) * 16384u; so = O_OV + (item - T_VAL) * 16384u; tot = min(tot, 16384u);
      ++kv;
    }
    so = RFL(so); tot = RFL(max(tot, 16u)); dst = RFL(dst);
    const rsrc_t rs = mkrs(sws + so, tot);
    const bool wide = (e >> 11) & 1u;
    const u32 lead = PLEAD, qs = tot > lead ? (tot - lead) & ~1023u : 0u;
    constexpr u32 ch = 1024u;
    bool told = false;
    for (u32 q = 0; q < tot; q += ch) {
      dma1(rs, c.lane * 16, q, ldsb + dst + q);
      if (q >= qs && !told) { told = true; if (c.lane == 0) lds_st(c.ctl + C_PSTART, k + 1u); }
      if (q + ch >= tot) cw = lds_ld(c.ctl + (c.lane == 0 ? C_QI + ((k + 1u) & 7u) : c.lane == 1 ? C_IFREE0 + ((k + 1u) & 3u) : C_DFREE0 + (kv & 3u)));
      if (wide) sm.landed<WIN2>(c, k * 32768u + q + ch);
      else if (WINM != WIN1 && item < T_MASK) sm.landed<WINM>(c, k * 32768u + q + ch);
      else sm.landed<WIN1>(c, k * 32768u + q + ch);
    }
  }
  sm.flush(c);
  STAMP(c, 8);
}

DI void role_sched(Ctx& c, const Params& p, int mi, int ni) {
  const int lane = c.lane;
  __builtin_amdgcn_s_setprio(3);
  u32 t = 0;
  if (lane == 0) t = __hip_atomic_fetch_add((GAS u32*)(c.ws + O_CALL + c.wg * 128), 1u, __ATOMIC_RELAXED, __HIP_MEMORY_SCOPE_AGENT) + 1u;
  c.call = RFL(t);
  if (lane == 0) lds_st(c.ctl + C_CALL, c.call);
  const rsrc_t rsL = mkrs(c.ws, WS_USED);
  u64 issued = 0; u32 n = 0, spins = 0;
  while (n < 64u) {
    if (2u * n >= RFL(lds_ld((LAS u32*)(c.lds + L_REL + 7 * 256))) + 2u * SCHED_Q) { if (!lds_expired(c, spins)) continue; }
    const u32x4 a = __builtin_amdgcn_raw_buffer_load_b128(rsL, lane * 16, (int)(O_FX + mi * 1024), SYS);
    const u32x2 b = __builtin_amdgcn_raw_buffer_load_b64(rsL, lane * 8, (int)(O_FW + ni * 512), SYS);
    const bool rdy = min(min(min(a.x, a.y), min(a.z, a.w)), min(b.x, b.y)) >= c.call;
    u64 m = __builtin_amdgcn_ballot_w64(rdy) & ~issued;
    if (m == 0) {
      if (!expired(c, spins)) { __builtin_amdgcn_s_sleep(1); CFENCE(); continue; }
      m = ~issued;
    }
    if ((m >> lane) & 1) lds_st(c.ctl + C_ORDER + n + __builtin_amdgcn_mbcnt_hi((u32)(m >> 32), __builtin_amdgcn_mbcnt_lo((u32)m, 0u)), lane + 1);
    issued |= m;
    n += (u32)__popcll(m);
    CFENCE();
  }
  STAMP(c, 9);
  if (c.wg == LEADER) {
    u32 mx;
    for (;;) {
      const u32x4 a = __builtin_amdgcn_raw_buffer_load_b128(rsL, lane * 16, (int)O_LT, SYS);
      const u32x4 b = __builtin_amdgcn_raw_buffer_load_b128(rsL, lane * 16, (int)(O_LT + 1024), SYS);
      mx = max(max(a.x, a.z), max(b.x, b.z));
      if (ALL(min(min(a.y, a.w), min(b.y, b.w)) >= c.call) || expired(c, spins)) break;
      CFENCE();
    }
    STAMP(c, 16);
    mx = wave_max(mx);
    if (lane < NR) __hip_atomic_store((GAS u64*)(*(char* LAS*)(c.lds + L_PEER + lane * 8) + O_GT + c.rank * 8), (u64)c.call << 32 | mx, __ATOMIC_RELAXED, __HIP_MEMORY_SCOPE_SYSTEM);
    if (lane == 0) lds_st(c.ctl + C_LEAD, 1u);
  }
  lds_wait(c, C_CMAX, 8u);
  const rsrc_t rsG = mkrs(c.ws + O_GT, 64u);
  u32 g;
  for (;;) {
    if (lane < 4) dma1(rsG, lane * 16, 0u, (u32)(size_t)c.lds + L_GT);
    asm volatile("s_waitcnt vmcnt(5)" ::: "memory");
    const u64 v = __hip_atomic_load((LAS u64*)(c.lds + L_GT) + (lane & 7), __ATOMIC_RELAXED, __HIP_MEMORY_SCOPE_WORKGROUP);
    g = (u32)v;
    if (ALL((u32)(v >> 32) >= c.call) || expired(c, spins)) break;
    __builtin_amdgcn_s_sleep(2);
  }
  g = wave_max(g);
  if (lane == 0) { lds_st(c.ctl + C_GMAX, g); CFENCE(); lds_st(c.ctl + C_GVALID, 1u); }
  STAMP(c, 10);
}

DI void role_loader(Ctx& c, u32 li, int mi, int ni) {
  __builtin_amdgcn_s_setprio(2);
  const rsrc_t rsL = mkrs(c.ws, WS_USED);
  const u32 voff = c.lane * 16, ldsb = (u32)(size_t)c.lds + L_RING;
  u32 pend = 0;
  lds_wait(c, C_PACKED, 8u);
  for (u32 m = li; m < 128u; m += 5u - NDEC) {
    const u32 slot = m % NS, kh = m & 1u;
    u32 o, spins = 0;
    for (;;) {
      const u32 vo = lds_ld(c.ctl + C_ORDER + (m >> 1)), vf = lds_ld((LAS u32*)(c.lds + L_REL + (c.lane & 7) * 256));
      o = RFL(vo);
      if (o != 0u && ALL(vf + (NS - 1u) >= m)) break;
      if (pend) { VMWAIT0(); if (c.lane == 0) lds_st(c.ctl + C_FULL0 + (pend - 1u) % NS, pend); pend = 0; continue; }
      if (lds_expired(c, spins)) break;
    }
    CFENCE();
    o = RFL(o);
    const u32 so = O_IMG + ((o - 1u) & 63u) * (96u * 8192u), sx = so + mi * 32768 + kh * 4096u, sw = so + 64 * 8192 + ni * 16384 + kh * 8192u, ld = ldsb + slot * SLOTB;
    dma4(rsL, voff, sw, ld + 16384u);
    if (pend) { asm volatile("s_waitcnt vmcnt(4)" ::: "memory"); if (c.lane == 0) lds_st(c.ctl + C_FULL0 + (pend - 1u) % NS, pend); }
    dma4(rsL, voff, sw + 4096u, ld + 20480u);
    dma4(rsL, voff, sx, ld); dma4(rsL, voff, sx + 8192u, ld + 4096u); dma4(rsL, voff, sx + 16384u, ld + 8192u); dma4(rsL, voff, sx + 24576u, ld + 12288u);
    pend = m + 1u;
  }
  VMWAIT0();
  if (c.lane == 0) lds_st(c.ctl + C_FULL0 + (pend - 1u) % NS, pend);
}

struct Geo { u32 nc, sz; };
DI Geo geo(u32 V) { return V <= 2048u ? Geo{row_bytes(V) >> 2, 8u * row_bytes(V)} : Geo{64u, (min(V, 2304u) + 3u) & ~3u}; }
struct Scale { u32 r; int kap; };
DI Scale split(float inv) { const u32 ib = __float_as_uint(inv), r = ((ib & 0x7FFFFFu) + (1u << 19)) >> 20; return Scale{r & 7u, (int)(ib >> 23) - 247 + (int)(r >> 3)}; }
DI u32 low_exp(u32 o, u32 kap32, u32& pos) {
  const u32 t = o + kap32, ge = (t >> 5) & B1;
  pos = ((t + 31u * B1) >> 6) & B1;
  return t & ((ge << 5) - ge);
}
DI u32 sure(u32 m, u32 r8, u32 pos, u32& up) {
  const u32 x = m + r8;
  up = (~x >> 3) & B1;
  return (((x ^ 8u * B1) + 15u * B1) >> 4) & pos;
}
DI u32 place(const Ctx& c, int wv, int cnt, u32 first, u32 sb, u32 extra, u32x4& hdr) {
  if (c.lane == 0) { lds_st(c.ctl + C_VSZ + wv, sb | extra); CFENCE(); lds_add(c.ctl + cnt, 1u); }
  lds_wait(c, cnt, 4u);
  const u32x4 z = *(LAS u32x4*)(c.ctl + C_VSZ + (wv & 4));
  hdr.x = z.x + first; hdr.y = z.y + (hdr.x & 0xFFFFu); hdr.z = z.z + (hdr.y & 0xFFFFu); hdr.w = z.w + (hdr.z & 0xFFFFu);
  return (((wv & 2) ? ((wv & 1) ? hdr.w : hdr.z) : ((wv & 1) ? hdr.y : hdr.x)) & 0xFFFFu) - sb;
}

DI void role_mfma(Ctx& c, const Params& p, int wv, int mi, int ni) {
  const int lane = c.lane, mw = wv >> 1, h = wv & 1;
  pack_inputs(c, p, wv);
  f32x4 acc[4][4];
#pragma unroll
  for (int g = 0; g < 4; ++g)
#pragma unroll
    for (int b = 0; b < 4; ++b) acc[g][b] = (f32x4){0.f, 0.f, 0.f, 0.f};
  if (wv == 0) STAMP(c, 15);
  bf16x8 x0, x1, x2, x3, wa0, wa1, wa2, wa3, wb0, wb1, wb2, wb3;
  u32 vx = (u32)(size_t)c.lds + L_RING + mw * 4096 + lane * 16, vw = (u32)(size_t)c.lds + L_RING + 16384 + h * 4096 + lane * 16;
  u32 vf = (u32)(size_t)(c.ctl + C_FULL0 + 2), done = 1, sl = 1, seen = 0, vs;
  const u32 relad = (u32)(size_t)c.lds + L_REL + wv * 256 + lane * 4;
#define MF(a, w, x) "v_mfma_f32_16x16x32_bf16 " a ", " w ", " x ", " a "\n\t"
#define HSTEP(c0, c1, c2, c3, n0, n1, n2, n3, m) do { \
    if (__builtin_expect(seen <= (m) + 1u, 0)) lds_wait(c, C_FULL0 + sl, (m) + 2u); \
    asm volatile(MF("%0", "%9", "%8") "ds_read_b128 %4, %13\n\t" MF("%1", "%10", "%8") "ds_read_b128 %5, %13 offset:1024\n\t" \
                 MF("%2", "%11", "%8") "ds_read_b128 %6, %13 offset:2048\n\t" MF("%3", "%12", "%8") "ds_read_b128 %7, %13 offset:3072" \
                 : "+v"(acc[0][0]), "+v"(acc[1][0]), "+v"(acc[2][0]), "+v"(acc[3][0]), "+v"(n0), "+v"(n1), "+v"(n2), "+v"(n3) \
                 : "v"(x0), "v"(c0), "v"(c1), "v"(c2), "v"(c3), "v"(vw) : "memory"); \
    const u32 d = sl == NS - 1u ? -(NS - 1u) * SLOTB : SLOTB, d4 = sl == NS - 2u ? -(NS - 1u) * 4u : 4u; \
    sl = sl == NS - 1u ? 0u : sl + 1u; \
    asm volatile("s_waitcnt lgkmcnt(7)\n\t" MF("%0", "%7", "%6") "ds_read_b128 %4, %11\n\t" MF("%1", "%8", "%6") "ds_read_b32 %5, %12\n\t" \
                 MF("%2", "%9", "%6") MF("%3", "%10", "%6") \
                 : "+v"(acc[0][1]), "+v"(acc[1][1]), "+v"(acc[2][1]), "+v"(acc[3][1]), "+v"(x0), "=&v"(vs) \
                 : "v"(x1), "v"(c0), "v"(c1), "v"(c2), "v"(c3), "v"(vx), "v"(vf) : "memory"); \
    asm volatile("s_waitcnt lgkmcnt(8)\n\t" MF("%0", "%8", "%7") "ds_read_b128 %4, %12 offset:1024\n\t" MF("%1", "%9", "%7") "v_add_u32 %5, %13, %5\n\t" \
                 MF("%2", "%10", "%7") "v_add_u32 %6, %14, %6\n\t" MF("%3", "%11", "%7") \
                 : "+v"(acc[0][2]), "+v"(acc[1][2]), "+v"(acc[2][2]), "+v"(acc[3][2]), "+v"(x1), "+v"(vw), "+v"(vf) \
                 : "v"(x2), "v"(c0), "v"(c1), "v"(c2), "v"(c3), "v"(vx), "s"(d), "s"(d4) : "memory"); \
    asm volatile("s_waitcnt lgkmcnt(8)\n\t" MF("%0", "%9", "%5") "ds_read_b128 %4, %6 offset:2048\n\tv_add_u32 %7, 1, %7\n\t" MF("%1", "%10", "%5") \
                 "s_waitcnt lgkmcnt(2)\n\tv_readfirstlane_b32 %8, %13\n\t" MF("%2", "%11", "%5") MF("%3", "%12", "%5") \
                 "ds_read_b128 %5, %6 offset:3072\n\tds_write_b32 %14, %7\n\tv_add_u32 %6, %15, %6" \
                 : "+v"(acc[0][3]), "+v"(acc[1][3]), "+v"(acc[2][3]), "+v"(acc[3][3]), "+v"(x2), "+v"(x3), "+v"(vx), "+v"(done), "=&s"(seen) \
                 : "v"(c0), "v"(c1), "v"(c2), "v"(c3), "v"(vs), "v"(relad), "s"(d) : "memory"); \
  } while (0)
  lds_wait(c, C_FULL0, 1u);
  asm volatile("ds_read_b128 %0, %9\n\tds_read_b128 %1, %9 offset:1024\n\tds_read_b128 %2, %9 offset:2048\n\tds_read_b128 %3, %9 offset:3072\n\t"
               "ds_read_b128 %4, %8\n\tds_read_b128 %5, %8 offset:1024\n\tds_read_b128 %6, %8 offset:2048\n\tds_read_b128 %7, %8 offset:3072\n\t"
               "ds_write_b32 %10, %11\n\ts_waitcnt lgkmcnt(0)"
               : "=&v"(wa0), "=&v"(wa1), "=&v"(wa2), "=&v"(wa3), "=&v"(x0), "=&v"(x1), "=&v"(x2), "=&v"(x3) : "v"(vx), "v"(vw), "v"(relad), "v"(done) : "memory");
  vx += SLOTB; vw += SLOTB;
  wb0 = wa0; wb1 = wa0; wb2 = wa0; wb3 = wa0;
  asm volatile("s_nop 1" : "+v"(vx), "+v"(vw), "+v"(wb0), "+v"(wb1), "+v"(wb2), "+v"(wb3));
  for (u32 m = 0; m < 126u; m += 2u) {
    HSTEP(wa0, wa1, wa2, wa3, wb0, wb1, wb2, wb3, m);
    HSTEP(wb0, wb1, wb2, wb3, wa0, wa1, wa2, wa3, m + 1u);
  }
  HSTEP(wa0, wa1, wa2, wa3, wb0, wb1, wb2, wb3, 126u);
#define MF4(b, x) asm volatile(MF("%0", "%5", "%4") MF("%1", "%6", "%4") MF("%2", "%7", "%4") MF("%3", "%8", "%4") \
    : "+v"(acc[0][b]), "+v"(acc[1][b]), "+v"(acc[2][b]), "+v"(acc[3][b]) : "v"(x), "v"(wb0), "v"(wb1), "v"(wb2), "v"(wb3))
  asm volatile("s_waitcnt lgkmcnt(0)" ::: "memory");
  MF4(0, x0); MF4(1, x1); MF4(2, x2); MF4(3, x3);
  asm volatile("s_nop 7\n\ts_nop 7" : "+v"(acc[0][2]), "+v"(acc[1][2]), "+v"(acc[2][2]), "+v"(acc[3][2]), "+v"(acc[0][3]), "+v"(acc[1][3]), "+v"(acc[2][3]), "+v"(acc[3][3]),
               "+v"(acc[0][0]), "+v"(acc[1][0]), "+v"(acc[2][0]), "+v"(acc[3][0]), "+v"(acc[0][1]), "+v"(acc[1][1]), "+v"(acc[2][1]), "+v"(acc[3][1]));
  if (wv == 0) STAMP(c, 11);
  const rsrc_t rsL = mkrs(c.ws, WS_USED);
  {
    float mx = 0.f;
#pragma unroll
    for (int e = 0; e < 64; e += 2) mx = fmaxf(fmaxf(mx, acc[e & 3][e >> 4][(e >> 2) & 3]), acc[(e & 3) + 1][e >> 4][(e >> 2) & 3]);
    const u32 wmx = wave_max(__float_as_uint(mx));
    if (lane == 0) {
      __hip_atomic_fetch_max(c.ctl + C_WGMAX, wmx, __ATOMIC_RELAXED, __HIP_MEMORY_SCOPE_WORKGROUP);
      CFENCE();
      if (lds_add(c.ctl + C_CMAX, 1u) == 7u)
        __hip_atomic_store((GAS u64*)(c.ws + O_LT + c.wg * 8), (u64)c.call << 32 | lds_ld(c.ctl + C_WGMAX), __ATOMIC_RELAXED, __HIP_MEMORY_SCOPE_AGENT);
    }
  }
  const u32 zs = wv >> 2, ht = c.wg * 2 + zs;
  u32 nz[2] = {0u, 0u};
#pragma unroll
  for (int e = 63; e >= 0; --e) nz[e >> 5] = __builtin_amdgcn_alignbit(nz[e >> 5], __float_as_uint(acc[e & 3][e >> 4][(e >> 2) & 3]), 31);
  nz[0] = ~nz[0]; nz[1] = ~nz[1];
  __builtin_amdgcn_raw_buffer_store_b64((u32x2){nz[0], nz[1]}, rsL, lane * 8, (int)(O_OM + c.wg * 16384 + wv * 512), SYS);
  const rsrc_t rsQ = mkrs(c.ws + O_MQ, 4u);
  u32 mqv = 0;
  if (wv == 7) mqv = __builtin_amdgcn_raw_ptr_buffer_atomic_add_i32(1, rsQ, lane * 4, 0, AGT);
  VMWAIT0();
  if (lane == 0) lds_add(c.ctl + C_CMASK, 1u);
  if (wv == 7) {
    lds_wait(c, C_CMASK, 8u);
    tell_peers(c, O_OF + c.rank * 8192 + F_MF + (RFL(mqv) - (c.call - 1u) * 768u) % 768u * 4u, c.call << 20 | (512u + c.wg) << 9 | 256u);
  }
  const u32 cnt = __popc(nz[0]) + __popc(nz[1]), inc = wave_incl_scan(cnt), V = __builtin_amdgcn_readlane(inc, 63);
  u32 o0 = ldsa(c.lds) + L_RING + wv * 4096 + inc - cnt;
  Geo G = geo(V);
  const bool in8 = (u32)lane < G.nc, in9 = 8u * G.nc + lane < G.sz >> 2;
  lds_wait(c, C_CMAX, 8u);
  if (wv == 0) STAMP(c, 56);
  LAS unsigned char* st = c.lds + L_RING + wv * 4096;
  u32 ob[9], base;
  bool hint;
  {
#pragma unroll
    for (int k = 0; k < 4; ++k) *(LAS u32x4*)(st + k * 1024 + lane * 16) = (u32x4){0u, 0u, 0u, 0u};
    u32 o = o0, se = 0, carry = 0;
#pragma unroll
    for (int z = 0; z < 16; z += 8) {
      u32 sel[8];
#pragma unroll
      for (int D = 0; D < 8; ++D) sel[D] = put_sel(c, nib4(nz[z >> 3], D));
#pragma unroll
      for (int D = 0; D < 8; ++D) {
        const int b = (z + D) >> 2, r = D & 3;
        const u32 h = prm(prm(__float_as_uint(acc[3][b][r]) << 1, __float_as_uint(acc[2][b][r]) << 1, 0x07030000u),
                          prm(__float_as_uint(acc[1][b][r]) << 1, __float_as_uint(acc[0][b][r]) << 1, 0x00000703u), 0x07060100u);
        se = __builtin_amdgcn_sad_u8(put4(o, carry, h, nib4(nz[z >> 3], D), sel[D]), 0u, se);
      }
    }
    or_at(o, carry);
    base = min(max(pick_top(__builtin_amdgcn_readlane(wave_incl_scan(se), 63), V), 13u) - 13u, 240u);
    if (V + lane < G.sz) st[V + lane] = (unsigned char)(base + 13u);
    CFENCE();
    if (wv == 0) STAMP(c, 57);
    if ((wv & 3) == 0) mqv = __builtin_amdgcn_raw_ptr_buffer_atomic_add_i32(1, rsQ, lane * 4, 0, AGT);
    u32 n[9], any = 0, l2a, l2b, l3, sb = 0;
    const u16x2 b2 = {(unsigned short)base, (unsigned short)base};
#pragma unroll
    for (u32 k = 0; k < 9u; ++k) ob[k] = *(LAS u32*)(st + (k * G.nc + lane) * 4u);
#pragma unroll
    for (u32 k = 0; k < 9u; ++k) {
      const u32 lo = __builtin_bit_cast(u32, __builtin_elementwise_sub_sat(__builtin_bit_cast(u16x2, ob[k] & 0x00FF00FFu), b2));
      const u32 hi = __builtin_bit_cast(u32, __builtin_elementwise_sub_sat(__builtin_bit_cast(u16x2, (ob[k] >> 8) & 0x00FF00FFu), b2));
      ob[k] = (k < 8u ? in8 : in9) ? lo | hi << 8 : 13u * B1;
      any |= ob[k];
      n[k] = (ob[k] + 2u * B1) & 15u * B1;
    }
    Lv g;
    hint = V <= 2304u && !ANY((any & 0xF0u * B1) != 0u) && enc_levels<9, 1024>(c, n, st + G.sz, 4096u - G.sz, g, l2a, l2b, l3);
    CFENCE();
    if (wv == 0) STAMP(c, 58);
    if (hint) {
      LAS u32* s = (LAS u32*)st + lane;
      if (in8) { s[0] = n[0] | n[1] << 2 | n[2] << 4 | n[3] << 6; s[G.nc] = n[4] | n[5] << 2 | n[6] << 4 | n[7] << 6; }
      if (in9) s[2u * G.nc] = n[8];
      s += (G.sz >> 2) - 6u * G.nc;
      if ((u32)lane < g.nc2) s[0] = l2a;
      if ((u32)lane < g.nc3) s[g.nc2] = l3;
      sb = (G.sz - 24u * G.nc + (g.nc2 + g.nc3) * 4u + 15u) & ~15u;
    }
    if ((wv & 3) == 0 && lane == 0) lds_st(c.ctl + C_MQ0 + zs, mqv);
    u32x4 hdr;
    const u32 omb = O_OM + piece_off(ht), woff = place(c, wv, C_CSZ0 + zs, 16u, sb, base << 16, hdr);
    if (wv == 0) STAMP(c, 59);
    if ((wv & 3) == 0 && lane == 0) st128<SYS>(hdr, rsL, omb);
    for (u32 k = lane * 16; k < sb; k += 1024u) st128<SYS>(*(LAS u32x4*)(st + k), rsL, k + (omb + woff));
    VMWAIT0();
    u32 t = 0;
    if (lane == 0) t = lds_add(c.ctl + C_CPRE0 + zs, 1u);
    if (RFL(t) == 3u) {
      const u32 mq = (RFL(lds_ld(c.ctl + C_MQ0 + zs)) - (c.call - 1u) * 768u) % 768u;
      tell_peers(c, O_OF + c.rank * 8192 + F_MF + mq * 4u, c.call << 20 | ht << 9 | (hdr.w & 0xFFFFu) >> 4);
    }
  }
  if (wv == 0) STAMP(c, 28);
  asm volatile("" : "+v"(nz[0]), "+v"(nz[1]), "+v"(o0), "+s"(G.nc));
  lds_wait(c, C_CPRE0 + zs, 4u);
#pragma unroll
  for (int k = 0; k < 4; ++k) *(LAS u32x4*)(st + k * 1024 + lane * 16) = (u32x4){0u, 0u, 0u, 0u};
  u32 sel0[16];
#pragma unroll
  for (int D = 0; D < 16; ++D) sel0[D] = put_sel(c, nib4(nz[D >> 3], D & 7));
#pragma unroll
  for (int g = 0; g < 4; ++g)
#pragma unroll
    for (int b = 0; b < 4; ++b) {
#pragma unroll
      for (int r = 0; r < 4; ++r) acc[g][b][r] = __builtin_amdgcn_fmed3f(acc[g][b][r], 0.f, __builtin_inff());
      asm volatile("" : "+v"(acc[g][b]));
    }
  lds_wait(c, C_GVALID, 1u);
  if (wv == 0) STAMP(c, 31);
  const u32 tm = g_ld_agent(c.ws + O_TMO);
  const float amax = __uint_as_float(RFL(lds_ld(c.ctl + C_GMAX)));
  const float inv = amax > 0.f ? 448.0f / amax : 0.f;
  const Scale sc = split(__uint_as_float(RFL(__float_as_uint(inv))));
  const u32 r8 = (8u - sc.r) * B1;
  u32 q[16];
#pragma unroll
  for (int b = 0; b < 4; ++b) {
    const f32x4 a0 = acc[0][b] * inv, a1 = acc[1][b] * inv, a2 = acc[2][b] * inv, a3 = acc[3][b] * inv;
#pragma unroll
    for (int r = 0; r < 4; ++r)
      q[4 * b + r] = (u32)__builtin_amdgcn_cvt_pk_fp8_f32(a2[r], a3[r], __builtin_amdgcn_cvt_pk_fp8_f32(a0[r], a1[r], 0, false), true);
  }
  {
    u32 o = o0, carry = 0, sb;
#pragma unroll
    for (int D = 0; D < 16; ++D) put4(o, carry, q[D], nib4(nz[D >> 3], D & 7), sel0[D]);
    or_at(o, carry);
    const int kap = min(max((int)base + sc.kap, -16), 16);
    if (V + lane < G.sz) st[V + lane] = (unsigned char)((((u32)(13 + kap) + (sc.r == 7u)) & 15u) << 3 | ((sc.r + 1u) & 7u));
    CFENCE();
    u32 d[9], nb[2] = {0u, 0u}, bad = 0, nca = 0;
    if (hint) {
      u32 sel[9];
#pragma unroll
      for (u32 k = 0; k < 9u; ++k) d[k] = *(LAS u32*)(st + (k * G.nc + lane) * 4u);
#pragma unroll
      for (u32 k = 0; k < 9u; ++k) {
        u32 pos, up;
        const u32 u = low_exp(ob[k], (u32)(kap + 32) * B1, pos), df = ((d[k] >> 3) & 15u * B1) + 32u * B1 - u;
        d[k] &= 7u * B1;
        const u32 su = sure(d[k], r8, pos, up), w = ((df ^ 32u * B1) & ~B1) | ((df ^ up) & su);
        bad |= (k < 8u ? in8 : in9) ? w : 0u;
        nb[k >> 3] |= to_nib(su) << (4u * (k & 7u));
        ob[k] = df & B1;
      }
      nb[0] = in8 ? ~nb[0] : 0u; nb[1] = in9 ? ~nb[1] & 15u : 0u;
#pragma unroll
      for (u32 k = 0; k < 9u; ++k) sel[k] = put_sel(c, nib4(nb[k >> 3], k & 7u));
      const u32 cn = __popc(nb[0]) + __popc(nb[1]), in2 = wave_incl_scan(cn), Va = __builtin_amdgcn_readlane(in2, 63);
      nca = (Va + 31u) >> 5;
      hint = !ANY(bad != 0u) && G.sz + nca * 32u + 8u <= 4096u;
      if (hint) {
        o = ldsa(st) + G.sz + in2 - cn; carry = 0;
#pragma unroll
        for (u32 k = 0; k < 9u; ++k) put4(o, carry, ob[k], nib4(nb[k >> 3], k & 7u), sel[k]);
        or_at(o, carry);
      }
    }
    CFENCE();
    if (hint) {
      u32 bits = 0;
#pragma unroll
      for (u32 k = 0; k < 8u; ++k) bits |= *(LAS u32*)(st + G.sz + (k * nca + lane) * 4u) << k;
      CFENCE();
      LAS u32* s = (LAS u32*)st + lane;
      if (in8) {
        s[0] = d[0] | d[1] << 3 | ((d[6] << 6) & 0xC0u * B1); s[G.nc] = d[2] | d[3] << 3 | ((d[7] << 6) & 0xC0u * B1);
        s[2u * G.nc] = d[4] | d[5] << 3 | ((d[6] << 4) & 0x40u * B1) | ((d[7] << 5) & 0x80u * B1);
      }
      if (in9) s[3u * G.nc] = d[8];
      if ((u32)lane < nca) s[(G.sz >> 2) - 5u * G.nc] = bits;
      sb = (G.sz - 20u * G.nc + nca * 4u + 15u) & ~15u;
    } else {
      const u32 nc = row_bytes(V) >> 2;
      for (u32 col = lane; col < nc; col += 64u) {
        LAS u32* s = (LAS u32*)st + col;
        const u32 d7 = s[7u * nc];
#pragma unroll
        for (u32 k = 0; k < 7u; ++k) s[k * nc] |= (d7 << (7u - k)) & 0x80u * B1;
      }
      sb = (28u * nc + 15u) & ~15u;
    }
    u32x4 hdr;
    const u32 ovb = O_OV + ht * 16384, woff = place(c, wv, C_CSZ2 + zs, 16u, sb, (u32)hint << 16, hdr);
    if ((wv & 3) == 0 && lane == 0) st128<SYS>(hdr, rsL, ovb);
    for (u32 k = lane * 16; k < sb; k += 1024u) st128<SYS>(*(LAS u32x4*)(st + k), rsL, k + (ovb + woff));
    VMWAIT0();
    u32 t = 0;
    if (lane == 0) t = lds_add(c.ctl + C_CVAL0 + zs, 1u);
    if (RFL(t) == 3u) tell_peers(c, O_OF + c.rank * 8192 + F_VF + ht * 4, c.call << 20 | (hdr.w & 0xFFFFu) >> 4);
  }
  if (wv == 0) STAMP(c, 12);
  const rsrc_t rsO = mkrs(p.out, (u32)MM * NN);
  const u32x4 nan4 = {0x7Fu * B1, 0x7Fu * B1, 0x7Fu * B1, 0x7Fu * B1};
  {
    const u32 obase = (u32)(mi * 256 + mw * 64 + (lane & 15)) * NN + c.rank * 2048 + ni * 128 + h * 64 + (lane >> 4) * 16;
    const bool bad = tm == c.call || lds_ld(c.ctl + C_ABORT) != 0u;
#pragma unroll
    for (int b = 0; b < 4; ++b) st128<OUTAUX>(bad ? nan4 : (u32x4){q[4 * b], q[4 * b + 1], q[4 * b + 2], q[4 * b + 3]}, rsO, obase + b * 16 * NN);
  }
}

DI void role_outdec(Ctx& c, const Params& p, int wv) {
  const int lane = c.lane;
  __builtin_amdgcn_s_setprio(0);
  c.call = lds_wait(c, C_CALL, 1u);
  lds_wait(c, C_GVALID, 1u);
  const float amax = __uint_as_float(RFL(lds_ld(c.ctl + C_GMAX)));
  const Scale sc = split(__uint_as_float(RFL(__float_as_uint(amax > 0.f ? 448.0f / amax : 0.f))));
  const u32 r8 = (8u - sc.r) * B1;
  const rsrc_t rsL = mkrs(c.ws, WS_USED), rsO = mkrs(p.out, (u32)MM * NN);
  const u32x4 nan4 = {0x7Fu * B1, 0x7Fu * B1, 0x7Fu * B1, 0x7Fu * B1};
  LAS unsigned char* st = c.lds + L_RING + (wv < 8 ? wv * 4096 : 98304 + (wv - 8) * 4096);
  const u32 src = (c.rank + 1 + wg_link(c.wg)) & 7;
  for (u32 kv = wv >> 2;; kv += 3u) {
    const u32 e = q_wait(c, C_QV, kv), dh = e & 0xFFFu, pb = ((e >> 12) & 2047u) * 32768u;
    if (dh >= 512u) break;
    const u32 dt = dh >> 1, dw = (dh & 1u) * 4u + (wv & 3), inb = O_MIN + src * 256 * 16384 + piece_off(dh);
    dma1(rsL, lane * 16, inb + 16u + (wv & 3) * 1280u, (u32)(size_t)st + 2560u);
    if (lane < 16) dma1(rsL, lane * 16, inb + 1040u + (wv & 3) * 1280u, (u32)(size_t)st + 3584u);
    const u32x2 m = __builtin_amdgcn_raw_buffer_load_b64(rsL, lane * 8, (int)(O_MIN + (src * 256 + dt) * 16384u + dw * 512u), AGT);
    const u32 he = __builtin_amdgcn_raw_buffer_load_b32(rsL, 0, (int)(inb + (wv & 3) * 4), AGT);
    const u32 dbase = (u32)(tile_mi(dt) * 256 + (dw >> 1) * 64 + (lane & 15)) * NN + src * 2048 + tile_ni(dt) * 128 + (dw & 1) * 64 + (lane >> 4) * 16;
    const u32 cn = __popc(m.x) + __popc(m.y), in2 = wave_incl_scan(cn), V2 = __builtin_amdgcn_readlane(in2, 63);
    const Geo G2 = geo(V2);
    const bool j8 = (u32)lane < G2.nc, j9 = 8u * G2.nc + lane < G2.sz >> 2;
    u32 u[9], pos[9];
    VMWAIT0();
    {
      LAS u32* s = (LAS u32*)(st + 2560) + lane;
      const u32 ra = s[0], rb = s[G2.nc], rc = s[2u * G2.nc], l1[3] = {j8 ? ra : ~0u, j8 ? rb : ~0u, j9 ? rc : ~0u};
      const u32 k32 = (u32)(min(max((int)((he >> 16) & 255u) + sc.kap, -16), 16) + 32) * B1;
      dec_levels<9, 1024>(c, l1, k32, u, st + 2560 + G2.sz - 24u * G2.nc, st, 2560u);
#pragma unroll
      for (u32 k = 0; k < 9u; ++k) u[k] = low_exp(u[k], 0u, pos[k]);
    }
    LAS unsigned char* sl = c.lds + L_STG + (kv & 3u) * 16384u;
    lds_wait(c, C_PROG, pb + 16u);
    const u32 hw = RFL(*(LAS u32*)(sl + (wv & 3) * 4)), hp = RFL(*(LAS u32*)(sl + ((wv + 3) & 3) * 4));
    const u32 wo = (wv & 3) ? min(hp & 0xFFF0u, 16384u - 3584u) : 16u;
    lds_wait(c, C_PROG, pb + (hw & 0xFFFFu));
    const u32 tm2 = g_ld_agent(c.ws + O_TMO);
    if ((hw >> 16) & 1u) {
      LAS u32* sg = (LAS u32*)(sl + wo) + lane;
      const u32 ra = sg[0], rb = sg[G2.nc], rc = sg[2u * G2.nc], bits = sg[(G2.sz >> 2) - 5u * G2.nc];
      u32 y[9], nb[2] = {0u, 0u};
      y[8] = sg[3u * G2.nc] & 7u * B1;
      y[0] = ra & 7u * B1; y[1] = (ra >> 3) & 7u * B1; y[2] = rb & 7u * B1; y[3] = (rb >> 3) & 7u * B1; y[4] = rc & 7u * B1; y[5] = (rc >> 3) & 7u * B1;
      y[6] = ((ra >> 6) & 3u * B1) | ((rc >> 4) & 4u * B1); y[7] = ((rb >> 6) & 3u * B1) | ((rc >> 5) & 4u * B1);
#pragma unroll
      for (u32 k = 0; k < 9u; ++k) {
        u32 up;
        const u32 su = sure(y[k], r8, pos[k], up);
        u[k] += up & su;
        nb[k >> 3] |= to_nib(su) << (4u * (k & 7u));
      }
      nb[0] = j8 ? ~nb[0] : 0u; nb[1] = j9 ? ~nb[1] & 15u : 0u;
      const u32 ca = __popc(nb[0]) + __popc(nb[1]), ia = wave_incl_scan(ca), nca = min((__builtin_amdgcn_readlane(ia, 63) + 31u) >> 5, 64u);
      if ((u32)lane < nca) {
#pragma unroll
        for (u32 k = 0; k < 8u; ++k) *(LAS u32*)(st + (k * nca + lane) * 4u) = (bits >> k) & B1;
      }
      CFENCE();
      u32 bb[9];
      get_all<9>(c, ldsa(st) + ia - ca, nb, bb);
#pragma unroll
      for (u32 k = 0; k < 9u; ++k) y[k] |= (u[k] + bb[k]) << 3;
      CFENCE();
      if (j8) {
#pragma unroll
        for (u32 k = 0; k < 8u; ++k) *(LAS u32*)(st + (k * G2.nc + lane) * 4u) = y[k];
      }
      if (j9) *(LAS u32*)(st + (8u * G2.nc + lane) * 4u) = y[8];
    } else {
      const u32 nc = row_bytes(V2) >> 2;
      for (u32 col = lane; col < nc; col += 64u) {
        LAS u32* sg = (LAS u32*)(sl + wo) + col;
        LAS u32* d = (LAS u32*)st + col;
        u32 d7 = 0;
#pragma unroll
        for (u32 k = 0; k < 7u; ++k) { const u32 v = sg[k * nc]; d7 |= (v & 0x80u * B1) >> (7u - k); d[k * nc] = v & 0x7Fu * B1; }
        d[7u * nc] = d7;
      }
    }
    CFENCE();
    if (lane == 0) lds_add(c.ctl + C_DFREE0 + (kv & 3u), 1u);
    CFENCE();
    u32 ov[16];
    const u32 mm[2] = {m.x, m.y};
    get_all<16>(c, ldsa(st) + in2 - cn, mm, ov);
    const bool bad = tm2 == c.call || lds_ld(c.ctl + C_ABORT) != 0u;
#pragma unroll
    for (int k = 0; k < 4; ++k) st128<OUTAUX>(bad ? nan4 : (u32x4){ov[4 * k], ov[4 * k + 1], ov[4 * k + 2], ov[4 * k + 3]}, rsO, dbase + k * 16 * NN);
    CFENCE();
  }
}

__global__ __launch_bounds__(1024, 4) void fused_gemm_ar(Params p) {
  extern __shared__ __attribute__((aligned(16))) unsigned char smem[];
  Ctx c;
  c.lds = (LAS unsigned char*)smem; c.ctl = (LAS u32*)c.lds; c.ws = p.ws[p.rank]; c.t0 = now(); c.call = 0;
  c.lane = threadIdx.x & 63; c.wg = blockIdx.x; c.rank = p.rank;
  const int wv = RFL(threadIdx.x >> 6), tid = threadIdx.x;
  if (tid >= 512) *(LAS u32*)(c.lds + L_REL + (tid - 512) * 4) = 0u;
  else if (tid < C_WORDS || (tid >= L_GT / 4 && tid < L_GT / 4 + 16)) c.ctl[tid] = 0u;
  else if (tid >= 192 && tid < 256) {
    const u32 nib = tid & 15; u32 s = 0, k = (tid - 192) >> 4;
    for (int g = 0; g < 4; ++g) { if ((nib >> g) & 1u) s |= k++ << (8 * g); else s |= 0x0Cu << (8 * g); }
    *(LAS u32*)(c.lds + L_LUTE + (tid - 192) * 4) = s;
  } else if (tid >= 288 && tid < 288 + NR) {
    u64 v = 0;
#pragma unroll
    for (int i = 0; i < NR; ++i) { u64 t = (u64)p.ws[i]; asm volatile("" : "+s"(t)); v = tid == 288 + i ? t : v; }
    *(LAS u64*)(c.lds + L_PEER + (tid - 288) * 8) = v;
  } else if (tid >= 256 && tid < 272) {
    const u32 nib = tid - 256; u32 s = 0x0C0C0C0Cu, k = 0;
    for (u32 g = 0; g < 4; ++g) if ((nib >> g) & 1u) { s = (s & ~(255u << (8 * k))) | g << (8 * k); ++k; }
    *(LAS u32*)(c.lds + L_LUTC + nib * 4) = s;
  }
  asm volatile("s_waitcnt lgkmcnt(0)" ::: "memory");
  __builtin_amdgcn_s_barrier();
  CFENCE();
  const int mi = tile_mi(c.wg), ni = tile_ni(c.wg);
  if (wv == 11) STAMP(c, 0);
  if (wv < 8) role_mfma(c, p, wv, mi, ni);
  else if (wv < 13 - NDEC) role_loader(c, wv - 8, mi, ni);
  else if (wv == 11) role_sched(c, p, mi, ni);
  else if (wv == 12) role_puller(c, p);
  else if (wv == 13) role_feeder(c);
  else role_decoder(c, wv == 10 ? 2 : wv - 14);
  if (wv < 12) role_outdec(c, p, wv);
}
}

extern "C" void kernel_launch(int rank, int nranks, ncclComm_t comm, hipStream_t stream,
                              void* const* d_in, const int* in_sizes, int n_in,
                              void* d_out, int out_size, const int* in_dtypes, int out_dtype,
                              void* const* d_out_all, void* const* d_ws_all, size_t ws_bytes) {
  if (nranks != NR || (unsigned)rank >= (unsigned)NR || n_in < 2 || ws_bytes < WS_USED || in_sizes[0] != MM * KS || in_sizes[1] != KS * NN || out_size != MM * NN) {
    fprintf(stderr, "kernel_launch: unexpected problem shape\n");
    return;
  }
  static bool init = false;
  if (!init) {
    hipError_t e = hipFuncSetAttribute((const void*)fused_gemm_ar, hipFuncAttributeMaxDynamicSharedMemorySize, (int)LDS_BYTES);
    if (e != hipSuccess) { fprintf(stderr, "hipFuncSetAttribute: %s\n", hipGetErrorString(e)); return; }
    init = true;
  }
  Params p;
  p.x = (const unsigned short*)d_in[0]; p.w = (const unsigned short*)d_in[1]; p.out = (unsigned char*)d_out;
  for (int i = 0; i < NR; ++i) p.ws[i] = (char*)d_ws_all[i];
  p.rank = rank; p.pad = 0;
  fused_gemm_ar<<<NWG, 1024, LDS_BYTES, stream>>>(p);
}
```

```cpp
#include <hip/hip_runtime.h>
#include <rccl/rccl.h>
#include <cstdio>

typedef unsigned u32;
typedef unsigned long long u64;
typedef u32 u32x4 __attribute__((ext_vector_type(4)));
typedef u32 u32x2 __attribute__((ext_vector_type(2)));
typedef unsigned short u16x2 __attribute__((ext_vector_type(2)));
typedef short bf16x8 __attribute__((ext_vector_type(8)));
typedef float f32x4 __attribute__((ext_vector_type(4)));
typedef __amdgpu_buffer_rsrc_t rsrc_t;
#define LAS __attribute__((address_space(3)))
#define GAS __attribute__((address_space(1)))
#define DI __device__ __forceinline__
#define RFL(x) __builtin_amdgcn_readfirstlane(x)
#define ALL(x) (__builtin_amdgcn_ballot_w64(!(x)) == 0ull)
#define ANY(x) (__builtin_amdgcn_ballot_w64(x) != 0ull)
#define CFENCE() asm volatile("" ::: "memory")
#define VMWAIT0() asm volatile("s_waitcnt vmcnt(0)" ::: "memory")

namespace {
constexpr int NR = 8, MM = 4096, KS = 512, NN = 16384, NWG = 256;
constexpr int NBLK = 768;
constexpr u32 SYS = 17, AGT = 16;
constexpr int LEADER = 16;
constexpr u64 DEADLINE = 100000000ull;
constexpr u32 O_CALL = 0;
constexpr u32 O_TMO = 32768;
constexpr u32 O_TICK = O_TMO + 128;
constexpr u32 O_MQ = O_TICK + 3072;
constexpr u32 O_GT = O_MQ + 128;
constexpr u32 O_LT = O_GT + 128;
constexpr u32 O_RF = O_LT + 2048;
constexpr u32 O_FX = O_RF + NR * NBLK * 4;
constexpr u32 O_FW = O_FX + 16 * 64 * 16;
constexpr u32 O_OF = O_FW + 16 * 64 * 8;
constexpr u32 F_MF = 0;
constexpr u32 F_MI = 4096;
constexpr u32 F_VF = 6144;
constexpr u32 O_DBG = 512 * 1024;
constexpr u32 O_OBX = 1u << 20;
constexpr u32 O_OBW = O_OBX + 512 * 8192;
constexpr u32 O_IMG = O_OBW + 2048 * 8192;
constexpr u32 O_OM = O_IMG + 64 * 96 * 8192;
constexpr u32 O_OV = O_OM + 256 * 16384;
constexpr u32 O_MIN = O_OV + 512 * 16384;
constexpr u32 WS_USED = O_MIN + NR * 256 * 16384;
__host__ __device__ constexpr u32 piece_off(u32 pc) { return (pc & 255u) * 16384u + (pc < 256u ? 4096u : 0u); }
static_assert(O_OF + NR * 8192 <= O_DBG, "control block overflow");
constexpr u32 T_MASK = NBLK, T_VAL = T_MASK + 512, T_END = T_VAL + 512;
enum { C_CALL = 0, C_ABORT, C_WGMAX, C_CMAX, C_CMASK, C_CVAL0, C_CVAL1, C_GVALID, C_GMAX, C_PSTART, C_PROG, C_PACKED,
       C_FULL0 = 12, C_LEAD = 17, C_CSZ0 = 18, C_CSZ2 = 20, C_CPRE0 = 22, C_DFREE0 = 24, C_IFREE0 = 28, C_VSZ = 32, C_QI = 40, C_QV = 48, C_MQ0 = 56, C_ORDER = 64, C_WORDS = 128 };
constexpr u32 L_LUTC = C_WORDS * 4;
constexpr u32 L_GT = L_LUTC + 128;
constexpr u32 L_PEER = L_GT + 64;
constexpr u32 L_LUTE = L_LUTC + 256;
constexpr u32 L_REL = 1024;
constexpr u32 L_RFC = L_REL + 2048;
constexpr u32 NS = 5, SLOTB = 24576;
constexpr u32 L_RING = L_RFC + 4096;
constexpr u32 L_STG = L_RING + 32768;
constexpr u32 L_IN = L_RING + NS * SLOTB;
constexpr u32 LDS_BYTES = L_IN + 4 * 8192;
static_assert(LDS_BYTES <= 163840 && LDS_BYTES > 81920 && L_STG + 5 * 16384 <= L_IN, "one workgroup per CU");
#define PLEAD 2048
#define WIN1 2
#define WIN2 4
#define T_HALF 352
#define PACK_T0 200
#define PACK_DT 100
#define NRAW 32
#define T_W1 728
#define T_W2 T_END
#define T_WA 240u
#define G0FREE 1
#define T_AH (T_END - 80u)
#define GATEM 12
#define SCHED_Q 3u
#define OUTAUX 0u
#define NDEC 3
#define DPRIO 1
#define NPUSH 12
#define KFIX 8u
#define T_W0 160

#define SPIN_FAST 16u
#define SLEEP_LONG 1
struct Params { const unsigned short* x; const unsigned short* w; unsigned char* out; char* ws[NR]; int rank; int pad; };

DI u64 now() { return __builtin_amdgcn_s_memrealtime(); }
DI rsrc_t mkrs(const void* p, u32 bytes) { return __builtin_amdgcn_make_buffer_rsrc((void*)p, (short)0, (int)bytes, 0x00020000); }
template <u32 AUX> DI void st128(u32x4 v, rsrc_t rs, u32 off) { __builtin_amdgcn_raw_buffer_store_b128(v, rs, (int)off, 0, AUX); }
DI void st32(u32 v, rsrc_t rs, u32 off) { __builtin_amdgcn_raw_buffer_store_b32(v, rs, (int)off, 0, SYS); }
DI u32 prm(u32 a, u32 b, u32 s) { return __builtin_amdgcn_perm(a, b, s); }
DI u32 bfi(u32 m, u32 a, u32 b) { return (m & a) | (~m & b); }
DI u32 lds_ld(LAS u32* p) { return __hip_atomic_load(p, __ATOMIC_RELAXED, __HIP_MEMORY_SCOPE_WORKGROUP); }
DI void lds_st(LAS u32* p, u32 v) { __hip_atomic_store(p, v, __ATOMIC_RELAXED, __HIP_MEMORY_SCOPE_WORKGROUP); }
DI u32 lds_add(LAS u32* p, u32 v) { return __hip_atomic_fetch_add(p, v, __ATOMIC_RELAXED, __HIP_MEMORY_SCOPE_WORKGROUP); }
DI void lds_or(LAS unsigned char* p, u32 v) { __hip_atomic_fetch_or((LAS u32*)p, v, __ATOMIC_RELAXED, __HIP_MEMORY_SCOPE_WORKGROUP); }
DI u32 g_ld_agent(const char* p) { return __hip_atomic_load((GAS u32*)p, __ATOMIC_RELAXED, __HIP_MEMORY_SCOPE_AGENT); }
DI u32 g_ld_sys(const char* p) { return __hip_atomic_load((GAS u32*)p, __ATOMIC_RELAXED, __HIP_MEMORY_SCOPE_SYSTEM); }
DI void g_st_agent(char* p, u32 v) { __hip_atomic_store((GAS u32*)p, v, __ATOMIC_RELAXED, __HIP_MEMORY_SCOPE_AGENT); }
DI void g_st_sys(char* p, u32 v) { __hip_atomic_store((GAS u32*)p, v, __ATOMIC_RELAXED, __HIP_MEMORY_SCOPE_SYSTEM); }

struct Ctx {
  LAS unsigned char* lds; LAS u32* ctl; char* ws; u64 t0; u32 call; int lane, wg, rank;
};
DI void give_up(const Ctx& c) { if (c.lane == 0) { lds_st(c.ctl + C_ABORT, 1u); if (c.call) g_st_agent(c.ws + O_TMO, c.call); } }
DI bool expired(const Ctx& c, u32& spins) {
  if ((++spins & 15u) != 0u) return false;
  if (RFL(lds_ld(c.ctl + C_ABORT)) != 0u || RFL(g_ld_agent(c.ws + O_TMO)) == c.call || now() - c.t0 > DEADLINE) { give_up(c); return true; }
  return false;
}
DI bool lds_expired(const Ctx& c, u32& spins) {
  if ((++spins & 63u) == 0u && (RFL(lds_ld(c.ctl + C_ABORT)) != 0u || now() - c.t0 > DEADLINE + DEADLINE / 4)) { give_up(c); return true; }
  if (spins < SPIN_FAST) __builtin_amdgcn_s_sleep(1); else __builtin_amdgcn_s_sleep(SLEEP_LONG);
  return false;
}
DI u32 lds_wait(const Ctx& c, int word, u32 target) {
  u32 v, spins = 0;
  while ((v = RFL(lds_ld(c.ctl + word))) < target) if (lds_expired(c, spins)) break;
  CFENCE();
  return v;
}
DI u32 q_wait(const Ctx& c, int q, u32 k) {
  u32 v, spins = 0;
  while (((((v = RFL(lds_ld(c.ctl + q + (k & 7u)))) >> 23) - (k + 1u)) & 256u) != 0u) if (lds_expired(c, spins)) { v = ~0u; break; }
  CFENCE();
  return RFL(v);
}
#define STAMP(c, i) do {} while (0)
DI void dma4(rsrc_t rs, u32 voff, u32 soff, u32 ldsdst) {
  u32 keep;
  asm volatile("s_nop 4\n\ts_mov_b32 %0, m0\n\ts_mov_b32 m0, %1\n\ts_nop 0\n\t"
               "buffer_load_dwordx4 %2, %3, %4 offen sc0 sc1 lds\n\t"
               "buffer_load_dwordx4 %2, %3, %4 offen offset:1024 sc0 sc1 lds\n\t"
               "buffer_load_dwordx4 %2, %3, %4 offen offset:2048 sc0 sc1 lds\n\t"
               "buffer_load_dwordx4 %2, %3, %4 offen offset:3072 sc0 sc1 lds\n\t"
               "s_mov_b32 m0, %0"
               : "=&s"(keep) : "s"(RFL(ldsdst)), "v"(voff), "s"(rs), "s"(RFL(soff)) : "memory");
}
DI void dma1(rsrc_t rs, u32 voff, u32 soff, u32 ldsdst) {
  u32 keep;
  asm volatile("s_nop 4\n\ts_mov_b32 %0, m0\n\ts_mov_b32 m0, %1\n\ts_nop 0\n\t"
               "buffer_load_dwordx4 %2, %3, %4 offen sc0 sc1 lds\n\t"
               "s_mov_b32 m0, %0"
               : "=&s"(keep) : "s"(RFL(ldsdst)), "v"(voff), "s"(rs), "s"(RFL(soff)) : "memory");
}
template <int CTRL, int ROWS> DI u32 dpp0(u32 v) { return (u32)__builtin_amdgcn_update_dpp(0, (int)v, CTRL, ROWS, 0xF, false); }
DI u32 wave_incl_scan(u32 v) {
  v += dpp0<0x111, 0xF>(v); v += dpp0<0x112, 0xF>(v); v += dpp0<0x114, 0xF>(v); v += dpp0<0x118, 0xF>(v);
  v += dpp0<0x142, 0xA>(v); v += dpp0<0x143, 0xC>(v);
  return v;
}
DI u32 wave_max(u32 v) {
  v = max(v, dpp0<0x111, 0xF>(v)); v = max(v, dpp0<0x112, 0xF>(v)); v = max(v, dpp0<0x114, 0xF>(v)); v = max(v, dpp0<0x118, 0xF>(v));
  v = max(v, dpp0<0x142, 0xA>(v)); v = max(v, dpp0<0x143, 0xC>(v));
  return __builtin_amdgcn_readlane(v, 63);
}
DI int tile_mi(int wg) { return ((wg & 7) >> 1) * 4 + ((wg >> 3) & 3); }
DI int tile_ni(int wg) { return (wg & 1) * 8 + (wg >> 5); }
DI int wg_late(int wg) { return (wg >> 4) & 1; }
DI int wg_link(int wg) { return (((wg & 15) | (wg >> 5) << 4) + 3 * wg_late(wg)) % 7; }
DI u32 link_wgs(int link, int late) { return (u32)(127 + 7 - (link + 7 - 3 * late) % 7) / 7u; }
DI u32 row_bytes(u32 V) { return (((V + 7u) >> 3) + 3u) & ~3u; }
DI void tell_peers(const Ctx& c, u32 off, u32 v) { if (c.lane < NR && c.lane != c.rank) g_st_sys(*(char* LAS*)(c.lds + L_PEER + c.lane * 8) + off, v); }
DI void image_flag(const Ctx& c, int kstep, int r) {
  if (c.lane == 0) g_st_agent(c.ws + (r < 64 ? O_FX + ((r >> 2) * 64 + kstep) * 16 + (r & 3) * 4 : O_FW + (((r - 64) >> 1) * 64 + kstep) * 8 + (r & 1) * 4), c.call);
}

constexpr u32 B1 = 0x01010101u;
DI u32 nib4(u32 m, int k) { return (k ? m >> (4 * k - 2) : m << 2) & 0x3Cu; }
DI u32 to_nib(u32 bits) { return __builtin_amdgcn_udot4(bits, 0x08040201u, 0u, false); }
DI u32 put_sel(const Ctx& c, u32 n4) { return *(LAS u32*)(c.lds + L_LUTC + n4); }
DI u32 ldsa(LAS unsigned char* p) { return (u32)(size_t)p; }
DI void or_at(u32 a, u32 v) { lds_or((LAS unsigned char*)(size_t)(a & ~3u), v); }
DI u32 put4(u32& a, u32& carry, u32 v, u32 n4, u32 sel) {
  const u32 c4 = prm(0u, v, sel);
  const u64 w = (u64)c4 << ((a << 3) & 24u);
  or_at(a, (u32)w | carry);
  carry = (u32)(w >> 32);
  a += __popc(n4);
  return c4;
}
template <u32 N> DI void get_all(const Ctx& c, u32 a, const u32* m, u32 (&x)[N]) {
  u32 sel[N], hi[N];
#pragma unroll
  for (u32 k = 0; k < N; ++k) {
    const u32 n4 = nib4(m[k >> 3], k & 7u);
    sel[k] = *(LAS u32*)(c.lds + L_LUTE + (((a << 6) & 0xC0u) | n4));
    LAS u32* d = (LAS u32*)(size_t)(a & ~3u);
    x[k] = d[0]; hi[k] = d[1];
    a += __popc(n4);
  }
  __builtin_amdgcn_sched_barrier(0);
#pragma unroll
  for (u32 k = 0; k < N; ++k) x[k] = prm(hi[k], x[k], sel[k]);
}
struct Lv { u32 rows2, nc2, nc3; };
template <u32 R> DI bool enc_tail(const Ctx& c, u32 V2, LAS unsigned char* s, u32 cap, Lv& g, u32& l2a, u32& l2b, u32& l3) {
  const u32 nc2 = (V2 + 4u * R - 1u) / (4u * R), b2 = R * nc2 * 4u;
  u32 e[R], sel[R], nb3 = 0;
  l2a = 0u; l2b = 0u;
#pragma unroll
  for (u32 r = 0; r < R; ++r) e[r] = *(LAS u32*)(s + (r * nc2 + c.lane) * 4u);
#pragma unroll
  for (u32 r = 0; r < R; ++r) {
    e[r] = (u32)c.lane < nc2 ? e[r] : 0u;
    const u32 lo3 = e[r] & 7u * B1, b3 = (e[r] >> 3) & B1;
    const u32 c2 = bfi((b3 << 8) - b3, prm(0x00000002u, 0x03000000u, lo3), prm(0u, 1u, lo3));
    if (r < 4u) l2a |= c2 << (2u * r); else l2b |= c2 << (2u * r - 8u);
    nb3 |= to_nib((c2 | c2 >> 1) & B1) << (4u * r);
  }
  nb3 = ~nb3 & (R == 8u ? ~0u : 0xFFFFu);
#pragma unroll
  for (u32 r = 0; r < R; ++r) sel[r] = put_sel(c, nib4(nb3, r));
  const u32 cn3 = __popc(nb3), in3 = wave_incl_scan(cn3), V3 = __builtin_amdgcn_readlane(in3, 63);
  g.rows2 = R; g.nc2 = nc2; g.nc3 = (V3 + 7u) >> 3;
  if (V3 > 512u || b2 + g.nc3 * 8u + 8u > cap) return false;
  u32 a = ldsa(s) + b2 + in3 - cn3, carry = 0;
#pragma unroll
  for (u32 r = 0; r < R; ++r) put4(a, carry, e[r], nib4(nb3, r), sel[r]);
  or_at(a, carry);
  CFENCE();
  l3 = *(LAS u32*)(s + b2 + c.lane * 4u) | *(LAS u32*)(s + b2 + (g.nc3 + c.lane) * 4u) << 4;
  return true;
}
template <u32 N, u32 MAX2> DI bool enc_levels(const Ctx& c, u32 (&n)[N], LAS unsigned char* s, u32 cap, Lv& g, u32& l2a, u32& l2b, u32& l3) {
  u32 nb[2] = {0u, 0u}, c1[N], sel[N];
#pragma unroll
  for (u32 k = 0; k < N; ++k) {
    const u32 m = ((n[k] + 3u * B1) >> 4) & B1;
    c1[k] = n[k] & (m + (m << 1));
    nb[k >> 3] |= to_nib(m) << (4u * (k & 7u));
  }
  nb[0] = ~nb[0]; nb[1] = ~nb[1] & (N == 16u ? ~0u : (1u << (4u * (N & 7u))) - 1u);
#pragma unroll
  for (u32 k = 0; k < N; ++k) sel[k] = put_sel(c, nib4(nb[k >> 3], k & 7u));
  const u32 cnt = __popc(nb[0]) + __popc(nb[1]), inc = wave_incl_scan(cnt), V2 = __builtin_amdgcn_readlane(inc, 63);
  if (V2 > MAX2 || V2 + 40u > cap) return false;
  u32 a = ldsa(s) + inc - cnt, carry = 0;
#pragma unroll
  for (u32 k = 0; k < N; ++k) put4(a, carry, n[k], nib4(nb[k >> 3], k & 7u), sel[k]);
  or_at(a, carry);
  CFENCE();
#pragma unroll
  for (u32 k = 0; k < N; ++k) n[k] = c1[k];
  if (MAX2 > 1024u && V2 > 1024u) return enc_tail<8>(c, V2, s, cap, g, l2a, l2b, l3);
  return enc_tail<4>(c, V2, s, cap, g, l2a, l2b, l3);
}
template <u32 R> DI void dec_tail(const Ctx& c, u32 V2, LAS unsigned char* in, LAS unsigned char* s, u32 cap) {
  const u32 nc2 = (V2 + 4u * R - 1u) / (4u * R), b2 = R * nc2 * 4u;
  const bool col = (u32)c.lane < nc2;
  u32 l2[2] = {*(LAS u32*)(in + c.lane * 4u), R == 8u ? *(LAS u32*)(in + (nc2 + c.lane) * 4u) : 0u}, nb3 = 0, e[R];
  const u32 l3 = *(LAS u32*)(in + R * nc2 + c.lane * 4u);
  l2[0] = col ? l2[0] : ~0u; l2[1] = col ? l2[1] : ~0u;
#pragma unroll
  for (u32 r = 0; r < R; ++r) nb3 |= (to_nib((l2[r >> 2] | l2[r >> 2] >> 1) & B1 << (2u * (r & 3u))) >> (2u * (r & 3u))) << (4u * r);
  nb3 = ~nb3 & (R == 8u ? ~0u : 0xFFFFu);
  const u32 cn3 = __popc(nb3), in3 = wave_incl_scan(cn3), V3 = __builtin_amdgcn_readlane(in3, 63), nc3 = (V3 + 7u) >> 3;
  if (V3 > 512u || b2 + nc3 * 8u + 8u > cap) return;
  if ((u32)c.lane < nc3) {
    *(LAS u32*)(s + b2 + c.lane * 4u) = ((l3 & 15u * B1) + 14u * B1) & 15u * B1;
    *(LAS u32*)(s + b2 + (nc3 + c.lane) * 4u) = (((l3 >> 4) & 15u * B1) + 14u * B1) & 15u * B1;
  }
  CFENCE();
  get_all<R>(c, ldsa(s) + b2 + in3 - cn3, &nb3, e);
  if (col) {
#pragma unroll
    for (u32 r = 0; r < R; ++r) *(LAS u32*)(s + (r * nc2 + c.lane) * 4u) = e[r] | prm(0u, 0x090A0E00u, (l2[r >> 2] >> (2u * (r & 3u))) & 3u * B1);
  }
}
template <u32 N, u32 MAX2> DI void dec_levels(const Ctx& c, const u32* l1, u32 add, u32 (&o)[N], LAS unsigned char* in, LAS unsigned char* s, u32 cap) {
  u32 nb[2] = {0u, 0u};
#pragma unroll
  for (u32 k = 0; k < N; ++k) nb[k >> 3] |= (to_nib((l1[k >> 2] | l1[k >> 2] >> 1) & B1 << (2u * (k & 3u))) >> (2u * (k & 3u))) << (4u * (k & 7u));
  nb[0] = ~nb[0]; nb[1] = ~nb[1] & (N == 16u ? ~0u : (1u << (4u * (N & 7u))) - 1u);
  const u32 cnt = __popc(nb[0]) + __popc(nb[1]), inc = wave_incl_scan(cnt), V2 = __builtin_amdgcn_readlane(inc, 63);
  if (V2 <= MAX2 && V2 + 40u <= cap) {
    if (MAX2 > 1024u && V2 > 1024u) dec_tail<8>(c, V2, in, s, cap); else dec_tail<4>(c, V2, in, s, cap);
  }
  CFENCE();
  get_all<N>(c, ldsa(s) + inc - cnt, nb, o);
#pragma unroll
  for (u32 k = 0; k < N; ++k) o[k] += prm(0u, 0x0D0C0B00u, (l1[k >> 2] >> (2u * (k & 3u))) & 3u * B1) + add;
}
DI u32 pick_top(u32 sum, u32 cnt) { return (u32)((float)sum * __builtin_amdgcn_rcpf((float)max(cnt, 1u)) + 1.83f); }

DI u32x4 piece(const u32 (&lo)[16], const u32 (&hi)[16], int t) {
  return (u32x4){prm(hi[2 * t], lo[2 * t], 0x05010400u), prm(hi[2 * t], lo[2 * t], 0x07030602u),
                 prm(hi[2 * t + 1], lo[2 * t + 1], 0x05010400u), prm(hi[2 * t + 1], lo[2 * t + 1], 0x07030602u)};
}
DI void pack_inputs(Ctx& c, const Params& p, int wv) {
  const int lane = c.lane;
  const rsrc_t rsL = mkrs(c.ws, WS_USED), rsX = mkrs(p.x, MM * KS * 2), rsW = mkrs(p.w, KS * NN * 2);
  LAS unsigned char* st = c.lds + L_RING + wv * 4096;
  const int hi7 = (c.wg & 15) | (c.wg >> 5) << 4, late = wg_late(c.wg);
  int q = wv == 0 ? hi7 * 2 + late : late ? 128 + wv * 128 + hi7 : 2560;
  if (q < 320) __builtin_amdgcn_s_setprio(3);
  if (q < NPUSH) {
    u32x4 v[8];
#pragma unroll
    for (int t = 0; t < 8; ++t) v[t] = __builtin_amdgcn_raw_buffer_load_b128(rsX, ((64 * q + (lane & 15)) * KS + 8 * (lane >> 4)) * 2 + 64 * (t >> 2), 16 * (t & 3) * KS * 2, 0);
#pragma unroll 1
    for (int l = 1; l <= NR; ++l) {
      const rsrc_t rsP = mkrs(p.ws[(c.rank + q + l) & 7], WS_USED);
#pragma unroll
      for (int t = 0; t < 8; ++t) st128<SYS>(v[t], rsP, O_IMG + (u32)(c.rank * 96 + q) * 8192u + t * 1024 + lane * 16);
      if (l == 6) asm volatile("s_waitcnt vmcnt(40)" ::: "memory");
    }
    c.call = lds_wait(c, C_CALL, 1u);
    VMWAIT0();
    tell_peers(c, O_FX + ((q >> 2) * 64 + c.rank) * 16 + (q & 3) * 4, c.call);
    image_flag(c, c.rank, q);
    q = late ? 1152 + hi7 : 2560;
  }
  for (; q < 2560; q = q < 256 ? (late ? 1152 + hi7 : 2560) : q + 1024) {
    int j = 0, r = q;
    j = q / 320; r = q - j * 320;
    if (j) { __builtin_amdgcn_s_setprio(0); while (now() - c.t0 < (u64)(PACK_T0 + PACK_DT * j)) __builtin_amdgcn_s_sleep(8); }
    const bool isx = r < 64;
    const int dest = (r - 64) & 7, ni = (r - 64) >> 4, kh = ((r - 64) >> 3) & 1;
    u32 lo[16], hi[16];
    *(LAS u32x4*)(st + lane * 16) = (u32x4){0u, 0u, 0u, 0u}; *(LAS u32x4*)(st + 1024 + lane * 16) = (u32x4){0u, 0u, 0u, 0u};
    if (isx) {
      u32x4 d[8];
      const int vo = ((64 * r + (lane & 15)) * KS + 64 * j + 8 * (lane >> 4)) * 2;
#pragma unroll
      for (int t = 0; t < 8; ++t) d[t] = __builtin_amdgcn_raw_buffer_load_b128(rsX, vo + 64 * (t >> 2), 16 * (t & 3) * KS * 2, 0);
      __builtin_amdgcn_sched_barrier(0);
#pragma unroll
      for (int t = 0; t < 8; ++t) {
        lo[2 * t] = prm(d[t].y, d[t].x, 0x06040200u); lo[2 * t + 1] = prm(d[t].w, d[t].z, 0x06040200u);
        hi[2 * t] = prm(d[t].y, d[t].x, 0x07050301u); hi[2 * t + 1] = prm(d[t].w, d[t].z, 0x07050301u);
      }
    } else {
      u32x2 d[2][8];
      const int vo = ((64 * j + 32 * kh + 8 * (lane >> 4)) * NN + dest * 2048 + ni * 128 + 4 * (lane & 15)) * 2;
#pragma unroll
      for (int k = 0; k < 8; ++k)
#pragma unroll
        for (int h = 0; h < 2; ++h) d[h][k] = __builtin_amdgcn_raw_buffer_load_b64(rsW, vo + 128 * h, k * NN * 2, 0);
      __builtin_amdgcn_sched_barrier(0);
#pragma unroll
      for (int h = 0; h < 2; ++h)
#pragma unroll
        for (int cc = 0; cc < 2; ++cc)
#pragma unroll
          for (int u = 0; u < 2; ++u) {
            const u32 a0 = d[h][4 * u][cc], a1 = d[h][4 * u + 1][cc], a2 = d[h][4 * u + 2][cc], a3 = d[h][4 * u + 3][cc];
            const u32 l0 = prm(a1, a0, 0x06020400u), l1 = prm(a3, a2, 0x06020400u), h0 = prm(a1, a0, 0x07030501u), h1 = prm(a3, a2, 0x07030501u);
            const int t = 4 * h + 2 * cc;
            lo[2 * t + u] = prm(l1, l0, 0x05040100u); lo[2 * t + 2 + u] = prm(l1, l0, 0x07060302u);
            hi[2 * t + u] = prm(h1, h0, 0x05040100u); hi[2 * t + 2 + u] = prm(h1, h0, 0x07060302u);
          }
    }
    const int lr = isx ? r : 64 + 2 * ni + kh;
    const bool own = isx || dest == c.rank, send = isx || dest != c.rank;
    if (own) {
      const u32 img = O_IMG + (u32)((j * 8 + c.rank) * 96 + lr) * 8192u + lane * 16;
#pragma unroll
      for (int t = 0; t < 8; ++t) st128<AGT>(piece(lo, hi, t), rsL, img + t * 1024);
    }
    u32 tot = 8192u;
    if (send) {
      const u32 slot = (isx ? O_OBX + (u32)(j * 64 + r) * 8192u : O_OBW + (u32)((dest * 8 + j) * 32 + 2 * ni + kh) * 8192u);
      u32 n[16], se = 0, any = 0, cnt = 0;
      if (q >= NRAW) {
#pragma unroll
      for (int k = 0; k < 16; ++k) { n[k] = bfi(B1, lo[k] >> 7, hi[k] << 1); se = __builtin_amdgcn_sad_u8(n[k], 0u, se); }
      const u32 base = min(max(pick_top(__builtin_amdgcn_readlane(wave_incl_scan(se), 63), 4096u), 13u) - 13u, 240u);
      const u32 bl = (base & 127u) * B1, bx = base < 128u ? 0x80u * B1 : 0u;
#pragma unroll
      for (int k = 0; k < 16; ++k) { n[k] = (((n[k] | 0x80u * B1) - bl) ^ (n[k] & 0x80u * B1)) ^ bx; any |= n[k]; }
      if (ANY((any & 0xF0u * B1) != 0u)) {
#pragma unroll
        for (int k = 0; k < 16; ++k) if (ANY((n[k] & 0xF0u * B1) != 0u)) {
#pragma unroll
          for (int b = 0; b < 4; ++b) {
            const u32 v = (n[k] >> (8 * b)) & 255u;
            const u64 m = __builtin_amdgcn_ballot_w64(v > 15u);
            const u32 pos = cnt + __builtin_amdgcn_mbcnt_hi((u32)(m >> 32), __builtin_amdgcn_mbcnt_lo((u32)m, 0u));
            if (v > 15u && pos < 31u) st32((u32)(k * 4 + b) << 14 | (u32)lane << 8 | ((v + base) & 255u), rsL, slot + 5124u + pos * 4u);
            cnt += (u32)__popcll(m);
          }
        }
      }
#pragma unroll
      for (int k = 0; k < 16; ++k) n[k] = ((n[k] & 15u * B1) + 2u * B1) & 15u * B1;
      Lv g; u32 l2a, l2b, l3;
      if (cnt <= 31u && enc_levels<16, 2048>(c, n, st, 2048u, g, l2a, l2b, l3)) {
#pragma unroll
        for (int t = 0; t < 4; ++t)
          st128<SYS>((u32x4){bfi(0x80u * B1, hi[4 * t], lo[4 * t]), bfi(0x80u * B1, hi[4 * t + 1], lo[4 * t + 1]),
                             bfi(0x80u * B1, hi[4 * t + 2], lo[4 * t + 2]), bfi(0x80u * B1, hi[4 * t + 3], lo[4 * t + 3])}, rsL, slot + t * 1024 + lane * 16);
        u32x4 c1;
#pragma unroll
        for (int t = 0; t < 4; ++t) c1[t] = n[4 * t] | n[4 * t + 1] << 2 | n[4 * t + 2] << 4 | n[4 * t + 3] << 6;
        st128<SYS>(c1, rsL, slot + 4096u + lane * 16);
        const u32 o2 = 5120u + ((cnt + 4u) & ~3u) * 4u;
        if (lane == 0) st32(base | cnt << 8, rsL, slot + 5120u);
        if ((u32)lane < g.nc2) { st32(l2a, rsL, slot + o2 + lane * 4); if (g.rows2 == 8u) st32(l2b, rsL, slot + o2 + (g.nc2 + lane) * 4u); }
        if ((u32)lane < g.nc3) st32(l3, rsL, slot + o2 + g.rows2 * g.nc2 + lane * 4);
        tot = (o2 + g.rows2 * g.nc2 + g.nc3 * 4u + 15u) & ~15u;
      } else VMWAIT0();
      }
      if (tot == 8192u) {
        asm volatile("" : "+v"(lo[0]), "+v"(lo[2]), "+v"(lo[4]), "+v"(lo[6]), "+v"(lo[8]), "+v"(lo[10]), "+v"(lo[12]), "+v"(lo[14]),
                          "+v"(lo[1]), "+v"(lo[3]), "+v"(lo[5]), "+v"(lo[7]), "+v"(lo[9]), "+v"(lo[11]), "+v"(lo[13]), "+v"(lo[15]));
#pragma unroll
        for (int t = 0; t < 8; ++t) st128<SYS>(piece(lo, hi, t), rsL, slot + t * 1024 + lane * 16);
      }
    }
    if (!c.call) c.call = lds_wait(c, C_CALL, 1u);
    VMWAIT0();
    if (send && (isx || lane == dest)) tell_peers(c, O_RF + (c.rank * NBLK + j * 96 + lr) * 4, c.call << 20 | tot >> 4);
    if (own) image_flag(c, j * 8 + c.rank, lr);
    if (q < 64) STAMP(c, 1);
    if (q >= 2048) STAMP(c, 2);
  }
  __builtin_amdgcn_s_setprio(0);
  c.call = lds_wait(c, C_CALL, 1u);
  if (lane == 0) lds_add(c.ctl + C_PACKED, 1u);
}

DI u32 ob_slot(int b, int me) { const int j = b / 96, r = b - j * 96; return r < 64 ? O_OBX + (u32)(j * 64 + r) * 8192u : O_OBW + (u32)((me * 8 + j) * 32 + r - 64) * 8192u; }
#define LEFT {0, 2, 4, 4}
struct Order {
  u32 lb[NBLK];
  constexpr Order() : lb() {
    constexpr int left[4] = LEFT, phase[4] = {6, 5, 1, 3};
    int n = 0, next[4] = {1, 1, 1, 1};
    for (int r = 0; r < 96; ++r) lb[n++] = (u32)r;
    for (int j = 1; j < 16; ++j) {
      bool go[4] = {};
      for (int g = 0; g < 4; ++g) go[g] = next[g] < 8 && (j > 7 || (j * (7 - left[g]) + phase[g]) / 7 >= next[g]) && !(j <= 7 && next[g] > 7 - left[g]);
      if (j < 8) for (int r = 0; r < 32; ++r) lb[n++] = (u32)(j * 96 + 64 + r);
      for (int u = 0; u < 4; ++u)
        for (int v = 0; v < 4; ++v) {
          const int t = j > 7 ? v : u, g = j > 7 ? u : v;
          if (go[g]) for (int b = 0; b < 4; ++b) lb[n++] = (u32)(next[g] * 96 + t * 16 + g * 4 + b);
        }
      for (int g = 0; g < 4; ++g) next[g] += go[g];
    }
  }
};
__constant__ Order ORDER = Order();
DI int pull_order(u32 B, int link) {
  const u32 lb = ORDER.lb[B], r = lb % 96u;
  return (int)(B >= 96u && r < 64u ? lb - r + ((r + 16u * link) & 63u) : lb);
}
DI u32 early_first(u32 t) { return (t & 15u) | (t >> 7) << 4 | ((t >> 4) & 7u) << 5; }
DI void role_feeder(Ctx& c) {
  const int lane = c.lane;
  __builtin_amdgcn_s_setprio(3);
  c.call = lds_wait(c, C_CALL, 1u);
  const int link = wg_link(c.wg), src = (c.rank + 1 + link) & 7;
  const u32 ne = link_wgs(link, 0), late = wg_late(c.wg), grp = (c.wg >> 3) & 3;
  GAS u32* tk = (GAS u32*)(c.ws + O_TICK + link * 128);
  u32 kv = 0, tbase = (c.call << 12) - KFIX * ne - NPUSH;
  const u32 stop = grp == 0u ? 0u : grp == 1u ? T_HALF : T_MASK;
  bool miss = true, out = false;
  const rsrc_t rsL = mkrs(c.ws, WS_USED);
  u32 fb = O_RF + src * NBLK * 4, B = 0;
  for (u32 k = 0;; ++k) {
    if (k == (late ? 0u : KFIX - 1u)) {
      if (lane < 2) __hip_atomic_fetch_max(tk + lane * 256, c.call << 12, __ATOMIC_RELAXED, __HIP_MEMORY_SCOPE_AGENT);
      VMWAIT0();
      if (late) lds_wait(c, C_PACKED, 8u);
    }
    const u32 ahead = B < T_AH ? 2u : 1u;
    if (k >= ahead) lds_wait(c, C_PSTART, k + 1u - ahead);
    if (k < KFIX && !late) {
      B = NPUSH + (u32)((c.wg & 15) | (c.wg >> 5) << 4) / 7u + k * ne;
      if (miss) { dma4(rsL, lane * 16, fb, (u32)(size_t)c.lds + L_RFC); VMWAIT0(); }
    } else for (;;) {
      if (!out && B >= stop) {
        if (grp != 1u) lds_wait(c, C_CMAX, 8u);
        STAMP(c, 6);
        out = true; tk += 256; tbase = (c.call << 12) - T_MASK; fb = O_OF + src * 8192; miss = true;
      }
      if (miss) dma4(rsL, lane * 16, fb, (u32)(size_t)c.lds + L_RFC);
      u32 t = 0;
      if (lane == 0) t = __hip_atomic_fetch_add(tk, 1u, __ATOMIC_RELAXED, __HIP_MEMORY_SCOPE_AGENT);
      VMWAIT0();
      B = RFL(t) - tbase;
      if (B < T_MASK || out) break;
    }
    if (B >= T_END) {
      if (lane < 3) lds_st(c.ctl + C_QI + ((k + lane) & 7u), (k + lane + 1u) << 23 | 0x7FFFFFu);
      if (kv >= 5u) lds_wait(c, C_DFREE0 + ((kv - 5u) & 3u), ((kv - 5u) & ~3u) + 4u);
      if (lane < 3) lds_st(c.ctl + C_QV + ((kv + lane) & 7u), (kv + lane + 1u) << 23 | 0x7FFFFFu);
      break;
    }
    u32 item, fo;
    if (B < T_MASK) { item = pull_order(B, link); fo = item * 4; }
    else if (B < T_VAL) { item = T_MASK; fo = F_MF + (B - T_MASK) * 4; }
    else {
      const u32 ht = early_first((B - T_VAL) >> 1) * 2u + (B & 1u);
      item = T_VAL + ht; fo = (lane & 1) ? (ht >> 1) * 4 : F_VF - F_MI + ht * 4;
      if (kv == 0u) fb += F_MI;
    }
    u32 v = B >= T_VAL && kv == 0u ? 0u : lds_ld((LAS u32*)(c.lds + L_RFC + fo)), spins = 0;
    miss = false;
    while (!ALL((int)((v & 0xFFF00000u) - (c.call << 20)) >= 0)) {
      miss = true;
      if (expired(c, spins)) break;
      if (spins > 1u) __builtin_amdgcn_s_sleep(1);
      v = g_ld_sys(c.ws + fb + fo);
    }
    if (item == T_MASK) { item += (v >> 10) & 511u; v &= 1023u; }
    if (lane == 0) {
      if (B >= T_VAL) lds_st(c.ctl + C_QV + (kv & 7u), (kv + 1u) << 23 | (k & 2047u) << 12 | (item - T_VAL));
      const u32 wide = B < T_W0 || (B >= T_HALF && B < T_MASK + T_WA);
      lds_st(c.ctl + C_QI + (k & 7u), (k + 1u) << 23 | item << 12 | wide << 11 | (v & 0x7FFu));
    }
    if (B >= T_VAL) ++kv;
  }
}
DI void role_decoder(Ctx& c, u32 d) {
  const int lane = c.lane, src = (c.rank + 1 + wg_link(c.wg)) & 7;
  __builtin_amdgcn_s_setprio(DPRIO);
  c.call = lds_wait(c, C_CALL, 1u);
  const rsrc_t rsL = mkrs(c.ws, WS_USED);
  for (u32 k = d;; k += NDEC) {
    const u32 e = q_wait(c, C_QI, k), lb = (e >> 12) & 2047u, tot = min((e & 0x7FFu) * 16u, 8192u);
    if (lb >= T_VAL) break;
    LAS unsigned char* sl = c.lds + L_IN + (k & 3u) * 8192u;
    if (lb >= T_MASK) {
      lds_wait(c, C_PROG, k * 32768u + tot);
      const u32 pc = lb - T_MASK, inb = O_MIN + src * 256 * 16384 + piece_off(pc) + lane * 16;
      const u32 hd = *(LAS u32*)(sl + (lane & 7) * 4);
      u32x4 v[8];
#pragma unroll
      for (int t = 0; t < 8; ++t)
        v[t] = *(LAS u32x4*)(sl + (pc >= 256u ? (t & 3) * 1024u : t ? min(__builtin_amdgcn_readlane(hd, t - 1) & 0x1FF0u, 7168u) : 32u) + lane * 16);
      CFENCE(); if (lane == 0) lds_add(c.ctl + C_IFREE0 + (k & 3u), 1u); CFENCE();
      if (pc < 256u && lane < 8) __builtin_amdgcn_raw_buffer_store_b32(hd, rsL, (int)(inb - lane * 12), 0, AGT);
#pragma unroll
      for (int t = 0; t < 8; ++t) if (t < 4 || pc < 256u) st128<AGT>(v[t], rsL, inb + (pc < 256u ? 32u : 0u) + t * 1024);
      VMWAIT0();
      if (lane == 0) __hip_atomic_fetch_add((GAS u32*)(c.ws + O_OF + src * 8192 + F_MI) + (pc & 255u), 1u << 19, __ATOMIC_RELAXED, __HIP_MEMORY_SCOPE_AGENT);
      continue;
    }
    const int j = lb / 96, r = lb - j * 96, kstep = j * 8 + src;
    const u32 img = O_IMG + (u32)(kstep * 96 + r) * 8192u + lane * 16;
    lds_wait(c, C_PROG, k * 32768u + tot);
    if (k < 2u) STAMP(c, 26 + k);
    if (tot == 8192u) {
      u32x4 v[8];
#pragma unroll
      for (int t = 0; t < 8; ++t) v[t] = *(LAS u32x4*)(sl + t * 1024 + lane * 16);
      CFENCE(); if (lane == 0) lds_add(c.ctl + C_IFREE0 + (k & 3u), 1u); CFENCE();
#pragma unroll
      for (int t = 0; t < 8; ++t) st128<AGT>(v[t], rsL, img + t * 1024);
    } else {
      const u32x4 c1 = *(LAS u32x4*)(sl + 4096 + lane * 16);
      const u32 exc = *(LAS u32*)(sl + 5120 + (lane & 31) * 4);
      u32x4 sm[4];
#pragma unroll
      for (int t = 0; t < 4; ++t) sm[t] = *(LAS u32x4*)(sl + t * 1024 + lane * 16);
      const u32 hd = RFL(exc), cnt = (hd >> 8) & 31u, base4 = min(hd & 255u, 240u) * B1;
      u32 n[16], lo[16], hi[16];
      const u32 l1[4] = {c1.x, c1.y, c1.z, c1.w};
      dec_levels<16, 2048>(c, l1, base4, n, sl + 5120u + ((cnt + 4u) & ~3u) * 4u, sl + 6144u, 2048u);
      CFENCE(); if (lane == 0) lds_add(c.ctl + C_IFREE0 + (k & 3u), 1u); CFENCE();
      for (u32 x = 1; x <= cnt; ++x) {
        const u32 rec = __builtin_amdgcn_readlane(exc, x);
        const u32 kk = (rec >> 16) & 15u, sh = ((rec >> 14) & 3u) * 8u;
        const u32 m = ((rec >> 8) & 63u) == (u32)lane ? 255u << sh : 0u, val = (rec & 255u) << sh;
#pragma unroll
        for (int i = 0; i < 16; ++i) n[i] = bfi(kk == (u32)i ? m : 0u, val, n[i]);
      }
#pragma unroll
      for (int i = 0; i < 16; ++i) { hi[i] = bfi(0x80u * B1, sm[i >> 2][i & 3], n[i] >> 1); lo[i] = bfi(0x80u * B1, n[i] << 7, sm[i >> 2][i & 3]); }
#pragma unroll
      for (int t = 0; t < 8; ++t) st128<AGT>(piece(lo, hi, t), rsL, img + t * 1024);
    }
    VMWAIT0();
    image_flag(c, kstep, r);
  }
}
constexpr int WMAX = WIN2 - 1;
#define WINM 3
struct Stream {
  u32 pub = 0, pend[WMAX] = {};
  template <int W> DI void landed(const Ctx& c, u32 prog) {
    asm volatile("s_waitcnt vmcnt(%0)" :: "n"(W - 1) : "memory");
    pub = max(pub, pend[WMAX - (W - 1)]);
    if (c.lane == 0) lds_st(c.ctl + C_PROG, pub);
    for (int k = 0; k + 1 < WMAX; ++k) pend[k] = pend[k + 1];
    pend[WMAX - 1] = prog;
  }
  DI void flush(const Ctx& c) {
    VMWAIT0();
    pub = pend[WMAX - 1];
    if (c.lane == 0) lds_st(c.ctl + C_PROG, pub);
  }
};
DI void role_puller(Ctx& c, const Params& p) {
  __builtin_amdgcn_s_setprio(3);
  const u32 ldsb = (u32)(size_t)c.lds;
  const char* sws = p.ws[(c.rank + 1 + wg_link(c.wg)) & 7];
  Stream sm;
  u32 kv = 0;
  u32 cw = 0;
  for (u32 k = 0;; ++k) {
    u32 e = __builtin_amdgcn_readlane(cw, 0), fi = __builtin_amdgcn_readlane(cw, 1), fd = __builtin_amdgcn_readlane(cw, 2);
    if ((e >> 23) != ((k + 1u) & 511u)) {
      e = RFL(lds_ld(c.ctl + C_QI + (k & 7u)));
      if ((e >> 23) != ((k + 1u) & 511u)) {
        sm.flush(c);
        e = q_wait(c, C_QI, k);
      }
    }
    CFENCE();
    const u32 item = (e >> 12) & 2047u;
    if ((~e & 0x7FFFFFu) == 0u) break;
    u32 so, tot = (e & 0x7FFu) * 16u, dst;
    if (item < T_VAL) {
      if (fi < k >> 2) { sm.flush(c); lds_wait(c, C_IFREE0 + (k & 3u), k >> 2); }
      dst = L_IN + (k & 3u) * 8192u;
      so = item < T_MASK ? ob_slot(item, c.rank) : O_OM + piece_off(item - T_MASK); tot = min(tot, 8192u);
    } else {
      if (fd < (kv & ~3u)) { sm.flush(c); lds_wait(c, C_DFREE0 + (kv & 3u), kv & ~3u); }
      dst = L_STG + (kv & 3u) * 16384u; so = O_OV + (item - T_VAL) * 16384u; tot = min(tot, 16384u);
      ++kv;
    }
    so = RFL(so); tot = RFL(max(tot, 16u)); dst = RFL(dst);
    const rsrc_t rs = mkrs(sws + so, tot);
    const bool wide = (e >> 11) & 1u;
    const u32 lead = PLEAD, qs = tot > lead ? (tot - lead) & ~1023u : 0u;
    constexpr u32 ch = 1024u;
    bool told = false;
    for (u32 q = 0; q < tot; q += ch) {
      dma1(rs, c.lane * 16, q, ldsb + dst + q);
      if (q >= qs && !told) { told = true; if (c.lane == 0) lds_st(c.ctl + C_PSTART, k + 1u); }
      if (q + ch >= tot) cw = lds_ld(c.ctl + (c.lane == 0 ? C_QI + ((k + 1u) & 7u) : c.lane == 1 ? C_IFREE0 + ((k + 1u) & 3u) : C_DFREE0 + (kv & 3u)));
      if (wide) sm.landed<WIN2>(c, k * 32768u + q + ch);
      else if (WINM != WIN1 && item < T_MASK) sm.landed<WINM>(c, k * 32768u + q + ch);
      else sm.landed<WIN1>(c, k * 32768u + q + ch);
    }
  }
  sm.flush(c);
  STAMP(c, 8);
}

DI void role_sched(Ctx& c, const Params& p, int mi, int ni) {
  const int lane = c.lane;
  __builtin_amdgcn_s_setprio(3);
  u32 t = 0;
  if (lane == 0) t = __hip_atomic_fetch_add((GAS u32*)(c.ws + O_CALL + c.wg * 128), 1u, __ATOMIC_RELAXED, __HIP_MEMORY_SCOPE_AGENT) + 1u;
  c.call = RFL(t);
  if (lane == 0) lds_st(c.ctl + C_CALL, c.call);
  const rsrc_t rsL = mkrs(c.ws, WS_USED);
  u64 issued = 0; u32 n = 0, spins = 0;
  while (n < 64u) {
    if (2u * n >= RFL(lds_ld((LAS u32*)(c.lds + L_REL + 7 * 256))) + 2u * SCHED_Q) { if (!lds_expired(c, spins)) continue; }
    const u32x4 a = __builtin_amdgcn_raw_buffer_load_b128(rsL, lane * 16, (int)(O_FX + mi * 1024), SYS);
    const u32x2 b = __builtin_amdgcn_raw_buffer_load_b64(rsL, lane * 8, (int)(O_FW + ni * 512), SYS);
    const bool rdy = min(min(min(a.x, a.y), min(a.z, a.w)), min(b.x, b.y)) >= c.call;
    u64 m = __builtin_amdgcn_ballot_w64(rdy) & ~issued;
    if (m == 0) {
      if (!expired(c, spins)) { __builtin_amdgcn_s_sleep(1); CFENCE(); continue; }
      m = ~issued;
    }
    if ((m >> lane) & 1) lds_st(c.ctl + C_ORDER + n + __builtin_amdgcn_mbcnt_hi((u32)(m >> 32), __builtin_amdgcn_mbcnt_lo((u32)m, 0u)), lane + 1);
    issued |= m;
    n += (u32)__popcll(m);
    CFENCE();
  }
  STAMP(c, 9);
  if (c.wg == LEADER) {
    u32 mx;
    for (;;) {
      const u32x4 a = __builtin_amdgcn_raw_buffer_load_b128(rsL, lane * 16, (int)O_LT, SYS);
      const u32x4 b = __builtin_amdgcn_raw_buffer_load_b128(rsL, lane * 16, (int)(O_LT + 1024), SYS);
      mx = max(max(a.x, a.z), max(b.x, b.z));
      if (ALL(min(min(a.y, a.w), min(b.y, b.w)) >= c.call) || expired(c, spins)) break;
      CFENCE();
    }
    STAMP(c, 16);
    mx = wave_max(mx);
    if (lane < NR) __hip_atomic_store((GAS u64*)(*(char* LAS*)(c.lds + L_PEER + lane * 8) + O_GT + c.rank * 8), (u64)c.call << 32 | mx, __ATOMIC_RELAXED, __HIP_MEMORY_SCOPE_SYSTEM);
    if (lane == 0) lds_st(c.ctl + C_LEAD, 1u);
  }
  lds_wait(c, C_CMAX, 8u);
  const rsrc_t rsG = mkrs(c.ws + O_GT, 64u);
  u32 g;
  for (;;) {
    if (lane < 4) dma1(rsG, lane * 16, 0u, (u32)(size_t)c.lds + L_GT);
    asm volatile("s_waitcnt vmcnt(5)" ::: "memory");
    const u64 v = __hip_atomic_load((LAS u64*)(c.lds + L_GT) + (lane & 7), __ATOMIC_RELAXED, __HIP_MEMORY_SCOPE_WORKGROUP);
    g = (u32)v;
    if (ALL((u32)(v >> 32) >= c.call) || expired(c, spins)) break;
    __builtin_amdgcn_s_sleep(2);
  }
  g = wave_max(g);
  if (lane == 0) { lds_st(c.ctl + C_GMAX, g); CFENCE(); lds_st(c.ctl + C_GVALID, 1u); }
  STAMP(c, 10);
}

DI void role_loader(Ctx& c, u32 li, int mi, int ni) {
  __builtin_amdgcn_s_setprio(2);
  const rsrc_t rsL = mkrs(c.ws, WS_USED);
  const u32 voff = c.lane * 16, ldsb = (u32)(size_t)c.lds + L_RING;
  u32 pend = 0;
  lds_wait(c, C_PACKED, 8u);
  for (u32 m = li; m < 128u; m += 5u - NDEC) {
    const u32 slot = m % NS, kh = m & 1u;
    u32 o, spins = 0;
    for (;;) {
      const u32 vo = lds_ld(c.ctl + C_ORDER + (m >> 1)), vf = lds_ld((LAS u32*)(c.lds + L_REL + (c.lane & 7) * 256));
      o = RFL(vo);
      if (o != 0u && ALL(vf + (NS - 1u) >= m)) break;
      if (pend) { VMWAIT0(); if (c.lane == 0) lds_st(c.ctl + C_FULL0 + (pend - 1u) % NS, pend); pend = 0; continue; }
      if (lds_expired(c, spins)) break;
    }
    CFENCE();
    o = RFL(o);
    const u32 so = O_IMG + ((o - 1u) & 63u) * (96u * 8192u), sx = so + mi * 32768 + kh * 4096u, sw = so + 64 * 8192 + ni * 16384 + kh * 8192u, ld = ldsb + slot * SLOTB;
    dma4(rsL, voff, sw, ld + 16384u);
    if (pend) { asm volatile("s_waitcnt vmcnt(4)" ::: "memory"); if (c.lane == 0) lds_st(c.ctl + C_FULL0 + (pend - 1u) % NS, pend); }
    dma4(rsL, voff, sw + 4096u, ld + 20480u);
    dma4(rsL, voff, sx, ld); dma4(rsL, voff, sx + 8192u, ld + 4096u); dma4(rsL, voff, sx + 16384u, ld + 8192u); dma4(rsL, voff, sx + 24576u, ld + 12288u);
    pend = m + 1u;
  }
  VMWAIT0();
  if (c.lane == 0) lds_st(c.ctl + C_FULL0 + (pend - 1u) % NS, pend);
}

struct Geo { u32 nc, sz; };
DI Geo geo(u32 V) { return V <= 2048u ? Geo{row_bytes(V) >> 2, 8u * row_bytes(V)} : Geo{64u, (min(V, 2304u) + 3u) & ~3u}; }
struct Scale { u32 r; int kap; };
DI Scale split(float inv) { const u32 ib = __float_as_uint(inv), r = ((ib & 0x7FFFFFu) + (1u << 19)) >> 20; return Scale{r & 7u, (int)(ib >> 23) - 247 + (int)(r >> 3)}; }
DI u32 low_exp(u32 o, u32 kap32, u32& pos) {
  const u32 t = o + kap32, ge = (t >> 5) & B1;
  pos = ((t + 31u * B1) >> 6) & B1;
  return t & ((ge << 5) - ge);
}
DI u32 sure(u32 m, u32 r8, u32 pos, u32& up) {
  const u32 x = m + r8;
  up = (~x >> 3) & B1;
  return (((x ^ 8u * B1) + 15u * B1) >> 4) & pos;
}
DI u32 place(const Ctx& c, int wv, int cnt, u32 first, u32 sb, u32 extra, u32x4& hdr) {
  if (c.lane == 0) { lds_st(c.ctl + C_VSZ + wv, sb | extra); CFENCE(); lds_add(c.ctl + cnt, 1u); }
  lds_wait(c, cnt, 4u);
  const u32x4 z = *(LAS u32x4*)(c.ctl + C_VSZ + (wv & 4));
  hdr.x = z.x + first; hdr.y = z.y + (hdr.x & 0xFFFFu); hdr.z = z.z + (hdr.y & 0xFFFFu); hdr.w = z.w + (hdr.z & 0xFFFFu);
  return (((wv & 2) ? ((wv & 1) ? hdr.w : hdr.z) : ((wv & 1) ? hdr.y : hdr.x)) & 0xFFFFu) - sb;
}

DI void role_mfma(Ctx& c, const Params& p, int wv, int mi, int ni) {
  const int lane = c.lane, mw = wv >> 1, h = wv & 1;
  pack_inputs(c, p, wv);
  f32x4 acc[4][4];
#pragma unroll
  for (int g = 0; g < 4; ++g)
#pragma unroll
    for (int b = 0; b < 4; ++b) acc[g][b] = (f32x4){0.f, 0.f, 0.f, 0.f};
  if (wv == 0) STAMP(c, 15);
  bf16x8 x0, x1, x2, x3, wa0, wa1, wa2, wa3, wb0, wb1, wb2, wb3;
  u32 vx = (u32)(size_t)c.lds + L_RING + mw * 4096 + lane * 16, vw = (u32)(size_t)c.lds + L_RING + 16384 + h * 4096 + lane * 16;
  u32 vf = (u32)(size_t)(c.ctl + C_FULL0 + 2), done = 1, sl = 1, seen = 0, vs;
  const u32 relad = (u32)(size_t)c.lds + L_REL + wv * 256 + lane * 4;
#define MF(a, w, x) "v_mfma_f32_16x16x32_bf16 " a ", " w ", " x ", " a "\n\t"
#define HSTEP(c0, c1, c2, c3, n0, n1, n2, n3, m) do { \
    if (__builtin_expect(seen <= (m) + 1u, 0)) lds_wait(c, C_FULL0 + sl, (m) + 2u); \
    asm volatile(MF("%0", "%9", "%8") "ds_read_b128 %4, %13\n\t" MF("%1", "%10", "%8") "ds_read_b128 %5, %13 offset:1024\n\t" \
                 MF("%2", "%11", "%8") "ds_read_b128 %6, %13 offset:2048\n\t" MF("%3", "%12", "%8") "ds_read_b128 %7, %13 offset:3072" \
                 : "+v"(acc[0][0]), "+v"(acc[1][0]), "+v"(acc[2][0]), "+v"(acc[3][0]), "+v"(n0), "+v"(n1), "+v"(n2), "+v"(n3) \
                 : "v"(x0), "v"(c0), "v"(c1), "v"(c2), "v"(c3), "v"(vw) : "memory"); \
    const u32 d = sl == NS - 1u ? -(NS - 1u) * SLOTB : SLOTB, d4 = sl == NS - 2u ? -(NS - 1u) * 4u : 4u; \
    sl = sl == NS - 1u ? 0u : sl + 1u; \
    asm volatile("s_waitcnt lgkmcnt(7)\n\t" MF("%0", "%7", "%6") "ds_read_b128 %4, %11\n\t" MF("%1", "%8", "%6") "ds_read_b32 %5, %12\n\t" \
                 MF("%2", "%9", "%6") MF("%3", "%10", "%6") \
                 : "+v"(acc[0][1]), "+v"(acc[1][1]), "+v"(acc[2][1]), "+v"(acc[3][1]), "+v"(x0), "=&v"(vs) \
                 : "v"(x1), "v"(c0), "v"(c1), "v"(c2), "v"(c3), "v"(vx), "v"(vf) : "memory"); \
    asm volatile("s_waitcnt lgkmcnt(8)\n\t" MF("%0", "%8", "%7") "ds_read_b128 %4, %12 offset:1024\n\t" MF("%1", "%9", "%7") "v_add_u32 %5, %13, %5\n\t" \
                 MF("%2", "%10", "%7") "v_add_u32 %6, %14, %6\n\t" MF("%3", "%11", "%7") \
                 : "+v"(acc[0][2]), "+v"(acc[1][2]), "+v"(acc[2][2]), "+v"(acc[3][2]), "+v"(x1), "+v"(vw), "+v"(vf) \
                 : "v"(x2), "v"(c0), "v"(c1), "v"(c2), "v"(c3), "v"(vx), "s"(d), "s"(d4) : "memory"); \
    asm volatile("s_waitcnt lgkmcnt(8)\n\t" MF("%0", "%9", "%5") "ds_read_b128 %4, %6 offset:2048\n\tv_add_u32 %7, 1, %7\n\t" MF("%1", "%10", "%5") \
                 "s_waitcnt lgkmcnt(2)\n\tv_readfirstlane_b32 %8, %13\n\t" MF("%2", "%11", "%5") MF("%3", "%12", "%5") \
                 "ds_read_b128 %5, %6 offset:3072\n\tds_write_b32 %14, %7\n\tv_add_u32 %6, %15, %6" \
                 : "+v"(acc[0][3]), "+v"(acc[1][3]), "+v"(acc[2][3]), "+v"(acc[3][3]), "+v"(x2), "+v"(x3), "+v"(vx), "+v"(done), "=&s"(seen) \
                 : "v"(c0), "v"(c1), "v"(c2), "v"(c3), "v"(vs), "v"(relad), "s"(d) : "memory"); \
  } while (0)
  lds_wait(c, C_FULL0, 1u);
  asm volatile("ds_read_b128 %0, %9\n\tds_read_b128 %1, %9 offset:1024\n\tds_read_b128 %2, %9 offset:2048\n\tds_read_b128 %3, %9 offset:3072\n\t"
               "ds_read_b128 %4, %8\n\tds_read_b128 %5, %8 offset:1024\n\tds_read_b128 %6, %8 offset:2048\n\tds_read_b128 %7, %8 offset:3072\n\t"
               "ds_write_b32 %10, %11\n\ts_waitcnt lgkmcnt(0)"
               : "=&v"(wa0), "=&v"(wa1), "=&v"(wa2), "=&v"(wa3), "=&v"(x0), "=&v"(x1), "=&v"(x2), "=&v"(x3) : "v"(vx), "v"(vw), "v"(relad), "v"(done) : "memory");
  vx += SLOTB; vw += SLOTB;
  wb0 = wa0; wb1 = wa0; wb2 = wa0; wb3 = wa0;
  asm volatile("s_nop 1" : "+v"(vx), "+v"(vw), "+v"(wb0), "+v"(wb1), "+v"(wb2), "+v"(wb3));
  for (u32 m = 0; m < 126u; m += 2u) {
    HSTEP(wa0, wa1, wa2, wa3, wb0, wb1, wb2, wb3, m);
    HSTEP(wb0, wb1, wb2, wb3, wa0, wa1, wa2, wa3, m + 1u);
  }
  HSTEP(wa0, wa1, wa2, wa3, wb0, wb1, wb2, wb3, 126u);
#define MF4(b, x) asm volatile(MF("%0", "%5", "%4") MF("%1", "%6", "%4") MF("%2", "%7", "%4") MF("%3", "%8", "%4") \
    : "+v"(acc[0][b]), "+v"(acc[1][b]), "+v"(acc[2][b]), "+v"(acc[3][b]) : "v"(x), "v"(wb0), "v"(wb1), "v"(wb2), "v"(wb3))
  asm volatile("s_waitcnt lgkmcnt(0)" ::: "memory");
  MF4(0, x0); MF4(1, x1); MF4(2, x2); MF4(3, x3);
  asm volatile("s_nop 7\n\ts_nop 7" : "+v"(acc[0][2]), "+v"(acc[1][2]), "+v"(acc[2][2]), "+v"(acc[3][2]), "+v"(acc[0][3]), "+v"(acc[1][3]), "+v"(acc[2][3]), "+v"(acc[3][3]),
               "+v"(acc[0][0]), "+v"(acc[1][0]), "+v"(acc[2][0]), "+v"(acc[3][0]), "+v"(acc[0][1]), "+v"(acc[1][1]), "+v"(acc[2][1]), "+v"(acc[3][1]));
  if (wv == 0) STAMP(c, 11);
  const rsrc_t rsL = mkrs(c.ws, WS_USED);
  {
    float mx = 0.f;
#pragma unroll
    for (int e = 0; e < 64; e += 2) mx = fmaxf(fmaxf(mx, acc[e & 3][e >> 4][(e >> 2) & 3]), acc[(e & 3) + 1][e >> 4][(e >> 2) & 3]);
    const u32 wmx = wave_max(__float_as_uint(mx));
    if (lane == 0) {
      __hip_atomic_fetch_max(c.ctl + C_WGMAX, wmx, __ATOMIC_RELAXED, __HIP_MEMORY_SCOPE_WORKGROUP);
      CFENCE();
      if (lds_add(c.ctl + C_CMAX, 1u) == 7u)
        __hip_atomic_store((GAS u64*)(c.ws + O_LT + c.wg * 8), (u64)c.call << 32 | lds_ld(c.ctl + C_WGMAX), __ATOMIC_RELAXED, __HIP_MEMORY_SCOPE_AGENT);
    }
  }
  const u32 zs = wv >> 2, ht = c.wg * 2 + zs;
  u32 nz[2] = {0u, 0u};
#pragma unroll
  for (int e = 63; e >= 0; --e) nz[e >> 5] = __builtin_amdgcn_alignbit(nz[e >> 5], __float_as_uint(acc[e & 3][e >> 4][(e >> 2) & 3]), 31);
  nz[0] = ~nz[0]; nz[1] = ~nz[1];
  __builtin_amdgcn_raw_buffer_store_b64((u32x2){nz[0], nz[1]}, rsL, lane * 8, (int)(O_OM + c.wg * 16384 + wv * 512), SYS);
  const rsrc_t rsQ = mkrs(c.ws + O_MQ, 4u);
  u32 mqv = 0;
  if (wv == 7) mqv = __builtin_amdgcn_raw_ptr_buffer_atomic_add_i32(1, rsQ, lane * 4, 0, AGT);
  VMWAIT0();
  if (lane == 0) lds_add(c.ctl + C_CMASK, 1u);
  if (wv == 7) {
    lds_wait(c, C_CMASK, 8u);
    tell_peers(c, O_OF + c.rank * 8192 + F_MF + (RFL(mqv) & 511u) * 4u, c.call << 20 | (256u + c.wg) << 10 | 256u);
  }
  const u32 cnt = __popc(nz[0]) + __popc(nz[1]), inc = wave_incl_scan(cnt), V = __builtin_amdgcn_readlane(inc, 63);
  u32 o0 = ldsa(c.lds) + L_RING + wv * 4096 + inc - cnt;
  Geo G = geo(V);
  const bool in8 = (u32)lane < G.nc, in9 = 8u * G.nc + lane < G.sz >> 2;
  lds_wait(c, C_CMAX, 8u);
  if (wv == 0) STAMP(c, 56);
  LAS unsigned char* st = c.lds + L_RING + wv * 4096;
  u32 ob[9], base;
  bool hint;
  {
#pragma unroll
    for (int k = 0; k < 4; ++k) *(LAS u32x4*)(st + k * 1024 + lane * 16) = (u32x4){0u, 0u, 0u, 0u};
    u32 o = o0, se = 0, carry = 0;
#pragma unroll
    for (int z = 0; z < 16; z += 8) {
      u32 sel[8];
#pragma unroll
      for (int D = 0; D < 8; ++D) sel[D] = put_sel(c, nib4(nz[z >> 3], D));
#pragma unroll
      for (int D = 0; D < 8; ++D) {
        const int b = (z + D) >> 2, r = D & 3;
        const u32 h = prm(prm(__float_as_uint(acc[3][b][r]) << 1, __float_as_uint(acc[2][b][r]) << 1, 0x07030000u),
                          prm(__float_as_uint(acc[1][b][r]) << 1, __float_as_uint(acc[0][b][r]) << 1, 0x00000703u), 0x07060100u);
        se = __builtin_amdgcn_sad_u8(put4(o, carry, h, nib4(nz[z >> 3], D), sel[D]), 0u, se);
      }
    }
    or_at(o, carry);
    base = min(max(pick_top(__builtin_amdgcn_readlane(wave_incl_scan(se), 63), V), 13u) - 13u, 240u);
    if (V + lane < G.sz) st[V + lane] = (unsigned char)(base + 13u);
    CFENCE();
    if (wv == 0) STAMP(c, 57);
    u32 n[9], any = 0, l2a, l2b, l3, sb = 0;
    const u16x2 b2 = {(unsigned short)base, (unsigned short)base};
#pragma unroll
    for (u32 k = 0; k < 9u; ++k) ob[k] = *(LAS u32*)(st + (k * G.nc + lane) * 4u);
#pragma unroll
    for (u32 k = 0; k < 9u; ++k) {
      const u32 lo = __builtin_bit_cast(u32, __builtin_elementwise_sub_sat(__builtin_bit_cast(u16x2, ob[k] & 0x00FF00FFu), b2));
      const u32 hi = __builtin_bit_cast(u32, __builtin_elementwise_sub_sat(__builtin_bit_cast(u16x2, (ob[k] >> 8) & 0x00FF00FFu), b2));
      ob[k] = (k < 8u ? in8 : in9) ? lo | hi << 8 : 13u * B1;
      any |= ob[k];
      n[k] = (ob[k] + 2u * B1) & 15u * B1;
    }
    Lv g;
    hint = V <= 2304u && !ANY((any & 0xF0u * B1) != 0u) && enc_levels<9, 1024>(c, n, st + G.sz, 4096u - G.sz, g, l2a, l2b, l3);
    CFENCE();
    if (wv == 0) STAMP(c, 58);
    if (hint) {
      LAS u32* s = (LAS u32*)st + lane;
      if (in8) { s[0] = n[0] | n[1] << 2 | n[2] << 4 | n[3] << 6; s[G.nc] = n[4] | n[5] << 2 | n[6] << 4 | n[7] << 6; }
      if (in9) s[2u * G.nc] = n[8];
      s += (G.sz >> 2) - 6u * G.nc;
      if ((u32)lane < g.nc2) s[0] = l2a;
      if ((u32)lane < g.nc3) s[g.nc2] = l3;
      sb = (G.sz - 24u * G.nc + (g.nc2 + g.nc3) * 4u + 15u) & ~15u;
      if (sb > 1008u) { hint = false; sb = 0; }
    }
    if (lane == 0) { lds_st(c.ctl + C_VSZ + wv, sb | base << 16); CFENCE(); lds_add(c.ctl + C_CSZ0, 1u); }
    lds_wait(c, C_CSZ0, 8u);
    const u32 z = lds_ld(c.ctl + C_VSZ + (lane & 7)), end = wave_incl_scan(lane < 8 ? z & 0xFFFFu : 0u) + 32u;
    const u32 omb = O_OM + piece_off(c.wg), woff = __builtin_amdgcn_readlane(end, wv) - sb;
    if (wv == 0) {
      if (lane < 8) __builtin_amdgcn_raw_buffer_store_b32(end | (z & 0xFFFF0000u), rsL, (int)(omb + lane * 4), 0, SYS);
      mqv = __builtin_amdgcn_raw_ptr_buffer_atomic_add_i32(1, rsQ, lane * 4, 0, AGT);
    }
    if ((u32)lane * 16u < sb) st128<SYS>(*(LAS u32x4*)(st + lane * 16), rsL, omb + woff + lane * 16);
    VMWAIT0();
    u32 t = 0;
    if (lane == 0) { if (wv == 0) lds_st(c.ctl + C_MQ0, mqv); CFENCE(); t = lds_add(c.ctl + C_CPRE0, 1u); }
    if (RFL(t) == 7u)
      tell_peers(c, O_OF + c.rank * 8192 + F_MF + (RFL(lds_ld(c.ctl + C_MQ0)) & 511u) * 4u, c.call << 20 | c.wg << 10 | __builtin_amdgcn_readlane(end, 7) >> 4);
  }
  if (wv == 0) STAMP(c, 28);
  asm volatile("" : "+v"(nz[0]), "+v"(nz[1]), "+v"(o0), "+s"(G.nc));
  lds_wait(c, C_CPRE0, 8u);
#pragma unroll
  for (int k = 0; k < 4; ++k) *(LAS u32x4*)(st + k * 1024 + lane * 16) = (u32x4){0u, 0u, 0u, 0u};
  u32 sel0[16];
#pragma unroll
  for (int D = 0; D < 16; ++D) sel0[D] = put_sel(c, nib4(nz[D >> 3], D & 7));
#pragma unroll
  for (int g = 0; g < 4; ++g)
#pragma unroll
    for (int b = 0; b < 4; ++b) {
#pragma unroll
      for (int r = 0; r < 4; ++r) acc[g][b][r] = __builtin_amdgcn_fmed3f(acc[g][b][r], 0.f, __builtin_inff());
      asm volatile("" : "+v"(acc[g][b]));
    }
  lds_wait(c, C_GVALID, 1u);
  if (wv == 0) STAMP(c, 31);
  const u32 tm = g_ld_agent(c.ws + O_TMO);
  const float amax = __uint_as_float(RFL(lds_ld(c.ctl + C_GMAX)));
  const float inv = amax > 0.f ? 448.0f / amax : 0.f;
  const Scale sc = split(__uint_as_float(RFL(__float_as_uint(inv))));
  const u32 r8 = (8u - sc.r) * B1;
  u32 q[16];
#pragma unroll
  for (int b = 0; b < 4; ++b) {
    const f32x4 a0 = acc[0][b] * inv, a1 = acc[1][b] * inv, a2 = acc[2][b] * inv, a3 = acc[3][b] * inv;
#pragma unroll
    for (int r = 0; r < 4; ++r)
      q[4 * b + r] = (u32)__builtin_amdgcn_cvt_pk_fp8_f32(a2[r], a3[r], __builtin_amdgcn_cvt_pk_fp8_f32(a0[r], a1[r], 0, false), true);
  }
  {
    u32 o = o0, carry = 0, sb;
#pragma unroll
    for (int D = 0; D < 16; ++D) put4(o, carry, q[D], nib4(nz[D >> 3], D & 7), sel0[D]);
    or_at(o, carry);
    const int kap = min(max((int)base + sc.kap, -16), 16);
    if (V + lane < G.sz) st[V + lane] = (unsigned char)((((u32)(13 + kap) + (sc.r == 7u)) & 15u) << 3 | ((sc.r + 1u) & 7u));
    CFENCE();
    u32 d[9], nb[2] = {0u, 0u}, bad = 0, nca = 0;
    if (hint) {
      u32 sel[9];
#pragma unroll
      for (u32 k = 0; k < 9u; ++k) d[k] = *(LAS u32*)(st + (k * G.nc + lane) * 4u);
#pragma unroll
      for (u32 k = 0; k < 9u; ++k) {
        u32 pos, up;
        const u32 u = low_exp(ob[k], (u32)(kap + 32) * B1, pos), df = ((d[k] >> 3) & 15u * B1) + 32u * B1 - u;
        d[k] &= 7u * B1;
        const u32 su = sure(d[k], r8, pos, up), w = ((df ^ 32u * B1) & ~B1) | ((df ^ up) & su);
        bad |= (k < 8u ? in8 : in9) ? w : 0u;
        nb[k >> 3] |= to_nib(su) << (4u * (k & 7u));
        ob[k] = df & B1;
      }
      nb[0] = in8 ? ~nb[0] : 0u; nb[1] = in9 ? ~nb[1] & 15u : 0u;
#pragma unroll
      for (u32 k = 0; k < 9u; ++k) sel[k] = put_sel(c, nib4(nb[k >> 3], k & 7u));
      const u32 cn = __popc(nb[0]) + __popc(nb[1]), in2 = wave_incl_scan(cn), Va = __builtin_amdgcn_readlane(in2, 63);
      nca = (Va + 31u) >> 5;
      hint = !ANY(bad != 0u) && G.sz + nca * 32u + 8u <= 4096u;
      if (hint) {
        o = ldsa(st) + G.sz + in2 - cn; carry = 0;
#pragma unroll
        for (u32 k = 0; k < 9u; ++k) put4(o, carry, ob[k], nib4(nb[k >> 3], k & 7u), sel[k]);
        or_at(o, carry);
      }
    }
    CFENCE();
    if (hint) {
      u32 bits = 0;
#pragma unroll
      for (u32 k = 0; k < 8u; ++k) bits |= *(LAS u32*)(st + G.sz + (k * nca + lane) * 4u) << k;
      CFENCE();
      LAS u32* s = (LAS u32*)st + lane;
      if (in8) {
        s[0] = d[0] | d[1] << 3 | ((d[6] << 6) & 0xC0u * B1); s[G.nc] = d[2] | d[3] << 3 | ((d[7] << 6) & 0xC0u * B1);
        s[2u * G.nc] = d[4] | d[5] << 3 | ((d[6] << 4) & 0x40u * B1) | ((d[7] << 5) & 0x80u * B1);
      }
      if (in9) s[3u * G.nc] = d[8];
      if ((u32)lane < nca) s[(G.sz >> 2) - 5u * G.nc] = bits;
      sb = (G.sz - 20u * G.nc + nca * 4u + 15u) & ~15u;
    } else {
      const u32 nc = row_bytes(V) >> 2;
      for (u32 col = lane; col < nc; col += 64u) {
        LAS u32* s = (LAS u32*)st + col;
        const u32 d7 = s[7u * nc];
#pragma unroll
        for (u32 k = 0; k < 7u; ++k) s[k * nc] |= (d7 << (7u - k)) & 0x80u * B1;
      }
      sb = (28u * nc + 15u) & ~15u;
    }
    u32x4 hdr;
    const u32 ovb = O_OV + ht * 16384, woff = place(c, wv, C_CSZ2 + zs, 16u, sb, (u32)hint << 16, hdr);
    if ((wv & 3) == 0 && lane == 0) st128<SYS>(hdr, rsL, ovb);
    for (u32 k = lane * 16; k < sb; k += 1024u) st128<SYS>(*(LAS u32x4*)(st + k), rsL, k + (ovb + woff));
    VMWAIT0();
    u32 t = 0;
    if (lane == 0) t = lds_add(c.ctl + C_CVAL0 + zs, 1u);
    if (RFL(t) == 3u) tell_peers(c, O_OF + c.rank * 8192 + F_VF + ht * 4, c.call << 20 | (hdr.w & 0xFFFFu) >> 4);
  }
  if (wv == 0) STAMP(c, 12);
  const rsrc_t rsO = mkrs(p.out, (u32)MM * NN);
  const u32x4 nan4 = {0x7Fu * B1, 0x7Fu * B1, 0x7Fu * B1, 0x7Fu * B1};
  {
    const u32 obase = (u32)(mi * 256 + mw * 64 + (lane & 15)) * NN + c.rank * 2048 + ni * 128 + h * 64 + (lane >> 4) * 16;
    const bool bad = tm == c.call || lds_ld(c.ctl + C_ABORT) != 0u;
#pragma unroll
    for (int b = 0; b < 4; ++b) st128<OUTAUX>(bad ? nan4 : (u32x4){q[4 * b], q[4 * b + 1], q[4 * b + 2], q[4 * b + 3]}, rsO, obase + b * 16 * NN);
  }
}

DI void role_outdec(Ctx& c, const Params& p, int wv) {
  const int lane = c.lane;
  __builtin_amdgcn_s_setprio(0);
  c.call = lds_wait(c, C_CALL, 1u);
  lds_wait(c, C_GVALID, 1u);
  const float amax = __uint_as_float(RFL(lds_ld(c.ctl + C_GMAX)));
  const Scale sc = split(__uint_as_float(RFL(__float_as_uint(amax > 0.f ? 448.0f / amax : 0.f))));
  const u32 r8 = (8u - sc.r) * B1;
  const rsrc_t rsL = mkrs(c.ws, WS_USED), rsO = mkrs(p.out, (u32)MM * NN);
  const u32x4 nan4 = {0x7Fu * B1, 0x7Fu * B1, 0x7Fu * B1, 0x7Fu * B1};
  LAS unsigned char* st = c.lds + L_RING + (wv < 8 ? wv * 4096 : 98304 + (wv - 8) * 4096);
  const u32 src = (c.rank + 1 + wg_link(c.wg)) & 7;
  for (u32 kv = wv >> 2;; kv += 3u) {
    const u32 e = q_wait(c, C_QV, kv), dh = e & 0xFFFu, pb = ((e >> 12) & 2047u) * 32768u;
    if (dh >= 512u) break;
    const u32 dt = dh >> 1, dw = (dh & 1u) * 4u + (wv & 3), inb = O_MIN + (src * 256 + dt) * 16384u;
    dma1(rsL, lane * 16, inb + 4128u + dw * 1024u, (u32)(size_t)st + 2560u);
    const u32x2 m = __builtin_amdgcn_raw_buffer_load_b64(rsL, lane * 8, (int)(inb + dw * 512u), AGT);
    const u32 he = __builtin_amdgcn_raw_buffer_load_b32(rsL, 0, (int)(inb + 4096u + dw * 4u), AGT);
    const u32 dbase = (u32)(tile_mi(dt) * 256 + (dw >> 1) * 64 + (lane & 15)) * NN + src * 2048 + tile_ni(dt) * 128 + (dw & 1) * 64 + (lane >> 4) * 16;
    const u32 cn = __popc(m.x) + __popc(m.y), in2 = wave_incl_scan(cn), V2 = __builtin_amdgcn_readlane(in2, 63);
    const Geo G2 = geo(V2);
    const bool j8 = (u32)lane < G2.nc, j9 = 8u * G2.nc + lane < G2.sz >> 2;
    u32 u[9], pos[9];
    VMWAIT0();
    {
      LAS u32* s = (LAS u32*)(st + 2560) + lane;
      const u32 ra = s[0], rb = s[G2.nc], rc = s[2u * G2.nc], l1[3] = {j8 ? ra : ~0u, j8 ? rb : ~0u, j9 ? rc : ~0u};
      const u32 k32 = (u32)(min(max((int)((he >> 16) & 255u) + sc.kap, -16), 16) + 32) * B1;
      dec_levels<9, 1024>(c, l1, k32, u, st + 2560 + G2.sz - 24u * G2.nc, st, 2560u);
#pragma unroll
      for (u32 k = 0; k < 9u; ++k) u[k] = low_exp(u[k], 0u, pos[k]);
    }
    LAS unsigned char* sl = c.lds + L_STG + (kv & 3u) * 16384u;
    lds_wait(c, C_PROG, pb + 16u);
    const u32 hw = RFL(*(LAS u32*)(sl + (wv & 3) * 4)), hp = RFL(*(LAS u32*)(sl + ((wv + 3) & 3) * 4));
    const u32 wo = (wv & 3) ? min(hp & 0xFFF0u, 16384u - 3584u) : 16u;
    lds_wait(c, C_PROG, pb + (hw & 0xFFFFu));
    const u32 tm2 = g_ld_agent(c.ws + O_TMO);
    if ((hw >> 16) & 1u) {
      LAS u32* sg = (LAS u32*)(sl + wo) + lane;
      const u32 ra = sg[0], rb = sg[G2.nc], rc = sg[2u * G2.nc], bits = sg[(G2.sz >> 2) - 5u * G2.nc];
      u32 y[9], nb[2] = {0u, 0u};
      y[8] = sg[3u * G2.nc] & 7u * B1;
      y[0] = ra & 7u * B1; y[1] = (ra >> 3) & 7u * B1; y[2] = rb & 7u * B1; y[3] = (rb >> 3) & 7u * B1; y[4] = rc & 7u * B1; y[5] = (rc >> 3) & 7u * B1;
      y[6] = ((ra >> 6) & 3u * B1) | ((rc >> 4) & 4u * B1); y[7] = ((rb >> 6) & 3u * B1) | ((rc >> 5) & 4u * B1);
#pragma unroll
      for (u32 k = 0; k < 9u; ++k) {
        u32 up;
        const u32 su = sure(y[k], r8, pos[k], up);
        u[k] += up & su;
        nb[k >> 3] |= to_nib(su) << (4u * (k & 7u));
      }
      nb[0] = j8 ? ~nb[0] : 0u; nb[1] = j9 ? ~nb[1] & 15u : 0u;
      const u32 ca = __popc(nb[0]) + __popc(nb[1]), ia = wave_incl_scan(ca), nca = min((__builtin_amdgcn_readlane(ia, 63) + 31u) >> 5, 64u);
      if ((u32)lane < nca) {
#pragma unroll
        for (u32 k = 0; k < 8u; ++k) *(LAS u32*)(st + (k * nca + lane) * 4u) = (bits >> k) & B1;
      }
      CFENCE();
      u32 bb[9];
      get_all<9>(c, ldsa(st) + ia - ca, nb, bb);
#pragma unroll
      for (u32 k = 0; k < 9u; ++k) y[k] |= (u[k] + bb[k]) << 3;
      CFENCE();
      if (j8) {
#pragma unroll
        for (u32 k = 0; k < 8u; ++k) *(LAS u32*)(st + (k * G2.nc + lane) * 4u) = y[k];
      }
      if (j9) *(LAS u32*)(st + (8u * G2.nc + lane) * 4u) = y[8];
    } else {
      const u32 nc = row_bytes(V2) >> 2;
      for (u32 col = lane; col < nc; col += 64u) {
        LAS u32* sg = (LAS u32*)(sl + wo) + col;
        LAS u32* d = (LAS u32*)st + col;
        u32 d7 = 0;
#pragma unroll
        for (u32 k = 0; k < 7u; ++k) { const u32 v = sg[k * nc]; d7 |= (v & 0x80u * B1) >> (7u - k); d[k * nc] = v & 0x7Fu * B1; }
        d[7u * nc] = d7;
      }
    }
    CFENCE();
    if (lane == 0) lds_add(c.ctl + C_DFREE0 + (kv & 3u), 1u);
    CFENCE();
    u32 ov[16];
    const u32 mm[2] = {m.x, m.y};
    get_all<16>(c, ldsa(st) + in2 - cn, mm, ov);
    const bool bad = tm2 == c.call || lds_ld(c.ctl + C_ABORT) != 0u;
#pragma unroll
    for (int k = 0; k < 4; ++k) st128<OUTAUX>(bad ? nan4 : (u32x4){ov[4 * k], ov[4 * k + 1], ov[4 * k + 2], ov[4 * k + 3]}, rsO, dbase + k * 16 * NN);
    CFENCE();
  }
}

__global__ __launch_bounds__(1024, 4) void fused_gemm_ar(Params p) {
  extern __shared__ __attribute__((aligned(16))) unsigned char smem[];
  Ctx c;
  c.lds = (LAS unsigned char*)smem; c.ctl = (LAS u32*)c.lds; c.ws = p.ws[p.rank]; c.t0 = now(); c.call = 0;
  c.lane = threadIdx.x & 63; c.wg = blockIdx.x; c.rank = p.rank;
  const int wv = RFL(threadIdx.x >> 6), tid = threadIdx.x;
  if (tid >= 512) *(LAS u32*)(c.lds + L_REL + (tid - 512) * 4) = 0u;
  else if (tid < C_WORDS || (tid >= L_GT / 4 && tid < L_GT / 4 + 16)) c.ctl[tid] = 0u;
  else if (tid >= 192 && tid < 256) {
    const u32 nib = tid & 15; u32 s = 0, k = (tid - 192) >> 4;
    for (int g = 0; g < 4; ++g) { if ((nib >> g) & 1u) s |= k++ << (8 * g); else s |= 0x0Cu << (8 * g); }
    *(LAS u32*)(c.lds + L_LUTE + (tid - 192) * 4) = s;
  } else if (tid >= 288 && tid < 288 + NR) {
    u64 v = 0;
#pragma unroll
    for (int i = 0; i < NR; ++i) { u64 t = (u64)p.ws[i]; asm volatile("" : "+s"(t)); v = tid == 288 + i ? t : v; }
    *(LAS u64*)(c.lds + L_PEER + (tid - 288) * 8) = v;
  } else if (tid >= 256 && tid < 272) {
    const u32 nib = tid - 256; u32 s = 0x0C0C0C0Cu, k = 0;
    for (u32 g = 0; g < 4; ++g) if ((nib >> g) & 1u) { s = (s & ~(255u << (8 * k))) | g << (8 * k); ++k; }
    *(LAS u32*)(c.lds + L_LUTC + nib * 4) = s;
  }
  asm volatile("s_waitcnt lgkmcnt(0)" ::: "memory");
  __builtin_amdgcn_s_barrier();
  CFENCE();
  const int mi = tile_mi(c.wg), ni = tile_ni(c.wg);
  if (wv == 11) STAMP(c, 0);
  if (wv < 8) role_mfma(c, p, wv, mi, ni);
  else if (wv < 13 - NDEC) role_loader(c, wv - 8, mi, ni);
  else if (wv == 11) role_sched(c, p, mi, ni);
  else if (wv == 12) role_puller(c, p);
  else if (wv == 13) role_feeder(c);
  else role_decoder(c, wv == 10 ? 2 : wv - 14);
  if (wv < 12) role_outdec(c, p, wv);
}
}

extern "C" void kernel_launch(int rank, int nranks, ncclComm_t comm, hipStream_t stream,
                              void* const* d_in, const int* in_sizes, int n_in,
                              void* d_out, int out_size, const int* in_dtypes, int out_dtype,
                              void* const* d_out_all, void* const* d_ws_all, size_t ws_bytes) {
  if (nranks != NR || (unsigned)rank >= (unsigned)NR || n_in < 2 || ws_bytes < WS_USED || in_sizes[0] != MM * KS || in_sizes[1] != KS * NN || out_size != MM * NN) {
    fprintf(stderr, "kernel_launch: unexpected problem shape\n");
    return;
  }
  static bool init = false;
  if (!init) {
    hipError_t e = hipFuncSetAttribute((const void*)fused_gemm_ar, hipFuncAttributeMaxDynamicSharedMemorySize, (int)LDS_BYTES);
    if (e != hipSuccess) { fprintf(stderr, "hipFuncSetAttribute: %s\n", hipGetErrorString(e)); return; }
    init = true;
  }
  Params p;
  p.x = (const unsigned short*)d_in[0]; p.w = (const unsigned short*)d_in[1]; p.out = (unsigned char*)d_out;
  for (int i = 0; i < NR; ++i) p.ws[i] = (char*)d_ws_all[i];
  p.rank = rank; p.pad = 0;
  fused_gemm_ar<<<NWG, 1024, LDS_BYTES, stream>>>(p);
}
```

```cpp
#include <hip/hip_runtime.h>
#include <rccl/rccl.h>
#include <cstdio>

typedef unsigned u32;
typedef unsigned long long u64;
typedef u32 u32x4 __attribute__((ext_vector_type(4)));
typedef u32 u32x2 __attribute__((ext_vector_type(2)));
typedef unsigned short u16x2 __attribute__((ext_vector_type(2)));
typedef short bf16x8 __attribute__((ext_vector_type(8)));
typedef float f32x4 __attribute__((ext_vector_type(4)));
typedef __amdgpu_buffer_rsrc_t rsrc_t;
#define LAS __attribute__((address_space(3)))
#define GAS __attribute__((address_space(1)))
#define DI __device__ __forceinline__
#define RFL(x) __builtin_amdgcn_readfirstlane(x)
#define ALL(x) (__builtin_amdgcn_ballot_w64(!(x)) == 0ull)
#define ANY(x) (__builtin_amdgcn_ballot_w64(x) != 0ull)
#define CFENCE() asm volatile("" ::: "memory")
#define VMWAIT0() asm volatile("s_waitcnt vmcnt(0)" ::: "memory")

namespace {
constexpr int NR = 8, MM = 4096, KS = 512, NN = 16384, NWG = 256;
constexpr int NBLK = 768;
constexpr u32 SYS = 17, AGT = 16;
constexpr int LEADER = 16;
constexpr u64 DEADLINE = 100000000ull;
constexpr u32 O_CALL = 0;
constexpr u32 O_TMO = 32768;
constexpr u32 O_TICK = O_TMO + 128;
constexpr u32 O_MQ = O_TICK + 2048;
constexpr u32 O_GT = O_MQ + 128;
constexpr u32 O_LT = O_GT + 128;
constexpr u32 O_RF = O_LT + 2048;
constexpr u32 O_FX = O_RF + NR * NBLK * 4;
constexpr u32 O_FW = O_FX + 16 * 64 * 16;
constexpr u32 O_OF = O_FW + 16 * 64 * 8;
constexpr u32 F_MF = 0;
constexpr u32 F_MI = 4096;
constexpr u32 F_VF = 6144;
constexpr u32 O_OBX = 1u << 20;
constexpr u32 O_OBW = O_OBX + 512 * 8192;
constexpr u32 O_IMG = O_OBW + 2048 * 8192;
constexpr u32 O_OM = O_IMG + 64 * 96 * 8192;
constexpr u32 O_OV = O_OM + 256 * 16384;
constexpr u32 O_MIN = O_OV + 512 * 16384;
constexpr u32 WS_USED = O_MIN + NR * 256 * 16384;
__host__ __device__ constexpr u32 piece_off(u32 pc) { return (pc & 255u) * 16384u + (pc < 256u ? 4096u : 0u); }
static_assert(O_OF + NR * 8192 <= O_OBX, "control block overflow");
constexpr u32 T_MASK = NBLK, T_VAL = T_MASK + 512, T_END = T_VAL + 512;
enum { C_CALL = 0, C_ABORT, C_WGMAX, C_CMAX, C_CMASK, C_CVAL0, C_CVAL1, C_GVALID, C_GMAX, C_PSTART, C_PROG, C_PACKED,
       C_FULL0 = 12, C_CSZ0 = 18, C_CSZ2 = 20, C_CPRE0 = 22, C_DFREE0 = 24, C_IFREE0 = 28, C_VSZ = 32, C_QI = 40, C_QV = 48, C_MQ0 = 56, C_ORDER = 64, C_WORDS = 128 };
constexpr u32 L_LUTC = C_WORDS * 4;
constexpr u32 L_GT = L_LUTC + 128;
constexpr u32 L_PEER = L_GT + 64;
constexpr u32 L_LUTE = L_LUTC + 256;
constexpr u32 L_REL = 1024;
constexpr u32 L_RFC = L_REL + 2048;
constexpr u32 NS = 5, SLOTB = 24576;
constexpr u32 L_RING = L_RFC + 4096;
constexpr u32 L_STG = L_RING + 32768;
constexpr u32 L_IN = L_RING + NS * SLOTB;
constexpr u32 LDS_BYTES = L_IN + 4 * 8192;
static_assert(LDS_BYTES <= 163840 && LDS_BYTES > 81920 && L_STG + 5 * 16384 <= L_IN, "one workgroup per CU");
constexpr int WIN1 = 2, WINM = 3, WIN2 = 4;
constexpr u32 PLEAD = 2048;
constexpr u32 T_AH = T_END - 80;
constexpr u32 KFIX = 2;
constexpr u32 T_W0 = 50;
constexpr u32 T_HALF = 352;
constexpr u32 T_WA = 240;
constexpr int NPUSH = 12, NRAW = 32;
constexpr u32 PACK_T0 = 200, PACK_DT = 100;
constexpr u32 SCHED_Q = 3;
constexpr u32 NDEC = 3;

struct Params { const unsigned short* x; const unsigned short* w; unsigned char* out; char* ws[NR]; int rank; int pad; };

DI u64 now() { return __builtin_amdgcn_s_memrealtime(); }
DI rsrc_t mkrs(const void* p, u32 bytes) { return __builtin_amdgcn_make_buffer_rsrc((void*)p, (short)0, (int)bytes, 0x00020000); }
template <u32 AUX> DI void st128(u32x4 v, rsrc_t rs, u32 off) { __builtin_amdgcn_raw_buffer_store_b128(v, rs, (int)off, 0, AUX); }
DI void st32(u32 v, rsrc_t rs, u32 off) { __builtin_amdgcn_raw_buffer_store_b32(v, rs, (int)off, 0, SYS); }
DI u32 prm(u32 a, u32 b, u32 s) { return __builtin_amdgcn_perm(a, b, s); }
DI u32 bfi(u32 m, u32 a, u32 b) { return (m & a) | (~m & b); }
DI u32 lds_ld(LAS u32* p) { return __hip_atomic_load(p, __ATOMIC_RELAXED, __HIP_MEMORY_SCOPE_WORKGROUP); }
DI void lds_st(LAS u32* p, u32 v) { __hip_atomic_store(p, v, __ATOMIC_RELAXED, __HIP_MEMORY_SCOPE_WORKGROUP); }
DI u32 lds_add(LAS u32* p, u32 v) { return __hip_atomic_fetch_add(p, v, __ATOMIC_RELAXED, __HIP_MEMORY_SCOPE_WORKGROUP); }
DI void lds_or(LAS unsigned char* p, u32 v) { __hip_atomic_fetch_or((LAS u32*)p, v, __ATOMIC_RELAXED, __HIP_MEMORY_SCOPE_WORKGROUP); }
DI u32 g_ld_agent(const char* p) { return __hip_atomic_load((GAS u32*)p, __ATOMIC_RELAXED, __HIP_MEMORY_SCOPE_AGENT); }
DI u32 g_ld_sys(const char* p) { return __hip_atomic_load((GAS u32*)p, __ATOMIC_RELAXED, __HIP_MEMORY_SCOPE_SYSTEM); }
DI void g_st_agent(char* p, u32 v) { __hip_atomic_store((GAS u32*)p, v, __ATOMIC_RELAXED, __HIP_MEMORY_SCOPE_AGENT); }
DI void g_st_sys(char* p, u32 v) { __hip_atomic_store((GAS u32*)p, v, __ATOMIC_RELAXED, __HIP_MEMORY_SCOPE_SYSTEM); }

struct Ctx {
  LAS unsigned char* lds; LAS u32* ctl; char* ws; u64 t0; u32 call; int lane, wg, rank;
};
DI void give_up(const Ctx& c) { if (c.lane == 0) { lds_st(c.ctl + C_ABORT, 1u); if (c.call) g_st_agent(c.ws + O_TMO, c.call); } }
DI bool expired(const Ctx& c, u32& spins) {
  if ((++spins & 15u) != 0u) return false;
  if (RFL(lds_ld(c.ctl + C_ABORT)) != 0u || RFL(g_ld_agent(c.ws + O_TMO)) == c.call || now() - c.t0 > DEADLINE) { give_up(c); return true; }
  return false;
}
DI bool lds_expired(const Ctx& c, u32& spins) {
  if ((++spins & 63u) == 0u && (RFL(lds_ld(c.ctl + C_ABORT)) != 0u || now() - c.t0 > DEADLINE + DEADLINE / 4)) { give_up(c); return true; }
  __builtin_amdgcn_s_sleep(1);
  return false;
}
DI u32 lds_wait(const Ctx& c, int word, u32 target) {
  u32 v, spins = 0;
  while ((v = RFL(lds_ld(c.ctl + word))) < target) if (lds_expired(c, spins)) break;
  CFENCE();
  return v;
}
DI u32 q_wait(const Ctx& c, int q, u32 k) {
  u32 v, spins = 0;
  while (((((v = RFL(lds_ld(c.ctl + q + (k & 7u)))) >> 23) - (k + 1u)) & 256u) != 0u) if (lds_expired(c, spins)) { v = ~0u; break; }
  CFENCE();
  return RFL(v);
}
DI void dma4(rsrc_t rs, u32 voff, u32 soff, u32 ldsdst) {
  u32 keep;
  asm volatile("s_nop 4\n\ts_mov_b32 %0, m0\n\ts_mov_b32 m0, %1\n\ts_nop 0\n\t"
               "buffer_load_dwordx4 %2, %3, %4 offen sc0 sc1 lds\n\t"
               "buffer_load_dwordx4 %2, %3, %4 offen offset:1024 sc0 sc1 lds\n\t"
               "buffer_load_dwordx4 %2, %3, %4 offen offset:2048 sc0 sc1 lds\n\t"
               "buffer_load_dwordx4 %2, %3, %4 offen offset:3072 sc0 sc1 lds\n\t"
               "s_mov_b32 m0, %0"
               : "=&s"(keep) : "s"(RFL(ldsdst)), "v"(voff), "s"(rs), "s"(RFL(soff)) : "memory");
}
DI void dma1(rsrc_t rs, u32 voff, u32 soff, u32 ldsdst) {
  u32 keep;
  asm volatile("s_nop 4\n\ts_mov_b32 %0, m0\n\ts_mov_b32 m0, %1\n\ts_nop 0\n\t"
               "buffer_load_dwordx4 %2, %3, %4 offen sc0 sc1 lds\n\t"
               "s_mov_b32 m0, %0"
               : "=&s"(keep) : "s"(RFL(ldsdst)), "v"(voff), "s"(rs), "s"(RFL(soff)) : "memory");
}
template <int CTRL, int ROWS> DI u32 dpp0(u32 v) { return (u32)__builtin_amdgcn_update_dpp(0, (int)v, CTRL, ROWS, 0xF, false); }
DI u32 wave_incl_scan(u32 v) {
  v += dpp0<0x111, 0xF>(v); v += dpp0<0x112, 0xF>(v); v += dpp0<0x114, 0xF>(v); v += dpp0<0x118, 0xF>(v);
  v += dpp0<0x142, 0xA>(v); v += dpp0<0x143, 0xC>(v);
  return v;
}
DI u32 wave_max(u32 v) {
  v = max(v, dpp0<0x111, 0xF>(v)); v = max(v, dpp0<0x112, 0xF>(v)); v = max(v, dpp0<0x114, 0xF>(v)); v = max(v, dpp0<0x118, 0xF>(v));
  v = max(v, dpp0<0x142, 0xA>(v)); v = max(v, dpp0<0x143, 0xC>(v));
  return __builtin_amdgcn_readlane(v, 63);
}
DI int tile_mi(int wg) { return ((wg & 7) >> 1) * 4 + ((wg >> 3) & 3); }
DI int tile_ni(int wg) { return (wg & 1) * 8 + (wg >> 5); }
DI int wg_late(int wg) { return (wg >> 4) & 1; }
DI int wg_link(int wg) { return (((wg & 15) | (wg >> 5) << 4) + 3 * wg_late(wg)) % 7; }
DI u32 link_wgs(int link, int late) { return (u32)(127 + 7 - (link + 7 - 3 * late) % 7) / 7u; }
DI u32 row_bytes(u32 V) { return (((V + 7u) >> 3) + 3u) & ~3u; }
DI void tell_peers(const Ctx& c, u32 off, u32 v) { if (c.lane < NR && c.lane != c.rank) g_st_sys(*(char* LAS*)(c.lds + L_PEER + c.lane * 8) + off, v); }
DI void image_flag(const Ctx& c, int kstep, int r) {
  if (c.lane == 0) g_st_agent(c.ws + (r < 64 ? O_FX + ((r >> 2) * 64 + kstep) * 16 + (r & 3) * 4 : O_FW + (((r - 64) >> 1) * 64 + kstep) * 8 + (r & 1) * 4), c.call);
}

constexpr u32 B1 = 0x01010101u;
DI u32 nib4(u32 m, int k) { return (k ? m >> (4 * k - 2) : m << 2) & 0x3Cu; }
DI u32 to_nib(u32 bits) { return __builtin_amdgcn_udot4(bits, 0x08040201u, 0u, false); }
DI u32 put_sel(const Ctx& c, u32 n4) { return *(LAS u32*)(c.lds + L_LUTC + n4); }
DI u32 ldsa(LAS unsigned char* p) { return (u32)(size_t)p; }
DI void or_at(u32 a, u32 v) { lds_or((LAS unsigned char*)(size_t)(a & ~3u), v); }
DI u32 put4(u32& a, u32& carry, u32 v, u32 n4, u32 sel) {
  const u32 c4 = prm(0u, v, sel);
  const u64 w = (u64)c4 << ((a << 3) & 24u);
  or_at(a, (u32)w | carry);
  carry = (u32)(w >> 32);
  a += __popc(n4);
  return c4;
}
template <u32 N> DI void get_all(const Ctx& c, u32 a, const u32* m, u32 (&x)[N]) {
  u32 sel[N], hi[N];
#pragma unroll
  for (u32 k = 0; k < N; ++k) {
    const u32 n4 = nib4(m[k >> 3], k & 7u);
    sel[k] = *(LAS u32*)(c.lds + L_LUTE + (((a << 6) & 0xC0u) | n4));
    LAS u32* d = (LAS u32*)(size_t)(a & ~3u);
    x[k] = d[0]; hi[k] = d[1];
    a += __popc(n4);
  }
  __builtin_amdgcn_sched_barrier(0);
#pragma unroll
  for (u32 k = 0; k < N; ++k) x[k] = prm(hi[k], x[k], sel[k]);
}
struct Lv { u32 rows2, nc2, nc3; };
template <u32 R> DI bool enc_tail(const Ctx& c, u32 V2, LAS unsigned char* s, u32 cap, Lv& g, u32& l2a, u32& l2b, u32& l3) {
  const u32 nc2 = (V2 + 4u * R - 1u) / (4u * R), b2 = R * nc2 * 4u;
  u32 e[R], sel[R], nb3 = 0;
  l2a = 0u; l2b = 0u;
#pragma unroll
  for (u32 r = 0; r < R; ++r) e[r] = *(LAS u32*)(s + (r * nc2 + c.lane) * 4u);
#pragma unroll
  for (u32 r = 0; r < R; ++r) {
    e[r] = (u32)c.lane < nc2 ? e[r] : 0u;
    const u32 lo3 = e[r] & 7u * B1, b3 = (e[r] >> 3) & B1;
    const u32 c2 = bfi((b3 << 8) - b3, prm(0x00000002u, 0x03000000u, lo3), prm(0u, 1u, lo3));
    if (r < 4u) l2a |= c2 << (2u * r); else l2b |= c2 << (2u * r - 8u);
    nb3 |= to_nib((c2 | c2 >> 1) & B1) << (4u * r);
  }
  nb3 = ~nb3 & (R == 8u ? ~0u : 0xFFFFu);
#pragma unroll
  for (u32 r = 0; r < R; ++r) sel[r] = put_sel(c, nib4(nb3, r));
  const u32 cn3 = __popc(nb3), in3 = wave_incl_scan(cn3), V3 = __builtin_amdgcn_readlane(in3, 63);
  g.rows2 = R; g.nc2 = nc2; g.nc3 = (V3 + 7u) >> 3;
  if (V3 > 512u || b2 + g.nc3 * 8u + 8u > cap) return false;
  u32 a = ldsa(s) + b2 + in3 - cn3, carry = 0;
#pragma unroll
  for (u32 r = 0; r < R; ++r) put4(a, carry, e[r], nib4(nb3, r), sel[r]);
  or_at(a, carry);
  CFENCE();
  l3 = *(LAS u32*)(s + b2 + c.lane * 4u) | *(LAS u32*)(s + b2 + (g.nc3 + c.lane) * 4u) << 4;
  return true;
}
template <u32 N, u32 MAX2> DI bool enc_levels(const Ctx& c, u32 (&n)[N], LAS unsigned char* s, u32 cap, Lv& g, u32& l2a, u32& l2b, u32& l3) {
  u32 nb[2] = {0u, 0u}, c1[N], sel[N];
#pragma unroll
  for (u32 k = 0; k < N; ++k) {
    const u32 m = ((n[k] + 3u * B1) >> 4) & B1;
    c1[k] = n[k] & (m + (m << 1));
    nb[k >> 3] |= to_nib(m) << (4u * (k & 7u));
  }
  nb[0] = ~nb[0]; nb[1] = ~nb[1] & (N == 16u ? ~0u : (1u << (4u * (N & 7u))) - 1u);
#pragma unroll
  for (u32 k = 0; k < N; ++k) sel[k] = put_sel(c, nib4(nb[k >> 3], k & 7u));
  const u32 cnt = __popc(nb[0]) + __popc(nb[1]), inc = wave_incl_scan(cnt), V2 = __builtin_amdgcn_readlane(inc, 63);
  if (V2 > MAX2 || V2 + 40u > cap) return false;
  u32 a = ldsa(s) + inc - cnt, carry = 0;
#pragma unroll
  for (u32 k = 0; k < N; ++k) put4(a, carry, n[k], nib4(nb[k >> 3], k & 7u), sel[k]);
  or_at(a, carry);
  CFENCE();
#pragma unroll
  for (u32 k = 0; k < N; ++k) n[k] = c1[k];
  if (MAX2 > 1024u && V2 > 1024u) return enc_tail<8>(c, V2, s, cap, g, l2a, l2b, l3);
  return enc_tail<4>(c, V2, s, cap, g, l2a, l2b, l3);
}
template <u32 R> DI void dec_tail(const Ctx& c, u32 V2, LAS unsigned char* in, LAS unsigned char* s, u32 cap) {
  const u32 nc2 = (V2 + 4u * R - 1u) / (4u * R), b2 = R * nc2 * 4u;
  const bool col = (u32)c.lane < nc2;
  u32 l2[2] = {*(LAS u32*)(in + c.lane * 4u), R == 8u ? *(LAS u32*)(in + (nc2 + c.lane) * 4u) : 0u}, nb3 = 0, e[R];
  const u32 l3 = *(LAS u32*)(in + R * nc2 + c.lane * 4u);
  l2[0] = col ? l2[0] : ~0u; l2[1] = col ? l2[1] : ~0u;
#pragma unroll
  for (u32 r = 0; r < R; ++r) nb3 |= (to_nib((l2[r >> 2] | l2[r >> 2] >> 1) & B1 << (2u * (r & 3u))) >> (2u * (r & 3u))) << (4u * r);
  nb3 = ~nb3 & (R == 8u ? ~0u : 0xFFFFu);
  const u32 cn3 = __popc(nb3), in3 = wave_incl_scan(cn3), V3 = __builtin_amdgcn_readlane(in3, 63), nc3 = (V3 + 7u) >> 3;
  if (V3 > 512u || b2 + nc3 * 8u + 8u > cap) return;
  if ((u32)c.lane < nc3) {
    *(LAS u32*)(s + b2 + c.lane * 4u) = ((l3 & 15u * B1) + 14u * B1) & 15u * B1;
    *(LAS u32*)(s + b2 + (nc3 + c.lane) * 4u) = (((l3 >> 4) & 15u * B1) + 14u * B1) & 15u * B1;
  }
  CFENCE();
  get_all<R>(c, ldsa(s) + b2 + in3 - cn3, &nb3, e);
  if (col) {
#pragma unroll
    for (u32 r = 0; r < R; ++r) *(LAS u32*)(s + (r * nc2 + c.lane) * 4u) = e[r] | prm(0u, 0x090A0E00u, (l2[r >> 2] >> (2u * (r & 3u))) & 3u * B1);
  }
}
template <u32 N, u32 MAX2> DI void dec_levels(const Ctx& c, const u32* l1, u32 add, u32 (&o)[N], LAS unsigned char* in, LAS unsigned char* s, u32 cap) {
  u32 nb[2] = {0u, 0u};
#pragma unroll
  for (u32 k = 0; k < N; ++k) nb[k >> 3] |= (to_nib((l1[k >> 2] | l1[k >> 2] >> 1) & B1 << (2u * (k & 3u))) >> (2u * (k & 3u))) << (4u * (k & 7u));
  nb[0] = ~nb[0]; nb[1] = ~nb[1] & (N == 16u ? ~0u : (1u << (4u * (N & 7u))) - 1u);
  const u32 cnt = __popc(nb[0]) + __popc(nb[1]), inc = wave_incl_scan(cnt), V2 = __builtin_amdgcn_readlane(inc, 63);
  if (V2 <= MAX2 && V2 + 40u <= cap) {
    if (MAX2 > 1024u && V2 > 1024u) dec_tail<8>(c, V2, in, s, cap); else dec_tail<4>(c, V2, in, s, cap);
  }
  CFENCE();
  get_all<N>(c, ldsa(s) + inc - cnt, nb, o);
#pragma unroll
  for (u32 k = 0; k < N; ++k) o[k] += prm(0u, 0x0D0C0B00u, (l1[k >> 2] >> (2u * (k & 3u))) & 3u * B1) + add;
}
DI u32 pick_top(u32 sum, u32 cnt) { return (u32)((float)sum * __builtin_amdgcn_rcpf((float)max(cnt, 1u)) + 1.83f); }

DI u32x4 piece(const u32 (&lo)[16], const u32 (&hi)[16], int t) {
  return (u32x4){prm(hi[2 * t], lo[2 * t], 0x05010400u), prm(hi[2 * t], lo[2 * t], 0x07030602u),
                 prm(hi[2 * t + 1], lo[2 * t + 1], 0x05010400u), prm(hi[2 * t + 1], lo[2 * t + 1], 0x07030602u)};
}
DI void pack_inputs(Ctx& c, const Params& p, int wv) {
  const int lane = c.lane;
  const rsrc_t rsL = mkrs(c.ws, WS_USED), rsX = mkrs(p.x, MM * KS * 2), rsW = mkrs(p.w, KS * NN * 2);
  LAS unsigned char* st = c.lds + L_RING + wv * 4096;
  const int hi7 = (c.wg & 15) | (c.wg >> 5) << 4, late = wg_late(c.wg);
  int q = wv == 0 ? hi7 * 2 + late : late ? 128 + wv * 128 + hi7 : 2560;
  if (q < 320) __builtin_amdgcn_s_setprio(3);
  if (q < NPUSH) {
    u32x4 v[8];
#pragma unroll
    for (int t = 0; t < 8; ++t) v[t] = __builtin_amdgcn_raw_buffer_load_b128(rsX, ((64 * q + (lane & 15)) * KS + 8 * (lane >> 4)) * 2 + 64 * (t >> 2), 16 * (t & 3) * KS * 2, 0);
#pragma unroll 1
    for (int l = 1; l <= NR; ++l) {
      const rsrc_t rsP = mkrs(p.ws[(c.rank + q + l) & 7], WS_USED);
#pragma unroll
      for (int t = 0; t < 8; ++t) st128<SYS>(v[t], rsP, O_IMG + (u32)(c.rank * 96 + q) * 8192u + t * 1024 + lane * 16);
      if (l == 6) asm volatile("s_waitcnt vmcnt(40)" ::: "memory");
    }
    c.call = lds_wait(c, C_CALL, 1u);
    VMWAIT0();
    tell_peers(c, O_FX + ((q >> 2) * 64 + c.rank) * 16 + (q & 3) * 4, c.call);
    image_flag(c, c.rank, q);
    q = late ? 1152 + hi7 : 2560;
  }
  for (; q < 2560; q = q < 256 ? (late ? 1152 + hi7 : 2560) : q + 1024) {
    const int j = q / 320, r = q - j * 320;
    if (j) { __builtin_amdgcn_s_setprio(0); while (now() - c.t0 < (u64)(PACK_T0 + PACK_DT * j)) __builtin_amdgcn_s_sleep(8); }
    const bool isx = r < 64;
    const int dest = (r - 64) & 7, ni = (r - 64) >> 4, kh = ((r - 64) >> 3) & 1;
    u32 lo[16], hi[16];
    *(LAS u32x4*)(st + lane * 16) = (u32x4){0u, 0u, 0u, 0u}; *(LAS u32x4*)(st + 1024 + lane * 16) = (u32x4){0u, 0u, 0u, 0u};
    if (isx) {
      u32x4 d[8];
      const int vo = ((64 * r + (lane & 15)) * KS + 64 * j + 8 * (lane >> 4)) * 2;
#pragma unroll
      for (int t = 0; t < 8; ++t) d[t] = __builtin_amdgcn_raw_buffer_load_b128(rsX, vo + 64 * (t >> 2), 16 * (t & 3) * KS * 2, 0);
      __builtin_amdgcn_sched_barrier(0);
#pragma unroll
      for (int t = 0; t < 8; ++t) {
        lo[2 * t] = prm(d[t].y, d[t].x, 0x06040200u); lo[2 * t + 1] = prm(d[t].w, d[t].z, 0x06040200u);
        hi[2 * t] = prm(d[t].y, d[t].x, 0x07050301u); hi[2 * t + 1] = prm(d[t].w, d[t].z, 0x07050301u);
      }
    } else {
      u32x2 d[2][8];
      const int vo = ((64 * j + 32 * kh + 8 * (lane >> 4)) * NN + dest * 2048 + ni * 128 + 4 * (lane & 15)) * 2;
#pragma unroll
      for (int k = 0; k < 8; ++k)
#pragma unroll
        for (int h = 0; h < 2; ++h) d[h][k] = __builtin_amdgcn_raw_buffer_load_b64(rsW, vo + 128 * h, k * NN * 2, 0);
      __builtin_amdgcn_sched_barrier(0);
#pragma unroll
      for (int h = 0; h < 2; ++h)
#pragma unroll
        for (int cc = 0; cc < 2; ++cc)
#pragma unroll
          for (int u = 0; u < 2; ++u) {
            const u32 a0 = d[h][4 * u][cc], a1 = d[h][4 * u + 1][cc], a2 = d[h][4 * u + 2][cc], a3 = d[h][4 * u + 3][cc];
            const u32 l0 = prm(a1, a0, 0x06020400u), l1 = prm(a3, a2, 0x06020400u), h0 = prm(a1, a0, 0x07030501u), h1 = prm(a3, a2, 0x07030501u);
            const int t = 4 * h + 2 * cc;
            lo[2 * t + u] = prm(l1, l0, 0x05040100u); lo[2 * t + 2 + u] = prm(l1, l0, 0x07060302u);
            hi[2 * t + u] = prm(h1, h0, 0x05040100u); hi[2 * t + 2 + u] = prm(h1, h0, 0x07060302u);
          }
    }
    const int lr = isx ? r : 64 + 2 * ni + kh;
    const bool own = isx || dest == c.rank, send = isx || dest != c.rank;
    if (own) {
      const u32 img = O_IMG + (u32)((j * 8 + c.rank) * 96 + lr) * 8192u + lane * 16;
#pragma unroll
      for (int t = 0; t < 8; ++t) st128<AGT>(piece(lo, hi, t), rsL, img + t * 1024);
    }
    u32 tot = 8192u;
    if (send) {
      const u32 slot = (isx ? O_OBX + (u32)(j * 64 + r) * 8192u : O_OBW + (u32)((dest * 8 + j) * 32 + 2 * ni + kh) * 8192u);
      u32 n[16], se = 0, any = 0, cnt = 0;
      if (q >= NRAW) {
#pragma unroll
      for (int k = 0; k < 16; ++k) { n[k] = bfi(B1, lo[k] >> 7, hi[k] << 1); se = __builtin_amdgcn_sad_u8(n[k], 0u, se); }
      const u32 base = min(max(pick_top(__builtin_amdgcn_readlane(wave_incl_scan(se), 63), 4096u), 13u) - 13u, 240u);
      const u32 bl = (base & 127u) * B1, bx = base < 128u ? 0x80u * B1 : 0u;
#pragma unroll
      for (int k = 0; k < 16; ++k) { n[k] = (((n[k] | 0x80u * B1) - bl) ^ (n[k] & 0x80u * B1)) ^ bx; any |= n[k]; }
      if (ANY((any & 0xF0u * B1) != 0u)) {
#pragma unroll
        for (int k = 0; k < 16; ++k) if (ANY((n[k] & 0xF0u * B1) != 0u)) {
#pragma unroll
          for (int b = 0; b < 4; ++b) {
            const u32 v = (n[k] >> (8 * b)) & 255u;
            const u64 m = __builtin_amdgcn_ballot_w64(v > 15u);
            const u32 pos = cnt + __builtin_amdgcn_mbcnt_hi((u32)(m >> 32), __builtin_amdgcn_mbcnt_lo((u32)m, 0u));
            if (v > 15u && pos < 31u) st32((u32)(k * 4 + b) << 14 | (u32)lane << 8 | ((v + base) & 255u), rsL, slot + 5124u + pos * 4u);
            cnt += (u32)__popcll(m);
          }
        }
      }
#pragma unroll
      for (int k = 0; k < 16; ++k) n[k] = ((n[k] & 15u * B1) + 2u * B1) & 15u * B1;
      Lv g; u32 l2a, l2b, l3;
      if (cnt <= 31u && enc_levels<16, 2048>(c, n, st, 2048u, g, l2a, l2b, l3)) {
#pragma unroll
        for (int t = 0; t < 4; ++t)
          st128<SYS>((u32x4){bfi(0x80u * B1, hi[4 * t], lo[4 * t]), bfi(0x80u * B1, hi[4 * t + 1], lo[4 * t + 1]),
                             bfi(0x80u * B1, hi[4 * t + 2], lo[4 * t + 2]), bfi(0x80u * B1, hi[4 * t + 3], lo[4 * t + 3])}, rsL, slot + t * 1024 + lane * 16);
        u32x4 c1;
#pragma unroll
        for (int t = 0; t < 4; ++t) c1[t] = n[4 * t] | n[4 * t + 1] << 2 | n[4 * t + 2] << 4 | n[4 * t + 3] << 6;
        st128<SYS>(c1, rsL, slot + 4096u + lane * 16);
        const u32 o2 = 5120u + ((cnt + 4u) & ~3u) * 4u;
        if (lane == 0) st32(base | cnt << 8, rsL, slot + 5120u);
        if ((u32)lane < g.nc2) { st32(l2a, rsL, slot + o2 + lane * 4); if (g.rows2 == 8u) st32(l2b, rsL, slot + o2 + (g.nc2 + lane) * 4u); }
        if ((u32)lane < g.nc3) st32(l3, rsL, slot + o2 + g.rows2 * g.nc2 + lane * 4);
        tot = (o2 + g.rows2 * g.nc2 + g.nc3 * 4u + 15u) & ~15u;
      } else VMWAIT0();
      }
      if (tot == 8192u) {
        asm volatile("" : "+v"(lo[0]), "+v"(lo[2]), "+v"(lo[4]), "+v"(lo[6]), "+v"(lo[8]), "+v"(lo[10]), "+v"(lo[12]), "+v"(lo[14]),
                          "+v"(lo[1]), "+v"(lo[3]), "+v"(lo[5]), "+v"(lo[7]), "+v"(lo[9]), "+v"(lo[11]), "+v"(lo[13]), "+v"(lo[15]));
#pragma unroll
        for (int t = 0; t < 8; ++t) st128<SYS>(piece(lo, hi, t), rsL, slot + t * 1024 + lane * 16);
      }
    }
    if (!c.call) c.call = lds_wait(c, C_CALL, 1u);
    VMWAIT0();
    if (send && (isx || lane == dest)) tell_peers(c, O_RF + (c.rank * NBLK + j * 96 + lr) * 4, c.call << 20 | tot >> 4);
    if (own) image_flag(c, j * 8 + c.rank, lr);
  }
  __builtin_amdgcn_s_setprio(0);
  c.call = lds_wait(c, C_CALL, 1u);
  if (lane == 0) lds_add(c.ctl + C_PACKED, 1u);
}

DI u32 ob_slot(int b, int me) { const int j = b / 96, r = b - j * 96; return r < 64 ? O_OBX + (u32)(j * 64 + r) * 8192u : O_OBW + (u32)((me * 8 + j) * 32 + r - 64) * 8192u; }
struct Order {
  u32 lb[NBLK];
  constexpr Order() : lb() {
    constexpr int left[4] = {0, 2, 4, 4}, phase[4] = {6, 5, 1, 3};
    int n = 0, next[4] = {1, 1, 1, 1};
    for (int r = 0; r < 96; ++r) lb[n++] = (u32)r;
    for (int j = 1; j < 16; ++j) {
      bool go[4] = {};
      for (int g = 0; g < 4; ++g) go[g] = next[g] < 8 && (j > 7 || (j * (7 - left[g]) + phase[g]) / 7 >= next[g]) && !(j <= 7 && next[g] > 7 - left[g]);
      if (j < 8) for (int r = 0; r < 32; ++r) lb[n++] = (u32)(j * 96 + 64 + r);
      for (int u = 0; u < 4; ++u)
        for (int v = 0; v < 4; ++v) {
          const int t = j > 7 ? v : u, g = j > 7 ? u : v;
          if (go[g]) for (int b = 0; b < 4; ++b) lb[n++] = (u32)(next[g] * 96 + t * 16 + g * 4 + b);
        }
      for (int g = 0; g < 4; ++g) next[g] += go[g];
    }
  }
};
__constant__ Order ORDER = Order();
DI int pull_order(u32 B, int link) {
  const u32 lb = ORDER.lb[B], r = lb % 96u;
  return (int)(B >= 96u && r < 64u ? lb - r + ((r + 16u * link) & 63u) : lb);
}
DI u32 early_first(u32 t) { return (t & 15u) | (t >> 7) << 4 | ((t >> 4) & 7u) << 5; }
DI void role_feeder(Ctx& c) {
  const int lane = c.lane;
  __builtin_amdgcn_s_setprio(3);
  c.call = lds_wait(c, C_CALL, 1u);
  const int link = wg_link(c.wg), src = (c.rank + 1 + link) & 7;
  const u32 ne = link_wgs(link, 0), late = wg_late(c.wg), grp = (c.wg >> 3) & 3;
  GAS u32* tk = (GAS u32*)(c.ws + O_TICK + link * 128);
  u32 kv = 0, tbase = (c.call << 12) - KFIX * ne - NPUSH;
  const u32 stop = grp == 0u ? 0u : grp == 1u ? T_HALF : T_MASK;
  bool miss = true, out = false;
  const rsrc_t rsL = mkrs(c.ws, WS_USED);
  u32 fb = O_RF + src * NBLK * 4, B = 0;
  for (u32 k = 0;; ++k) {
    if (k == (late ? 0u : KFIX - 1u)) {
      if (lane < 2) __hip_atomic_fetch_max(tk + lane * 256, c.call << 12, __ATOMIC_RELAXED, __HIP_MEMORY_SCOPE_AGENT);
      VMWAIT0();
    }
    const u32 ahead = B < T_AH ? 2u : 1u;
    if (k >= ahead) lds_wait(c, C_PSTART, k + 1u - ahead);
    if (k < KFIX && !late) {
      B = NPUSH + (u32)((c.wg & 15) | (c.wg >> 5) << 4) / 7u + k * ne;
      if (miss) { dma4(rsL, lane * 16, fb, (u32)(size_t)c.lds + L_RFC); VMWAIT0(); }
    } else for (;;) {
      if (!out && B >= stop) {
        if (grp != 1u) lds_wait(c, C_CMAX, 8u);
        out = true; tk += 256; tbase = (c.call << 12) - T_MASK; fb = O_OF + src * 8192; miss = true;
      }
      if (miss) dma4(rsL, lane * 16, fb, (u32)(size_t)c.lds + L_RFC);
      u32 t = 0;
      if (lane == 0) t = __hip_atomic_fetch_add(tk, 1u, __ATOMIC_RELAXED, __HIP_MEMORY_SCOPE_AGENT);
      VMWAIT0();
      B = RFL(t) - tbase;
      if (B < T_MASK || out) break;
    }
    if (B >= T_END) {
      if (lane < 3) lds_st(c.ctl + C_QI + ((k + lane) & 7u), (k + lane + 1u) << 23 | 0x7FFFFFu);
      if (kv >= 5u) lds_wait(c, C_DFREE0 + ((kv - 5u) & 3u), ((kv - 5u) & ~3u) + 4u);
      if (lane < 3) lds_st(c.ctl + C_QV + ((kv + lane) & 7u), (kv + lane + 1u) << 23 | 0x7FFFFFu);
      break;
    }
    u32 item, fo;
    if (B < T_MASK) { item = pull_order(B, link); fo = item * 4; }
    else if (B < T_VAL) { item = T_MASK; fo = F_MF + (B - T_MASK) * 4; }
    else {
      const u32 ht = early_first((B - T_VAL) >> 1) * 2u + (B & 1u);
      item = T_VAL + ht; fo = (lane & 1) ? (ht >> 1) * 4 : F_VF - F_MI + ht * 4;
      if (kv == 0u) fb += F_MI;
    }
    u32 v = B >= T_VAL && kv == 0u ? (c.call - 1u) << 20 : lds_ld((LAS u32*)(c.lds + L_RFC + fo)), spins = 0;
    miss = false;
    while (!ALL((int)((v & 0xFFF00000u) - (c.call << 20)) >= 0)) {
      miss = true;
      if (expired(c, spins)) break;
      if (spins > 1u) __builtin_amdgcn_s_sleep(1);
      v = g_ld_sys(c.ws + fb + fo);
    }
    if (item == T_MASK) { item += (v >> 10) & 511u; v &= 1023u; }
    if (lane == 0) {
      if (B >= T_VAL) lds_st(c.ctl + C_QV + (kv & 7u), (kv + 1u) << 23 | (k & 2047u) << 12 | (item - T_VAL));
      const u32 wide = B < T_W0 || (B >= T_HALF && B < T_MASK + T_WA);
      lds_st(c.ctl + C_QI + (k & 7u), (k + 1u) << 23 | item << 12 | wide << 11 | (v & 0x7FFu));
    }
    if (B >= T_VAL) ++kv;
  }
}
DI void role_decoder(Ctx& c, u32 d) {
  const int lane = c.lane, src = (c.rank + 1 + wg_link(c.wg)) & 7;
  __builtin_amdgcn_s_setprio(1);
  c.call = lds_wait(c, C_CALL, 1u);
  const rsrc_t rsL = mkrs(c.ws, WS_USED);
  for (u32 k = d;; k += NDEC) {
    const u32 e = q_wait(c, C_QI, k), lb = (e >> 12) & 2047u, tot = min((e & 0x7FFu) * 16u, 8192u);
    if (lb >= T_VAL) break;
    LAS unsigned char* sl = c.lds + L_IN + (k & 3u) * 8192u;
    if (lb >= T_MASK) {
      lds_wait(c, C_PROG, k * 32768u + tot);
      const u32 pc = lb - T_MASK, inb = O_MIN + src * 256 * 16384 + piece_off(pc) + lane * 16;
      const u32 hd = *(LAS u32*)(sl + (lane & 7) * 4);
      u32x4 v[8];
#pragma unroll
      for (int t = 0; t < 8; ++t)
        v[t] = *(LAS u32x4*)(sl + (pc >= 256u ? (t & 3) * 1024u : t ? min(__builtin_amdgcn_readlane(hd, t - 1) & 0x1FF0u, 7168u) : 32u) + lane * 16);
      CFENCE(); if (lane == 0) lds_add(c.ctl + C_IFREE0 + (k & 3u), 1u); CFENCE();
      if (pc < 256u && lane < 8) __builtin_amdgcn_raw_buffer_store_b32(hd, rsL, (int)(inb - lane * 12), 0, AGT);
#pragma unroll
      for (int t = 0; t < 8; ++t) if (t < 4 || pc < 256u) st128<AGT>(v[t], rsL, inb + (pc < 256u ? 32u : 0u) + t * 1024);
      VMWAIT0();
      if (lane == 0) __hip_atomic_fetch_add((GAS u32*)(c.ws + O_OF + src * 8192 + F_MI) + (pc & 255u), 1u << 19, __ATOMIC_RELAXED, __HIP_MEMORY_SCOPE_AGENT);
      continue;
    }
    const int j = lb / 96, r = lb - j * 96, kstep = j * 8 + src;
    const u32 img = O_IMG + (u32)(kstep * 96 + r) * 8192u + lane * 16;
    lds_wait(c, C_PROG, k * 32768u + tot);
    if (tot == 8192u) {
      u32x4 v[8];
#pragma unroll
      for (int t = 0; t < 8; ++t) v[t] = *(LAS u32x4*)(sl + t * 1024 + lane * 16);
      CFENCE(); if (lane == 0) lds_add(c.ctl + C_IFREE0 + (k & 3u), 1u); CFENCE();
#pragma unroll
      for (int t = 0; t < 8; ++t) st128<AGT>(v[t], rsL, img + t * 1024);
    } else {
      const u32x4 c1 = *(LAS u32x4*)(sl + 4096 + lane * 16);
      const u32 exc = *(LAS u32*)(sl + 5120 + (lane & 31) * 4);
      u32x4 sm[4];
#pragma unroll
      for (int t = 0; t < 4; ++t) sm[t] = *(LAS u32x4*)(sl + t * 1024 + lane * 16);
      const u32 hd = RFL(exc), cnt = (hd >> 8) & 31u, base4 = min(hd & 255u, 240u) * B1;
      u32 n[16], lo[16], hi[16];
      const u32 l1[4] = {c1.x, c1.y, c1.z, c1.w};
      dec_levels<16, 2048>(c, l1, base4, n, sl + 5120u + ((cnt + 4u) & ~3u) * 4u, sl + 6144u, 2048u);
      CFENCE(); if (lane == 0) lds_add(c.ctl + C_IFREE0 + (k & 3u), 1u); CFENCE();
      for (u32 x = 1; x <= cnt; ++x) {
        const u32 rec = __builtin_amdgcn_readlane(exc, x);
        const u32 kk = (rec >> 16) & 15u, sh = ((rec >> 14) & 3u) * 8u;
        const u32 m = ((rec >> 8) & 63u) == (u32)lane ? 255u << sh : 0u, val = (rec & 255u) << sh;
#pragma unroll
        for (int i = 0; i < 16; ++i) n[i] = bfi(kk == (u32)i ? m : 0u, val, n[i]);
      }
#pragma unroll
      for (int i = 0; i < 16; ++i) { hi[i] = bfi(0x80u * B1, sm[i >> 2][i & 3], n[i] >> 1); lo[i] = bfi(0x80u * B1, n[i] << 7, sm[i >> 2][i & 3]); }
#pragma unroll
      for (int t = 0; t < 8; ++t) st128<AGT>(piece(lo, hi, t), rsL, img + t * 1024);
    }
    VMWAIT0();
    image_flag(c, kstep, r);
  }
}
constexpr int WMAX = WIN2 - 1;
struct Stream {
  u32 pub = 0, pend[WMAX] = {};
  template <int W> DI void landed(const Ctx& c, u32 prog) {
    asm volatile("s_waitcnt vmcnt(%0)" :: "n"(W - 1) : "memory");
    pub = max(pub, pend[WMAX - (W - 1)]);
    if (c.lane == 0) lds_st(c.ctl + C_PROG, pub);
    for (int k = 0; k + 1 < WMAX; ++k) pend[k] = pend[k + 1];
    pend[WMAX - 1] = prog;
  }
  DI void flush(const Ctx& c) {
    VMWAIT0();
    pub = pend[WMAX - 1];
    if (c.lane == 0) lds_st(c.ctl + C_PROG, pub);
  }
};
DI void role_puller(Ctx& c, const Params& p) {
  __builtin_amdgcn_s_setprio(3);
  const u32 ldsb = (u32)(size_t)c.lds;
  const char* sws = p.ws[(c.rank + 1 + wg_link(c.wg)) & 7];
  Stream sm;
  u32 kv = 0;
  u32 cw = 0;
  for (u32 k = 0;; ++k) {
    u32 e = __builtin_amdgcn_readlane(cw, 0), fi = __builtin_amdgcn_readlane(cw, 1), fd = __builtin_amdgcn_readlane(cw, 2);
    if ((e >> 23) != ((k + 1u) & 511u)) {
      e = RFL(lds_ld(c.ctl + C_QI + (k & 7u)));
      if ((e >> 23) != ((k + 1u) & 511u)) {
        sm.flush(c);
        e = q_wait(c, C_QI, k);
      }
    }
    CFENCE();
    const u32 item = (e >> 12) & 2047u;
    if ((~e & 0x7FFFFFu) == 0u) break;
    u32 so, tot = (e & 0x7FFu) * 16u, dst;
    if (item < T_VAL) {
      if (fi < k >> 2) { sm.flush(c); lds_wait(c, C_IFREE0 + (k & 3u), k >> 2); }
      dst = L_IN + (k & 3u) * 8192u;
      so = item < T_MASK ? ob_slot(item, c.rank) : O_OM + piece_off(item - T_MASK); tot = min(tot, 8192u);
    } else {
      if (fd < (kv & ~3u)) { sm.flush(c); lds_wait(c, C_DFREE0 + (kv & 3u), kv & ~3u); }
      dst = L_STG + (kv & 3u) * 16384u; so = O_OV + (item - T_VAL) * 16384u; tot = min(tot, 16384u);
      ++kv;
    }
    so = RFL(so); tot = RFL(max(tot, 16u)); dst = RFL(dst);
    const rsrc_t rs = mkrs(sws + so, tot);
    const bool wide = (e >> 11) & 1u;
    const u32 qs = tot > PLEAD ? (tot - PLEAD) & ~1023u : 0u;
    for (u32 q = 0; q < tot; q += 1024u) {
      dma1(rs, c.lane * 16, q, ldsb + dst + q);
      if (q == qs && c.lane == 0) lds_st(c.ctl + C_PSTART, k + 1u);
      if (q + 1024u >= tot) cw = lds_ld(c.ctl + (c.lane == 0 ? C_QI + ((k + 1u) & 7u) : c.lane == 1 ? C_IFREE0 + ((k + 1u) & 3u) : C_DFREE0 + (kv & 3u)));
      if (wide) sm.landed<WIN2>(c, k * 32768u + q + 1024u);
      else if (item < T_MASK) sm.landed<WINM>(c, k * 32768u + q + 1024u);
      else sm.landed<WIN1>(c, k * 32768u + q + 1024u);
    }
  }
  sm.flush(c);
}

DI void role_sched(Ctx& c, const Params& p, int mi, int ni) {
  const int lane = c.lane;
  __builtin_amdgcn_s_setprio(3);
  u32 t = 0;
  if (lane == 0) t = __hip_atomic_fetch_add((GAS u32*)(c.ws + O_CALL + c.wg * 128), 1u, __ATOMIC_RELAXED, __HIP_MEMORY_SCOPE_AGENT) + 1u;
  c.call = RFL(t);
  if (lane == 0) lds_st(c.ctl + C_CALL, c.call);
  const rsrc_t rsL = mkrs(c.ws, WS_USED);
  u64 issued = 0; u32 n = 0, spins = 0;
  while (n < 64u) {
    if (2u * n >= RFL(lds_ld((LAS u32*)(c.lds + L_REL + 7 * 256))) + 2u * SCHED_Q) { if (!lds_expired(c, spins)) continue; }
    const u32x4 a = __builtin_amdgcn_raw_buffer_load_b128(rsL, lane * 16, (int)(O_FX + mi * 1024), SYS);
    const u32x2 b = __builtin_amdgcn_raw_buffer_load_b64(rsL, lane * 8, (int)(O_FW + ni * 512), SYS);
    const bool rdy = min(min(min(a.x, a.y), min(a.z, a.w)), min(b.x, b.y)) >= c.call;
    u64 m = __builtin_amdgcn_ballot_w64(rdy) & ~issued;
    if (m == 0) {
      if (!expired(c, spins)) { __builtin_amdgcn_s_sleep(1); CFENCE(); continue; }
      m = ~issued;
    }
    if ((m >> lane) & 1) lds_st(c.ctl + C_ORDER + n + __builtin_amdgcn_mbcnt_hi((u32)(m >> 32), __builtin_amdgcn_mbcnt_lo((u32)m, 0u)), lane + 1);
    issued |= m;
    n += (u32)__popcll(m);
    CFENCE();
  }
  if (c.wg == LEADER) {
    u32 mx;
    for (;;) {
      const u32x4 a = __builtin_amdgcn_raw_buffer_load_b128(rsL, lane * 16, (int)O_LT, SYS);
      const u32x4 b = __builtin_amdgcn_raw_buffer_load_b128(rsL, lane * 16, (int)(O_LT + 1024), SYS);
      mx = max(max(a.x, a.z), max(b.x, b.z));
      if (ALL(min(min(a.y, a.w), min(b.y, b.w)) >= c.call) || expired(c, spins)) break;
      CFENCE();
    }
    mx = wave_max(mx);
    if (lane < NR) __hip_atomic_store((GAS u64*)(*(char* LAS*)(c.lds + L_PEER + lane * 8) + O_GT + c.rank * 8), (u64)c.call << 32 | mx, __ATOMIC_RELAXED, __HIP_MEMORY_SCOPE_SYSTEM);
  }
  lds_wait(c, C_CMAX, 8u);
  const rsrc_t rsG = mkrs(c.ws + O_GT, 64u);
  u32 g;
  for (;;) {
    if (lane < 4) dma1(rsG, lane * 16, 0u, (u32)(size_t)c.lds + L_GT);
    asm volatile("s_waitcnt vmcnt(5)" ::: "memory");
    const u64 v = __hip_atomic_load((LAS u64*)(c.lds + L_GT) + (lane & 7), __ATOMIC_RELAXED, __HIP_MEMORY_SCOPE_WORKGROUP);
    g = (u32)v;
    if (ALL((u32)(v >> 32) >= c.call) || expired(c, spins)) break;
    __builtin_amdgcn_s_sleep(2);
  }
  g = wave_max(g);
  if (lane == 0) { lds_st(c.ctl + C_GMAX, g); CFENCE(); lds_st(c.ctl + C_GVALID, 1u); }
}

DI void role_loader(Ctx& c, u32 li, int mi, int ni) {
  __builtin_amdgcn_s_setprio(2);
  const rsrc_t rsL = mkrs(c.ws, WS_USED);
  const u32 voff = c.lane * 16, ldsb = (u32)(size_t)c.lds + L_RING;
  u32 pend = 0;
  lds_wait(c, C_PACKED, 8u);
  for (u32 m = li; m < 128u; m += 5u - NDEC) {
    const u32 slot = m % NS, kh = m & 1u;
    u32 o, spins = 0;
    for (;;) {
      const u32 vo = lds_ld(c.ctl + C_ORDER + (m >> 1)), vf = lds_ld((LAS u32*)(c.lds + L_REL + (c.lane & 7) * 256));
      o = RFL(vo);
      if (o != 0u && ALL(vf + (NS - 1u) >= m)) break;
      if (pend) { VMWAIT0(); if (c.lane == 0) lds_st(c.ctl + C_FULL0 + (pend - 1u) % NS, pend); pend = 0; continue; }
      if (lds_expired(c, spins)) break;
    }
    CFENCE();
    o = RFL(o);
    const u32 so = O_IMG + ((o - 1u) & 63u) * (96u * 8192u), sx = so + mi * 32768 + kh * 4096u, sw = so + 64 * 8192 + ni * 16384 + kh * 8192u, ld = ldsb + slot * SLOTB;
    dma4(rsL, voff, sw, ld + 16384u);
    if (pend) { asm volatile("s_waitcnt vmcnt(4)" ::: "memory"); if (c.lane == 0) lds_st(c.ctl + C_FULL0 + (pend - 1u) % NS, pend); }
    dma4(rsL, voff, sw + 4096u, ld + 20480u);
    dma4(rsL, voff, sx, ld); dma4(rsL, voff, sx + 8192u, ld + 4096u); dma4(rsL, voff, sx + 16384u, ld + 8192u); dma4(rsL, voff, sx + 24576u, ld + 12288u);
    pend = m + 1u;
  }
  VMWAIT0();
  if (c.lane == 0) lds_st(c.ctl + C_FULL0 + (pend - 1u) % NS, pend);
}

struct Geo { u32 nc, sz; };
DI Geo geo(u32 V) { return V <= 2048u ? Geo{row_bytes(V) >> 2, 8u * row_bytes(V)} : Geo{64u, (min(V, 2304u) + 3u) & ~3u}; }
struct Scale { u32 r; int kap; };
DI Scale split(float inv) { const u32 ib = __float_as_uint(inv), r = ((ib & 0x7FFFFFu) + (1u << 19)) >> 20; return Scale{r & 7u, (int)(ib >> 23) - 247 + (int)(r >> 3)}; }
DI u32 low_exp(u32 o, u32 kap32, u32& pos) {
  const u32 t = o + kap32, ge = (t >> 5) & B1;
  pos = ((t + 31u * B1) >> 6) & B1;
  return t & ((ge << 5) - ge);
}
DI u32 sure(u32 m, u32 r8, u32 pos, u32& up) {
  const u32 x = m + r8;
  up = (~x >> 3) & B1;
  return (((x ^ 8u * B1) + 15u * B1) >> 4) & pos;
}
DI u32 place(const Ctx& c, int wv, int cnt, u32 first, u32 sb, u32 extra, u32x4& hdr) {
  if (c.lane == 0) { lds_st(c.ctl + C_VSZ + wv, sb | extra); CFENCE(); lds_add(c.ctl + cnt, 1u); }
  lds_wait(c, cnt, 4u);
  const u32x4 z = *(LAS u32x4*)(c.ctl + C_VSZ + (wv & 4));
  hdr.x = z.x + first; hdr.y = z.y + (hdr.x & 0xFFFFu); hdr.z = z.z + (hdr.y & 0xFFFFu); hdr.w = z.w + (hdr.z & 0xFFFFu);
  return (((wv & 2) ? ((wv & 1) ? hdr.w : hdr.z) : ((wv & 1) ? hdr.y : hdr.x)) & 0xFFFFu) - sb;
}

DI void role_mfma(Ctx& c, const Params& p, int wv, int mi, int ni) {
  const int lane = c.lane, mw = wv >> 1, h = wv & 1;
  pack_inputs(c, p, wv);
  f32x4 acc[4][4];
#pragma unroll
  for (int g = 0; g < 4; ++g)
#pragma unroll
    for (int b = 0; b < 4; ++b) acc[g][b] = (f32x4){0.f, 0.f, 0.f, 0.f};
  bf16x8 x0, x1, x2, x3, wa0, wa1, wa2, wa3, wb0, wb1, wb2, wb3;
  u32 vx = (u32)(size_t)c.lds + L_RING + mw * 4096 + lane * 16, vw = (u32)(size_t)c.lds + L_RING + 16384 + h * 4096 + lane * 16;
  u32 vf = (u32)(size_t)(c.ctl + C_FULL0 + 2), done = 1, sl = 1, seen = 0, vs;
  const u32 relad = (u32)(size_t)c.lds + L_REL + wv * 256 + lane * 4;
#define MF(a, w, x) "v_mfma_f32_16x16x32_bf16 " a ", " w ", " x ", " a "\n\t"
#define HSTEP(c0, c1, c2, c3, n0, n1, n2, n3, m) do { \
    if (__builtin_expect(seen <= (m) + 1u, 0)) lds_wait(c, C_FULL0 + sl, (m) + 2u); \
    asm volatile(MF("%0", "%9", "%8") "ds_read_b128 %4, %13\n\t" MF("%1", "%10", "%8") "ds_read_b128 %5, %13 offset:1024\n\t" \
                 MF("%2", "%11", "%8") "ds_read_b128 %6, %13 offset:2048\n\t" MF("%3", "%12", "%8") "ds_read_b128 %7, %13 offset:3072" \
                 : "+v"(acc[0][0]), "+v"(acc[1][0]), "+v"(acc[2][0]), "+v"(acc[3][0]), "+v"(n0), "+v"(n1), "+v"(n2), "+v"(n3) \
                 : "v"(x0), "v"(c0), "v"(c1), "v"(c2), "v"(c3), "v"(vw) : "memory"); \
    const u32 d = sl == NS - 1u ? -(NS - 1u) * SLOTB : SLOTB, d4 = sl == NS - 2u ? -(NS - 1u) * 4u : 4u; \
    sl = sl == NS - 1u ? 0u : sl + 1u; \
    asm volatile("s_waitcnt lgkmcnt(7)\n\t" MF("%0", "%7", "%6") "ds_read_b128 %4, %11\n\t" MF("%1", "%8", "%6") "ds_read_b32 %5, %12\n\t" \
                 MF("%2", "%9", "%6") MF("%3", "%10", "%6") \
                 : "+v"(acc[0][1]), "+v"(acc[1][1]), "+v"(acc[2][1]), "+v"(acc[3][1]), "+v"(x0), "=&v"(vs) \
                 : "v"(x1), "v"(c0), "v"(c1), "v"(c2), "v"(c3), "v"(vx), "v"(vf) : "memory"); \
    asm volatile("s_waitcnt lgkmcnt(8)\n\t" MF("%0", "%8", "%7") "ds_read_b128 %4, %12 offset:1024\n\t" MF("%1", "%9", "%7") "v_add_u32 %5, %13, %5\n\t" \
                 MF("%2", "%10", "%7") "v_add_u32 %6, %14, %6\n\t" MF("%3", "%11", "%7") \
                 : "+v"(acc[0][2]), "+v"(acc[1][2]), "+v"(acc[2][2]), "+v"(acc[3][2]), "+v"(x1), "+v"(vw), "+v"(vf) \
                 : "v"(x2), "v"(c0), "v"(c1), "v"(c2), "v"(c3), "v"(vx), "s"(d), "s"(d4) : "memory"); \
    asm volatile("s_waitcnt lgkmcnt(8)\n\t" MF("%0", "%9", "%5") "ds_read_b128 %4, %6 offset:2048\n\tv_add_u32 %7, 1, %7\n\t" MF("%1", "%10", "%5") \
                 "s_waitcnt lgkmcnt(2)\n\tv_readfirstlane_b32 %8, %13\n\t" MF("%2", "%11", "%5") MF("%3", "%12", "%5") \
                 "ds_read_b128 %5, %6 offset:3072\n\tds_write_b32 %14, %7\n\tv_add_u32 %6, %15, %6" \
                 : "+v"(acc[0][3]), "+v"(acc[1][3]), "+v"(acc[2][3]), "+v"(acc[3][3]), "+v"(x2), "+v"(x3), "+v"(vx), "+v"(done), "=&s"(seen) \
                 : "v"(c0), "v"(c1), "v"(c2), "v"(c3), "v"(vs), "v"(relad), "s"(d) : "memory"); \
  } while (0)
  lds_wait(c, C_FULL0, 1u);
  asm volatile("ds_read_b128 %0, %9\n\tds_read_b128 %1, %9 offset:1024\n\tds_read_b128 %2, %9 offset:2048\n\tds_read_b128 %3, %9 offset:3072\n\t"
               "ds_read_b128 %4, %8\n\tds_read_b128 %5, %8 offset:1024\n\tds_read_b128 %6, %8 offset:2048\n\tds_read_b128 %7, %8 offset:3072\n\t"
               "ds_write_b32 %10, %11\n\ts_waitcnt lgkmcnt(0)"
               : "=&v"(wa0), "=&v"(wa1), "=&v"(wa2), "=&v"(wa3), "=&v"(x0), "=&v"(x1), "=&v"(x2), "=&v"(x3) : "v"(vx), "v"(vw), "v"(relad), "v"(done) : "memory");
  vx += SLOTB; vw += SLOTB;
  wb0 = wa0; wb1 = wa0; wb2 = wa0; wb3 = wa0;
  asm volatile("s_nop 1" : "+v"(vx), "+v"(vw), "+v"(wb0), "+v"(wb1), "+v"(wb2), "+v"(wb3));
  for (u32 m = 0; m < 126u; m += 2u) {
    HSTEP(wa0, wa1, wa2, wa3, wb0, wb1, wb2, wb3, m);
    HSTEP(wb0, wb1, wb2, wb3, wa0, wa1, wa2, wa3, m + 1u);
  }
  HSTEP(wa0, wa1, wa2, wa3, wb0, wb1, wb2, wb3, 126u);
#define MF4(b, x) asm volatile(MF("%0", "%5", "%4") MF("%1", "%6", "%4") MF("%2", "%7", "%4") MF("%3", "%8", "%4") \
    : "+v"(acc[0][b]), "+v"(acc[1][b]), "+v"(acc[2][b]), "+v"(acc[3][b]) : "v"(x), "v"(wb0), "v"(wb1), "v"(wb2), "v"(wb3))
  asm volatile("s_waitcnt lgkmcnt(0)" ::: "memory");
  MF4(0, x0); MF4(1, x1); MF4(2, x2); MF4(3, x3);
  asm volatile("s_nop 7\n\ts_nop 7" : "+v"(acc[0][2]), "+v"(acc[1][2]), "+v"(acc[2][2]), "+v"(acc[3][2]), "+v"(acc[0][3]), "+v"(acc[1][3]), "+v"(acc[2][3]), "+v"(acc[3][3]),
               "+v"(acc[0][0]), "+v"(acc[1][0]), "+v"(acc[2][0]), "+v"(acc[3][0]), "+v"(acc[0][1]), "+v"(acc[1][1]), "+v"(acc[2][1]), "+v"(acc[3][1]));
  const rsrc_t rsL = mkrs(c.ws, WS_USED);
  {
    float mx = 0.f;
#pragma unroll
    for (int e = 0; e < 64; e += 2) mx = fmaxf(fmaxf(mx, acc[e & 3][e >> 4][(e >> 2) & 3]), acc[(e & 3) + 1][e >> 4][(e >> 2) & 3]);
    const u32 wmx = wave_max(__float_as_uint(mx));
    if (lane == 0) {
      __hip_atomic_fetch_max(c.ctl + C_WGMAX, wmx, __ATOMIC_RELAXED, __HIP_MEMORY_SCOPE_WORKGROUP);
      CFENCE();
      if (lds_add(c.ctl + C_CMAX, 1u) == 7u)
        __hip_atomic_store((GAS u64*)(c.ws + O_LT + c.wg * 8), (u64)c.call << 32 | lds_ld(c.ctl + C_WGMAX), __ATOMIC_RELAXED, __HIP_MEMORY_SCOPE_AGENT);
    }
  }
  const u32 zs = wv >> 2, ht = c.wg * 2 + zs;
  u32 nz[2] = {0u, 0u};
#pragma unroll
  for (int e = 63; e >= 0; --e) nz[e >> 5] = __builtin_amdgcn_alignbit(nz[e >> 5], __float_as_uint(acc[e & 3][e >> 4][(e >> 2) & 3]), 31);
  nz[0] = ~nz[0]; nz[1] = ~nz[1];
  __builtin_amdgcn_raw_buffer_store_b64((u32x2){nz[0], nz[1]}, rsL, lane * 8, (int)(O_OM + c.wg * 16384 + wv * 512), SYS);
  const rsrc_t rsQ = mkrs(c.ws + O_MQ, 4u);
  u32 mqv = 0;
  if (wv == 7) mqv = __builtin_amdgcn_raw_ptr_buffer_atomic_add_i32(1, rsQ, lane * 4, 0, AGT);
  VMWAIT0();
  if (lane == 0) lds_add(c.ctl + C_CMASK, 1u);
  if (wv == 7) {
    lds_wait(c, C_CMASK, 8u);
    tell_peers(c, O_OF + c.rank * 8192 + F_MF + (RFL(mqv) & 511u) * 4u, c.call << 20 | (256u + c.wg) << 10 | 256u);
  }
  const u32 cnt = __popc(nz[0]) + __popc(nz[1]), inc = wave_incl_scan(cnt), V = __builtin_amdgcn_readlane(inc, 63);
  u32 o0 = ldsa(c.lds) + L_RING + wv * 4096 + inc - cnt;
  Geo G = geo(V);
  const bool in8 = (u32)lane < G.nc, in9 = 8u * G.nc + lane < G.sz >> 2;
  lds_wait(c, C_CMAX, 8u);
  LAS unsigned char* st = c.lds + L_RING + wv * 4096;
  u32 ob[9], base;
  bool hint;
  {
#pragma unroll
    for (int k = 0; k < 4; ++k) *(LAS u32x4*)(st + k * 1024 + lane * 16) = (u32x4){0u, 0u, 0u, 0u};
    u32 o = o0, se = 0, carry = 0;
#pragma unroll
    for (int z = 0; z < 16; z += 8) {
      u32 sel[8];
#pragma unroll
      for (int D = 0; D < 8; ++D) sel[D] = put_sel(c, nib4(nz[z >> 3], D));
#pragma unroll
      for (int D = 0; D < 8; ++D) {
        const int b = (z + D) >> 2, r = D & 3;
        const u32 h = prm(prm(__float_as_uint(acc[3][b][r]) << 1, __float_as_uint(acc[2][b][r]) << 1, 0x07030000u),
                          prm(__float_as_uint(acc[1][b][r]) << 1, __float_as_uint(acc[0][b][r]) << 1, 0x00000703u), 0x07060100u);
        se = __builtin_amdgcn_sad_u8(put4(o, carry, h, nib4(nz[z >> 3], D), sel[D]), 0u, se);
      }
    }
    or_at(o, carry);
    base = min(max(pick_top(__builtin_amdgcn_readlane(wave_incl_scan(se), 63), V), 13u) - 13u, 240u);
    if (V + lane < G.sz) st[V + lane] = (unsigned char)(base + 13u);
    CFENCE();
    u32 n[9], any = 0, l2a, l2b, l3, sb = 0;
    const u16x2 b2 = {(unsigned short)base, (unsigned short)base};
#pragma unroll
    for (u32 k = 0; k < 9u; ++k) ob[k] = *(LAS u32*)(st + (k * G.nc + lane) * 4u);
#pragma unroll
    for (u32 k = 0; k < 9u; ++k) {
      const u32 lo = __builtin_bit_cast(u32, __builtin_elementwise_sub_sat(__builtin_bit_cast(u16x2, ob[k] & 0x00FF00FFu), b2));
      const u32 hi = __builtin_bit_cast(u32, __builtin_elementwise_sub_sat(__builtin_bit_cast(u16x2, (ob[k] >> 8) & 0x00FF00FFu), b2));
      ob[k] = (k < 8u ? in8 : in9) ? lo | hi << 8 : 13u * B1;
      any |= ob[k];
      n[k] = (ob[k] + 2u * B1) & 15u * B1;
    }
    Lv g;
    hint = V <= 2304u && !ANY((any & 0xF0u * B1) != 0u) && enc_levels<9, 1024>(c, n, st + G.sz, 4096u - G.sz, g, l2a, l2b, l3);
    CFENCE();
    if (hint) {
      LAS u32* s = (LAS u32*)st + lane;
      if (in8) { s[0] = n[0] | n[1] << 2 | n[2] << 4 | n[3] << 6; s[G.nc] = n[4] | n[5] << 2 | n[6] << 4 | n[7] << 6; }
      if (in9) s[2u * G.nc] = n[8];
      s += (G.sz >> 2) - 6u * G.nc;
      if ((u32)lane < g.nc2) s[0] = l2a;
      if ((u32)lane < g.nc3) s[g.nc2] = l3;
      sb = (G.sz - 24u * G.nc + (g.nc2 + g.nc3) * 4u + 15u) & ~15u;
      if (sb > 1008u) { hint = false; sb = 0; }
    }
    if (lane == 0) { lds_st(c.ctl + C_VSZ + wv, sb | base << 16); CFENCE(); lds_add(c.ctl + C_CSZ0, 1u); }
    lds_wait(c, C_CSZ0, 8u);
    const u32 z = lds_ld(c.ctl + C_VSZ + (lane & 7)), end = wave_incl_scan(lane < 8 ? z & 0xFFFFu : 0u) + 32u;
    const u32 omb = O_OM + piece_off(c.wg), woff = __builtin_amdgcn_readlane(end, wv) - sb;
    if (wv == 0) {
      if (lane < 8) __builtin_amdgcn_raw_buffer_store_b32(end | (z & 0xFFFF0000u), rsL, (int)(omb + lane * 4), 0, SYS);
      mqv = __builtin_amdgcn_raw_ptr_buffer_atomic_add_i32(1, rsQ, lane * 4, 0, AGT);
    }
    if ((u32)lane * 16u < sb) st128<SYS>(*(LAS u32x4*)(st + lane * 16), rsL, omb + woff + lane * 16);
    VMWAIT0();
    u32 t = 0;
    if (lane == 0) { if (wv == 0) lds_st(c.ctl + C_MQ0, mqv); CFENCE(); t = lds_add(c.ctl + C_CPRE0, 1u); }
    if (RFL(t) == 7u)
      tell_peers(c, O_OF + c.rank * 8192 + F_MF + (RFL(lds_ld(c.ctl + C_MQ0)) & 511u) * 4u, c.call << 20 | c.wg << 10 | __builtin_amdgcn_readlane(end, 7) >> 4);
  }
  asm volatile("" : "+v"(nz[0]), "+v"(nz[1]), "+v"(o0), "+s"(G.nc));
  lds_wait(c, C_CPRE0, 8u);
#pragma unroll
  for (int k = 0; k < 4; ++k) *(LAS u32x4*)(st + k * 1024 + lane * 16) = (u32x4){0u, 0u, 0u, 0u};
  u32 sel0[16];
#pragma unroll
  for (int D = 0; D < 16; ++D) sel0[D] = put_sel(c, nib4(nz[D >> 3], D & 7));
#pragma unroll
  for (int g = 0; g < 4; ++g)
#pragma unroll
    for (int b = 0; b < 4; ++b) {
#pragma unroll
      for (int r = 0; r < 4; ++r) acc[g][b][r] = __builtin_amdgcn_fmed3f(acc[g][b][r], 0.f, __builtin_inff());
      asm volatile("" : "+v"(acc[g][b]));
    }
  lds_wait(c, C_GVALID, 1u);
  const u32 tm = g_ld_agent(c.ws + O_TMO);
  const float amax = __uint_as_float(RFL(lds_ld(c.ctl + C_GMAX)));
  const float inv = amax > 0.f ? 448.0f / amax : 0.f;
  const Scale sc = split(__uint_as_float(RFL(__float_as_uint(inv))));
  const u32 r8 = (8u - sc.r) * B1;
  u32 q[16];
#pragma unroll
  for (int b = 0; b < 4; ++b) {
    const f32x4 a0 = acc[0][b] * inv, a1 = acc[1][b] * inv, a2 = acc[2][b] * inv, a3 = acc[3][b] * inv;
#pragma unroll
    for (int r = 0; r < 4; ++r)
      q[4 * b + r] = (u32)__builtin_amdgcn_cvt_pk_fp8_f32(a2[r], a3[r], __builtin_amdgcn_cvt_pk_fp8_f32(a0[r], a1[r], 0, false), true);
  }
  {
    u32 o = o0, carry = 0, sb;
#pragma unroll
    for (int D = 0; D < 16; ++D) put4(o, carry, q[D], nib4(nz[D >> 3], D & 7), sel0[D]);
    or_at(o, carry);
    const int kap = min(max((int)base + sc.kap, -16), 16);
    if (V + lane < G.sz) st[V + lane] = (unsigned char)((((u32)(13 + kap) + (sc.r == 7u)) & 15u) << 3 | ((sc.r + 1u) & 7u));
    CFENCE();
    u32 d[9], nb[2] = {0u, 0u}, bad = 0, nca = 0;
    if (hint) {
      u32 sel[9];
#pragma unroll
      for (u32 k = 0; k < 9u; ++k) d[k] = *(LAS u32*)(st + (k * G.nc + lane) * 4u);
#pragma unroll
      for (u32 k = 0; k < 9u; ++k) {
        u32 pos, up;
        const u32 u = low_exp(ob[k], (u32)(kap + 32) * B1, pos), df = ((d[k] >> 3) & 15u * B1) + 32u * B1 - u;
        d[k] &= 7u * B1;
        const u32 su = sure(d[k], r8, pos, up), w = ((df ^ 32u * B1) & ~B1) | ((df ^ up) & su);
        bad |= (k < 8u ? in8 : in9) ? w : 0u;
        nb[k >> 3] |= to_nib(su) << (4u * (k & 7u));
        ob[k] = df & B1;
      }
      nb[0] = in8 ? ~nb[0] : 0u; nb[1] = in9 ? ~nb[1] & 15u : 0u;
#pragma unroll
      for (u32 k = 0; k < 9u; ++k) sel[k] = put_sel(c, nib4(nb[k >> 3], k & 7u));
      const u32 cn = __popc(nb[0]) + __popc(nb[1]), in2 = wave_incl_scan(cn), Va = __builtin_amdgcn_readlane(in2, 63);
      nca = (Va + 31u) >> 5;
      hint = !ANY(bad != 0u) && G.sz + nca * 32u + 8u <= 4096u;
      if (hint) {
        o = ldsa(st) + G.sz + in2 - cn; carry = 0;
#pragma unroll
        for (u32 k = 0; k < 9u; ++k) put4(o, carry, ob[k], nib4(nb[k >> 3], k & 7u), sel[k]);
        or_at(o, carry);
      }
    }
    CFENCE();
    if (hint) {
      u32 bits = 0;
#pragma unroll
      for (u32 k = 0; k < 8u; ++k) bits |= *(LAS u32*)(st + G.sz + (k * nca + lane) * 4u) << k;
      CFENCE();
      LAS u32* s = (LAS u32*)st + lane;
      if (in8) {
        s[0] = d[0] | d[1] << 3 | ((d[6] << 6) & 0xC0u * B1); s[G.nc] = d[2] | d[3] << 3 | ((d[7] << 6) & 0xC0u * B1);
        s[2u * G.nc] = d[4] | d[5] << 3 | ((d[6] << 4) & 0x40u * B1) | ((d[7] << 5) & 0x80u * B1);
      }
      if (in9) s[3u * G.nc] = d[8];
      if ((u32)lane < nca) s[(G.sz >> 2) - 5u * G.nc] = bits;
      sb = (G.sz - 20u * G.nc + nca * 4u + 15u) & ~15u;
    } else {
      const u32 nc = row_bytes(V) >> 2;
      for (u32 col = lane; col < nc; col += 64u) {
        LAS u32* s = (LAS u32*)st + col;
        const u32 d7 = s[7u * nc];
#pragma unroll
        for (u32 k = 0; k < 7u; ++k) s[k * nc] |= (d7 << (7u - k)) & 0x80u * B1;
      }
      sb = (28u * nc + 15u) & ~15u;
    }
    u32x4 hdr;
    const u32 ovb = O_OV + ht * 16384, woff = place(c, wv, C_CSZ2 + zs, 16u, sb, (u32)hint << 16, hdr);
    if ((wv & 3) == 0 && lane == 0) st128<SYS>(hdr, rsL, ovb);
    for (u32 k = lane * 16; k < sb; k += 1024u) st128<SYS>(*(LAS u32x4*)(st + k), rsL, k + (ovb + woff));
    VMWAIT0();
    u32 t = 0;
    if (lane == 0) t = lds_add(c.ctl + C_CVAL0 + zs, 1u);
    if (RFL(t) == 3u) tell_peers(c, O_OF + c.rank * 8192 + F_VF + ht * 4, c.call << 20 | (hdr.w & 0xFFFFu) >> 4);
  }
  const rsrc_t rsO = mkrs(p.out, (u32)MM * NN);
  const u32x4 nan4 = {0x7Fu * B1, 0x7Fu * B1, 0x7Fu * B1, 0x7Fu * B1};
  {
    const u32 obase = (u32)(mi * 256 + mw * 64 + (lane & 15)) * NN + c.rank * 2048 + ni * 128 + h * 64 + (lane >> 4) * 16;
    const bool bad = tm == c.call || lds_ld(c.ctl + C_ABORT) != 0u;
#pragma unroll
    for (int b = 0; b < 4; ++b) st128<0>(bad ? nan4 : (u32x4){q[4 * b], q[4 * b + 1], q[4 * b + 2], q[4 * b + 3]}, rsO, obase + b * 16 * NN);
  }
}

DI void role_outdec(Ctx& c, const Params& p, int wv) {
  const int lane = c.lane;
  __builtin_amdgcn_s_setprio(0);
  c.call = lds_wait(c, C_CALL, 1u);
  lds_wait(c, C_GVALID, 1u);
  const float amax = __uint_as_float(RFL(lds_ld(c.ctl + C_GMAX)));
  const Scale sc = split(__uint_as_float(RFL(__float_as_uint(amax > 0.f ? 448.0f / amax : 0.f))));
  const u32 r8 = (8u - sc.r) * B1;
  const rsrc_t rsL = mkrs(c.ws, WS_USED), rsO = mkrs(p.out, (u32)MM * NN);
  const u32x4 nan4 = {0x7Fu * B1, 0x7Fu * B1, 0x7Fu * B1, 0x7Fu * B1};
  LAS unsigned char* st = c.lds + L_RING + (wv < 8 ? wv * 4096 : 98304 + (wv - 8) * 4096);
  const u32 src = (c.rank + 1 + wg_link(c.wg)) & 7;
  for (u32 kv = wv >> 2;; kv += 3u) {
    const u32 e = q_wait(c, C_QV, kv), dh = e & 0xFFFu, pb = ((e >> 12) & 2047u) * 32768u;
    if (dh >= 512u) break;
    const u32 dt = dh >> 1, dw = (dh & 1u) * 4u + (wv & 3), inb = O_MIN + (src * 256 + dt) * 16384u;
    dma1(rsL, lane * 16, inb + 4128u + dw * 1024u, (u32)(size_t)st + 2560u);
    const u32x2 m = __builtin_amdgcn_raw_buffer_load_b64(rsL, lane * 8, (int)(inb + dw * 512u), AGT);
    const u32 he = __builtin_amdgcn_raw_buffer_load_b32(rsL, 0, (int)(inb + 4096u + dw * 4u), AGT);
    const u32 dbase = (u32)(tile_mi(dt) * 256 + (dw >> 1) * 64 + (lane & 15)) * NN + src * 2048 + tile_ni(dt) * 128 + (dw & 1) * 64 + (lane >> 4) * 16;
    const u32 cn = __popc(m.x) + __popc(m.y), in2 = wave_incl_scan(cn), V2 = __builtin_amdgcn_readlane(in2, 63);
    const Geo G2 = geo(V2);
    const bool j8 = (u32)lane < G2.nc, j9 = 8u * G2.nc + lane < G2.sz >> 2;
    u32 u[9], pos[9];
    VMWAIT0();
    {
      LAS u32* s = (LAS u32*)(st + 2560) + lane;
      const u32 ra = s[0], rb = s[G2.nc], rc = s[2u * G2.nc], l1[3] = {j8 ? ra : ~0u, j8 ? rb : ~0u, j9 ? rc : ~0u};
      const u32 k32 = (u32)(min(max((int)((he >> 16) & 255u) + sc.kap, -16), 16) + 32) * B1;
      dec_levels<9, 1024>(c, l1, k32, u, st + 2560 + G2.sz - 24u * G2.nc, st, 2560u);
#pragma unroll
      for (u32 k = 0; k < 9u; ++k) u[k] = low_exp(u[k], 0u, pos[k]);
    }
    LAS unsigned char* sl = c.lds + L_STG + (kv & 3u) * 16384u;
    lds_wait(c, C_PROG, pb + 16u);
    const u32 hw = RFL(*(LAS u32*)(sl + (wv & 3) * 4)), hp = RFL(*(LAS u32*)(sl + ((wv + 3) & 3) * 4));
    const u32 wo = (wv & 3) ? min(hp & 0xFFF0u, 16384u - 3584u) : 16u;
    lds_wait(c, C_PROG, pb + (hw & 0xFFFFu));
    const u32 tm2 = g_ld_agent(c.ws + O_TMO);
    if ((hw >> 16) & 1u) {
      LAS u32* sg = (LAS u32*)(sl + wo) + lane;
      const u32 ra = sg[0], rb = sg[G2.nc], rc = sg[2u * G2.nc], bits = sg[(G2.sz >> 2) - 5u * G2.nc];
      u32 y[9], nb[2] = {0u, 0u};
      y[8] = sg[3u * G2.nc] & 7u * B1;
      y[0] = ra & 7u * B1; y[1] = (ra >> 3) & 7u * B1; y[2] = rb & 7u * B1; y[3] = (rb >> 3) & 7u * B1; y[4] = rc & 7u * B1; y[5] = (rc >> 3) & 7u * B1;
      y[6] = ((ra >> 6) & 3u * B1) | ((rc >> 4) & 4u * B1); y[7] = ((rb >> 6) & 3u * B1) | ((rc >> 5) & 4u * B1);
#pragma unroll
      for (u32 k = 0; k < 9u; ++k) {
        u32 up;
        const u32 su = sure(y[k], r8, pos[k], up);
        u[k] += up & su;
        nb[k >> 3] |= to_nib(su) << (4u * (k & 7u));
      }
      nb[0] = j8 ? ~nb[0] : 0u; nb[1] = j9 ? ~nb[1] & 15u : 0u;
      const u32 ca = __popc(nb[0]) + __popc(nb[1]), ia = wave_incl_scan(ca), nca = min((__builtin_amdgcn_readlane(ia, 63) + 31u) >> 5, 64u);
      if ((u32)lane < nca) {
#pragma unroll
        for (u32 k = 0; k < 8u; ++k) *(LAS u32*)(st + (k * nca + lane) * 4u) = (bits >> k) & B1;
      }
      CFENCE();
      u32 bb[9];
      get_all<9>(c, ldsa(st) + ia - ca, nb, bb);
#pragma unroll
      for (u32 k = 0; k < 9u; ++k) y[k] |= (u[k] + bb[k]) << 3;
      CFENCE();
      if (j8) {
#pragma unroll
        for (u32 k = 0; k < 8u; ++k) *(LAS u32*)(st + (k * G2.nc + lane) * 4u) = y[k];
      }
      if (j9) *(LAS u32*)(st + (8u * G2.nc + lane) * 4u) = y[8];
    } else {
      const u32 nc = row_bytes(V2) >> 2;
      for (u32 col = lane; col < nc; col += 64u) {
        LAS u32* sg = (LAS u32*)(sl + wo) + col;
        LAS u32* d = (LAS u32*)st + col;
        u32 d7 = 0;
#pragma unroll
        for (u32 k = 0; k < 7u; ++k) { const u32 v = sg[k * nc]; d7 |= (v & 0x80u * B1) >> (7u - k); d[k * nc] = v & 0x7Fu * B1; }
        d[7u * nc] = d7;
      }
    }
    CFENCE();
    if (lane == 0) lds_add(c.ctl + C_DFREE0 + (kv & 3u), 1u);
    CFENCE();
    u32 ov[16];
    const u32 mm[2] = {m.x, m.y};
    get_all<16>(c, ldsa(st) + in2 - cn, mm, ov);
    const bool bad = tm2 == c.call || lds_ld(c.ctl + C_ABORT) != 0u;
#pragma unroll
    for (int k = 0; k < 4; ++k) st128<0>(bad ? nan4 : (u32x4){ov[4 * k], ov[4 * k + 1], ov[4 * k + 2], ov[4 * k + 3]}, rsO, dbase + k * 16 * NN);
    CFENCE();
  }
}

__global__ __launch_bounds__(1024, 4) void fused_gemm_ar(Params p) {
  extern __shared__ __attribute__((aligned(16))) unsigned char smem[];
  Ctx c;
  c.lds = (LAS unsigned char*)smem; c.ctl = (LAS u32*)c.lds; c.ws = p.ws[p.rank]; c.t0 = now(); c.call = 0;
  c.lane = threadIdx.x & 63; c.wg = blockIdx.x; c.rank = p.rank;
  const int wv = RFL(threadIdx.x >> 6), tid = threadIdx.x;
  if (tid >= 512) *(LAS u32*)(c.lds + L_REL + (tid - 512) * 4) = 0u;
  else if (tid < C_WORDS || (tid >= L_GT / 4 && tid < L_GT / 4 + 16)) c.ctl[tid] = 0u;
  else if (tid >= 192 && tid < 256) {
    const u32 nib = tid & 15; u32 s = 0, k = (tid - 192) >> 4;
    for (int g = 0; g < 4; ++g) { if ((nib >> g) & 1u) s |= k++ << (8 * g); else s |= 0x0Cu << (8 * g); }
    *(LAS u32*)(c.lds + L_LUTE + (tid - 192) * 4) = s;
  } else if (tid >= 288 && tid < 288 + NR) {
    u64 v = 0;
#pragma unroll
    for (int i = 0; i < NR; ++i) { u64 t = (u64)p.ws[i]; asm volatile("" : "+s"(t)); v = tid == 288 + i ? t : v; }
    *(LAS u64*)(c.lds + L_PEER + (tid - 288) * 8) = v;
  } else if (tid >= 256 && tid < 272) {
    const u32 nib = tid - 256; u32 s = 0x0C0C0C0Cu, k = 0;
    for (u32 g = 0; g < 4; ++g) if ((nib >> g) & 1u) { s = (s & ~(255u << (8 * k))) | g << (8 * k); ++k; }
    *(LAS u32*)(c.lds + L_LUTC + nib * 4) = s;
  }
  asm volatile("s_waitcnt lgkmcnt(0)" ::: "memory");
  __builtin_amdgcn_s_barrier();
  CFENCE();
  const int mi = tile_mi(c.wg), ni = tile_ni(c.wg);
  if (wv < 8) role_mfma(c, p, wv, mi, ni);
  else if (wv < 13 - NDEC) role_loader(c, wv - 8, mi, ni);
  else if (wv == 11) role_sched(c, p, mi, ni);
  else if (wv == 12) role_puller(c, p);
  else if (wv == 13) role_feeder(c);
  else role_decoder(c, wv == 10 ? 2 : wv - 14);
  if (wv < 12) role_outdec(c, p, wv);
}
}

extern "C" void kernel_launch(int rank, int nranks, ncclComm_t comm, hipStream_t stream,
                              void* const* d_in, const int* in_sizes, int n_in,
                              void* d_out, int out_size, const int* in_dtypes, int out_dtype,
                              void* const* d_out_all, void* const* d_ws_all, size_t ws_bytes) {
  if (nranks != NR || (unsigned)rank >= (unsigned)NR || n_in < 2 || ws_bytes < WS_USED || in_sizes[0] != MM * KS || in_sizes[1] != KS * NN || out_size != MM * NN) {
    fprintf(stderr, "kernel_launch: unexpected problem shape\n");
    return;
  }
  static bool init = false;
  if (!init) {
    hipError_t e = hipFuncSetAttribute((const void*)fused_gemm_ar, hipFuncAttributeMaxDynamicSharedMemorySize, (int)LDS_BYTES);
    if (e != hipSuccess) { fprintf(stderr, "hipFuncSetAttribute: %s\n", hipGetErrorString(e)); return; }
    init = true;
  }
  Params p;
  p.x = (const unsigned short*)d_in[0]; p.w = (const unsigned short*)d_in[1]; p.out = (unsigned char*)d_out;
  for (int i = 0; i < NR; ++i) p.ws[i] = (char*)d_ws_all[i];
  p.rank = rank; p.pad = 0;
  fused_gemm_ar<<<NWG, 1024, LDS_BYTES, stream>>>(p);
}
```
